```python
import numpy as np

kernel_name = "gpu_gemm_rs_m8192_k8192_n8192_bf16_none"


ABSMAX_RTOL = 0.008

COMPUTE_DTYPE = "bfloat16"
EPILOGUE = "none"
IN_QUANT = None
OUT_QUANT = None
IN_DTYPES = {"x": "bfloat16", "w_mat": "bfloat16"}
WORLD_SIZE = 8

def _to_bf16(x):
    u = np.ascontiguousarray(x, dtype=np.float32).view(np.uint32)
    return ((u + 0x7FFF + ((u >> 16) & 1)) & 0xFFFF0000).view(np.float32)

def setup_inputs():

    x = _to_bf16(np.random.randn(8192, 8192).astype(np.float32))
    w_mat = _to_bf16((np.random.randn(8192, 8192) / np.sqrt(8192)).astype(np.float32))
    return {"x": x, "w_mat": w_mat}

def _epilogue(y):
    return y

def reference(x, w_mat):
    y = (x @ w_mat).astype(np.float32)
    return _epilogue(y)
```

```cpp
#include <hip/hip_runtime.h>
#include <rccl/rccl.h>
#include <cstdio>
#include <cstdint>

#define HIP_CHECK(expr) do { hipError_t _e = (expr); if (_e != hipSuccess) { \
    fprintf(stderr, "HIP error %s at %s:%d\n", hipGetErrorString(_e), __FILE__, __LINE__); \
    return; } } while (0)

#define LAS __attribute__((address_space(3)))
typedef short bf16x8 __attribute__((ext_vector_type(8)));
typedef short s16x4 __attribute__((ext_vector_type(4)));
typedef float f32x4 __attribute__((ext_vector_type(4)));
typedef float f32x2 __attribute__((ext_vector_type(2)));
typedef unsigned u32x4 __attribute__((ext_vector_type(4)));
typedef __bf16 bf16x2_t __attribute__((ext_vector_type(2)));
typedef __amdgpu_buffer_rsrc_t rsrc_t;

namespace {

constexpr int NR = 8;
constexpr int MM = 8192, NN = 8192, KL = 1024;
constexpr int NWG = 256, NTHR = 512;
#define NGEMM 224
constexpr int BM = 256, BK = 64, HALF = 128, HTB = HALF * BK * 2, STAGE_BYTES = 8 * HTB;
constexpr int NT = KL / BK;
constexpr int NTILES = (MM / BM) * (NN / BM);
constexpr unsigned TILE_B = BM * BM * 2;
constexpr unsigned SLOT_B = 16u << 20;
constexpr size_t CTRL_OFF = (size_t)256 << 20;
constexpr size_t WS_NEED = CTRL_OFF + (1u << 20);
constexpr unsigned C_TICKET = 0, C_RED = 128, C_TMO = 256, C_FLAGS = 4096;
constexpr int NUNITS = 128 * 4;
constexpr unsigned long long DEADLINE = 30000000ull;
constexpr int LDS_BYTES = STAGE_BYTES + 256;
constexpr int AUX_SYS = 17;

struct Params {
    const void* x; const void* w; void* out;
    char* ws[NR];
    int rank;
};

__device__ __forceinline__ int lds_byte(int r, int c) { const int st = (r >> 4) * 2 + (c >> 5), rr = r & 15, cc = c & 31, ob = rr * 64 + cc * 2; return st * 1024 + (ob ^ (((ob >> 9) & 1) << 5)); }
__device__ __forceinline__ void stage_rc(int b, int& R, int& C) { const int st = b / 1024, sb = b % 1024, swz = sb ^ (((sb >> 9) & 1) << 5); R = (st >> 1) * 16 + swz / 64; C = (st & 1) * 32 + (swz % 64) / 2; }
__device__ __forceinline__ unsigned cvtpk(float lo, float hi) { f32x2 v = {lo, hi}; bf16x2_t b = __builtin_convertvector(v, bf16x2_t); return __builtin_bit_cast(unsigned, b); }
__device__ __forceinline__ rsrc_t mk_rsrc(const void* p, unsigned bytes) { return __builtin_amdgcn_make_buffer_rsrc((void*)p, 0, bytes, 0x00020000); }

__device__ __forceinline__ void tile_of(int i, int& pm, int& pn) { pn = i >> 5; pm = i & 31; }

#define TRD(dst, addr, off) asm volatile("ds_read_b64_tr_b16 %0, %1 offset:%c2" : "=&v"(dst) : "v"(addr), "i"(off) : "memory")
#define PKB(x) (bf16x8){(x)[0][0], (x)[0][1], (x)[0][2], (x)[0][3], (x)[1][0], (x)[1][1], (x)[1][2], (x)[1][3]}

__global__ __launch_bounds__(NTHR, 2) void gemm_rs_kernel(Params p) {
    extern __shared__ __attribute__((aligned(16))) unsigned char smem[];
    LAS unsigned char* lds = (LAS unsigned char*)smem;
    LAS unsigned* lds_ctl = (LAS unsigned*)(lds + STAGE_BYTES);
    const int tid = threadIdx.x, wid = __builtin_amdgcn_readfirstlane(tid >> 6), lane = tid & 63;
    const int wr = wid >> 2, wc = wid & 3, fr = lane & 15, fq = lane >> 4;
    const int wg = blockIdx.x, rank = p.rank;
    char* myws = p.ws[rank];
    unsigned* ctl = (unsigned*)(myws + CTRL_OFF);

    unsigned call = 0;
    if (tid == 0) {
        const unsigned t = __hip_atomic_fetch_add(ctl + C_TICKET / 4, 1u, __ATOMIC_RELAXED, __HIP_MEMORY_SCOPE_AGENT);
        lds_ctl[0] = t / NWG + 1;
    }

    if (wg < NGEMM) {
        const rsrc_t rx = mk_rsrc(p.x, MM * KL * 2), rw = mk_rsrc(p.w, KL * NN * 2);
        unsigned voffA, voffB;
        { int R, C; stage_rc(tid * 16, R, C); voffA = (unsigned)(R * KL + C) * 2u; }
        { const int row = tid >> 4, chp = tid & 15, ch = chp ^ (((row & 3) << 2) | (((row >> 3) & 1) << 1)); voffB = (unsigned)(row * NN + ch * 8) * 2u; }
        constexpr unsigned A_I = 64 * KL * 2, B_I = 32 * NN * 2;
        constexpr unsigned A_K = BK * 2, B_K = BK * NN * 2;
        constexpr unsigned A_H = HALF * KL * 2, B_H = HALF * 2;
        constexpr unsigned A_T = BM * KL * 2, B_T = BM * 2;
        const unsigned ldsw = (unsigned)wid * 1024u;
        const int aoff = lds_byte(wr * 64 + fr, fq * 8);
        unsigned boff[2];
        { const int q = (lane & 15) >> 2, pp = lane & 3, g = fq;
#pragma unroll
          for (int n = 0; n < 2; ++n) boff[n] = (unsigned)(size_t)lds + 4 * HTB + 256 * (8 * g + q) + 16 * (((wc ^ q) << 2) | ((n ^ (g & 1)) << 1) | (pp >> 1)) + 8 * (pp & 1); }
#define SA(b, h) (((b) * 2 + (h)) * HTB)
#define SB(b, h) ((4 + (b) * 2 + (h)) * HTB)
#define STAGE_A(bufoff, soff) do { _Pragma("unroll") for (int _i = 0; _i < 2; ++_i) \
        __builtin_amdgcn_raw_ptr_buffer_load_lds(rx, (LAS void*)(lds + (bufoff) + ldsw + _i * 8192), 16, voffA, (soff) + _i * A_I, 0, 0); } while (0)
#define STAGE_B(bufoff, soff) do { _Pragma("unroll") for (int _i = 0; _i < 2; ++_i) \
        __builtin_amdgcn_raw_ptr_buffer_load_lds(rw, (LAS void*)(lds + (bufoff) + ldsw + _i * 8192), 16, voffB, (soff) + _i * B_I, 0, 0); } while (0)
#define LDA(dst, b, h) do { _Pragma("unroll") for (int m = 0; m < 4; ++m) _Pragma("unroll") for (int k = 0; k < 2; ++k) dst[m][k] = *(const LAS bf16x8*)(lds + SA(b, h) + aoff + m * 2048 + k * 1024); } while (0)
#define LDB(dst, b, h) do { _Pragma("unroll") for (int n = 0; n < 2; ++n) _Pragma("unroll") for (int k = 0; k < 2; ++k) { \
        TRD(dst[n][k][0], boff[n], ((b) * 2 + (h)) * HTB + k * 8192); TRD(dst[n][k][1], boff[n], ((b) * 2 + (h)) * HTB + k * 8192 + 1024); } } while (0)
#define MMA(ai, bj, At, Bt) do { __builtin_amdgcn_s_setprio(1); _Pragma("unroll") for (int m = 0; m < 4; ++m) _Pragma("unroll") for (int n = 0; n < 2; ++n) _Pragma("unroll") for (int k = 0; k < 2; ++k) \
        acc[ai][bj][m][n] = __builtin_amdgcn_mfma_f32_16x16x32_bf16(PKB(Bt[n][k]), At[m][k], acc[ai][bj][m][n], 0, 0, 0); __builtin_amdgcn_s_setprio(0); } while (0)
#define WAIT_B(X, Y) asm volatile("s_waitcnt lgkmcnt(0)" : "+v"(X[0][0][0]), "+v"(X[0][0][1]), "+v"(X[0][1][0]), "+v"(X[0][1][1]), "+v"(X[1][0][0]), "+v"(X[1][0][1]), "+v"(X[1][1][0]), "+v"(X[1][1][1]), \
        "+v"(Y[0][0][0]), "+v"(Y[0][0][1]), "+v"(Y[0][1][0]), "+v"(Y[0][1][1]), "+v"(Y[1][0][0]), "+v"(Y[1][0][1]), "+v"(Y[1][1][0]), "+v"(Y[1][1][1]) :: "memory")
#define WAIT_V(n) asm volatile("s_waitcnt vmcnt(" #n ")" ::: "memory")
#define WAIT_L(n) asm volatile("s_waitcnt lgkmcnt(" #n ")" ::: "memory")
#define BAR __builtin_amdgcn_s_barrier()
#define SCHED __builtin_amdgcn_sched_barrier(0)
        int cur = wg, pm, pn;
        tile_of(cur, pm, pn);
        f32x4 acc[2][2][4][2];
#pragma unroll
        for (int a = 0; a < 2; ++a)
#pragma unroll
            for (int b = 0; b < 2; ++b)
#pragma unroll
                for (int m = 0; m < 4; ++m)
#pragma unroll
                    for (int n = 0; n < 2; ++n) acc[a][b][m][n] = (f32x4){0.f, 0.f, 0.f, 0.f};
        bf16x8 At[4][2]; s16x4 B0[2][2][2], B1[2][2][2];
        unsigned cA = (unsigned)pm * A_T, cB = (unsigned)pn * B_T;
        STAGE_B(SB(0, 0), cB); STAGE_B(SB(0, 1), cB + B_H); STAGE_A(SA(0, 0), cA); STAGE_A(SA(0, 1), cA + A_H);
        if (wr == 1) BAR;
        WAIT_V(2); BAR;
        STAGE_B(SB(1, 0), cB + B_K); STAGE_A(SA(1, 0), cA + A_K); STAGE_B(SB(1, 1), cB + B_H + B_K);
        WAIT_V(6); BAR;
        for (;;) {
            const int nxt = cur + NGEMM;
            const bool has_next = nxt < NTILES;
            int npm = pm, npn = pn;
            if (has_next) tile_of(nxt, npm, npn);
            const unsigned nA = (unsigned)npm * A_T, nB = (unsigned)npn * B_T;
            for (int t = 0; t < NT; t += 2) {
                const bool last = (t == NT - 2);
                const unsigned a1 = cA + (unsigned)(t + 1) * A_K;
                const unsigned a2 = last ? nA : cA + (unsigned)(t + 2) * A_K, b2 = last ? nB : cB + (unsigned)(t + 2) * B_K;
                const unsigned a3 = a2 + A_K, b3 = b2 + B_K;
                LDB(B0, 0, 0); LDB(B1, 0, 1); SCHED; LDA(At, 0, 0); STAGE_A(SA(1, 1), a1 + A_H);
                WAIT_V(8); WAIT_B(B0, B1); BAR; MMA(0, 0, At, B0); MMA(0, 1, At, B1); BAR; SCHED;
                LDA(At, 0, 1); STAGE_B(SB(0, 0), b2); STAGE_B(SB(0, 1), b2 + B_H); STAGE_A(SA(0, 0), a2);
                WAIT_V(8); WAIT_L(0); BAR; MMA(1, 0, At, B0); MMA(1, 1, At, B1); BAR; SCHED;
                LDB(B0, 1, 0); LDB(B1, 1, 1); SCHED; LDA(At, 1, 0); STAGE_A(SA(0, 1), a2 + A_H);
                WAIT_V(8); WAIT_B(B0, B1); BAR; MMA(0, 0, At, B0); MMA(0, 1, At, B1); BAR; SCHED;
                LDA(At, 1, 1); STAGE_B(SB(1, 0), b3); STAGE_B(SB(1, 1), b3 + B_H); STAGE_A(SA(1, 0), a3);
                WAIT_V(8); WAIT_L(0); BAR; MMA(1, 0, At, B0); MMA(1, 1, At, B1); BAR; SCHED;
            }
            if (wr == 0) BAR;
            {
                call = lds_ctl[0];
                const int owner = pm >> 2, tl = pn * 4 + (pm & 3);
                char* dws = p.ws[(unsigned)owner < (unsigned)NR ? owner : 0];
                const rsrc_t rd = mk_rsrc(dws + ((size_t)((call & 1) * NR + rank) * SLOT_B), SLOT_B);
                const unsigned so = (unsigned)tl * TILE_B + (unsigned)wid * 16384u;
#pragma unroll
                for (int ai = 0; ai < 2; ++ai)
#pragma unroll
                    for (int bj = 0; bj < 2; ++bj)
#pragma unroll
                        for (int m = 0; m < 4; ++m) {
                            const f32x4 v0 = acc[ai][bj][m][0], v1 = acc[ai][bj][m][1];
                            u32x4 w; w.x = cvtpk(v0[0], v0[1]); w.y = cvtpk(v0[2], v0[3]); w.z = cvtpk(v1[0], v1[1]); w.w = cvtpk(v1[2], v1[3]);
                            __builtin_amdgcn_raw_buffer_store_b128(w, rd, lane * 16, so + ((ai * 2 + bj) * 4 + m) * 1024, AUX_SYS);
                        }
                WAIT_V(0); BAR;
                if (tid == 0) {
                    unsigned* f = (unsigned*)(dws + CTRL_OFF + C_FLAGS + (size_t)(tl * NR + rank) * 128);
                    __hip_atomic_store(f, call, __ATOMIC_RELAXED, __HIP_MEMORY_SCOPE_SYSTEM);
                }
            }
            if (!has_next) break;
#pragma unroll
            for (int a = 0; a < 2; ++a)
#pragma unroll
                for (int b = 0; b < 2; ++b)
#pragma unroll
                    for (int m = 0; m < 4; ++m)
#pragma unroll
                        for (int n = 0; n < 2; ++n) acc[a][b][m][n] = (f32x4){0.f, 0.f, 0.f, 0.f};
            cur = nxt; pm = npm; pn = npn; cA = nA; cB = nB;
            if (wr == 1) BAR;
        }
        WAIT_V(0); BAR;
#undef SA
#undef SB
    } else {
        __syncthreads();
    }

    call = lds_ctl[0];
    {
        const rsrc_t rs = mk_rsrc(myws + (size_t)(call & 1) * NR * SLOT_B, NR * SLOT_B);
        const rsrc_t ro = mk_rsrc(p.out, (MM / NR) * NN * 2);
        const unsigned base = (call - 1) * (unsigned)(NUNITS + NWG);
        for (;;) {
            __syncthreads();
            if (wid == 0) {
                unsigned u = 0;
                if (lane == 0) u = __hip_atomic_fetch_add(ctl + C_RED / 4, 1u, __ATOMIC_RELAXED, __HIP_MEMORY_SCOPE_AGENT) - base;
                u = __builtin_amdgcn_readfirstlane(u);
                unsigned bad = 0;
                if (u < (unsigned)NUNITS) {
                    const unsigned* f = (const unsigned*)(myws + CTRL_OFF + C_FLAGS + (size_t)((u >> 2) * NR + (lane & 7)) * 128);
                    const unsigned long long t0 = __builtin_amdgcn_s_memrealtime();
                    for (;;) {
                        const unsigned v = __hip_atomic_load(f, __ATOMIC_RELAXED, __HIP_MEMORY_SCOPE_SYSTEM);
                        if (__all((int)(v - call) >= 0)) break;
                        if (__hip_atomic_load(ctl + C_TMO / 4, __ATOMIC_RELAXED, __HIP_MEMORY_SCOPE_AGENT) == call
                            || __builtin_amdgcn_s_memrealtime() - t0 > DEADLINE) {
                            if (lane == 0) __hip_atomic_store(ctl + C_TMO / 4, call, __ATOMIC_RELAXED, __HIP_MEMORY_SCOPE_AGENT);
                            break;
                        }
                        __builtin_amdgcn_s_sleep(4);
                    }
                    asm volatile("s_waitcnt vmcnt(0)" ::: "memory");
                    bad = __hip_atomic_load(ctl + C_TMO / 4, __ATOMIC_RELAXED, __HIP_MEMORY_SCOPE_AGENT) == call;
                }
                if (lane == 0) { lds_ctl[1] = u; lds_ctl[2] = bad; }
            }
            __syncthreads();
            const unsigned u = lds_ctl[1], bad = lds_ctl[2];
            if (u >= (unsigned)NUNITS) break;
            const unsigned tl = u >> 2, qt = u & 3;
            const unsigned so = tl * TILE_B + (unsigned)wid * 16384u + qt * 4096u;
            const int pn = tl >> 2, rt = tl & 3, ai = qt >> 1, bj = qt & 1;
            const unsigned orow = rt * 256 + ai * 128 + wr * 64 + fr;
            const unsigned ocol = pn * 256 + bj * 128 + wc * 32 + ((fq & 1) * 16 + (fq >> 1) * 8);
            u32x4 d[4][NR];
            if (!bad) {
#pragma unroll
                for (int m = 0; m < 4; ++m)
#pragma unroll
                    for (int s = 0; s < NR; ++s)
                        d[m][s] = __builtin_amdgcn_raw_buffer_load_b128(rs, lane * 16, so + s * SLOT_B + m * 1024, AUX_SYS);
            }
#pragma unroll
            for (int m = 0; m < 4; ++m) {
                u32x4 o;
                if (!bad) {
                    float a[8];
#pragma unroll
                    for (int e = 0; e < 8; ++e) a[e] = 0.f;
#pragma unroll
                    for (int s = 0; s < NR; ++s)
#pragma unroll
                        for (int j = 0; j < 4; ++j) {
                            a[2 * j] += __uint_as_float(d[m][s][j] << 16);
                            a[2 * j + 1] += __uint_as_float(d[m][s][j] & 0xffff0000u);
                        }
                    unsigned r0 = cvtpk(a[0], a[1]), r1 = cvtpk(a[2], a[3]), r2 = cvtpk(a[4], a[5]), r3 = cvtpk(a[6], a[7]);
                    auto s02 = __builtin_amdgcn_permlane16_swap(r0, r2, false, false);
                    auto s13 = __builtin_amdgcn_permlane16_swap(r1, r3, false, false);
                    o.x = s02[0]; o.y = s13[0]; o.z = s02[1]; o.w = s13[1];
                } else {
                    o = (u32x4){0x7fc07fc0u, 0x7fc07fc0u, 0x7fc07fc0u, 0x7fc07fc0u};
                }
                __builtin_amdgcn_raw_buffer_store_b128(o, ro, ((orow + m * 16) * NN + ocol) * 2, 0, 0);
            }
        }
    }
}

}

extern "C" void kernel_launch(int rank, int nranks, ncclComm_t comm,
                              hipStream_t stream,
                              void* const* d_in, const int* in_sizes, int n_in,
                              void* d_out, int out_size,
                              const int* in_dtypes, int out_dtype,
                              void* const* d_out_all,
                              void* const* d_ws_all, size_t ws_bytes) {
    (void)comm; (void)in_dtypes; (void)out_dtype; (void)d_out_all;
    if (nranks != NR || n_in < 2 || in_sizes[0] != MM * KL || in_sizes[1] != KL * NN || out_size != (MM / NR) * NN
        || ws_bytes < WS_NEED || (unsigned)rank >= (unsigned)NR) {
        fprintf(stderr, "kernel_launch: unexpected shapes\n");
        return;
    }
    static bool inited[NR] = {};
    if (!inited[rank]) {
        HIP_CHECK(hipFuncSetAttribute((const void*)gemm_rs_kernel, hipFuncAttributeMaxDynamicSharedMemorySize, LDS_BYTES));
        inited[rank] = true;
    }
    Params p;
    p.x = d_in[0]; p.w = d_in[1]; p.out = d_out; p.rank = rank;
    for (int i = 0; i < NR; ++i) p.ws[i] = (char*)d_ws_all[i];
    hipLaunchKernelGGL(gemm_rs_kernel, dim3(NWG), dim3(NTHR), LDS_BYTES, stream, p);
    HIP_CHECK(hipGetLastError());
}
```

```cpp
#include <hip/hip_runtime.h>
#include <rccl/rccl.h>
#include <cstdio>
#include <cstdint>

#define HIP_CHECK(expr) do { hipError_t _e = (expr); if (_e != hipSuccess) { \
    fprintf(stderr, "HIP error %s at %s:%d\n", hipGetErrorString(_e), __FILE__, __LINE__); \
    return; } } while (0)

#define LAS __attribute__((address_space(3)))
typedef short bf16x8 __attribute__((ext_vector_type(8)));
typedef short s16x4 __attribute__((ext_vector_type(4)));
typedef float f32x4 __attribute__((ext_vector_type(4)));
typedef float f32x2 __attribute__((ext_vector_type(2)));
typedef unsigned u32x4 __attribute__((ext_vector_type(4)));
typedef __bf16 bf16x2_t __attribute__((ext_vector_type(2)));
typedef __amdgpu_buffer_rsrc_t rsrc_t;

namespace {

constexpr int NR = 8;
constexpr int MM = 8192, NN = 8192, KL = 1024;
constexpr int NWG = 256, NTHR = 512;
#define NGEMM 224
#define PULL 1
#define RWIN 8
#define MCH 2
#define JOIN 0
constexpr int BM = 256, BK = 64, HALF = 128, HTB = HALF * BK * 2, STAGE_BYTES = 8 * HTB;
constexpr int NT = KL / BK;
constexpr int NTILES = (MM / BM) * (NN / BM);
constexpr unsigned TILE_B = BM * BM * 2;
constexpr unsigned SLOT_B = 16u << 20;
constexpr size_t CTRL_OFF = (size_t)256 << 20;
constexpr size_t WS_NEED = CTRL_OFF + (1u << 20);
constexpr unsigned C_TICKET = 0, C_RED = 128, C_TMO = 256, C_FLAGS = 4096;
constexpr int UPT = 16 / MCH;
constexpr int NUNITS = 128 * UPT;
constexpr int NCLAIM = JOIN ? NWG : NWG - NGEMM;
constexpr unsigned long long DEADLINE = 30000000ull;
constexpr int LDS_BYTES = STAGE_BYTES + 256;
constexpr int AUX_SYS = 17;

#define STAMP(slot) do {} while (0)
struct Params {
    const void* x; const void* w; void* out;
    char* ws[NR];
    int rank;
};

__device__ __forceinline__ int lds_byte(int r, int c) { const int st = (r >> 4) * 2 + (c >> 5), rr = r & 15, cc = c & 31, ob = rr * 64 + cc * 2; return st * 1024 + (ob ^ (((ob >> 9) & 1) << 5)); }
__device__ __forceinline__ void stage_rc(int b, int& R, int& C) { const int st = b / 1024, sb = b % 1024, swz = sb ^ (((sb >> 9) & 1) << 5); R = (st >> 1) * 16 + swz / 64; C = (st & 1) * 32 + (swz % 64) / 2; }
__device__ __forceinline__ unsigned cvtpk(float lo, float hi) { f32x2 v = {lo, hi}; bf16x2_t b = __builtin_convertvector(v, bf16x2_t); return __builtin_bit_cast(unsigned, b); }
__device__ __forceinline__ rsrc_t mk_rsrc(const void* p, unsigned bytes) { return __builtin_amdgcn_make_buffer_rsrc((void*)p, 0, bytes, 0x00020000); }

__device__ __forceinline__ void tile_of(int i, int& pm, int& pn) { pn = i >> 5; pm = i & 31; }

#define TRD(dst, addr, off) asm volatile("ds_read_b64_tr_b16 %0, %1 offset:%c2" : "=&v"(dst) : "v"(addr), "i"(off) : "memory")
#define PKB(x) (bf16x8){(x)[0][0], (x)[0][1], (x)[0][2], (x)[0][3], (x)[1][0], (x)[1][1], (x)[1][2], (x)[1][3]}

__global__ __launch_bounds__(NTHR, 2) void gemm_rs_kernel(Params p) {
    extern __shared__ __attribute__((aligned(16))) unsigned char smem[];
    LAS unsigned char* lds = (LAS unsigned char*)smem;
    LAS unsigned* lds_ctl = (LAS unsigned*)(lds + STAGE_BYTES);
    const int tid = threadIdx.x, wid = __builtin_amdgcn_readfirstlane(tid >> 6), lane = tid & 63;
    const int wr = wid >> 2, wc = wid & 3, fr = lane & 15, fq = lane >> 4;
    const int wg = blockIdx.x, rank = p.rank;
    char* myws = p.ws[rank];
    unsigned* ctl = (unsigned*)(myws + CTRL_OFF);

    unsigned call = 0;
    int dslot = 0;
    STAMP(dslot); ++dslot;
    if (tid == 0) {
        const unsigned t = __hip_atomic_fetch_add(ctl + C_TICKET / 4, 1u, __ATOMIC_RELAXED, __HIP_MEMORY_SCOPE_AGENT);
        lds_ctl[0] = t / NWG + 1;
    }

    if (wg < NGEMM) {
        const rsrc_t rx = mk_rsrc(p.x, MM * KL * 2), rw = mk_rsrc(p.w, KL * NN * 2);
        unsigned voffA, voffB;
        { int R, C; stage_rc(tid * 16, R, C); voffA = (unsigned)(R * KL + C) * 2u; }
        { const int row = tid >> 4, chp = tid & 15, ch = chp ^ (((row & 3) << 2) | (((row >> 3) & 1) << 1)); voffB = (unsigned)(row * NN + ch * 8) * 2u; }
        constexpr unsigned A_I = 64 * KL * 2, B_I = 32 * NN * 2;
        constexpr unsigned A_K = BK * 2, B_K = BK * NN * 2;
        constexpr unsigned A_H = HALF * KL * 2, B_H = HALF * 2;
        constexpr unsigned A_T = BM * KL * 2, B_T = BM * 2;
        const unsigned ldsw = (unsigned)wid * 1024u;
        const int aoff = lds_byte(wr * 64 + fr, fq * 8);
        unsigned boff[2];
        { const int q = (lane & 15) >> 2, pp = lane & 3, g = fq;
#pragma unroll
          for (int n = 0; n < 2; ++n) boff[n] = (unsigned)(size_t)lds + 4 * HTB + 256 * (8 * g + q) + 16 * (((wc ^ q) << 2) | ((n ^ (g & 1)) << 1) | (pp >> 1)) + 8 * (pp & 1); }
#define SA(b, h) (((b) * 2 + (h)) * HTB)
#define SB(b, h) ((4 + (b) * 2 + (h)) * HTB)
#define STAGE_A(bufoff, soff) do { _Pragma("unroll") for (int _i = 0; _i < 2; ++_i) \
        __builtin_amdgcn_raw_ptr_buffer_load_lds(rx, (LAS void*)(lds + (bufoff) + ldsw + _i * 8192), 16, voffA, (soff) + _i * A_I, 0, 0); } while (0)
#define STAGE_B(bufoff, soff) do { _Pragma("unroll") for (int _i = 0; _i < 2; ++_i) \
        __builtin_amdgcn_raw_ptr_buffer_load_lds(rw, (LAS void*)(lds + (bufoff) + ldsw + _i * 8192), 16, voffB, (soff) + _i * B_I, 0, 0); } while (0)
#define LDA(dst, b, h) do { _Pragma("unroll") for (int m = 0; m < 4; ++m) _Pragma("unroll") for (int k = 0; k < 2; ++k) dst[m][k] = *(const LAS bf16x8*)(lds + SA(b, h) + aoff + m * 2048 + k * 1024); } while (0)
#define LDB(dst, b, h) do { _Pragma("unroll") for (int n = 0; n < 2; ++n) _Pragma("unroll") for (int k = 0; k < 2; ++k) { \
        TRD(dst[n][k][0], boff[n], ((b) * 2 + (h)) * HTB + k * 8192); TRD(dst[n][k][1], boff[n], ((b) * 2 + (h)) * HTB + k * 8192 + 1024); } } while (0)
#define MMA(ai, bj, At, Bt) do { __builtin_amdgcn_s_setprio(1); _Pragma("unroll") for (int m = 0; m < 4; ++m) _Pragma("unroll") for (int n = 0; n < 2; ++n) _Pragma("unroll") for (int k = 0; k < 2; ++k) \
        acc[ai][bj][m][n] = __builtin_amdgcn_mfma_f32_16x16x32_bf16(PKB(Bt[n][k]), At[m][k], acc[ai][bj][m][n], 0, 0, 0); __builtin_amdgcn_s_setprio(0); } while (0)
#define WAIT_B(X, Y) asm volatile("s_waitcnt lgkmcnt(0)" : "+v"(X[0][0][0]), "+v"(X[0][0][1]), "+v"(X[0][1][0]), "+v"(X[0][1][1]), "+v"(X[1][0][0]), "+v"(X[1][0][1]), "+v"(X[1][1][0]), "+v"(X[1][1][1]), \
        "+v"(Y[0][0][0]), "+v"(Y[0][0][1]), "+v"(Y[0][1][0]), "+v"(Y[0][1][1]), "+v"(Y[1][0][0]), "+v"(Y[1][0][1]), "+v"(Y[1][1][0]), "+v"(Y[1][1][1]) :: "memory")
#define WAIT_V(n) asm volatile("s_waitcnt vmcnt(" #n ")" ::: "memory")
#define WAIT_L(n) asm volatile("s_waitcnt lgkmcnt(" #n ")" ::: "memory")
#define BAR __builtin_amdgcn_s_barrier()
#define SCHED __builtin_amdgcn_sched_barrier(0)
        int cur = wg, pm, pn;
        tile_of(cur, pm, pn);
        f32x4 acc[2][2][4][2];
#pragma unroll
        for (int a = 0; a < 2; ++a)
#pragma unroll
            for (int b = 0; b < 2; ++b)
#pragma unroll
                for (int m = 0; m < 4; ++m)
#pragma unroll
                    for (int n = 0; n < 2; ++n) acc[a][b][m][n] = (f32x4){0.f, 0.f, 0.f, 0.f};
        bf16x8 At[4][2]; s16x4 B0[2][2][2], B1[2][2][2];
        unsigned cA = (unsigned)pm * A_T, cB = (unsigned)pn * B_T;
        STAGE_B(SB(0, 0), cB); STAGE_B(SB(0, 1), cB + B_H); STAGE_A(SA(0, 0), cA); STAGE_A(SA(0, 1), cA + A_H);
        if (wr == 1) BAR;
        WAIT_V(2); BAR;
        STAGE_B(SB(1, 0), cB + B_K); STAGE_A(SA(1, 0), cA + A_K); STAGE_B(SB(1, 1), cB + B_H + B_K);
        WAIT_V(6); BAR;
        for (;;) {
            const int nxt = cur + NGEMM;
            const bool has_next = nxt < NTILES;
            int npm = pm, npn = pn;
            if (has_next) tile_of(nxt, npm, npn);
            const unsigned nA = (unsigned)npm * A_T, nB = (unsigned)npn * B_T;
            for (int t = 0; t < NT; t += 2) {
                const bool last = (t == NT - 2);
                const unsigned a1 = cA + (unsigned)(t + 1) * A_K;
                const unsigned a2 = last ? nA : cA + (unsigned)(t + 2) * A_K, b2 = last ? nB : cB + (unsigned)(t + 2) * B_K;
                const unsigned a3 = a2 + A_K, b3 = b2 + B_K;
                LDB(B0, 0, 0); LDB(B1, 0, 1); SCHED; LDA(At, 0, 0); STAGE_A(SA(1, 1), a1 + A_H);
                WAIT_V(8); WAIT_B(B0, B1); BAR; MMA(0, 0, At, B0); MMA(0, 1, At, B1); BAR; SCHED;
                LDA(At, 0, 1); STAGE_B(SB(0, 0), b2); STAGE_B(SB(0, 1), b2 + B_H); STAGE_A(SA(0, 0), a2);
                WAIT_V(8); WAIT_L(0); BAR; MMA(1, 0, At, B0); MMA(1, 1, At, B1); BAR; SCHED;
                LDB(B0, 1, 0); LDB(B1, 1, 1); SCHED; LDA(At, 1, 0); STAGE_A(SA(0, 1), a2 + A_H);
                WAIT_V(8); WAIT_B(B0, B1); BAR; MMA(0, 0, At, B0); MMA(0, 1, At, B1); BAR; SCHED;
                LDA(At, 1, 1); STAGE_B(SB(1, 0), b3); STAGE_B(SB(1, 1), b3 + B_H); STAGE_A(SA(1, 0), a3);
                WAIT_V(8); WAIT_L(0); BAR; MMA(1, 0, At, B0); MMA(1, 1, At, B1); BAR; SCHED;
            }
            if (wr == 0) BAR;
            STAMP(dslot); ++dslot;
            {
                call = lds_ctl[0];
                const int owner = pm >> 2, tl = pn * 4 + (pm & 3);
                char* dws = p.ws[(unsigned)owner < (unsigned)NR ? owner : 0];
                const rsrc_t rd = PULL ? mk_rsrc(myws + ((size_t)((call & 1) * NR + owner) * SLOT_B), SLOT_B)
                                       : mk_rsrc(dws + ((size_t)((call & 1) * NR + rank) * SLOT_B), SLOT_B);
                const unsigned so = (unsigned)tl * TILE_B + (unsigned)wid * 16384u;
#pragma unroll
                for (int ai = 0; ai < 2; ++ai)
#pragma unroll
                    for (int bj = 0; bj < 2; ++bj)
#pragma unroll
                        for (int m = 0; m < 4; ++m) {
                            const f32x4 v0 = acc[ai][bj][m][0], v1 = acc[ai][bj][m][1];
                            u32x4 w; w.x = cvtpk(v0[0], v0[1]); w.y = cvtpk(v0[2], v0[3]); w.z = cvtpk(v1[0], v1[1]); w.w = cvtpk(v1[2], v1[3]);
                            __builtin_amdgcn_raw_buffer_store_b128(w, rd, lane * 16, so + ((ai * 2 + bj) * 4 + m) * 1024, AUX_SYS);
                        }
                WAIT_V(0); BAR;
                STAMP(dslot); ++dslot;
                if (tid == 0) {
                    unsigned* f = (unsigned*)(dws + CTRL_OFF + C_FLAGS + (size_t)(tl * NR + rank) * 128);
                    __hip_atomic_store(f, call, __ATOMIC_RELAXED, __HIP_MEMORY_SCOPE_SYSTEM);
                }
            }
            if (!has_next) break;
#pragma unroll
            for (int a = 0; a < 2; ++a)
#pragma unroll
                for (int b = 0; b < 2; ++b)
#pragma unroll
                    for (int m = 0; m < 4; ++m)
#pragma unroll
                        for (int n = 0; n < 2; ++n) acc[a][b][m][n] = (f32x4){0.f, 0.f, 0.f, 0.f};
            cur = nxt; pm = npm; pn = npn; cA = nA; cB = nB;
            if (wr == 1) BAR;
        }
        WAIT_V(0); BAR;
#undef SA
#undef SB
    } else {
        __syncthreads();
    }

    call = lds_ctl[0];
    dslot = 16;
    STAMP(dslot); ++dslot;
    if (JOIN || wg >= NGEMM) {
        rsrc_t rs[NR];
#pragma unroll
        for (int s = 0; s < NR; ++s) {
            const int src = (rank + s) & (NR - 1);
            rs[s] = PULL ? mk_rsrc(p.ws[src] + (size_t)((call & 1) * NR + rank) * SLOT_B, SLOT_B)
                         : mk_rsrc(myws + (size_t)((call & 1) * NR + src) * SLOT_B, SLOT_B);
        }
        const rsrc_t ro = mk_rsrc(p.out, (MM / NR) * NN * 2);
        const unsigned base = (call - 1) * (unsigned)(NUNITS + NCLAIM);
        for (;;) {
            __syncthreads();
            if (wid == 0) {
                unsigned u = 0;
                if (lane == 0) u = __hip_atomic_fetch_add(ctl + C_RED / 4, 1u, __ATOMIC_RELAXED, __HIP_MEMORY_SCOPE_AGENT) - base;
                u = __builtin_amdgcn_readfirstlane(u);
                unsigned bad = 0;
                if (u < (unsigned)NUNITS) {
                    const unsigned* f = (const unsigned*)(myws + CTRL_OFF + C_FLAGS + (size_t)((u / UPT) * NR + (lane & 7)) * 128);
                    const unsigned long long t0 = __builtin_amdgcn_s_memrealtime();
                    for (;;) {
                        const unsigned v = __hip_atomic_load(f, __ATOMIC_RELAXED, __HIP_MEMORY_SCOPE_SYSTEM);
                        if (__all((int)(v - call) >= 0)) break;
                        if (__hip_atomic_load(ctl + C_TMO / 4, __ATOMIC_RELAXED, __HIP_MEMORY_SCOPE_AGENT) == call
                            || __builtin_amdgcn_s_memrealtime() - t0 > DEADLINE) {
                            if (lane == 0) __hip_atomic_store(ctl + C_TMO / 4, call, __ATOMIC_RELAXED, __HIP_MEMORY_SCOPE_AGENT);
                            break;
                        }
                        __builtin_amdgcn_s_sleep(4);
                    }
                    asm volatile("s_waitcnt vmcnt(0)" ::: "memory");
                    bad = __hip_atomic_load(ctl + C_TMO / 4, __ATOMIC_RELAXED, __HIP_MEMORY_SCOPE_AGENT) == call;
                }
                if (lane == 0) { lds_ctl[1] = u; lds_ctl[2] = bad; }
            }
            __syncthreads();
            const unsigned u = lds_ctl[1], bad = lds_ctl[2];
            STAMP(dslot); ++dslot;
            if (u >= (unsigned)NUNITS) { STAMP(63); break; }
            const unsigned tl = u / UPT, c0 = (u % UPT) * MCH;
            const unsigned so = tl * TILE_B + (unsigned)wid * 16384u + c0 * 1024u;
            const int pn = tl >> 2, rt = tl & 3, ai = c0 >> 3, bj = (c0 >> 2) & 1, m0 = c0 & 3;
            const unsigned orow = rt * 256 + ai * 128 + wr * 64 + m0 * 16 + fr;
            const unsigned ocol = pn * 256 + bj * 128 + wc * 32 + ((fq & 1) * 16 + (fq >> 1) * 8);
            const unsigned ooff = (orow * NN + ocol) * 2;
            if (bad) {
#pragma unroll
                for (int m = 0; m < MCH; ++m)
                    __builtin_amdgcn_raw_buffer_store_b128((u32x4){0x7fc07fc0u, 0x7fc07fc0u, 0x7fc07fc0u, 0x7fc07fc0u}, ro, ooff, m * 16 * NN * 2, 0);
                continue;
            }
            constexpr int NI = MCH * NR;
            u32x4 d[NI];
            float a[8];
#pragma unroll
            for (int e = 0; e < RWIN && e < NI; ++e) d[e] = __builtin_amdgcn_raw_buffer_load_b128(rs[e % NR], lane * 16, so + (e / NR) * 1024, AUX_SYS);
#pragma unroll
            for (int e = 0; e < NI; ++e) {
                if (e + RWIN < NI) d[e + RWIN] = __builtin_amdgcn_raw_buffer_load_b128(rs[(e + RWIN) % NR], lane * 16, so + ((e + RWIN) / NR) * 1024, AUX_SYS);
                __builtin_amdgcn_sched_barrier(0);
                const u32x4 v = d[e];
                if (e % NR == 0) {
#pragma unroll
                    for (int j = 0; j < 4; ++j) { a[2 * j] = __uint_as_float(v[j] << 16); a[2 * j + 1] = __uint_as_float(v[j] & 0xffff0000u); }
                } else {
#pragma unroll
                    for (int j = 0; j < 4; ++j) { a[2 * j] += __uint_as_float(v[j] << 16); a[2 * j + 1] += __uint_as_float(v[j] & 0xffff0000u); }
                }
                if (e % NR == NR - 1) {
                    unsigned r0 = cvtpk(a[0], a[1]), r1 = cvtpk(a[2], a[3]), r2 = cvtpk(a[4], a[5]), r3 = cvtpk(a[6], a[7]);
                    auto s02 = __builtin_amdgcn_permlane16_swap(r0, r2, false, false);
                    auto s13 = __builtin_amdgcn_permlane16_swap(r1, r3, false, false);
                    u32x4 o; o.x = s02[0]; o.y = s13[0]; o.z = s02[1]; o.w = s13[1];
                    __builtin_amdgcn_raw_buffer_store_b128(o, ro, ooff, (e / NR) * 16 * NN * 2, 0);
                }
                __builtin_amdgcn_sched_barrier(0);
            }
        }
    }
}

}

extern "C" void kernel_launch(int rank, int nranks, ncclComm_t comm,
                              hipStream_t stream,
                              void* const* d_in, const int* in_sizes, int n_in,
                              void* d_out, int out_size,
                              const int* in_dtypes, int out_dtype,
                              void* const* d_out_all,
                              void* const* d_ws_all, size_t ws_bytes) {
    (void)comm; (void)in_dtypes; (void)out_dtype; (void)d_out_all;
    if (nranks != NR || n_in < 2 || in_sizes[0] != MM * KL || in_sizes[1] != KL * NN || out_size != (MM / NR) * NN
        || ws_bytes < WS_NEED || (unsigned)rank >= (unsigned)NR) {
        fprintf(stderr, "kernel_launch: unexpected shapes\n");
        return;
    }
    static bool inited[NR] = {};
    if (!inited[rank]) {
        HIP_CHECK(hipFuncSetAttribute((const void*)gemm_rs_kernel, hipFuncAttributeMaxDynamicSharedMemorySize, LDS_BYTES));
        inited[rank] = true;
    }
    Params p;
    p.x = d_in[0]; p.w = d_in[1]; p.out = d_out; p.rank = rank;
    for (int i = 0; i < NR; ++i) p.ws[i] = (char*)d_ws_all[i];
    hipLaunchKernelGGL(gemm_rs_kernel, dim3(NWG), dim3(NTHR), LDS_BYTES, stream, p);
    HIP_CHECK(hipGetLastError());
}
```

```cpp
#include <hip/hip_runtime.h>
#include <rccl/rccl.h>
#include <cstdio>
#include <cstdint>

#define HIP_CHECK(expr) do { hipError_t _e = (expr); if (_e != hipSuccess) { \
    fprintf(stderr, "HIP error %s at %s:%d\n", hipGetErrorString(_e), __FILE__, __LINE__); \
    return; } } while (0)

#define LAS __attribute__((address_space(3)))
typedef short bf16x8 __attribute__((ext_vector_type(8)));
typedef short s16x4 __attribute__((ext_vector_type(4)));
typedef float f32x4 __attribute__((ext_vector_type(4)));
typedef float f32x2 __attribute__((ext_vector_type(2)));
typedef unsigned u32x4 __attribute__((ext_vector_type(4)));
typedef __bf16 bf16x2_t __attribute__((ext_vector_type(2)));
typedef __amdgpu_buffer_rsrc_t rsrc_t;

namespace {

constexpr int NR = 8;
constexpr int MM = 8192, NN = 8192, KL = 1024;
constexpr int NWG = 256, NTHR = 512;
#define NGEMM 224
#define RWIN 7
#define GAUX 0
#define FORCE_RAW 0
#define JOIN 0
constexpr int BM = 256, BK = 64, HALF = 128, HTB = HALF * BK * 2, STAGE_BYTES = 8 * HTB;
constexpr int NT = KL / BK;
constexpr int NTILES = (MM / BM) * (NN / BM);
constexpr unsigned TILE_B = BM * BM * 2;
constexpr unsigned SLOT_B = 16u << 20;
constexpr size_t CTRL_OFF = (size_t)256 << 20;
constexpr size_t WS_NEED = CTRL_OFF + (1u << 20);
constexpr unsigned C_TICKET = 0, C_RED = 128, C_TMO = 256, C_FLAGS = 4096;
constexpr unsigned C_HDR = 256 * 1024;
constexpr unsigned MAXEXC = 7;
constexpr unsigned RAWMODE = 255;
constexpr int UPT = 4;
constexpr int NUNITS = 128 * UPT;
constexpr int NCLAIM = JOIN ? NWG : NWG - NGEMM;
constexpr unsigned long long DEADLINE = 30000000ull;
constexpr int LDS_BYTES = STAGE_BYTES + 256 + 8 * 32;
constexpr int AUX_SYS = 17;

#define STAMP(slot) do {} while (0)
struct Params {
    const void* x; const void* w; void* out;
    char* ws[NR];
    int rank;
};

__device__ __forceinline__ int lds_byte(int r, int c) { const int st = (r >> 4) * 2 + (c >> 5), rr = r & 15, cc = c & 31, ob = rr * 64 + cc * 2; return st * 1024 + (ob ^ (((ob >> 9) & 1) << 5)); }
__device__ __forceinline__ void stage_rc(int b, int& R, int& C) { const int st = b / 1024, sb = b % 1024, swz = sb ^ (((sb >> 9) & 1) << 5); R = (st >> 1) * 16 + swz / 64; C = (st & 1) * 32 + (swz % 64) / 2; }
__device__ __forceinline__ unsigned cvtpk(float lo, float hi) { f32x2 v = {lo, hi}; bf16x2_t b = __builtin_convertvector(v, bf16x2_t); return __builtin_bit_cast(unsigned, b); }
__device__ __forceinline__ rsrc_t mk_rsrc(const void* p, unsigned bytes) { return __builtin_amdgcn_make_buffer_rsrc((void*)p, 0, bytes, 0x00020000); }

template <int AUX> __device__ __forceinline__ void st128(u32x4 v, rsrc_t r, unsigned voff, unsigned soff) {
    __builtin_amdgcn_raw_buffer_store_b128(v, r, voff, soff, AUX);
    asm volatile("s_nop 1" :: "v"(v) : "memory");
}
__device__ __forceinline__ unsigned bfi(unsigned m, unsigned a, unsigned b) { return (a & m) | (b & ~m); }
__device__ __forceinline__ unsigned vperm(unsigned hi, unsigned lo, unsigned sel) { return __builtin_amdgcn_perm(hi, lo, sel); }
__device__ __forceinline__ float wave_max_nonneg(float v) {
    int x = __float_as_int(v);
    x = max(x, __builtin_amdgcn_update_dpp(0, x, 0xB1, 0xf, 0xf, true));
    x = max(x, __builtin_amdgcn_update_dpp(0, x, 0x4E, 0xf, 0xf, true));
    x = max(x, __builtin_amdgcn_update_dpp(0, x, 0x141, 0xf, 0xf, true));
    x = max(x, __builtin_amdgcn_update_dpp(0, x, 0x140, 0xf, 0xf, true));
    const int a = __builtin_amdgcn_readlane(x, 0), b = __builtin_amdgcn_readlane(x, 16), c = __builtin_amdgcn_readlane(x, 32), d = __builtin_amdgcn_readlane(x, 48);
    return __int_as_float(max(max(a, b), max(c, d)));
}

__device__ __forceinline__ void tile_of(int i, int& pm, int& pn) { pn = i >> 5; pm = i & 31; }

#define TRD(dst, addr, off) asm volatile("ds_read_b64_tr_b16 %0, %1 offset:%c2" : "=&v"(dst) : "v"(addr), "i"(off) : "memory")
#define PKB(x) (bf16x8){(x)[0][0], (x)[0][1], (x)[0][2], (x)[0][3], (x)[1][0], (x)[1][1], (x)[1][2], (x)[1][3]}

__global__ __launch_bounds__(NTHR, 2) void gemm_rs_kernel(Params p) {
    extern __shared__ __attribute__((aligned(16))) unsigned char smem[];
    LAS unsigned char* lds = (LAS unsigned char*)smem;
    LAS unsigned* lds_ctl = (LAS unsigned*)(lds + STAGE_BYTES);
    const int tid = threadIdx.x, wid = __builtin_amdgcn_readfirstlane(tid >> 6), lane = tid & 63;
    const int wr = wid >> 2, wc = wid & 3, fr = lane & 15, fq = lane >> 4;
    const int wg = blockIdx.x, rank = p.rank;
    char* myws = p.ws[rank];
    unsigned* ctl = (unsigned*)(myws + CTRL_OFF);

    unsigned call = 0;
    int dslot = 0;
    STAMP(dslot); ++dslot;
    if (tid == 0) {
        const unsigned t = __hip_atomic_fetch_add(ctl + C_TICKET / 4, 1u, __ATOMIC_RELAXED, __HIP_MEMORY_SCOPE_AGENT);
        lds_ctl[0] = t / NWG + 1;
    }

    if (wg < NGEMM) {
        const rsrc_t rx = mk_rsrc(p.x, MM * KL * 2), rw = mk_rsrc(p.w, KL * NN * 2);
        unsigned voffA, voffB;
        { int R, C; stage_rc(tid * 16, R, C); voffA = (unsigned)(R * KL + C) * 2u; }
        { const int row = tid >> 4, chp = tid & 15, ch = chp ^ (((row & 3) << 2) | (((row >> 3) & 1) << 1)); voffB = (unsigned)(row * NN + ch * 8) * 2u; }
        constexpr unsigned A_I = 64 * KL * 2, B_I = 32 * NN * 2;
        constexpr unsigned A_K = BK * 2, B_K = BK * NN * 2;
        constexpr unsigned A_H = HALF * KL * 2, B_H = HALF * 2;
        constexpr unsigned A_T = BM * KL * 2, B_T = BM * 2;
        const unsigned ldsw = (unsigned)wid * 1024u;
        const int aoff = lds_byte(wr * 64 + fr, fq * 8);
        unsigned boff[2];
        { const int q = (lane & 15) >> 2, pp = lane & 3, g = fq;
#pragma unroll
          for (int n = 0; n < 2; ++n) boff[n] = (unsigned)(size_t)lds + 4 * HTB + 256 * (8 * g + q) + 16 * (((wc ^ q) << 2) | ((n ^ (g & 1)) << 1) | (pp >> 1)) + 8 * (pp & 1); }
#define SA(b, h) (((b) * 2 + (h)) * HTB)
#define SB(b, h) ((4 + (b) * 2 + (h)) * HTB)
#define STAGE_A(bufoff, soff) do { _Pragma("unroll") for (int _i = 0; _i < 2; ++_i) \
        __builtin_amdgcn_raw_ptr_buffer_load_lds(rx, (LAS void*)(lds + (bufoff) + ldsw + _i * 8192), 16, voffA, (soff) + _i * A_I, 0, GAUX); } while (0)
#define STAGE_B(bufoff, soff) do { _Pragma("unroll") for (int _i = 0; _i < 2; ++_i) \
        __builtin_amdgcn_raw_ptr_buffer_load_lds(rw, (LAS void*)(lds + (bufoff) + ldsw + _i * 8192), 16, voffB, (soff) + _i * B_I, 0, GAUX); } while (0)
#define LDA(dst, b, h) do { _Pragma("unroll") for (int m = 0; m < 4; ++m) _Pragma("unroll") for (int k = 0; k < 2; ++k) dst[m][k] = *(const LAS bf16x8*)(lds + SA(b, h) + aoff + m * 2048 + k * 1024); } while (0)
#define LDB(dst, b, h) do { _Pragma("unroll") for (int n = 0; n < 2; ++n) _Pragma("unroll") for (int k = 0; k < 2; ++k) { \
        TRD(dst[n][k][0], boff[n], ((b) * 2 + (h)) * HTB + k * 8192); TRD(dst[n][k][1], boff[n], ((b) * 2 + (h)) * HTB + k * 8192 + 1024); } } while (0)
#define MMA(ai, bj, At, Bt) do { __builtin_amdgcn_s_setprio(1); _Pragma("unroll") for (int m = 0; m < 4; ++m) _Pragma("unroll") for (int n = 0; n < 2; ++n) _Pragma("unroll") for (int k = 0; k < 2; ++k) \
        acc[ai][bj][m][n] = __builtin_amdgcn_mfma_f32_16x16x32_bf16(PKB(Bt[n][k]), At[m][k], acc[ai][bj][m][n], 0, 0, 0); __builtin_amdgcn_s_setprio(0); } while (0)
#define WAIT_B(X, Y) asm volatile("s_waitcnt lgkmcnt(0)" : "+v"(X[0][0][0]), "+v"(X[0][0][1]), "+v"(X[0][1][0]), "+v"(X[0][1][1]), "+v"(X[1][0][0]), "+v"(X[1][0][1]), "+v"(X[1][1][0]), "+v"(X[1][1][1]), \
        "+v"(Y[0][0][0]), "+v"(Y[0][0][1]), "+v"(Y[0][1][0]), "+v"(Y[0][1][1]), "+v"(Y[1][0][0]), "+v"(Y[1][0][1]), "+v"(Y[1][1][0]), "+v"(Y[1][1][1]) :: "memory")
#define WAIT_V(n) asm volatile("s_waitcnt vmcnt(" #n ")" ::: "memory")
#define WAIT_L(n) asm volatile("s_waitcnt lgkmcnt(" #n ")" ::: "memory")
#define BAR __builtin_amdgcn_s_barrier()
#define SCHED __builtin_amdgcn_sched_barrier(0)
        int cur = wg, pm, pn;
        tile_of(cur, pm, pn);
        f32x4 acc[2][2][4][2];
#pragma unroll
        for (int a = 0; a < 2; ++a)
#pragma unroll
            for (int b = 0; b < 2; ++b)
#pragma unroll
                for (int m = 0; m < 4; ++m)
#pragma unroll
                    for (int n = 0; n < 2; ++n) acc[a][b][m][n] = (f32x4){0.f, 0.f, 0.f, 0.f};
        bf16x8 At[4][2]; s16x4 B0[2][2][2], B1[2][2][2];
        unsigned cA = (unsigned)pm * A_T, cB = (unsigned)pn * B_T;
        STAGE_B(SB(0, 0), cB); STAGE_B(SB(0, 1), cB + B_H); STAGE_A(SA(0, 0), cA); STAGE_A(SA(0, 1), cA + A_H);
        if (wr == 1) BAR;
        WAIT_V(2); BAR;
        STAGE_B(SB(1, 0), cB + B_K); STAGE_A(SA(1, 0), cA + A_K); STAGE_B(SB(1, 1), cB + B_H + B_K);
        WAIT_V(6); BAR;
        for (;;) {
            const int nxt = cur + NGEMM;
            const bool has_next = nxt < NTILES;
            int npm = pm, npn = pn;
            if (has_next) tile_of(nxt, npm, npn);
            const unsigned nA = (unsigned)npm * A_T, nB = (unsigned)npn * B_T;
            for (int t = 0; t < NT; t += 2) {
                const bool last = (t == NT - 2);
                const unsigned a1 = cA + (unsigned)(t + 1) * A_K;
                const unsigned a2 = last ? nA : cA + (unsigned)(t + 2) * A_K, b2 = last ? nB : cB + (unsigned)(t + 2) * B_K;
                const unsigned a3 = a2 + A_K, b3 = b2 + B_K;
                LDB(B0, 0, 0); LDB(B1, 0, 1); SCHED; LDA(At, 0, 0); STAGE_A(SA(1, 1), a1 + A_H);
                WAIT_V(8); WAIT_B(B0, B1); BAR; MMA(0, 0, At, B0); MMA(0, 1, At, B1); BAR; SCHED;
                LDA(At, 0, 1); STAGE_B(SB(0, 0), b2); STAGE_B(SB(0, 1), b2 + B_H); STAGE_A(SA(0, 0), a2);
                WAIT_V(8); WAIT_L(0); BAR; MMA(1, 0, At, B0); MMA(1, 1, At, B1); BAR; SCHED;
                LDB(B0, 1, 0); LDB(B1, 1, 1); SCHED; LDA(At, 1, 0); STAGE_A(SA(0, 1), a2 + A_H);
                WAIT_V(8); WAIT_B(B0, B1); BAR; MMA(0, 0, At, B0); MMA(0, 1, At, B1); BAR; SCHED;
                LDA(At, 1, 1); STAGE_B(SB(1, 0), b3); STAGE_B(SB(1, 1), b3 + B_H); STAGE_A(SA(1, 0), a3);
                WAIT_V(8); WAIT_L(0); BAR; MMA(1, 0, At, B0); MMA(1, 1, At, B1); BAR; SCHED;
            }
            if (wr == 0) BAR;
            STAMP(dslot); ++dslot;
            {
                call = lds_ctl[0];
                const int owner = pm >> 2, tl = pn * 4 + (pm & 3);
                char* dws = p.ws[(unsigned)owner < (unsigned)NR ? owner : 0];
                const rsrc_t rd = mk_rsrc(myws + ((size_t)((call & 1) * NR + owner) * SLOT_B), SLOT_B);
                const unsigned so = (unsigned)tl * TILE_B + (unsigned)wid * 16384u;
                LAS unsigned* hdr = lds_ctl + 64 + wid * 8;
                float mx = 0.f;
#pragma unroll
                for (int ai = 0; ai < 2; ++ai)
#pragma unroll
                    for (int bj = 0; bj < 2; ++bj)
#pragma unroll
                        for (int m = 0; m < 4; ++m)
#pragma unroll
                            for (int n = 0; n < 2; ++n) {
                                const f32x4 v = acc[ai][bj][m][n];
                                mx = fmaxf(fmaxf(mx, fabsf(v[0])), fabsf(v[1])); mx = fmaxf(fmaxf(mx, fabsf(v[2])), fabsf(v[3]));
                            }
                mx = wave_max_nonneg(mx);
                const unsigned base = (cvtpk(mx, mx) >> 8) & 0x7fu;
                const unsigned bq = base * 0x01010101u | 0x80808080u;
                unsigned cnt = FORCE_RAW ? RAWMODE : 0;
#pragma unroll
                for (int g = 0; g < 4; ++g) {
                    unsigned L[8], P[4], ex = 0;
#pragma unroll
                    for (int m = 0; m < 4; ++m) {
                        const f32x4 v0 = acc[g >> 1][g & 1][m][0], v1 = acc[g >> 1][g & 1][m][1];
                        const unsigned w0 = cvtpk(v0[0], v0[1]), w1 = cvtpk(v0[2], v0[3]), w2 = cvtpk(v1[0], v1[1]), w3 = cvtpk(v1[2], v1[3]);
                        L[2 * m] = vperm(w1, w0, 0x06040200u); L[2 * m + 1] = vperm(w3, w2, 0x06040200u);
                        const unsigned h01 = vperm(w1, w0, 0x07050301u), h23 = vperm(w3, w2, 0x07050301u);
                        const unsigned d01 = bq - (h01 & 0x7f7f7f7fu), d23 = bq - (h23 & 0x7f7f7f7fu);
                        ex |= (d01 ^ 0x80808080u) | (d23 ^ 0x80808080u);
                        P[m] = bfi(0x0f0f0f0fu, bfi(0x07070707u, d01, h01 >> 4), bfi(0x70707070u, d23 << 4, h23));
                    }
                    st128<AUX_SYS>((u32x4){L[0], L[1], L[2], L[3]}, rd, lane * 16, so + g * 3072);
                    st128<AUX_SYS>((u32x4){L[4], L[5], L[6], L[7]}, rd, lane * 16, so + g * 3072 + 1024);
                    st128<AUX_SYS>((u32x4){P[0], P[1], P[2], P[3]}, rd, lane * 16, so + g * 3072 + 2048);
                    if (__any((ex & 0xf8f8f8f8u) != 0)) {
#pragma unroll
                        for (int m = 0; m < 4; ++m)
#pragma unroll
                            for (int j = 0; j < 8; ++j) {
                                const float v = acc[g >> 1][g & 1][m][j >> 2][j & 3];
                                const unsigned hb = (cvtpk(v, v) >> 8) & 0xffu;
                                const bool e = base - (hb & 0x7fu) > 7u;
                                const unsigned long long bal = __ballot(e);
                                if (bal) {
                                    const unsigned idx = cnt + __builtin_amdgcn_mbcnt_hi((unsigned)(bal >> 32), __builtin_amdgcn_mbcnt_lo((unsigned)bal, 0u));
                                    if (e && idx < MAXEXC) hdr[1 + idx] = (unsigned)((g * 4 + m) << 17 | j << 8) | (unsigned)lane << 11 | hb;
                                    cnt = min(cnt + (unsigned)__popcll(bal), RAWMODE);
                                }
                            }
                    }
                }
                if (cnt > MAXEXC) {
                    cnt = RAWMODE;
                    asm volatile("s_waitcnt vmcnt(0)" ::: "memory");
#pragma unroll
                    for (int j = 0; j < 16; ++j) {
                        const f32x4 v0 = acc[j >> 3][(j >> 2) & 1][j & 3][0], v1 = acc[j >> 3][(j >> 2) & 1][j & 3][1];
                        u32x4 w; w.x = cvtpk(v0[0], v0[1]); w.y = cvtpk(v0[2], v0[3]); w.z = cvtpk(v1[0], v1[1]); w.w = cvtpk(v1[2], v1[3]);
                        st128<AUX_SYS>(w, rd, lane * 16, so + j * 1024);
                    }
                }
                if (lane == 0) hdr[0] = base | cnt << 8;
                asm volatile("s_waitcnt lgkmcnt(0)" ::: "memory");
                if (lane < 8) {
                    unsigned* hg = (unsigned*)(dws + CTRL_OFF + C_HDR + (size_t)((((call & 1) * 128 + tl) * 8 + wid) * NR + rank) * 32);
                    __hip_atomic_store(hg + lane, hdr[lane], __ATOMIC_RELAXED, __HIP_MEMORY_SCOPE_SYSTEM);
                }
                WAIT_V(0); BAR;
                STAMP(dslot); ++dslot;
                if (tid == 0) {
                    unsigned* f = (unsigned*)(dws + CTRL_OFF + C_FLAGS + (size_t)(tl * NR + rank) * 128);
                    __hip_atomic_store(f, call, __ATOMIC_RELAXED, __HIP_MEMORY_SCOPE_SYSTEM);
                }
            }
            if (!has_next) break;
#pragma unroll
            for (int a = 0; a < 2; ++a)
#pragma unroll
                for (int b = 0; b < 2; ++b)
#pragma unroll
                    for (int m = 0; m < 4; ++m)
#pragma unroll
                        for (int n = 0; n < 2; ++n) acc[a][b][m][n] = (f32x4){0.f, 0.f, 0.f, 0.f};
            cur = nxt; pm = npm; pn = npn; cA = nA; cB = nB;
            if (wr == 1) BAR;
        }
        WAIT_V(0); BAR;
#undef SA
#undef SB
    } else {
        __syncthreads();
    }

    call = lds_ctl[0];
    dslot = 16;
    STAMP(dslot); ++dslot;
    if (JOIN || wg >= NGEMM) {
        rsrc_t rs[NR];
#pragma unroll
        for (int s = 0; s < NR; ++s) rs[s] = mk_rsrc(p.ws[(rank + s) & (NR - 1)] + (size_t)((call & 1) * NR + rank) * SLOT_B, SLOT_B);
        const rsrc_t ro = mk_rsrc(p.out, (MM / NR) * NN * 2);
        const unsigned base = (call - 1) * (unsigned)(NUNITS + NCLAIM);
        for (;;) {
            __syncthreads();
            if (wid == 0) {
                unsigned u = 0;
                if (lane == 0) u = __hip_atomic_fetch_add(ctl + C_RED / 4, 1u, __ATOMIC_RELAXED, __HIP_MEMORY_SCOPE_AGENT) - base;
                u = __builtin_amdgcn_readfirstlane(u);
                unsigned bad = 0;
                if (u < (unsigned)NUNITS) {
                    const unsigned* f = (const unsigned*)(myws + CTRL_OFF + C_FLAGS + (size_t)((u / UPT) * NR + (lane & 7)) * 128);
                    const unsigned long long t0 = __builtin_amdgcn_s_memrealtime();
                    for (;;) {
                        const unsigned v = __hip_atomic_load(f, __ATOMIC_RELAXED, __HIP_MEMORY_SCOPE_SYSTEM);
                        if (__all((int)(v - call) >= 0)) break;
                        if (__hip_atomic_load(ctl + C_TMO / 4, __ATOMIC_RELAXED, __HIP_MEMORY_SCOPE_AGENT) == call
                            || __builtin_amdgcn_s_memrealtime() - t0 > DEADLINE) {
                            if (lane == 0) __hip_atomic_store(ctl + C_TMO / 4, call, __ATOMIC_RELAXED, __HIP_MEMORY_SCOPE_AGENT);
                            break;
                        }
                        __builtin_amdgcn_s_sleep(4);
                    }
                    asm volatile("s_waitcnt vmcnt(0)" ::: "memory");
                    bad = __hip_atomic_load(ctl + C_TMO / 4, __ATOMIC_RELAXED, __HIP_MEMORY_SCOPE_AGENT) == call;
                }
                if (lane == 0) { lds_ctl[1] = u; lds_ctl[2] = bad; }
            }
            __syncthreads();
            const unsigned u = lds_ctl[1], bad = lds_ctl[2];
            STAMP(dslot); ++dslot;
            if (u >= (unsigned)NUNITS) { STAMP(63); break; }
            const unsigned tl = u / UPT, g = u % UPT;
            const unsigned sw = tl * TILE_B + (unsigned)wid * 16384u, so = sw + g * 3072u;
            const int pn = tl >> 2, rt = tl & 3, ai = g >> 1, bj = g & 1;
            const unsigned orow = rt * 256 + ai * 128 + wr * 64 + fr;
            const unsigned ocol = pn * 256 + bj * 128 + wc * 32 + ((fq & 1) * 16 + (fq >> 1) * 8);
            const unsigned ooff = (orow * NN + ocol) * 2;
            if (bad) {
#pragma unroll
                for (int m = 0; m < 4; ++m)
                    st128<0>((u32x4){0x7fc07fc0u, 0x7fc07fc0u, 0x7fc07fc0u, 0x7fc07fc0u}, ro, ooff, m * 16 * NN * 2);
                continue;
            }
            const unsigned hv = __hip_atomic_load((const unsigned*)(myws + CTRL_OFF + C_HDR + (size_t)(((call & 1) * 128 + tl) * 8 + wid) * 256) + lane,
                                                  __ATOMIC_RELAXED, __HIP_MEMORY_SCOPE_SYSTEM);
            u32x4 d[NR][3];
            float a[4][8];
#pragma unroll
            for (int s = 0; s < RWIN; ++s)
#pragma unroll
                for (int q = 0; q < 3; ++q) d[s][q] = __builtin_amdgcn_raw_buffer_load_b128(rs[s], lane * 16, so + q * 1024, AUX_SYS);
#pragma unroll
            for (int s = 0; s < NR; ++s) {
                if (s + RWIN < NR) {
#pragma unroll
                    for (int q = 0; q < 3; ++q) d[s + RWIN][q] = __builtin_amdgcn_raw_buffer_load_b128(rs[s + RWIN], lane * 16, so + q * 1024, AUX_SYS);
                }
                __builtin_amdgcn_sched_barrier(0);
                const int hl = ((rank + s) & (NR - 1)) * 8;
                const unsigned h0 = __builtin_amdgcn_readlane(hv, hl), cnt = h0 >> 8;
                if (cnt == RAWMODE) {
#pragma unroll
                    for (int m = 0; m < 4; ++m) {
                        const u32x4 v = __builtin_amdgcn_raw_buffer_load_b128(rs[s], lane * 16, sw + (g * 4 + m) * 1024, AUX_SYS);
#pragma unroll
                        for (int j = 0; j < 4; ++j) {
                            const float lo = __uint_as_float(v[j] << 16), hi = __uint_as_float(v[j] & 0xffff0000u);
                            a[m][2 * j] = s == 0 ? lo : a[m][2 * j] + lo; a[m][2 * j + 1] = s == 0 ? hi : a[m][2 * j + 1] + hi;
                        }
                    }
                } else {
                    const unsigned bq = (h0 & 0x7fu) * 0x01010101u | 0x80808080u;
                    unsigned H[8];
#pragma unroll
                    for (int m = 0; m < 4; ++m) {
                        const unsigned P = d[s][2][m];
                        H[2 * m] = bfi(0x7f7f7f7fu, bq - (P & 0x07070707u), P << 4);
                        H[2 * m + 1] = bfi(0x7f7f7f7fu, bq - ((P >> 4) & 0x07070707u), P);
                    }
                    for (unsigned e = 0; e < cnt; ++e) {
                        const unsigned en = __builtin_amdgcn_readlane(hv, hl + 1 + e);
                        if ((en >> 19) == g) {
                            const unsigned k = (en >> 17 & 3) * 2 + (en >> 10 & 1), sh = (en >> 8 & 3) * 8;
                            const unsigned mk = (unsigned)lane == (en >> 11 & 63) ? 0xffu << sh : 0u, val = (en & 0xffu) << sh;
#pragma unroll
                            for (int kk = 0; kk < 8; ++kk)
                                if (k == (unsigned)kk) H[kk] = bfi(mk, val, H[kk]);
                        }
                    }
#pragma unroll
                    for (int m = 0; m < 4; ++m)
#pragma unroll
                        for (int j = 0; j < 8; ++j) {
                            const unsigned Lw = d[s][m >> 1][(m & 1) * 2 + (j >> 2)];
                            const float v = __uint_as_float(vperm(H[2 * m + (j >> 2)], Lw, (4u + (j & 3)) << 24 | (unsigned)(j & 3) << 16 | 0x0c0cu));
                            a[m][j] = s == 0 ? v : a[m][j] + v;
                        }
                }
#pragma unroll
                for (int m = 0; m < 4; ++m)
                    asm volatile("" : "+v"(a[m][0]), "+v"(a[m][1]), "+v"(a[m][2]), "+v"(a[m][3]), "+v"(a[m][4]), "+v"(a[m][5]), "+v"(a[m][6]), "+v"(a[m][7]));
                __builtin_amdgcn_sched_barrier(0);
            }
#pragma unroll
            for (int m = 0; m < 4; ++m) {
                unsigned r0 = cvtpk(a[m][0], a[m][1]), r1 = cvtpk(a[m][2], a[m][3]), r2 = cvtpk(a[m][4], a[m][5]), r3 = cvtpk(a[m][6], a[m][7]);
                auto s02 = __builtin_amdgcn_permlane16_swap(r0, r2, false, false);
                auto s13 = __builtin_amdgcn_permlane16_swap(r1, r3, false, false);
                u32x4 o; o.x = s02[0]; o.y = s13[0]; o.z = s02[1]; o.w = s13[1];
                st128<0>(o, ro, ooff, m * 16 * NN * 2);
            }
        }
    }
}

}

extern "C" void kernel_launch(int rank, int nranks, ncclComm_t comm,
                              hipStream_t stream,
                              void* const* d_in, const int* in_sizes, int n_in,
                              void* d_out, int out_size,
                              const int* in_dtypes, int out_dtype,
                              void* const* d_out_all,
                              void* const* d_ws_all, size_t ws_bytes) {
    (void)comm; (void)in_dtypes; (void)out_dtype; (void)d_out_all;
    if (nranks != NR || n_in < 2 || in_sizes[0] != MM * KL || in_sizes[1] != KL * NN || out_size != (MM / NR) * NN
        || ws_bytes < WS_NEED || (unsigned)rank >= (unsigned)NR) {
        fprintf(stderr, "kernel_launch: unexpected shapes\n");
        return;
    }
    static bool inited[NR] = {};
    if (!inited[rank]) {
        HIP_CHECK(hipFuncSetAttribute((const void*)gemm_rs_kernel, hipFuncAttributeMaxDynamicSharedMemorySize, LDS_BYTES));
        inited[rank] = true;
    }
    Params p;
    p.x = d_in[0]; p.w = d_in[1]; p.out = d_out; p.rank = rank;
    for (int i = 0; i < NR; ++i) p.ws[i] = (char*)d_ws_all[i];
    hipLaunchKernelGGL(gemm_rs_kernel, dim3(NWG), dim3(NTHR), LDS_BYTES, stream, p);
    HIP_CHECK(hipGetLastError());
}
```

```cpp
#include <hip/hip_runtime.h>
#include <rccl/rccl.h>
#include <cstdio>
#include <cstdint>

#define HIP_CHECK(expr) do { hipError_t _e = (expr); if (_e != hipSuccess) { \
    fprintf(stderr, "HIP error %s at %s:%d\n", hipGetErrorString(_e), __FILE__, __LINE__); \
    return; } } while (0)

#define LAS __attribute__((address_space(3)))
typedef short bf16x8 __attribute__((ext_vector_type(8)));
typedef short s16x4 __attribute__((ext_vector_type(4)));
typedef float f32x4 __attribute__((ext_vector_type(4)));
typedef float f32x2 __attribute__((ext_vector_type(2)));
typedef unsigned u32x4 __attribute__((ext_vector_type(4)));
typedef unsigned short u16x2 __attribute__((ext_vector_type(2)));
typedef __bf16 bf16x2_t __attribute__((ext_vector_type(2)));
typedef __amdgpu_buffer_rsrc_t rsrc_t;

namespace {

constexpr int NR = 8;
constexpr int MM = 8192, NN = 8192, KL = 1024;
constexpr int NWG = 256, NTHR = 512;
#define PN1 13
#define UPJ 2
#define GPJ 4
#define FORCE_RAW 0
#define NOGEMM 0
constexpr int NGEMM = 16 * PN1, NCOMM = NWG - NGEMM, NCW = NCOMM * 8;
constexpr int PN2 = 16 - PN1;
constexpr int HTB = 16384, STAGE_BYTES = 8 * HTB;
constexpr int NT = 64;
constexpr unsigned UNIT_B = 4096;
constexpr size_t MIB = 1u << 20;
constexpr size_t OFF_PACK = 0;
constexpr unsigned PACK_B = 32u << 20;
constexpr size_t OFF_STG = 64 * MIB;
constexpr unsigned STG_B = 32u << 20;
constexpr size_t OFF_CP = 128 * MIB;
constexpr unsigned CP_B = 32u << 20;
constexpr size_t OFF_IH = 160 * MIB;
constexpr unsigned IH_B = 2 * 16 * 8 * 64 * 16;
constexpr size_t OFF_RDY = 161 * MIB;
constexpr size_t OFF_CH = 162 * MIB;
constexpr size_t CTRL_OFF = 256 * MIB;
constexpr size_t WS_NEED = CTRL_OFF + MIB;
constexpr unsigned C_TICKET = 0, C_TMO = 256;
constexpr unsigned MAXEXC = 2, RAWMODE = 255, POISON = 254;
constexpr unsigned long long DEADLINE = 30000000ull;
constexpr int LDS_BYTES = STAGE_BYTES + 256;
constexpr int AUX_SYS = 17;
constexpr int NJ1 = (NT / UPJ) * (16 + PN1) * NR, NJ_IN = (NT / UPJ) * 32 * NR;
constexpr int JPS = 4 / GPJ;
constexpr int NJ_OUT = 128 * 8 * JPS;

#define STAMP(slot) do {} while (0)
#define STAMPW(slot) do {} while (0)
struct Params {
    const void* x; const void* w; void* out;
    char* ws[NR];
    int rank;
};

__device__ __forceinline__ unsigned cvtpk(float lo, float hi) { f32x2 v = {lo, hi}; bf16x2_t b = __builtin_convertvector(v, bf16x2_t); return __builtin_bit_cast(unsigned, b); }
__device__ __forceinline__ rsrc_t mk_rsrc(const void* p, unsigned bytes) { return __builtin_amdgcn_make_buffer_rsrc((void*)p, 0, bytes, 0x00020000); }

template <int AUX> __device__ __forceinline__ void st128(u32x4 v, rsrc_t r, unsigned voff, unsigned soff) {
    __builtin_amdgcn_raw_buffer_store_b128(v, r, voff, soff, AUX);
    asm volatile("s_nop 1" :: "v"(v) : "memory");
}
__device__ __forceinline__ void gst128_sys(void* ptr, u32x4 v) {
    asm volatile("global_store_dwordx4 %0, %1, off sc0 sc1\n\ts_nop 1" :: "v"(ptr), "v"(v) : "memory");
}
__device__ __forceinline__ unsigned gld32_sys(const void* ptr) {
    unsigned v;
    asm volatile("global_load_dword %0, %1, off sc0 sc1\n\ts_waitcnt vmcnt(0)" : "=&v"(v) : "v"(ptr) : "memory");
    return v;
}
__device__ __forceinline__ unsigned bfi(unsigned m, unsigned a, unsigned b) { return (a & m) | (b & ~m); }
__device__ __forceinline__ unsigned vperm(unsigned hi, unsigned lo, unsigned sel) { return __builtin_amdgcn_perm(hi, lo, sel); }
__device__ __forceinline__ unsigned rfl(unsigned v) { return __builtin_amdgcn_readfirstlane(v); }
__device__ __forceinline__ unsigned wave_max_u(int x) {
    x = max(x, __builtin_amdgcn_update_dpp(0, x, 0xB1, 0xf, 0xf, true));
    x = max(x, __builtin_amdgcn_update_dpp(0, x, 0x4E, 0xf, 0xf, true));
    x = max(x, __builtin_amdgcn_update_dpp(0, x, 0x141, 0xf, 0xf, true));
    x = max(x, __builtin_amdgcn_update_dpp(0, x, 0x140, 0xf, 0xf, true));
    const int a = __builtin_amdgcn_readlane(x, 0), b = __builtin_amdgcn_readlane(x, 16), c = __builtin_amdgcn_readlane(x, 32), d = __builtin_amdgcn_readlane(x, 48);
    return (unsigned)max(max(a, b), max(c, d));
}
__device__ __forceinline__ bool timed_out(unsigned* tmo, unsigned call, unsigned long long t0) {
    if (__hip_atomic_load(tmo, __ATOMIC_RELAXED, __HIP_MEMORY_SCOPE_AGENT) == call) return true;
    if (__builtin_amdgcn_s_memrealtime() - t0 > DEADLINE) { __hip_atomic_store(tmo, call, __ATOMIC_RELAXED, __HIP_MEMORY_SCOPE_AGENT); return true; }
    return false;
}

__device__ __forceinline__ void encode_unit(const unsigned (&w)[4][4], rsrc_t r, unsigned so, int lane, unsigned& h1, unsigned& e0, unsigned& e1) {
    u16x2 mx = {0, 0};
#pragma unroll
    for (int m = 0; m < 4; ++m)
#pragma unroll
        for (int q = 0; q < 4; ++q) mx = __builtin_elementwise_max(mx, __builtin_bit_cast(u16x2, w[m][q] & 0x7f007f00u));
    const unsigned base = wave_max_u((int)max(mx[0], mx[1])) >> 8;
    const unsigned bq = base * 0x01010101u | 0x80808080u;
    unsigned L[8], P[4], ex = 0;
#pragma unroll
    for (int m = 0; m < 4; ++m) {
        L[2 * m] = vperm(w[m][1], w[m][0], 0x06040200u); L[2 * m + 1] = vperm(w[m][3], w[m][2], 0x06040200u);
        const unsigned h01 = vperm(w[m][1], w[m][0], 0x07050301u), h23 = vperm(w[m][3], w[m][2], 0x07050301u);
        const unsigned d01 = bq - (h01 & 0x7f7f7f7fu), d23 = bq - (h23 & 0x7f7f7f7fu);
        ex |= d01 | d23;
        P[m] = bfi(0x0f0f0f0fu, bfi(0x07070707u, d01, h01 >> 4), bfi(0x70707070u, d23 << 4, h23));
    }
    unsigned cnt = FORCE_RAW ? RAWMODE : 0;
    e0 = 0; e1 = 0;
    if (__any((ex & 0x78787878u) != 0)) {
#pragma unroll
        for (int m = 0; m < 4; ++m)
#pragma unroll
            for (int j = 0; j < 8; ++j) {
                const unsigned hb = (w[m][j >> 1] >> (8 + 16 * (j & 1))) & 0xffu;
                unsigned long long bal = __ballot(base - (hb & 0x7fu) > 7u);
                while (bal && cnt <= MAXEXC) {
                    const unsigned l = (unsigned)__builtin_ctzll(bal);
                    const unsigned en = (unsigned)(m << 17 | j << 8) | l << 11 | (unsigned)__builtin_amdgcn_readlane((int)hb, (int)l);
                    if (cnt == 0) e0 = en; else e1 = en;
                    ++cnt; bal &= bal - 1;
                }
            }
        if (cnt > MAXEXC) cnt = RAWMODE;
    }
    if (cnt == RAWMODE) {
#pragma unroll
        for (int m = 0; m < 4; ++m) st128<AUX_SYS>((u32x4){w[m][0], w[m][1], w[m][2], w[m][3]}, r, lane * 16, so + m * 1024);
    } else {
        st128<AUX_SYS>((u32x4){L[0], L[1], L[2], L[3]}, r, lane * 16, so);
        st128<AUX_SYS>((u32x4){L[4], L[5], L[6], L[7]}, r, lane * 16, so + 1024);
        st128<AUX_SYS>((u32x4){P[0], P[1], P[2], P[3]}, r, lane * 16, so + 2048);
    }
    h1 = base | cnt << 8;
}
__device__ __forceinline__ void decode_high(u32x4 P, unsigned h1, unsigned e0, unsigned e1, int lane, unsigned (&H)[8]) {
    const unsigned bq = (h1 & 0x7fu) * 0x01010101u | 0x80808080u;
#pragma unroll
    for (int m = 0; m < 4; ++m) {
        H[2 * m] = bfi(0x7f7f7f7fu, bq - (P[m] & 0x07070707u), P[m] << 4);
        H[2 * m + 1] = bfi(0x7f7f7f7fu, bq - ((P[m] >> 4) & 0x07070707u), P[m]);
    }
    const unsigned cnt = h1 >> 8;
    for (unsigned e = 0; e < cnt; ++e) {
        const unsigned en = e ? e1 : e0;
        const unsigned k = (en >> 17 & 3) * 2 + (en >> 10 & 1), sh = (en >> 8 & 3) * 8;
        const unsigned mk = (unsigned)lane == (en >> 11 & 63) ? 0xffu << sh : 0u, val = (en & 0xffu) << sh;
#pragma unroll
        for (int kk = 0; kk < 8; ++kk)
            if (k == (unsigned)kk) H[kk] = bfi(mk, val, H[kk]);
    }
}

#define TRD(dst, addr, off) asm volatile("ds_read_b64_tr_b16 %0, %1 offset:%c2" : "=&v"(dst) : "v"(addr), "i"(off) : "memory")
#define PKB(x) (bf16x8){(x)[0][0], (x)[0][1], (x)[0][2], (x)[0][3], (x)[1][0], (x)[1][1], (x)[1][2], (x)[1][3]}
#define WAIT_V(n) asm volatile("s_waitcnt vmcnt(" #n ")" ::: "memory")
#define WAIT_L(n) asm volatile("s_waitcnt lgkmcnt(" #n ")" ::: "memory")
#define BAR __builtin_amdgcn_s_barrier()
#define SCHED __builtin_amdgcn_sched_barrier(0)

__global__ __launch_bounds__(NTHR, 2) void gemm_rs_kernel(Params p) {
    extern __shared__ __attribute__((aligned(16))) unsigned char smem[];
    LAS unsigned char* lds = (LAS unsigned char*)smem;
    LAS unsigned* lds_ctl = (LAS unsigned*)(lds + STAGE_BYTES);
    const int tid = threadIdx.x, wid = __builtin_amdgcn_readfirstlane(tid >> 6), lane = tid & 63;
    const int wr = wid >> 2, wc = wid & 3, fr = lane & 15, fq = lane >> 4;
    const int wg = blockIdx.x, rank = p.rank;
    const int bi = rank & 1, bj_ = (rank >> 1) & 1, bk = rank >> 2;
    char* myws = p.ws[rank];
    unsigned* ctl = (unsigned*)(myws + CTRL_OFF);
    unsigned* tmo = ctl + C_TMO / 4;

    int dslot = 0;
    STAMP(dslot); ++dslot;
    if (tid == 0) {
        const unsigned t = __hip_atomic_fetch_add(ctl + C_TICKET / 4, 1u, __ATOMIC_RELAXED, __HIP_MEMORY_SCOPE_AGENT);
        lds_ctl[0] = t / NWG + 1;
    }
    __syncthreads();
    const unsigned call = rfl(lds_ctl[0]);
    const unsigned par = call & 1;

    if (wg < NGEMM) {
        {
            const rsrc_t rx = mk_rsrc(p.x, MM * KL * 2), rw = mk_rsrc(p.w, KL * NN * 2);
            const rsrc_t rp = mk_rsrc(myws + OFF_PACK + (size_t)par * PACK_B, PACK_B);
            for (int idx = wg; idx < 1024; idx += NGEMM) {
                const int g8 = idx >> 7, mat = (idx >> 6) & 1, kk = (idx >> 5) & 1, blk = idx & 31, t = g8 * 8 + wid;
                unsigned w[4][4];
                if (mat == 0) {
                    BAR;
                    const unsigned vo = (unsigned)((blk * 256 + (tid >> 3)) * (KL * 2) + kk * 1024 + g8 * 128 + (((tid & 7) ^ ((tid >> 4) & 7)) << 4));
#pragma unroll
                    for (int i = 0; i < 4; ++i)
                        __builtin_amdgcn_raw_ptr_buffer_load_lds(rx, (LAS void*)(lds + i * 8192 + wid * 1024), 16, vo, i * 64 * KL * 2, 0, 0);
                    WAIT_V(0); BAR;
#pragma unroll
                    for (int c = 0; c < 4; ++c) {
                        const u32x4 v = *(const LAS u32x4*)(lds + (c * 64 + lane) * 128 + ((wid ^ ((lane >> 1) & 7)) << 4));
                        w[c][0] = v[0]; w[c][1] = v[1]; w[c][2] = v[2]; w[c][3] = v[3];
                    }
                } else {
#pragma unroll
                    for (int c = 0; c < 4; ++c) {
                        const int k8 = (c & 1) * 4 + fq, ch = fr ^ (((k8 & 3) << 2) | ((rank & 1) << 1));
                        const u32x4 v = __builtin_amdgcn_raw_buffer_load_b128(rw, (unsigned)(k8 * (NN * 2) + (c >> 1) * 256 + ch * 16),
                                                                              (unsigned)((kk * 512 + t * 8) * (NN * 2) + blk * 512), 0);
                        w[c][0] = v[0]; w[c][1] = v[1]; w[c][2] = v[2]; w[c][3] = v[3];
                    }
                }
                unsigned h1, e0, e1;
                encode_unit(w, rp, (unsigned)(((mat * 2 + kk) * 32 + blk) * 64 + t) * UNIT_B, lane, h1, e0, e1);
                WAIT_V(0);
                const int cb = mat == 0 ? ((blk >> 2) * 2 + (blk & 1)) : (blk & 15);
                const int c0 = mat == 0 ? ((blk >> 1) & 1) + 4 * kk : 2 * (blk >> 4) + 4 * kk, c1 = c0 + (mat == 0 ? 2 : 1);
                if (lane < 2) {
                    char* dst = (lane == 0 ? p.ws[c0 & 7] : p.ws[c1 & 7]) + OFF_IH + (size_t)par * IH_B + (size_t)((((mat * 16 + cb) * NR + rank) * 64 + t) * 16);
                    gst128_sys(dst, (u32x4){call, h1, e0, e1});
                }
            }
        }
        STAMP(dslot); ++dslot;
        const int pm = 2 * (wg & 7) + ((wg >> 3) & 1), pn = wg >> 4;
        const rsrc_t ra = mk_rsrc(myws + OFF_STG, STG_B), rb = mk_rsrc(myws + OFF_STG + STG_B, STG_B);
        const rsrc_t rc = mk_rsrc(myws + OFF_CP, CP_B);
        const unsigned voff = (unsigned)(wid * 1024 + lane * 16);
        constexpr unsigned S_K = HTB, S_H = NT * HTB, S_T = 2 * NT * HTB;
        const unsigned ldsw = (unsigned)wid * 1024u;
        const int aoff = fq * 2048 + (wr * 64 + fr) * 16;
        unsigned boff[2];
        { const int q = (lane & 15) >> 2, pp = lane & 3, g = fq;
#pragma unroll
          for (int n = 0; n < 2; ++n) boff[n] = (unsigned)(size_t)lds + 4 * HTB + 256 * (8 * g + q) + 16 * (((wc ^ q) << 2) | ((n ^ (g & 1)) << 1) | (pp >> 1)) + 8 * (pp & 1); }
        const unsigned* rdyf = (const unsigned*)(myws + OFF_RDY);
#define SA(b, h) (((b) * 2 + (h)) * HTB)
#define SB(b, h) ((4 + (b) * 2 + (h)) * HTB)
#define STAGE(rs, bufoff, soff) do { _Pragma("unroll") for (int _i = 0; _i < 2; ++_i) \
        __builtin_amdgcn_raw_ptr_buffer_load_lds(rs, (LAS void*)(lds + (bufoff) + ldsw + _i * 8192), 16, voff, (soff) + _i * 8192, 0, AUX_SYS); } while (0)
#define LDA(dst, b, h) do { _Pragma("unroll") for (int m = 0; m < 4; ++m) _Pragma("unroll") for (int k = 0; k < 2; ++k) dst[m][k] = *(const LAS bf16x8*)(lds + SA(b, h) + aoff + m * 256 + k * 8192); } while (0)
#define LDB(dst, b, h) do { _Pragma("unroll") for (int n = 0; n < 2; ++n) _Pragma("unroll") for (int k = 0; k < 2; ++k) { \
        TRD(dst[n][k][0], boff[n], ((b) * 2 + (h)) * HTB + k * 8192); TRD(dst[n][k][1], boff[n], ((b) * 2 + (h)) * HTB + k * 8192 + 1024); } } while (0)
#define MMA(ai, bj, At, Bt) do { __builtin_amdgcn_s_setprio(1); _Pragma("unroll") for (int m = 0; m < 4; ++m) _Pragma("unroll") for (int n = 0; n < 2; ++n) _Pragma("unroll") for (int k = 0; k < 2; ++k) \
        acc[ai][bj][m][n] = __builtin_amdgcn_mfma_f32_16x16x32_bf16(PKB(Bt[n][k]), At[m][k], acc[ai][bj][m][n], 0, 0, 0); __builtin_amdgcn_s_setprio(0); } while (0)
#define WAIT_B(X, Y) asm volatile("s_waitcnt lgkmcnt(0)" : "+v"(X[0][0][0]), "+v"(X[0][0][1]), "+v"(X[0][1][0]), "+v"(X[0][1][1]), "+v"(X[1][0][0]), "+v"(X[1][0][1]), "+v"(X[1][1][0]), "+v"(X[1][1][1]), \
        "+v"(Y[0][0][0]), "+v"(Y[0][0][1]), "+v"(Y[0][1][0]), "+v"(Y[0][1][1]), "+v"(Y[1][0][0]), "+v"(Y[1][0][1]), "+v"(Y[1][1][0]), "+v"(Y[1][1][1]) :: "memory")
#define WAIT_B1(X) asm volatile("s_waitcnt lgkmcnt(0)" : "+v"(X[0][0][0]), "+v"(X[0][0][1]), "+v"(X[0][1][0]), "+v"(X[0][1][1]), "+v"(X[1][0][0]), "+v"(X[1][0][1]), "+v"(X[1][1][0]), "+v"(X[1][1][1]) :: "memory")
#define GATE(tt) do { if ((int)(tt) >= rdy) { \
            const unsigned long long _t0 = __builtin_amdgcn_s_memrealtime(); \
            for (;;) { \
                const int _t = min(rdy + ((lane >> 3) & 3), NT - 1); \
                const unsigned _v = gld32_sys(rdyf + (((lane >> 5) * 16 + (lane < 32 ? pm : pn)) * 64 + _t) * NR + (lane & 7)); \
                const unsigned long long _b = __ballot((int)(_v - call) >= 0); \
                const unsigned _m = ~((unsigned)_b & (unsigned)(_b >> 32)); \
                rdy += _m ? (int)(__builtin_ctz(_m) >> 3) : 4; \
                if ((int)(tt) < rdy) break; \
                if (timed_out(tmo, call, _t0)) { rdy = NT; break; } \
                __builtin_amdgcn_s_sleep(8); \
            } } } while (0)
        const int owner = pm >> 1;
        char* chd = p.ws[owner] + OFF_CH;
        f32x4 acc[2][2][4][2];
#pragma unroll
        for (int a = 0; a < 2; ++a)
#pragma unroll
            for (int b = 0; b < 2; ++b)
#pragma unroll
                for (int m = 0; m < 4; ++m)
#pragma unroll
                    for (int n = 0; n < 2; ++n) acc[a][b][m][n] = (f32x4){0.f, 0.f, 0.f, 0.f};
        bf16x8 At[4][2]; s16x4 B0[2][2][2], B1[2][2][2];
        int rdy = 0;
        const unsigned cA = (unsigned)pm * S_T, cB = (unsigned)pn * S_T;
        GATE(0);
        STAGE(rb, SB(0, 0), cB); STAGE(rb, SB(0, 1), cB + S_H); STAGE(ra, SA(0, 0), cA); STAGE(ra, SA(0, 1), cA + S_H);
        GATE(1);
        if (wr == 1) BAR;
        WAIT_V(2); BAR;
        STAGE(rb, SB(1, 0), cB + S_K); STAGE(ra, SA(1, 0), cA + S_K); STAGE(rb, SB(1, 1), cB + S_H + S_K);
        WAIT_V(6); BAR;
        STAMP(dslot); ++dslot;
        for (int t = 0; t < (NOGEMM ? 0 : NT); t += 2) {
            const int t2 = min(t + 2, NT - 1), t3 = min(t + 3, NT - 1);
            const unsigned a1 = cA + (unsigned)(t + 1) * S_K;
            const unsigned a2 = cA + (unsigned)t2 * S_K, b2 = cB + (unsigned)t2 * S_K, a3 = cA + (unsigned)t3 * S_K, b3 = cB + (unsigned)t3 * S_K;
            LDB(B0, 0, 0); LDB(B1, 0, 1); SCHED; LDA(At, 0, 0); STAGE(ra, SA(1, 1), a1 + S_H);
            WAIT_V(8); WAIT_B(B0, B1); BAR; MMA(0, 0, At, B0); MMA(0, 1, At, B1); BAR; SCHED;
            GATE(t2);
            LDA(At, 0, 1); STAGE(rb, SB(0, 0), b2); STAGE(rb, SB(0, 1), b2 + S_H); STAGE(ra, SA(0, 0), a2);
            WAIT_V(8); WAIT_L(0); BAR; MMA(1, 0, At, B0); MMA(1, 1, At, B1); BAR; SCHED;
            LDB(B0, 1, 0); LDB(B1, 1, 1); SCHED; LDA(At, 1, 0); STAGE(ra, SA(0, 1), a2 + S_H);
            WAIT_V(8); WAIT_B(B0, B1); BAR; MMA(0, 0, At, B0); MMA(0, 1, At, B1); BAR; SCHED;
            GATE(t3);
            LDA(At, 1, 1); STAGE(rb, SB(1, 0), b3); STAGE(rb, SB(1, 1), b3 + S_H); STAGE(ra, SA(1, 0), a3);
            WAIT_V(8); WAIT_L(0); BAR; MMA(1, 0, At, B0); MMA(1, 1, At, B1); BAR; SCHED;
        }
        if (wr == 0) BAR;
        STAMP(dslot); ++dslot;
#define EPI_GROUP(A, pnx, g, H1, E0, E1) do { unsigned _w[4][4]; \
            _Pragma("unroll") for (int m = 0; m < 4; ++m) { \
                _w[m][0] = cvtpk(A[m][0][0], A[m][0][1]); _w[m][1] = cvtpk(A[m][0][2], A[m][0][3]); \
                _w[m][2] = cvtpk(A[m][1][0], A[m][1][1]); _w[m][3] = cvtpk(A[m][1][2], A[m][1][3]); } \
            encode_unit(_w, rc, (unsigned)(((pm * 16 + (pnx)) * 8 + wid) * 4 + (g)) * UNIT_B, lane, H1, E0, E1); } while (0)
#define CH_ADDR(pnx, g) (chd + (size_t)(((((bi * 2 + (pm & 1)) * 32 + bj_ * 16 + (pnx)) * 8 + wid) * 2 + bk) * 4 + (g)) * 16)
        {
            unsigned hv1 = 0, hv2 = 0, hv3 = 0;
#pragma unroll
            for (int g = 0; g < 4; ++g) {
                unsigned h1, e0, e1;
                EPI_GROUP(acc[g >> 1][g & 1], pn, g, h1, e0, e1);
                if (lane == g) { hv1 = h1; hv2 = e0; hv3 = e1; }
            }
            WAIT_V(0);
            if (__hip_atomic_load(tmo, __ATOMIC_RELAXED, __HIP_MEMORY_SCOPE_AGENT) == call) hv1 = POISON << 8;
            if (lane < 4) gst128_sys(CH_ADDR(pn, lane), (u32x4){call, hv1, hv2, hv3});
        }
        STAMP(dslot); ++dslot;
        for (int e = pn; e < 4 * PN2; e += PN1) {
            const int ai = e & 1, bj = (e >> 1) & 1, pn2 = PN1 + (e >> 2);
            {
                const unsigned long long t0 = __builtin_amdgcn_s_memrealtime();
                const u32x4* f = (const u32x4*)(rdyf + ((16 + pn2) * 64) * NR) + lane;
                for (;;) {
                    u32x4 v0, v1;
                    asm volatile("global_load_dwordx4 %0, %2, off sc0 sc1\n\tglobal_load_dwordx4 %1, %2, off offset:1024 sc0 sc1\n\ts_waitcnt vmcnt(0)"
                                 : "=&v"(v0), "=&v"(v1) : "v"(f) : "memory");
                    bool ok = true;
#pragma unroll
                    for (int q = 0; q < 4; ++q) ok = ok && (int)(v0[q] - call) >= 0 && (int)(v1[q] - call) >= 0;
                    if (__all(ok)) break;
                    if (timed_out(tmo, call, t0)) break;
                    __builtin_amdgcn_s_sleep(8);
                }
            }
            BAR;
            STAMP(dslot); ++dslot;
            const unsigned gA = (unsigned)(2 * pm + ai) * S_H, gB = (unsigned)(2 * pn2 + bj) * S_H;
#pragma unroll
            for (int m = 0; m < 4; ++m)
#pragma unroll
                for (int n = 0; n < 2; ++n) acc[0][0][m][n] = (f32x4){0.f, 0.f, 0.f, 0.f};
#pragma unroll
            for (int q = 0; q < 3; ++q) { STAGE(ra, q * HTB, gA + q * S_K); STAGE(rb, (4 + q) * HTB, gB + q * S_K); }
            for (int t = 0; t < (NOGEMM ? 0 : NT); t += 4) {
#pragma unroll
                for (int q = 0; q < 4; ++q) {
                    const unsigned tn = (unsigned)min(t + q + 3, NT - 1) * S_K;
                    WAIT_V(8); BAR;
                    STAGE(ra, ((q + 3) & 3) * HTB, gA + tn); STAGE(rb, (4 + ((q + 3) & 3)) * HTB, gB + tn);
                    LDB(B0, q >> 1, q & 1); SCHED; LDA(At, q >> 1, q & 1);
                    WAIT_B1(B0);
                    MMA(0, 0, At, B0);
                    SCHED;
                }
            }
            WAIT_V(0);
            unsigned h1, e0, e1;
            EPI_GROUP(acc[0][0], pn2, ai * 2 + bj, h1, e0, e1);
            WAIT_V(0);
            if (__hip_atomic_load(tmo, __ATOMIC_RELAXED, __HIP_MEMORY_SCOPE_AGENT) == call) h1 = POISON << 8;
            if (lane == 0) gst128_sys(CH_ADDR(pn2, ai * 2 + bj), (u32x4){call, h1, e0, e1});
            STAMP(dslot); ++dslot;
        }
#undef SA
#undef SB
    } else {
        const int cw = (wg - NGEMM) * 8 + wid;
        {
            const rsrc_t rs = mk_rsrc(myws + OFF_STG, 2 * STG_B);
            const int s = cw & 7;
            const rsrc_t rp = mk_rsrc(p.ws[s] + OFF_PACK + (size_t)par * PACK_B, PACK_B);
            const unsigned* ih = (const unsigned*)(myws + OFF_IH + (size_t)par * IH_B);
            unsigned* rdyf = (unsigned*)(myws + OFF_RDY);
            for (int q = cw; q < NJ_IN; q += NCW) {
                int r = q >> 3, tp, e;
                if (q < NJ1) { tp = r / (16 + PN1); e = r % (16 + PN1); } else { r -= NJ1 >> 3; tp = r / PN2; e = 16 + PN1 + r % PN2; }
                const int mat = e >> 4, blk = e & 15, t = tp * UPJ;
                const int gblk = mat == 0 ? (blk >> 1) * 4 + bi * 2 + (blk & 1) : bj_ * 16 + blk;
                const unsigned* hp = ih + (((mat * 16 + blk) * NR + s) * 64 + t) * 4 + min(lane, UPJ * 4 - 1);
                unsigned hv;
                bool bad = false;
                {
                    const bool chk = lane < UPJ * 4 && (lane & 3) == 0;
                    hv = gld32_sys(hp);
                    if (!__all(!chk || (int)(hv - call) >= 0)) {
                        const unsigned long long t0 = __builtin_amdgcn_s_memrealtime();
                        for (;;) {
                            __builtin_amdgcn_s_sleep(4);
                            hv = gld32_sys(hp);
                            if (__all(!chk || (int)(hv - call) >= 0)) break;
                            if (timed_out(tmo, call, t0)) { bad = true; break; }
                        }
                    }
                }
                if (bad) break;
                const unsigned so = (unsigned)(((mat * 2 + bk) * 32 + gblk) * 64 + t) * UNIT_B;
                u32x4 d[UPJ][4];
#pragma unroll
                for (int u = 0; u < UPJ; ++u) {
#pragma unroll
                    for (int c = 0; c < 3; ++c) d[u][c] = __builtin_amdgcn_raw_buffer_load_b128(rp, lane * 16, so + u * UNIT_B + c * 1024, AUX_SYS);
                    if ((unsigned)__builtin_amdgcn_readlane((int)hv, u * 4 + 1) >> 8 == RAWMODE)
                        d[u][3] = __builtin_amdgcn_raw_buffer_load_b128(rp, lane * 16, so + u * UNIT_B + 3072, AUX_SYS);
                }
#pragma unroll
                for (int u = 0; u < UPJ; ++u) {
                    const unsigned h1 = __builtin_amdgcn_readlane((int)hv, u * 4 + 1);
                    const unsigned dst = (unsigned)mat * STG_B + (unsigned)(2 * blk * 64 + t + u) * HTB + (unsigned)s * 2048u;
                    if (h1 >> 8 != RAWMODE) {
                        unsigned H[8];
                        decode_high(d[u][2], h1, __builtin_amdgcn_readlane((int)hv, u * 4 + 2), __builtin_amdgcn_readlane((int)hv, u * 4 + 3), lane, H);
#pragma unroll
                        for (int m = 0; m < 4; ++m) {
                            const unsigned l0 = d[u][m >> 1][(m & 1) * 2], l1 = d[u][m >> 1][(m & 1) * 2 + 1];
                            st128<AUX_SYS>((u32x4){vperm(H[2 * m], l0, 0x05010400u), vperm(H[2 * m], l0, 0x07030602u),
                                                   vperm(H[2 * m + 1], l1, 0x05010400u), vperm(H[2 * m + 1], l1, 0x07030602u)},
                                           rs, lane * 16, dst + (m >> 1) * (64 * HTB) + (m & 1) * 1024);
                        }
                    } else {
#pragma unroll
                        for (int m = 0; m < 4; ++m) st128<AUX_SYS>(d[u][m], rs, lane * 16, dst + (m >> 1) * (64 * HTB) + (m & 1) * 1024);
                    }
                }
                WAIT_V(0);
                if (lane < UPJ) __hip_atomic_store(rdyf + ((mat * 16 + blk) * 64 + t + lane) * NR + s, call, __ATOMIC_RELAXED, __HIP_MEMORY_SCOPE_SYSTEM);
                if (q == cw) STAMPW(8);
                if (q < NJ1 && q + NCW >= NJ1) STAMPW(16);
            }
        }
        STAMPW(24);
        {
            const rsrc_t ro = mk_rsrc(p.out, (MM / NR) * NN * 2);
            for (int q = cw; q < NJ_OUT; q += NCW) {
                const int part = q % JPS, w8 = (q / JPS) & 7;
                int y = q / (JPS * 8), pn;
                if (y < 8 * PN1) pn = y >> 3; else { y -= 8 * PN1; pn = PN1 + (y >> 3); }
                const int ti = y & 1, tj = (y >> 1) & 1, sub = (y >> 2) & 1;
                const int rt = ti * 2 + sub, ct = tj * 16 + pn;
                const int g0 = part * GPJ;
                const unsigned* hp = (const unsigned*)(myws + OFF_CH) + ((rt * 32 + ct) * 8 + w8) * 32 + (lane & 31);
                const bool mine = (unsigned)(((lane >> 2) & 3) - g0) < (unsigned)GPJ && (lane & 3) == 0;
                unsigned hv = gld32_sys(hp);
                if (!__all(!mine || (int)(hv - call) >= 0)) {
                    const unsigned long long t0 = __builtin_amdgcn_s_memrealtime();
                    for (;;) {
                        __builtin_amdgcn_s_sleep(4);
                        hv = gld32_sys(hp);
                        if (__all(!mine || (int)(hv - call) >= 0)) break;
                        if (timed_out(tmo, call, t0)) break;
                    }
                }
                bool bad = __hip_atomic_load(tmo, __ATOMIC_RELAXED, __HIP_MEMORY_SCOPE_AGENT) == call;
                bad = bad || __any(((lane & 3) == 1) && (unsigned)(((lane >> 2) & 3) - g0) < (unsigned)GPJ && (hv >> 8) == POISON);
                const int wr8 = w8 >> 2, wc8 = w8 & 3;
                const unsigned so = (unsigned)((((rank * 2 + sub) * 16 + pn) * 8 + w8) * 4 + g0) * UNIT_B;
                u32x4 d[GPJ][2][4];
                if (!bad) {
#pragma unroll
                    for (int g = 0; g < GPJ; ++g)
#pragma unroll
                        for (int k = 0; k < 2; ++k) {
                            const rsrc_t rc = mk_rsrc(p.ws[(ti + 2 * tj + 4 * k) & 7] + OFF_CP, CP_B);
#pragma unroll
                            for (int c = 0; c < 3; ++c) d[g][k][c] = __builtin_amdgcn_raw_buffer_load_b128(rc, lane * 16, so + g * UNIT_B + c * 1024, AUX_SYS);
                            if ((unsigned)__builtin_amdgcn_readlane((int)hv, (k * 4 + g0 + g) * 4 + 1) >> 8 == RAWMODE)
                                d[g][k][3] = __builtin_amdgcn_raw_buffer_load_b128(rc, lane * 16, so + g * UNIT_B + 3072, AUX_SYS);
                        }
                }
#pragma unroll
                for (int g = 0; g < GPJ; ++g) {
                    const int gg = g0 + g, ai = gg >> 1, bj = gg & 1;
                    const unsigned orow = rt * 256 + ai * 128 + wr8 * 64 + fr;
                    const unsigned ocol = ct * 256 + bj * 128 + wc8 * 32 + ((fq & 1) * 16 + (fq >> 1) * 8);
                    const unsigned ooff = (orow * NN + ocol) * 2;
                    if (bad) {
#pragma unroll
                        for (int m = 0; m < 4; ++m)
                            st128<0>((u32x4){0x7fc07fc0u, 0x7fc07fc0u, 0x7fc07fc0u, 0x7fc07fc0u}, ro, ooff, m * 16 * NN * 2);
                        continue;
                    }
                    float a[4][8];
#pragma unroll
                    for (int k = 0; k < 2; ++k) {
                        const int hl = (k * 4 + gg) * 4;
                        const unsigned h1 = __builtin_amdgcn_readlane((int)hv, hl + 1);
                        if (h1 >> 8 == RAWMODE) {
#pragma unroll
                            for (int m = 0; m < 4; ++m)
#pragma unroll
                                for (int j = 0; j < 4; ++j) {
                                    const unsigned v = d[g][k][m][j];
                                    const float lo = __uint_as_float(v << 16), hi = __uint_as_float(v & 0xffff0000u);
                                    a[m][2 * j] = k == 0 ? lo : a[m][2 * j] + lo; a[m][2 * j + 1] = k == 0 ? hi : a[m][2 * j + 1] + hi;
                                }
                        } else {
                            unsigned H[8];
                            decode_high(d[g][k][2], h1, __builtin_amdgcn_readlane((int)hv, hl + 2), __builtin_amdgcn_readlane((int)hv, hl + 3), lane, H);
#pragma unroll
                            for (int m = 0; m < 4; ++m)
#pragma unroll
                                for (int j = 0; j < 8; ++j) {
                                    const unsigned Lw = d[g][k][m >> 1][(m & 1) * 2 + (j >> 2)];
                                    const float v = __uint_as_float(vperm(H[2 * m + (j >> 2)], Lw, (4u + (j & 3)) << 24 | (unsigned)(j & 3) << 16 | 0x0c0cu));
                                    a[m][j] = k == 0 ? v : a[m][j] + v;
                                }
                        }
                    }
#pragma unroll
                    for (int m = 0; m < 4; ++m) {
                        unsigned r0 = cvtpk(a[m][0], a[m][1]), r1 = cvtpk(a[m][2], a[m][3]), r2 = cvtpk(a[m][4], a[m][5]), r3 = cvtpk(a[m][6], a[m][7]);
                        auto s02 = __builtin_amdgcn_permlane16_swap(r0, r2, false, false);
                        auto s13 = __builtin_amdgcn_permlane16_swap(r1, r3, false, false);
                        u32x4 o; o.x = s02[0]; o.y = s13[0]; o.z = s02[1]; o.w = s13[1];
                        st128<0>(o, ro, ooff, m * 16 * NN * 2);
                    }
                }
                if (q == cw) STAMPW(32);
            }
        }
        STAMPW(40);
    }
}

}

extern "C" void kernel_launch(int rank, int nranks, ncclComm_t comm,
                              hipStream_t stream,
                              void* const* d_in, const int* in_sizes, int n_in,
                              void* d_out, int out_size,
                              const int* in_dtypes, int out_dtype,
                              void* const* d_out_all,
                              void* const* d_ws_all, size_t ws_bytes) {
    (void)comm; (void)in_dtypes; (void)out_dtype; (void)d_out_all;
    if (nranks != NR || n_in < 2 || in_sizes[0] != MM * KL || in_sizes[1] != KL * NN || out_size != (MM / NR) * NN
        || ws_bytes < WS_NEED || (unsigned)rank >= (unsigned)NR) {
        fprintf(stderr, "kernel_launch: unexpected shapes\n");
        return;
    }
    static bool inited[NR] = {};
    if (!inited[rank]) {
        HIP_CHECK(hipFuncSetAttribute((const void*)gemm_rs_kernel, hipFuncAttributeMaxDynamicSharedMemorySize, LDS_BYTES));
        inited[rank] = true;
    }
    Params p;
    p.x = d_in[0]; p.w = d_in[1]; p.out = d_out; p.rank = rank;
    for (int i = 0; i < NR; ++i) p.ws[i] = (char*)d_ws_all[i];
    hipLaunchKernelGGL(gemm_rs_kernel, dim3(NWG), dim3(NTHR), LDS_BYTES, stream, p);
    HIP_CHECK(hipGetLastError());
}
```

```cpp
#include <hip/hip_runtime.h>
#include <rccl/rccl.h>
#include <cstdio>
#include <cstdint>

#define HIP_CHECK(expr) do { hipError_t _e = (expr); if (_e != hipSuccess) { \
    fprintf(stderr, "HIP error %s at %s:%d\n", hipGetErrorString(_e), __FILE__, __LINE__); \
    return; } } while (0)

#define LAS __attribute__((address_space(3)))
typedef short bf16x8 __attribute__((ext_vector_type(8)));
typedef short s16x4 __attribute__((ext_vector_type(4)));
typedef float f32x4 __attribute__((ext_vector_type(4)));
typedef float f32x2 __attribute__((ext_vector_type(2)));
typedef unsigned u32x4 __attribute__((ext_vector_type(4)));
typedef unsigned short u16x2 __attribute__((ext_vector_type(2)));
typedef __bf16 bf16x2_t __attribute__((ext_vector_type(2)));
typedef __amdgpu_buffer_rsrc_t rsrc_t;

namespace {

constexpr int NR = 8;
constexpr int MM = 8192, NN = 8192, KL = 1024;
constexpr int NWG = 256, NTHR = 512;
#define PN1 13
#define LOOK 8
#define CW_IN 2
#define CW_OUT 2
#define FORCE_RAW 0
#define NOGEMM 0
#define GAUX 17
#define OAUX 0
constexpr int NGEMM = 16 * PN1, NCOMM = NWG - NGEMM, NCW = NCOMM * 8;
constexpr int PN2 = 16 - PN1;
constexpr int HTB = 16384, STAGE_BYTES = 8 * HTB;
constexpr int NT = 64;
constexpr unsigned UNIT_B = 4096;
constexpr size_t MIB = 1u << 20;
constexpr size_t OFF_PACK = 0;
constexpr unsigned PACK_B = 32u << 20;
constexpr size_t OFF_STG = 64 * MIB;
constexpr unsigned STG_B = 32u << 20;
constexpr size_t OFF_CP = 128 * MIB;
constexpr unsigned CP_B = 32u << 20;
constexpr size_t OFF_IH = 160 * MIB;
constexpr unsigned IH_B = 2 * 16 * 8 * 64 * 16;
constexpr size_t OFF_RDY = 161 * MIB;
constexpr size_t OFF_CH = 162 * MIB;
constexpr size_t CTRL_OFF = 256 * MIB;
constexpr size_t WS_NEED = CTRL_OFF + MIB;
constexpr unsigned C_TMO = 256, C_CNT = 4096;
constexpr unsigned MAXEXC = 2, RAWMODE = 255, POISON = 254;
constexpr unsigned long long DEADLINE = 30000000ull;
constexpr int LDS_AUX = STAGE_BYTES;
constexpr int LDS_BYTES = STAGE_BYTES + 8192;
constexpr int AUX_SYS = 17;
constexpr int E1 = 16 + PN1;
constexpr int NJ1 = (NT / 2) * E1, NJ_IN = (NT / 2) * 32;
constexpr int W_IN = NCOMM * CW_IN / (NR - 1);
constexpr int NJO1 = 64 * PN1, NJ_OUT = 1024, W_OUT = NCOMM * CW_OUT / 4;

#define STAMP(slot) do {} while (0)
#define STAMPW(slot) do {} while (0)
#define DIAG_BLK0 do {} while (0)
#define DIAG_BLK1(t0) do {} while (0)
struct Params {
    const void* x; const void* w; void* out;
    char* ws[NR];
    int rank;
};

__device__ __forceinline__ unsigned cvtpk(float lo, float hi) { f32x2 v = {lo, hi}; bf16x2_t b = __builtin_convertvector(v, bf16x2_t); return __builtin_bit_cast(unsigned, b); }
__device__ __forceinline__ rsrc_t mk_rsrc(const void* p, unsigned bytes) { return __builtin_amdgcn_make_buffer_rsrc((void*)p, 0, bytes, 0x00020000); }

template <int AUX> __device__ __forceinline__ void st128(u32x4 v, rsrc_t r, unsigned voff, unsigned soff) {
    __builtin_amdgcn_raw_buffer_store_b128(v, r, voff, soff, AUX);
    asm volatile("s_nop 1" :: "v"(v) : "memory");
}
__device__ __forceinline__ void gst128_sys(void* ptr, u32x4 v) {
    asm volatile("global_store_dwordx4 %0, %1, off sc0 sc1\n\ts_nop 1" :: "v"(ptr), "v"(v) : "memory");
}
__device__ __forceinline__ unsigned gld32_sys(const void* ptr) {
    unsigned v;
    asm volatile("global_load_dword %0, %1, off sc0 sc1\n\ts_waitcnt vmcnt(0)" : "=&v"(v) : "v"(ptr) : "memory");
    return v;
}
__device__ __forceinline__ void gld32x2_sys(const void* pa, const void* pb, unsigned& a, unsigned& b) {
    asm volatile("global_load_dword %0, %2, off sc0 sc1\n\tglobal_load_dword %1, %3, off sc0 sc1\n\ts_waitcnt vmcnt(0)"
                 : "=&v"(a), "=&v"(b) : "v"(pa), "v"(pb) : "memory");
}
template <int OFF> __device__ __forceinline__ void gst128s_sys(const void* base, unsigned voff, u32x4 v) {
    asm volatile("global_store_dwordx4 %0, %1, %2 offset:%c3 sc0 sc1\n\ts_nop 1" :: "v"(voff), "v"(v), "s"(base), "i"(OFF) : "memory");
}
__device__ __forceinline__ unsigned bfi(unsigned m, unsigned a, unsigned b) { return (a & m) | (b & ~m); }
__device__ __forceinline__ unsigned vperm(unsigned hi, unsigned lo, unsigned sel) { return __builtin_amdgcn_perm(hi, lo, sel); }
__device__ __forceinline__ unsigned rfl(unsigned v) { return __builtin_amdgcn_readfirstlane(v); }
__device__ __forceinline__ unsigned wave_max_u(int x) {
    x = max(x, __builtin_amdgcn_update_dpp(0, x, 0xB1, 0xf, 0xf, true));
    x = max(x, __builtin_amdgcn_update_dpp(0, x, 0x4E, 0xf, 0xf, true));
    x = max(x, __builtin_amdgcn_update_dpp(0, x, 0x141, 0xf, 0xf, true));
    x = max(x, __builtin_amdgcn_update_dpp(0, x, 0x140, 0xf, 0xf, true));
    const int a = __builtin_amdgcn_readlane(x, 0), b = __builtin_amdgcn_readlane(x, 16), c = __builtin_amdgcn_readlane(x, 32), d = __builtin_amdgcn_readlane(x, 48);
    return (unsigned)max(max(a, b), max(c, d));
}
__device__ __forceinline__ bool timed_out(unsigned* tmo, unsigned call, unsigned long long t0) {
    if (__hip_atomic_load(tmo, __ATOMIC_RELAXED, __HIP_MEMORY_SCOPE_AGENT) == call) return true;
    if (__builtin_amdgcn_s_memrealtime() - t0 > DEADLINE) { __hip_atomic_store(tmo, call, __ATOMIC_RELAXED, __HIP_MEMORY_SCOPE_AGENT); return true; }
    return false;
}

__device__ __forceinline__ void encode_unit(const unsigned (&w)[4][4], rsrc_t r, unsigned so, int lane, unsigned& h1, unsigned& e0, unsigned& e1) {
    u16x2 mx = {0, 0};
#pragma unroll
    for (int m = 0; m < 4; ++m)
#pragma unroll
        for (int q = 0; q < 4; ++q) mx = __builtin_elementwise_max(mx, __builtin_bit_cast(u16x2, w[m][q] & 0x7f007f00u));
    const unsigned base = wave_max_u((int)max(mx[0], mx[1])) >> 8;
    const unsigned bq = base * 0x01010101u | 0x80808080u;
    unsigned L[8], P[4], ex = 0;
#pragma unroll
    for (int m = 0; m < 4; ++m) {
        L[2 * m] = vperm(w[m][1], w[m][0], 0x06040200u); L[2 * m + 1] = vperm(w[m][3], w[m][2], 0x06040200u);
        const unsigned h01 = vperm(w[m][1], w[m][0], 0x07050301u), h23 = vperm(w[m][3], w[m][2], 0x07050301u);
        const unsigned d01 = bq - (h01 & 0x7f7f7f7fu), d23 = bq - (h23 & 0x7f7f7f7fu);
        ex |= d01 | d23;
        P[m] = bfi(0x0f0f0f0fu, bfi(0x07070707u, d01, h01 >> 4), bfi(0x70707070u, d23 << 4, h23));
    }
    unsigned cnt = FORCE_RAW ? RAWMODE : 0;
    e0 = 0; e1 = 0;
    if (__any((ex & 0x78787878u) != 0)) {
#pragma unroll
        for (int m = 0; m < 4; ++m)
#pragma unroll
            for (int j = 0; j < 8; ++j) {
                const unsigned hb = (w[m][j >> 1] >> (8 + 16 * (j & 1))) & 0xffu;
                unsigned long long bal = __ballot(base - (hb & 0x7fu) > 7u);
                while (bal && cnt <= MAXEXC) {
                    const unsigned l = (unsigned)__builtin_ctzll(bal);
                    const unsigned en = (unsigned)(m << 17 | j << 8) | l << 11 | (unsigned)__builtin_amdgcn_readlane((int)hb, (int)l);
                    if (cnt == 0) e0 = en; else e1 = en;
                    ++cnt; bal &= bal - 1;
                }
            }
        if (cnt > MAXEXC) cnt = RAWMODE;
    }
    if (cnt == RAWMODE) {
#pragma unroll
        for (int m = 0; m < 4; ++m) st128<AUX_SYS>((u32x4){w[m][0], w[m][1], w[m][2], w[m][3]}, r, lane * 16, so + m * 1024);
    } else {
        st128<AUX_SYS>((u32x4){L[0], L[1], L[2], L[3]}, r, lane * 16, so);
        st128<AUX_SYS>((u32x4){L[4], L[5], L[6], L[7]}, r, lane * 16, so + 1024);
        st128<AUX_SYS>((u32x4){P[0], P[1], P[2], P[3]}, r, lane * 16, so + 2048);
    }
    h1 = base | cnt << 8;
}
__device__ __forceinline__ void decode_high(u32x4 P, unsigned h1, unsigned e0, unsigned e1, int lane, unsigned (&H)[8]) {
    const unsigned bq = (h1 & 0x7fu) * 0x01010101u | 0x80808080u;
#pragma unroll
    for (int m = 0; m < 4; ++m) {
        H[2 * m] = bfi(0x7f7f7f7fu, bq - (P[m] & 0x07070707u), P[m] << 4);
        H[2 * m + 1] = bfi(0x7f7f7f7fu, bq - ((P[m] >> 4) & 0x07070707u), P[m]);
    }
    const unsigned cnt = h1 >> 8;
    for (unsigned e = 0; e < cnt; ++e) {
        const unsigned en = e ? e1 : e0;
        const unsigned k = (en >> 17 & 3) * 2 + (en >> 10 & 1), sh = (en >> 8 & 3) * 8;
        const unsigned mk = (unsigned)lane == (en >> 11 & 63) ? 0xffu << sh : 0u, val = (en & 0xffu) << sh;
#pragma unroll
        for (int kk = 0; kk < 8; ++kk)
            if (k == (unsigned)kk) H[kk] = bfi(mk, val, H[kk]);
    }
}

#define TRD(dst, addr, off) asm volatile("ds_read_b64_tr_b16 %0, %1 offset:%c2" : "=&v"(dst) : "v"(addr), "i"(off) : "memory")
#define PKB(x) (bf16x8){(x)[0][0], (x)[0][1], (x)[0][2], (x)[0][3], (x)[1][0], (x)[1][1], (x)[1][2], (x)[1][3]}
#define WAIT_V(n) asm volatile("s_waitcnt vmcnt(" #n ")" ::: "memory")
#define WAIT_L(n) asm volatile("s_waitcnt lgkmcnt(" #n ")" ::: "memory")
#define BAR __builtin_amdgcn_s_barrier()
#define SCHED __builtin_amdgcn_sched_barrier(0)
#define DSR128(dst, addr, off) asm volatile("ds_read_b128 %0, %1 offset:%c2" : "=&v"(dst) : "v"(addr), "i"(off) : "memory")
#define DSR32(dst, addr) asm volatile("ds_read_b32 %0, %1" : "=&v"(dst) : "v"(addr) : "memory")
#define DMA16(rs, ldsoff, voff, soff) __builtin_amdgcn_raw_ptr_buffer_load_lds(rs, (LAS void*)(lds + (ldsoff)), 16, voff, soff, 0, AUX_SYS)
#define DMA4(rs, ldsoff, voff) __builtin_amdgcn_raw_ptr_buffer_load_lds(rs, (LAS void*)(lds + (ldsoff)), 4, voff, 0, 0, AUX_SYS)

__global__ __launch_bounds__(NTHR, 2) void gemm_rs_kernel(Params p) {
    extern __shared__ __attribute__((aligned(16))) unsigned char smem[];
    LAS unsigned char* lds = (LAS unsigned char*)smem;
    const int tid = threadIdx.x, wid = __builtin_amdgcn_readfirstlane(tid >> 6), lane = tid & 63;
    const int wr = wid >> 2, wc = wid & 3, fr = lane & 15, fq = lane >> 4;
    const int wg = blockIdx.x, rank = p.rank;
    const int bi = rank & 1, bj_ = (rank >> 1) & 1, bk = rank >> 2;
    char* myws = p.ws[rank];
    unsigned* ctl = (unsigned*)(myws + CTRL_OFF);
    unsigned* tmo = ctl + C_TMO / 4;

    int dslot = 0;
    STAMP(dslot); ++dslot;
    const unsigned ldsa = (unsigned)(size_t)lds;
    unsigned* cntp = ctl + C_CNT / 4 + wg * 8 + wid;

    if (wg < NGEMM) {
        const rsrc_t rx = mk_rsrc(p.x, MM * KL * 2), rw = mk_rsrc(p.w, KL * NN * 2);
        const unsigned vx = (unsigned)((tid >> 3) * (KL * 2) + (((tid & 7) ^ ((tid >> 4) & 7)) << 4));
        const unsigned vw = (unsigned)(fq * (NN * 2) + ((fr ^ ((fq << 2) | ((rank & 1) << 1))) << 4));
        auto pk_issue = [&](int idx, int buf) {
            const int g8 = idx >> 7, mat = (idx >> 6) & 1, kk = (idx >> 5) & 1, blk = idx & 31;
            if (mat == 0) {
#pragma unroll
                for (int i = 0; i < 4; ++i)
                    __builtin_amdgcn_raw_ptr_buffer_load_lds(rx, (LAS void*)(lds + buf + i * 8192 + wid * 1024), 16, vx,
                                                             (unsigned)((blk * 256 + i * 64) * (KL * 2) + kk * 1024 + g8 * 128), 0, 0);
            } else {
#pragma unroll
                for (int c = 0; c < 4; ++c)
                    __builtin_amdgcn_raw_ptr_buffer_load_lds(rw, (LAS void*)(lds + buf + wid * 4096 + c * 1024), 16, vw,
                                                             (unsigned)((kk * 512 + (g8 * 8 + wid) * 8 + (c & 1) * 4) * (NN * 2) + blk * 512 + (c >> 1) * 256), 0, 0);
            }
        };
        pk_issue(wg, 0);
        const unsigned call = rfl(gld32_sys(cntp)) + 1;
        if (lane == 0) __hip_atomic_store(cntp, call, __ATOMIC_RELAXED, __HIP_MEMORY_SCOPE_AGENT);
        const unsigned par = call & 1;
        unsigned* rdyw = (unsigned*)(myws + OFF_RDY);
        {
            const rsrc_t rp = mk_rsrc(myws + OFF_PACK + (size_t)par * PACK_B, PACK_B);
            const rsrc_t rs = mk_rsrc(myws + OFF_STG, 2 * STG_B);
            int buf = 0;
            for (int idx = wg; idx < 1024; idx += NGEMM, buf ^= 32768) {
                const int g8 = idx >> 7, mat = (idx >> 6) & 1, kk = (idx >> 5) & 1, blk = idx & 31, t = g8 * 8 + wid;
                BAR;
                if (idx + NGEMM < 1024) pk_issue(idx + NGEMM, buf ^ 32768);
                u32x4 v[4];
                if (mat == 0) {
                    const unsigned a = ldsa + buf + lane * 128 + ((wid ^ ((lane >> 1) & 7)) << 4);
                    DSR128(v[0], a, 0); DSR128(v[1], a, 8192); DSR128(v[2], a, 16384); DSR128(v[3], a, 24576);
                } else {
                    const unsigned a = ldsa + buf + wid * 4096 + lane * 16;
                    DSR128(v[0], a, 0); DSR128(v[1], a, 1024); DSR128(v[2], a, 2048); DSR128(v[3], a, 3072);
                }
                asm volatile("s_waitcnt lgkmcnt(0)" : "+v"(v[0]), "+v"(v[1]), "+v"(v[2]), "+v"(v[3]) :: "memory");
                unsigned w[4][4];
#pragma unroll
                for (int c = 0; c < 4; ++c) { w[c][0] = v[c][0]; w[c][1] = v[c][1]; w[c][2] = v[c][2]; w[c][3] = v[c][3]; }
                unsigned h1, e0, e1;
                encode_unit(w, rp, (unsigned)(((mat * 2 + kk) * 32 + blk) * 64 + t) * UNIT_B, lane, h1, e0, e1);
                const int cb = mat == 0 ? ((blk >> 2) * 2 + (blk & 1)) : (blk & 15);
                const int c0 = mat == 0 ? ((blk >> 1) & 1) + 4 * kk : 2 * (blk >> 4) + 4 * kk, c1 = c0 + (mat == 0 ? 2 : 1);
                const bool own = c0 == rank || c1 == rank;
                if (own) {
                    const unsigned dst = (unsigned)mat * STG_B + (unsigned)(2 * cb * 64 + t) * HTB + (unsigned)rank * 2048u;
#pragma unroll
                    for (int m = 0; m < 4; ++m) st128<AUX_SYS>(v[m], rs, lane * 16, dst + (m >> 1) * (64 * HTB) + (m & 1) * 1024);
                }
                char* const w0 = p.ws[c0 & 7];
                char* const w1 = p.ws[c1 & 7];
                WAIT_V(0);
                if (lane < 2) {
                    const int c = lane == 0 ? c0 : c1;
                    if (c != rank) gst128_sys((lane == 0 ? w0 : w1) + OFF_IH + (size_t)par * IH_B + (size_t)((((mat * 16 + cb) * NR + rank) * 64 + t) * 16), (u32x4){call, h1, e0, e1});
                    else __hip_atomic_store(rdyw + ((mat * 16 + cb) * 64 + t) * NR + rank, call, __ATOMIC_RELAXED, __HIP_MEMORY_SCOPE_SYSTEM);
                }
            }
            BAR;
        }
        STAMP(dslot); ++dslot;
        const int pm = 2 * (wg & 7) + ((wg >> 3) & 1), pn = wg >> 4;
        const rsrc_t ra = mk_rsrc(myws + OFF_STG, STG_B), rb = mk_rsrc(myws + OFF_STG + STG_B, STG_B);
        const rsrc_t rc = mk_rsrc(myws + OFF_CP, CP_B);
        const unsigned voff = (unsigned)(wid * 1024 + lane * 16);
        constexpr unsigned S_K = HTB, S_H = NT * HTB, S_T = 2 * NT * HTB;
        const unsigned ldsw = (unsigned)wid * 1024u;
        const int aoff = fq * 2048 + (wr * 64 + fr) * 16;
        unsigned boff[2];
        { const int q = (lane & 15) >> 2, pp = lane & 3, g = fq;
#pragma unroll
          for (int n = 0; n < 2; ++n) boff[n] = (unsigned)(size_t)lds + 4 * HTB + 256 * (8 * g + q) + 16 * (((wc ^ q) << 2) | ((n ^ (g & 1)) << 1) | (pp >> 1)) + 8 * (pp & 1); }
        const unsigned* rdyf = (const unsigned*)(myws + OFF_RDY);
#define SA(b, h) (((b) * 2 + (h)) * HTB)
#define SB(b, h) ((4 + (b) * 2 + (h)) * HTB)
#define STAGE(rs, bufoff, soff) do { _Pragma("unroll") for (int _i = 0; _i < 2; ++_i) \
        __builtin_amdgcn_raw_ptr_buffer_load_lds(rs, (LAS void*)(lds + (bufoff) + ldsw + _i * 8192), 16, voff, (soff) + _i * 8192, 0, GAUX); } while (0)
#define LDA(dst, b, h) do { _Pragma("unroll") for (int m = 0; m < 4; ++m) _Pragma("unroll") for (int k = 0; k < 2; ++k) dst[m][k] = *(const LAS bf16x8*)(lds + SA(b, h) + aoff + m * 256 + k * 8192); } while (0)
#define LDB(dst, b, h) do { _Pragma("unroll") for (int n = 0; n < 2; ++n) _Pragma("unroll") for (int k = 0; k < 2; ++k) { \
        TRD(dst[n][k][0], boff[n], ((b) * 2 + (h)) * HTB + k * 8192); TRD(dst[n][k][1], boff[n], ((b) * 2 + (h)) * HTB + k * 8192 + 1024); } } while (0)
#define MMA(ai, bj, At, Bt) do { __builtin_amdgcn_s_setprio(1); _Pragma("unroll") for (int m = 0; m < 4; ++m) _Pragma("unroll") for (int n = 0; n < 2; ++n) _Pragma("unroll") for (int k = 0; k < 2; ++k) \
        acc[ai][bj][m][n] = __builtin_amdgcn_mfma_f32_16x16x32_bf16(PKB(Bt[n][k]), At[m][k], acc[ai][bj][m][n], 0, 0, 0); __builtin_amdgcn_s_setprio(0); } while (0)
#define WAIT_B(X, Y) asm volatile("s_waitcnt lgkmcnt(0)" : "+v"(X[0][0][0]), "+v"(X[0][0][1]), "+v"(X[0][1][0]), "+v"(X[0][1][1]), "+v"(X[1][0][0]), "+v"(X[1][0][1]), "+v"(X[1][1][0]), "+v"(X[1][1][1]), \
        "+v"(Y[0][0][0]), "+v"(Y[0][0][1]), "+v"(Y[0][1][0]), "+v"(Y[0][1][1]), "+v"(Y[1][0][0]), "+v"(Y[1][0][1]), "+v"(Y[1][1][0]), "+v"(Y[1][1][1]) :: "memory")
#define WAIT_B1(X) asm volatile("s_waitcnt lgkmcnt(0)" : "+v"(X[0][0][0]), "+v"(X[0][0][1]), "+v"(X[0][1][0]), "+v"(X[0][1][1]), "+v"(X[1][0][0]), "+v"(X[1][0][1]), "+v"(X[1][1][0]), "+v"(X[1][1][1]) :: "memory")
        const rsrc_t rr = mk_rsrc(myws + OFF_RDY, 2 * 16 * 64 * 32);
        const unsigned pvo = (unsigned)(((lane >> 5) * 16 + (lane < 32 ? pm : pn)) * 2048 + (lane & 1) * 16);
        const unsigned pla = ldsa + LDS_AUX + wid * 1024 + lane * 16;
        int rdy = 0, pbase = 0;
        bool pol = false;
        u32x4 fv;
#define COUNT_RDY(v) do { const bool _ok = (int)((v)[0] - call) >= 0 && (int)((v)[1] - call) >= 0 && (int)((v)[2] - call) >= 0 && (int)((v)[3] - call) >= 0; \
            const unsigned long long _b = __ballot(_ok); \
            unsigned _m = (unsigned)_b & (unsigned)(_b >> 32); _m = ~(_m & (_m >> 1)) & 0x55555555u; \
            rdy = min(pbase + (_m ? (int)(__builtin_ctz(_m) >> 1) : 16), NT); } while (0)
#define FLAGREAD do { if (pol) DSR128(fv, pla, 0); } while (0)
#define GATE(tt) do { \
            if (pol) { asm volatile("s_waitcnt lgkmcnt(0)" : "+v"(fv) :: "memory"); COUNT_RDY(fv); } \
            if ((int)(tt) >= rdy) { \
                const unsigned long long _t0 = __builtin_amdgcn_s_memrealtime(); \
                DIAG_BLK0; \
                for (;;) { \
                    pbase = rdy; \
                    const char* _p = (const char*)rdyf + pvo + min(rdy + ((lane >> 1) & 15), NT - 1) * 32; \
                    u32x4 _v; \
                    asm volatile("global_load_dwordx4 %0, %1, off sc0 sc1\n\ts_waitcnt vmcnt(0)" : "=&v"(_v) : "v"(_p) : "memory"); \
                    COUNT_RDY(_v); \
                    if ((int)(tt) < rdy) break; \
                    if (timed_out(tmo, call, _t0)) { rdy = NT; break; } \
                    __builtin_amdgcn_s_sleep(8); \
                } DIAG_BLK1(_t0); } \
            pol = rdy < min((int)(tt) + LOOK, NT); \
            if (pol) { pbase = rdy; DMA16(rr, LDS_AUX + wid * 1024, pvo + min(rdy + ((lane >> 1) & 15), NT - 1) * 32, 0); } } while (0)
#define WAIT_VP do { if (pol) WAIT_V(9); else WAIT_V(8); } while (0)
        const int owner = pm >> 1;
        char* chd = p.ws[owner] + OFF_CH;
        f32x4 acc[2][2][4][2];
#pragma unroll
        for (int a = 0; a < 2; ++a)
#pragma unroll
            for (int b = 0; b < 2; ++b)
#pragma unroll
                for (int m = 0; m < 4; ++m)
#pragma unroll
                    for (int n = 0; n < 2; ++n) acc[a][b][m][n] = (f32x4){0.f, 0.f, 0.f, 0.f};
        bf16x8 At[4][2]; s16x4 B0[2][2][2], B1[2][2][2];
        const unsigned cA = (unsigned)pm * S_T, cB = (unsigned)pn * S_T;
        if (!NOGEMM) { GATE(1); }
        STAGE(rb, SB(0, 0), cB); STAGE(rb, SB(0, 1), cB + S_H); STAGE(ra, SA(0, 0), cA); STAGE(ra, SA(0, 1), cA + S_H);
        if (wr == 1) BAR;
        WAIT_V(2); BAR;
        STAGE(rb, SB(1, 0), cB + S_K); STAGE(ra, SA(1, 0), cA + S_K); STAGE(rb, SB(1, 1), cB + S_H + S_K);
        WAIT_V(6); BAR;
        STAMP(dslot); ++dslot;
        for (int t = 0; t < (NOGEMM ? 0 : NT); t += 2) {
            const int t2 = min(t + 2, NT - 1), t3 = min(t + 3, NT - 1);
            const unsigned a1 = cA + (unsigned)(t + 1) * S_K;
            const unsigned a2 = cA + (unsigned)t2 * S_K, b2 = cB + (unsigned)t2 * S_K, a3 = cA + (unsigned)t3 * S_K, b3 = cB + (unsigned)t3 * S_K;
            LDB(B0, 0, 0); LDB(B1, 0, 1); SCHED; LDA(At, 0, 0); STAGE(ra, SA(1, 1), a1 + S_H);
            WAIT_V(8); WAIT_B(B0, B1); FLAGREAD; BAR; MMA(0, 0, At, B0); MMA(0, 1, At, B1); BAR; SCHED;
            GATE(t2);
            LDA(At, 0, 1); STAGE(rb, SB(0, 0), b2); STAGE(rb, SB(0, 1), b2 + S_H); STAGE(ra, SA(0, 0), a2);
            WAIT_VP; WAIT_L(0); BAR; MMA(1, 0, At, B0); MMA(1, 1, At, B1); BAR; SCHED;
            LDB(B0, 1, 0); LDB(B1, 1, 1); SCHED; LDA(At, 1, 0); STAGE(ra, SA(0, 1), a2 + S_H);
            WAIT_V(8); WAIT_B(B0, B1); FLAGREAD; BAR; MMA(0, 0, At, B0); MMA(0, 1, At, B1); BAR; SCHED;
            GATE(t3);
            LDA(At, 1, 1); STAGE(rb, SB(1, 0), b3); STAGE(rb, SB(1, 1), b3 + S_H); STAGE(ra, SA(1, 0), a3);
            WAIT_VP; WAIT_L(0); BAR; MMA(1, 0, At, B0); MMA(1, 1, At, B1); BAR; SCHED;
        }
        if (wr == 0) BAR;
        STAMP(dslot); ++dslot;
#define EPI_GROUP(A, pnx, g, H1, E0, E1) do { unsigned _w[4][4]; \
            _Pragma("unroll") for (int m = 0; m < 4; ++m) { \
                _w[m][0] = cvtpk(A[m][0][0], A[m][0][1]); _w[m][1] = cvtpk(A[m][0][2], A[m][0][3]); \
                _w[m][2] = cvtpk(A[m][1][0], A[m][1][1]); _w[m][3] = cvtpk(A[m][1][2], A[m][1][3]); } \
            encode_unit(_w, rc, (unsigned)(((pm * 16 + (pnx)) * 8 + wid) * 4 + (g)) * UNIT_B, lane, H1, E0, E1); } while (0)
#define CH_ADDR(pnx, g) (chd + (size_t)(((((bi * 2 + (pm & 1)) * 32 + bj_ * 16 + (pnx)) * 8 + wid) * 2 + bk) * 4 + (g)) * 16)
        {
            unsigned hv1 = 0, hv2 = 0, hv3 = 0;
#pragma unroll
            for (int g = 0; g < 4; ++g) {
                unsigned h1, e0, e1;
                EPI_GROUP(acc[g >> 1][g & 1], pn, g, h1, e0, e1);
                if (lane == g) { hv1 = h1; hv2 = e0; hv3 = e1; }
            }
            WAIT_V(0);
            if (__hip_atomic_load(tmo, __ATOMIC_RELAXED, __HIP_MEMORY_SCOPE_AGENT) == call) hv1 = POISON << 8;
            if (lane < 4) gst128_sys(CH_ADDR(pn, lane), (u32x4){call, hv1, hv2, hv3});
        }
        STAMP(dslot); ++dslot;
        for (int e = pn; e < 4 * PN2; e += PN1) {
            const int ai = e & 1, bj = (e >> 1) & 1, pn2 = PN1 + (e >> 2);
            if (!NOGEMM) {
                const unsigned long long t0 = __builtin_amdgcn_s_memrealtime();
                const u32x4* f = (const u32x4*)(rdyf + ((16 + pn2) * 64) * NR) + lane;
                for (;;) {
                    u32x4 v0, v1;
                    asm volatile("global_load_dwordx4 %0, %2, off sc0 sc1\n\tglobal_load_dwordx4 %1, %2, off offset:1024 sc0 sc1\n\ts_waitcnt vmcnt(0)"
                                 : "=&v"(v0), "=&v"(v1) : "v"(f) : "memory");
                    bool ok = true;
#pragma unroll
                    for (int q = 0; q < 4; ++q) ok = ok && (int)(v0[q] - call) >= 0 && (int)(v1[q] - call) >= 0;
                    if (__all(ok)) break;
                    if (timed_out(tmo, call, t0)) break;
                    __builtin_amdgcn_s_sleep(16);
                }
            }
            BAR;
            STAMP(dslot); ++dslot;
            const unsigned gA = (unsigned)(2 * pm + ai) * S_H, gB = (unsigned)(2 * pn2 + bj) * S_H;
#pragma unroll
            for (int m = 0; m < 4; ++m)
#pragma unroll
                for (int n = 0; n < 2; ++n) acc[0][0][m][n] = (f32x4){0.f, 0.f, 0.f, 0.f};
#pragma unroll
            for (int q = 0; q < 3; ++q) { STAGE(ra, q * HTB, gA + q * S_K); STAGE(rb, (4 + q) * HTB, gB + q * S_K); }
            for (int t = 0; t < (NOGEMM ? 0 : NT); t += 4) {
#pragma unroll
                for (int q = 0; q < 4; ++q) {
                    const unsigned tn = (unsigned)min(t + q + 3, NT - 1) * S_K;
                    WAIT_V(8); BAR;
                    STAGE(ra, ((q + 3) & 3) * HTB, gA + tn); STAGE(rb, (4 + ((q + 3) & 3)) * HTB, gB + tn);
                    LDB(B0, q >> 1, q & 1); SCHED; LDA(At, q >> 1, q & 1);
                    WAIT_B1(B0);
                    MMA(0, 0, At, B0);
                    SCHED;
                }
            }
            WAIT_V(0);
            unsigned h1, e0, e1;
            EPI_GROUP(acc[0][0], pn2, ai * 2 + bj, h1, e0, e1);
            WAIT_V(0);
            if (__hip_atomic_load(tmo, __ATOMIC_RELAXED, __HIP_MEMORY_SCOPE_AGENT) == call) h1 = POISON << 8;
            if (lane == 0) gst128_sys(CH_ADDR(pn2, ai * 2 + bj), (u32x4){call, h1, e0, e1});
            STAMP(dslot); ++dslot;
        }
#undef SA
#undef SB
    } else {
        const unsigned call = rfl(gld32_sys(cntp)) + 1;
        if (lane == 0) __hip_atomic_store(cntp, call, __ATOMIC_RELAXED, __HIP_MEMORY_SCOPE_AGENT);
        const unsigned par = call & 1;
        const rsrc_t rh = mk_rsrc(myws, (unsigned)WS_NEED);
        const unsigned lw = (unsigned)wid * 16384u, lh = LDS_AUX + (unsigned)wid * 1024u;
        const unsigned vl16 = (unsigned)lane * 16u;
#define PULL_UNIT(rp, slot, so, h1) do { DMA16(rp, slot, vl16, so); DMA16(rp, (slot) + 1024, vl16, (so) + 1024); DMA16(rp, (slot) + 2048, vl16, (so) + 2048); \
            if ((h1) >> 8 == RAWMODE) DMA16(rp, (slot) + 3072, vl16, (so) + 3072); } while (0)
#define READ_UNITS(d, slot) do { const unsigned _a = ldsa + (slot) + vl16; \
            DSR128(d[0][0], _a, 0); DSR128(d[0][1], _a, 1024); DSR128(d[0][2], _a, 2048); DSR128(d[0][3], _a, 3072); \
            DSR128(d[1][0], _a, 4096); DSR128(d[1][1], _a, 5120); DSR128(d[1][2], _a, 6144); DSR128(d[1][3], _a, 7168); } while (0)
#define WAIT_UNITS(d, hv) asm volatile("s_waitcnt lgkmcnt(0)" : "+v"(d[0][0]), "+v"(d[0][1]), "+v"(d[0][2]), "+v"(d[0][3]), \
            "+v"(d[1][0]), "+v"(d[1][1]), "+v"(d[1][2]), "+v"(d[1][3]), "+v"(hv) :: "memory")
        if (const int cw = (wg - NGEMM) * CW_IN + wid; wid < CW_IN && cw < W_IN * (NR - 1)) {
            const int s = (rank + 1 + cw % (NR - 1)) & 7;
            const rsrc_t rp = mk_rsrc(p.ws[s] + OFF_PACK + (size_t)par * PACK_B, PACK_B);
            const unsigned hoff = (unsigned)OFF_IH + par * IH_B + (unsigned)s * 1024u + (unsigned)min(lane, 7) * 4u;
            unsigned* rdyf = (unsigned*)(myws + OFF_RDY);
            auto jblk = [&](int j) { return j < NJ1 ? j % E1 : E1 + (j - NJ1) % PN2; };
            auto jstep = [&](int j) { return 2 * (j < NJ1 ? j / E1 : (j - NJ1) / PN2); };
            auto hdr = [&](int j) { j = min(j, NJ_IN - 1); return hoff + (unsigned)(jblk(j) * 8192 + jstep(j) * 16); };
            auto ready = [&](unsigned hv) { return __all((lane & 3) != 0 || lane >= 8 || (int)(hv - call) >= 0) != 0; };
            auto pull = [&](int j, unsigned hv, unsigned slot) {
                const int e = jblk(j), mat = e >> 4, blk = e & 15;
                const int gblk = mat == 0 ? (blk >> 1) * 4 + bi * 2 + (blk & 1) : bj_ * 16 + blk;
                const unsigned so = (unsigned)(((mat * 2 + bk) * 32 + gblk) * 64 + jstep(j)) * UNIT_B;
                PULL_UNIT(rp, slot, so, (unsigned)__builtin_amdgcn_readlane((int)hv, 1));
                PULL_UNIT(rp, slot + 4096, so + UNIT_B, (unsigned)__builtin_amdgcn_readlane((int)hv, 5));
            };
            auto publish = [&](int j) {
                if (lane < 2) __hip_atomic_store(rdyf + (jblk(j) * 64 + jstep(j) + lane) * NR + s, call, __ATOMIC_RELAXED, __HIP_MEMORY_SCOPE_SYSTEM);
            };
            int jj = cw / (NR - 1);
            while (jj < NJ_IN) {
                unsigned hvA, hvB;
                gld32x2_sys(myws + hdr(jj), myws + hdr(jj + W_IN), hvA, hvB);
                if (!ready(hvA)) {
                    const unsigned long long t0 = __builtin_amdgcn_s_memrealtime();
                    bool bad = false;
                    for (;;) {
                        __builtin_amdgcn_s_sleep(4);
                        gld32x2_sys(myws + hdr(jj), myws + hdr(jj + W_IN), hvA, hvB);
                        if (ready(hvA)) break;
                        if (timed_out(tmo, call, t0)) { bad = true; break; }
                    }
                    if (bad) break;
                }
                unsigned sl = lw, hs = lh;
                pull(jj, hvA, sl);
                DMA4(rh, hs, hdr(jj + 2 * W_IN));
                int pend = -1;
                for (;;) {
                    const int nx = jj + W_IN;
                    const bool more = nx < NJ_IN && ready(hvB);
                    if (more) { pull(nx, hvB, sl ^ 8192); WAIT_V(6); } else WAIT_V(0);
                    if (pend >= 0) publish(pend);
                    unsigned hvC;
                    u32x4 d[2][4];
                    DSR32(hvC, ldsa + hs + lane * 4);
                    READ_UNITS(d, sl);
                    WAIT_UNITS(d, hvC);
                    hs ^= 256;
                    if (more) DMA4(rh, hs, hdr(nx + 2 * W_IN));
                    const int e = jblk(jj);
                    const char* dst = myws + OFF_STG + (size_t)((unsigned)(e >> 4) * STG_B + (unsigned)(2 * (e & 15) * 64 + jstep(jj)) * HTB + (unsigned)s * 2048u);
#pragma unroll
                    for (int u = 0; u < 2; ++u) {
                        const unsigned h1 = __builtin_amdgcn_readlane((int)hvA, u * 4 + 1);
                        u32x4 o[4];
                        if (h1 >> 8 != RAWMODE) {
                            unsigned H[8];
                            decode_high(d[u][2], h1, __builtin_amdgcn_readlane((int)hvA, u * 4 + 2), __builtin_amdgcn_readlane((int)hvA, u * 4 + 3), lane, H);
#pragma unroll
                            for (int m = 0; m < 4; ++m) {
                                const unsigned l0 = d[u][m >> 1][(m & 1) * 2], l1 = d[u][m >> 1][(m & 1) * 2 + 1];
                                o[m] = (u32x4){vperm(H[2 * m], l0, 0x05010400u), vperm(H[2 * m], l0, 0x07030602u),
                                               vperm(H[2 * m + 1], l1, 0x05010400u), vperm(H[2 * m + 1], l1, 0x07030602u)};
                            }
                        } else {
#pragma unroll
                            for (int m = 0; m < 4; ++m) o[m] = d[u][m];
                        }
                        const char* d0 = dst + u * HTB;
                        gst128s_sys<0>(d0, vl16, o[0]); gst128s_sys<1024>(d0, vl16, o[1]);
                        gst128s_sys<0>(d0 + 64 * HTB, vl16, o[2]); gst128s_sys<1024>(d0 + 64 * HTB, vl16, o[3]);
                    }
                    if (jj < W_IN) STAMPW(8);
                    if (jj < NJ1 && nx >= NJ1) STAMPW(16);
                    pend = jj; jj = nx; hvA = hvB; hvB = hvC; sl ^= 8192;
                    if (!more) break;
                }
                WAIT_V(0);
                publish(pend);
            }
        }
        STAMPW(24);
        if (const int cw = (wg - NGEMM) * CW_OUT + wid; wid < CW_OUT) {
            const rsrc_t ro = mk_rsrc(p.out, (MM / NR) * NN * 2);
            const int ti = cw & 1, tj = (cw >> 1) & 1;
            const rsrc_t rc0 = mk_rsrc(p.ws[ti + 2 * tj] + OFF_CP, CP_B), rc1 = mk_rsrc(p.ws[ti + 2 * tj + 4] + OFF_CP, CP_B);
            auto hdr = [&](int j) {
                j = min(j, NJ_OUT - 1);
                const int y = j >> 5, rt = ti * 2 + (y & 1), ct = tj * 16 + (y >> 1);
                return lane < 32 ? (unsigned)OFF_CH + (unsigned)(((rt * 32 + ct) * 8 + ((j >> 2) & 7)) * 128 + lane * 4) : (unsigned)CTRL_OFF + C_TMO;
            };
            auto ready = [&](unsigned hv, int j) { return __all(lane >= 32 || (lane & 15) != (j & 3) * 4 || (int)(hv - call) >= 0) != 0; };
            auto pull = [&](int j, unsigned hv, unsigned slot) {
                const int y = j >> 5, g = j & 3;
                const unsigned so = (unsigned)((((rank * 2 + (y & 1)) * 16 + (y >> 1)) * 8 + ((j >> 2) & 7)) * 4 + g) * UNIT_B;
                PULL_UNIT(rc0, slot, so, (unsigned)__builtin_amdgcn_readlane((int)hv, g * 4 + 1));
                PULL_UNIT(rc1, slot + 4096, so, (unsigned)__builtin_amdgcn_readlane((int)hv, 16 + g * 4 + 1));
            };
            int jj = cw >> 2;
            bool dead = false;
            while (jj < NJ_OUT) {
                unsigned hvA, hvB;
                gld32x2_sys(myws + hdr(jj), myws + hdr(jj + W_OUT), hvA, hvB);
                if (!dead && !ready(hvA, jj)) {
                    const unsigned long long t0 = __builtin_amdgcn_s_memrealtime();
                    for (;;) {
                        __builtin_amdgcn_s_sleep(4);
                        gld32x2_sys(myws + hdr(jj), myws + hdr(jj + W_OUT), hvA, hvB);
                        if (ready(hvA, jj)) break;
                        if (timed_out(tmo, call, t0)) { dead = true; break; }
                    }
                }
                unsigned sl = lw, hs = lh;
                if (!dead) pull(jj, hvA, sl);
                DMA4(rh, hs, hdr(jj + 2 * W_OUT));
                for (;;) {
                    const int nx = jj + W_OUT;
                    const bool more = !dead && nx < NJ_OUT && ready(hvB, nx);
                    if (more) { pull(nx, hvB, sl ^ 8192); WAIT_V(6); } else WAIT_V(0);
                    unsigned hvC;
                    u32x4 d[2][4];
                    DSR32(hvC, ldsa + hs + lane * 4);
                    READ_UNITS(d, sl);
                    WAIT_UNITS(d, hvC);
                    hs ^= 256;
                    if (more) DMA4(rh, hs, hdr(nx + 2 * W_OUT));
                    const int gg = jj & 3, w8 = (jj >> 2) & 7, y = jj >> 5;
                    const bool bad = dead || (unsigned)__builtin_amdgcn_readlane((int)hvA, 32) == call
                                     || (unsigned)__builtin_amdgcn_readlane((int)hvA, gg * 4 + 1) >> 8 == POISON
                                     || (unsigned)__builtin_amdgcn_readlane((int)hvA, 16 + gg * 4 + 1) >> 8 == POISON;
                    const unsigned orow = (ti * 2 + (y & 1)) * 256 + (gg >> 1) * 128 + (w8 >> 2) * 64 + fr;
                    const unsigned ocol = (tj * 16 + (y >> 1)) * 256 + (gg & 1) * 128 + (w8 & 3) * 32 + ((fq & 1) * 16 + (fq >> 1) * 8);
                    const unsigned ooff = ((orow & 1023u) * NN + ocol) * 2;
                    if (bad) {
#pragma unroll
                        for (int m = 0; m < 4; ++m)
                            st128<0>((u32x4){0x7fc07fc0u, 0x7fc07fc0u, 0x7fc07fc0u, 0x7fc07fc0u}, ro, ooff, m * 16 * NN * 2);
                    } else {
                        float a[4][8];
#pragma unroll
                        for (int k = 0; k < 2; ++k) {
                            const int hl = (k * 4 + gg) * 4;
                            const unsigned h1 = __builtin_amdgcn_readlane((int)hvA, hl + 1);
                            if (h1 >> 8 == RAWMODE) {
#pragma unroll
                                for (int m = 0; m < 4; ++m)
#pragma unroll
                                    for (int j = 0; j < 4; ++j) {
                                        const unsigned v = d[k][m][j];
                                        const float lo = __uint_as_float(v << 16), hi = __uint_as_float(v & 0xffff0000u);
                                        a[m][2 * j] = k == 0 ? lo : a[m][2 * j] + lo; a[m][2 * j + 1] = k == 0 ? hi : a[m][2 * j + 1] + hi;
                                    }
                            } else {
                                unsigned H[8];
                                decode_high(d[k][2], h1, __builtin_amdgcn_readlane((int)hvA, hl + 2), __builtin_amdgcn_readlane((int)hvA, hl + 3), lane, H);
#pragma unroll
                                for (int m = 0; m < 4; ++m)
#pragma unroll
                                    for (int j = 0; j < 8; ++j) {
                                        const unsigned Lw = d[k][m >> 1][(m & 1) * 2 + (j >> 2)];
                                        const float v = __uint_as_float(vperm(H[2 * m + (j >> 2)], Lw, (4u + (j & 3)) << 24 | (unsigned)(j & 3) << 16 | 0x0c0cu));
                                        a[m][j] = k == 0 ? v : a[m][j] + v;
                                    }
                            }
                        }
#pragma unroll
                        for (int m = 0; m < 4; ++m) {
                            unsigned r0 = cvtpk(a[m][0], a[m][1]), r1 = cvtpk(a[m][2], a[m][3]), r2 = cvtpk(a[m][4], a[m][5]), r3 = cvtpk(a[m][6], a[m][7]);
                            auto s02 = __builtin_amdgcn_permlane16_swap(r0, r2, false, false);
                            auto s13 = __builtin_amdgcn_permlane16_swap(r1, r3, false, false);
                            u32x4 o; o.x = s02[0]; o.y = s13[0]; o.z = s02[1]; o.w = s13[1];
                            st128<OAUX>(o, ro, ooff, m * 16 * NN * 2);
                        }
                    }
                    if (jj < W_OUT) STAMPW(32);
                    jj = nx; hvA = hvB; hvB = hvC; sl ^= 8192;
                    if (!more) break;
                }
            }
        }
        STAMPW(40);
    }
}

}

extern "C" void kernel_launch(int rank, int nranks, ncclComm_t comm,
                              hipStream_t stream,
                              void* const* d_in, const int* in_sizes, int n_in,
                              void* d_out, int out_size,
                              const int* in_dtypes, int out_dtype,
                              void* const* d_out_all,
                              void* const* d_ws_all, size_t ws_bytes) {
    (void)comm; (void)in_dtypes; (void)out_dtype; (void)d_out_all;
    if (nranks != NR || n_in < 2 || in_sizes[0] != MM * KL || in_sizes[1] != KL * NN || out_size != (MM / NR) * NN
        || ws_bytes < WS_NEED || (unsigned)rank >= (unsigned)NR) {
        fprintf(stderr, "kernel_launch: unexpected shapes\n");
        return;
    }
    static bool inited[NR] = {};
    if (!inited[rank]) {
        HIP_CHECK(hipFuncSetAttribute((const void*)gemm_rs_kernel, hipFuncAttributeMaxDynamicSharedMemorySize, LDS_BYTES));
        inited[rank] = true;
    }
    Params p;
    p.x = d_in[0]; p.w = d_in[1]; p.out = d_out; p.rank = rank;
    for (int i = 0; i < NR; ++i) p.ws[i] = (char*)d_ws_all[i];
    hipLaunchKernelGGL(gemm_rs_kernel, dim3(NWG), dim3(NTHR), LDS_BYTES, stream, p);
    HIP_CHECK(hipGetLastError());
}
```

```cpp
#include <hip/hip_runtime.h>
#include <rccl/rccl.h>
#include <cstdio>
#include <cstdint>

#define HIP_CHECK(expr) do { hipError_t _e = (expr); if (_e != hipSuccess) { \
    fprintf(stderr, "HIP error %s at %s:%d\n", hipGetErrorString(_e), __FILE__, __LINE__); \
    return; } } while (0)

#define LAS __attribute__((address_space(3)))
typedef short bf16x8 __attribute__((ext_vector_type(8)));
typedef short s16x4 __attribute__((ext_vector_type(4)));
typedef float f32x4 __attribute__((ext_vector_type(4)));
typedef float f32x2 __attribute__((ext_vector_type(2)));
typedef unsigned u32x4 __attribute__((ext_vector_type(4)));
typedef unsigned short u16x2 __attribute__((ext_vector_type(2)));
typedef __bf16 bf16x2_t __attribute__((ext_vector_type(2)));
typedef __amdgpu_buffer_rsrc_t rsrc_t;

namespace {

constexpr int NR = 8;
constexpr int MM = 8192, NN = 8192, KL = 1024;
constexpr int NWG = 256, NTHR = 512;
#define PN1 13
#define LOOK 8
#define CW 2
#define PKSLEEP 100
#define FORCE_RAW 0
#define NOGEMM 0
#define GAUX 17
#define OAUX 0
constexpr int NGEMM = 16 * PN1, NCOMM = NWG - NGEMM, NCW = NCOMM * CW;
constexpr int PN2 = 16 - PN1;
constexpr int HTB = 16384, STAGE_BYTES = 8 * HTB;
constexpr int NT = 64;
constexpr unsigned UNIT_B = 4096;
constexpr size_t MIB = 1u << 20;
constexpr size_t OFF_PACK = 0;
constexpr unsigned PACK_B = 32u << 20;
constexpr size_t OFF_STG = 64 * MIB;
constexpr unsigned STG_B = 32u << 20;
constexpr size_t OFF_CP = 128 * MIB;
constexpr unsigned CP_B = 32u << 20;
constexpr size_t OFF_IH = 160 * MIB;
constexpr unsigned IH_B = 2 * 16 * 8 * 64 * 16;
constexpr size_t OFF_RDY = 161 * MIB;
constexpr size_t OFF_CH = 162 * MIB;
constexpr size_t CTRL_OFF = 256 * MIB;
constexpr size_t WS_NEED = CTRL_OFF + MIB;
constexpr unsigned C_TMO = 256, C_CNT = 4096;
constexpr unsigned C_TKT = 16384;
constexpr unsigned MAXEXC = 2, RAWMODE = 255, POISON = 254;
constexpr unsigned long long DEADLINE = 30000000ull;
constexpr int LDS_AUX = STAGE_BYTES;
constexpr int LDS_BAD = STAGE_BYTES + 8192;
constexpr int LDS_BYTES = LDS_BAD + 256;
constexpr int AUX_SYS = 17;
constexpr int E1 = 16 + PN1;
constexpr int NJ1 = (NT / 2) * E1, NJ_IN = (NT / 2) * 32;
#define NFB 4
#define SKD 2
constexpr int EF = 8 + NFB, ES = E1 - EF;
constexpr int JD_OUT = 2048, JD_END = JD_OUT + 1024;
__host__ __device__ constexpr int in_job(int j) {
    int i = 0, r = 0;
    if (j < SKD * EF) { i = j % EF; r = j / EF; }
    else if (j < SKD * EF + (NT / 2 - SKD) * E1) { const int q = j - SKD * EF; i = q % E1; r = SKD + q / E1 - (i < EF ? 0 : SKD); }
    else if (j < NJ1) { const int q = j - (SKD * EF + (NT / 2 - SKD) * E1); i = EF + q % ES; r = NT / 2 - SKD + q / ES; }
    else { const int q = j - NJ1; i = E1 + q % PN2; r = q / PN2; }
    const int e = i < 8 ? 2 * i : i < EF ? 8 + i : i < 8 + EF ? 2 * (i - EF) + 1 : i;
    return e * 64 + 2 * r;
}
__host__ __device__ constexpr int out_tile(int o) { const int y = o >> 5; return y < NFB ? 2 * y : y < 2 * NFB ? 2 * (y - NFB) + 1 : y; }
#define W_IN (NCW / (NR - 1))

#define STAMP(slot) do {} while (0)
#define STAMPW(slot) do {} while (0)
#define STAMPJ(row, j) do {} while (0)
#define JC(j) (j)
#define JCODE(j) in_job(j)
#define DIAG_BLK0 do {} while (0)
#define DIAG_BLK1(t0) do {} while (0)
struct Params {
    const void* x; const void* w; void* out;
    char* ws[NR];
    int rank;
};

__device__ __forceinline__ unsigned cvtpk(float lo, float hi) { f32x2 v = {lo, hi}; bf16x2_t b = __builtin_convertvector(v, bf16x2_t); return __builtin_bit_cast(unsigned, b); }
__device__ __forceinline__ rsrc_t mk_rsrc(const void* p, unsigned bytes) { return __builtin_amdgcn_make_buffer_rsrc((void*)p, 0, bytes, 0x00020000); }

template <int AUX> __device__ __forceinline__ void st128(u32x4 v, rsrc_t r, unsigned voff, unsigned soff) {
    __builtin_amdgcn_raw_buffer_store_b128(v, r, voff, soff, AUX);
    asm volatile("s_nop 1" :: "v"(v) : "memory");
}
__device__ __forceinline__ void gst128_sys(void* ptr, u32x4 v) {
    asm volatile("global_store_dwordx4 %0, %1, off sc0 sc1\n\ts_nop 1" :: "v"(ptr), "v"(v) : "memory");
}
__device__ __forceinline__ unsigned gld32_sys(const void* ptr) {
    unsigned v;
    asm volatile("global_load_dword %0, %1, off sc0 sc1\n\ts_waitcnt vmcnt(0)" : "=&v"(v) : "v"(ptr) : "memory");
    return v;
}
__device__ __forceinline__ void gld32x2_sys(const void* pa, const void* pb, unsigned& a, unsigned& b) {
    asm volatile("global_load_dword %0, %2, off sc0 sc1\n\tglobal_load_dword %1, %3, off sc0 sc1\n\ts_waitcnt vmcnt(0)"
                 : "=&v"(a), "=&v"(b) : "v"(pa), "v"(pb) : "memory");
}
template <int OFF> __device__ __forceinline__ void gst128s_sys(const void* base, unsigned voff, u32x4 v) {
    asm volatile("global_store_dwordx4 %0, %1, %2 offset:%c3 sc0 sc1\n\ts_nop 1" :: "v"(voff), "v"(v), "s"(base), "i"(OFF) : "memory");
}
__device__ __forceinline__ unsigned bfi(unsigned m, unsigned a, unsigned b) { return (a & m) | (b & ~m); }
__device__ __forceinline__ unsigned vperm(unsigned hi, unsigned lo, unsigned sel) { return __builtin_amdgcn_perm(hi, lo, sel); }
__device__ __forceinline__ unsigned rfl(unsigned v) { return __builtin_amdgcn_readfirstlane(v); }
__device__ __forceinline__ unsigned wave_max_u(int x) {
    x = max(x, __builtin_amdgcn_update_dpp(0, x, 0xB1, 0xf, 0xf, true));
    x = max(x, __builtin_amdgcn_update_dpp(0, x, 0x4E, 0xf, 0xf, true));
    x = max(x, __builtin_amdgcn_update_dpp(0, x, 0x141, 0xf, 0xf, true));
    x = max(x, __builtin_amdgcn_update_dpp(0, x, 0x140, 0xf, 0xf, true));
    const int a = __builtin_amdgcn_readlane(x, 0), b = __builtin_amdgcn_readlane(x, 16), c = __builtin_amdgcn_readlane(x, 32), d = __builtin_amdgcn_readlane(x, 48);
    return (unsigned)max(max(a, b), max(c, d));
}
__device__ __forceinline__ bool timed_out(unsigned* tmo, unsigned call, unsigned long long t0) {
    if (__hip_atomic_load(tmo, __ATOMIC_RELAXED, __HIP_MEMORY_SCOPE_AGENT) == call) return true;
    if (__builtin_amdgcn_s_memrealtime() - t0 > DEADLINE) { __hip_atomic_store(tmo, call, __ATOMIC_RELAXED, __HIP_MEMORY_SCOPE_AGENT); return true; }
    return false;
}

__device__ __forceinline__ void encode_unit(const unsigned (&w)[4][4], const char* dst, int lane, unsigned& h1, unsigned& e0, unsigned& e1) {
    u16x2 mx = {0, 0};
#pragma unroll
    for (int m = 0; m < 4; ++m)
#pragma unroll
        for (int q = 0; q < 4; ++q) mx = __builtin_elementwise_max(mx, __builtin_bit_cast(u16x2, w[m][q] & 0x7f007f00u));
    const unsigned base = wave_max_u((int)max(mx[0], mx[1])) >> 8;
    const unsigned bq = base * 0x01010101u | 0x80808080u;
    unsigned L[8], P[4], ex = 0;
#pragma unroll
    for (int m = 0; m < 4; ++m) {
        L[2 * m] = vperm(w[m][1], w[m][0], 0x06040200u); L[2 * m + 1] = vperm(w[m][3], w[m][2], 0x06040200u);
        const unsigned h01 = vperm(w[m][1], w[m][0], 0x07050301u), h23 = vperm(w[m][3], w[m][2], 0x07050301u);
        const unsigned d01 = bq - (h01 & 0x7f7f7f7fu), d23 = bq - (h23 & 0x7f7f7f7fu);
        ex |= d01 | d23;
        P[m] = bfi(0x0f0f0f0fu, bfi(0x07070707u, d01, h01 >> 4), bfi(0x70707070u, d23 << 4, h23));
    }
    unsigned cnt = FORCE_RAW ? RAWMODE : 0;
    e0 = 0; e1 = 0;
    if (__any((ex & 0x78787878u) != 0)) {
#pragma unroll
        for (int m = 0; m < 4; ++m)
#pragma unroll
            for (int j = 0; j < 8; ++j) {
                const unsigned hb = (w[m][j >> 1] >> (8 + 16 * (j & 1))) & 0xffu;
                unsigned long long bal = __ballot(base - (hb & 0x7fu) > 7u);
                while (bal && cnt <= MAXEXC) {
                    const unsigned l = (unsigned)__builtin_ctzll(bal);
                    const unsigned en = (unsigned)(m << 17 | j << 8) | l << 11 | (unsigned)__builtin_amdgcn_readlane((int)hb, (int)l);
                    if (cnt == 0) e0 = en; else e1 = en;
                    ++cnt; bal &= bal - 1;
                }
            }
        if (cnt > MAXEXC) cnt = RAWMODE;
    }
    const unsigned vo = (unsigned)lane * 16u;
    if (cnt == RAWMODE) {
        gst128s_sys<0>(dst, vo, (u32x4){w[0][0], w[0][1], w[0][2], w[0][3]});
        gst128s_sys<1024>(dst, vo, (u32x4){w[1][0], w[1][1], w[1][2], w[1][3]});
        gst128s_sys<2048>(dst, vo, (u32x4){w[2][0], w[2][1], w[2][2], w[2][3]});
        gst128s_sys<3072>(dst, vo, (u32x4){w[3][0], w[3][1], w[3][2], w[3][3]});
    } else {
        gst128s_sys<0>(dst, vo, (u32x4){L[0], L[1], L[2], L[3]});
        gst128s_sys<1024>(dst, vo, (u32x4){L[4], L[5], L[6], L[7]});
        gst128s_sys<2048>(dst, vo, (u32x4){P[0], P[1], P[2], P[3]});
    }
    h1 = base | cnt << 8;
}
__device__ __forceinline__ void decode_high(u32x4 P, unsigned h1, unsigned e0, unsigned e1, int lane, unsigned (&H)[8]) {
    const unsigned bq = (h1 & 0x7fu) * 0x01010101u | 0x80808080u;
#pragma unroll
    for (int m = 0; m < 4; ++m) {
        H[2 * m] = bfi(0x7f7f7f7fu, bq - (P[m] & 0x07070707u), P[m] << 4);
        H[2 * m + 1] = bfi(0x7f7f7f7fu, bq - ((P[m] >> 4) & 0x07070707u), P[m]);
    }
    const unsigned cnt = h1 >> 8;
    for (unsigned e = 0; e < cnt; ++e) {
        const unsigned en = e ? e1 : e0;
        const unsigned k = (en >> 17 & 3) * 2 + (en >> 10 & 1), sh = (en >> 8 & 3) * 8;
        const unsigned mk = (unsigned)lane == (en >> 11 & 63) ? 0xffu << sh : 0u, val = (en & 0xffu) << sh;
#pragma unroll
        for (int kk = 0; kk < 8; ++kk)
            if (k == (unsigned)kk) H[kk] = bfi(mk, val, H[kk]);
    }
}

#define TRD(dst, addr, off) asm volatile("ds_read_b64_tr_b16 %0, %1 offset:%c2" : "=&v"(dst) : "v"(addr), "i"(off) : "memory")
#define PKB(x) (bf16x8){(x)[0][0], (x)[0][1], (x)[0][2], (x)[0][3], (x)[1][0], (x)[1][1], (x)[1][2], (x)[1][3]}
#define WAIT_V(n) asm volatile("s_waitcnt vmcnt(" #n ")" ::: "memory")
#define WAIT_L(n) asm volatile("s_waitcnt lgkmcnt(" #n ")" ::: "memory")
#define BAR __builtin_amdgcn_s_barrier()
#define SCHED __builtin_amdgcn_sched_barrier(0)
#define DSR128(dst, addr, off) asm volatile("ds_read_b128 %0, %1 offset:%c2" : "=&v"(dst) : "v"(addr), "i"(off) : "memory")
#define DSR32(dst, addr) asm volatile("ds_read_b32 %0, %1" : "=&v"(dst) : "v"(addr) : "memory")
#define DMA16(rs, ldsoff, voff, soff) __builtin_amdgcn_raw_ptr_buffer_load_lds(rs, (LAS void*)(lds + (ldsoff)), 16, voff, soff, 0, AUX_SYS)
#define DMA4(rs, ldsoff, voff) __builtin_amdgcn_raw_ptr_buffer_load_lds(rs, (LAS void*)(lds + (ldsoff)), 4, voff, 0, 0, AUX_SYS)
#define TICKET(tk, ptr) asm volatile("s_mov_b64 exec, 1\n\tglobal_atomic_add %0, %1, %2, %3 sc0 sc1\n\ts_mov_b64 exec, -1" : "=&v"(tk) : "v"(0u), "v"(1u), "s"(ptr) : "memory")
__device__ __forceinline__ unsigned ticket_value(unsigned tk) {
    unsigned r;
    asm volatile("v_readfirstlane_b32 %0, %1" : "=s"(r) : "v"(tk) : "memory");
    return r;
}

__global__ __launch_bounds__(NTHR, 2) void gemm_rs_kernel(Params p) {
    extern __shared__ __attribute__((aligned(16))) unsigned char smem[];
    LAS unsigned char* lds = (LAS unsigned char*)smem;
    const int tid = threadIdx.x, wid = __builtin_amdgcn_readfirstlane(tid >> 6), lane = tid & 63;
    const int wr = wid >> 2, wc = wid & 3, fr = lane & 15, fq = lane >> 4;
    const int wg = blockIdx.x, rank = p.rank;
    const int bi = rank & 1, bj_ = (rank >> 1) & 1, bk = rank >> 2;
    char* myws = p.ws[rank];
    unsigned* ctl = (unsigned*)(myws + CTRL_OFF);
    unsigned* tmo = ctl + C_TMO / 4;

    int dslot = 0;
    STAMP(dslot); ++dslot;
    const unsigned ldsa = (unsigned)(size_t)lds;
    unsigned* cntp = ctl + C_CNT / 4 + wg * 8 + wid;

    if (wg < NGEMM) {
        const rsrc_t rx = mk_rsrc(p.x, MM * KL * 2), rw = mk_rsrc(p.w, KL * NN * 2);
        const unsigned vx = (unsigned)((tid >> 3) * (KL * 2) + (((tid & 7) ^ ((tid >> 4) & 7)) << 4));
        const unsigned vw = (unsigned)(fq * (NN * 2) + ((fr ^ ((fq << 2) | ((rank & 1) << 1))) << 4));
        auto pk_issue = [&](int idx, int buf) {
            const int g8 = idx >> 7, mat = (idx >> 6) & 1, kk = (idx >> 5) & 1, blk = idx & 31;
            if (mat == 0) {
#pragma unroll
                for (int i = 0; i < 4; ++i)
                    __builtin_amdgcn_raw_ptr_buffer_load_lds(rx, (LAS void*)(lds + buf + i * 8192 + wid * 1024), 16, vx,
                                                             (unsigned)((blk * 256 + i * 64) * (KL * 2) + kk * 1024 + g8 * 128), 0, 0);
            } else {
#pragma unroll
                for (int c = 0; c < 4; ++c)
                    __builtin_amdgcn_raw_ptr_buffer_load_lds(rw, (LAS void*)(lds + buf + wid * 4096 + c * 1024), 16, vw,
                                                             (unsigned)((kk * 512 + (g8 * 8 + wid) * 8 + (c & 1) * 4) * (NN * 2) + blk * 512 + (c >> 1) * 256), 0, 0);
            }
        };
        pk_issue(wg, 0);
        volatile LAS unsigned* const badf = (volatile LAS unsigned*)(lds + LDS_BAD);
        if (tid == 0) asm volatile("ds_write_b32 %0, %1" :: "v"(ldsa + LDS_BAD), "v"(0u) : "memory");
        const unsigned call = rfl(gld32_sys(cntp)) + 1;
        if (lane == 0) __hip_atomic_store(cntp, call, __ATOMIC_RELAXED, __HIP_MEMORY_SCOPE_AGENT);
        const unsigned par = call & 1;
        unsigned* rdyw = (unsigned*)(myws + OFF_RDY);
        {
            const char* const pk = myws + OFF_PACK + (size_t)par * PACK_B;
            const rsrc_t rs = mk_rsrc(myws + OFF_STG, 2 * STG_B);
            int buf = 0;
            for (int idx = wg; idx < 1024; idx += NGEMM, buf ^= 32768) {
                const int g8 = idx >> 7, mat = (idx >> 6) & 1, kk = (idx >> 5) & 1, blk = idx & 31, t = g8 * 8 + wid;
                BAR;
                if (idx >= NGEMM) __builtin_amdgcn_s_sleep(PKSLEEP);
                if (idx + NGEMM < 1024) pk_issue(idx + NGEMM, buf ^ 32768);
                u32x4 v[4];
                if (mat == 0) {
                    const unsigned a = ldsa + buf + lane * 128 + ((wid ^ ((lane >> 1) & 7)) << 4);
                    DSR128(v[0], a, 0); DSR128(v[1], a, 8192); DSR128(v[2], a, 16384); DSR128(v[3], a, 24576);
                } else {
                    const unsigned a = ldsa + buf + wid * 4096 + lane * 16;
                    DSR128(v[0], a, 0); DSR128(v[1], a, 1024); DSR128(v[2], a, 2048); DSR128(v[3], a, 3072);
                }
                asm volatile("s_waitcnt lgkmcnt(0)" : "+v"(v[0]), "+v"(v[1]), "+v"(v[2]), "+v"(v[3]) :: "memory");
                unsigned w[4][4];
#pragma unroll
                for (int c = 0; c < 4; ++c) { w[c][0] = v[c][0]; w[c][1] = v[c][1]; w[c][2] = v[c][2]; w[c][3] = v[c][3]; }
                unsigned h1, e0, e1;
                encode_unit(w, pk + (unsigned)(((mat * 2 + kk) * 32 + blk) * 64 + t) * UNIT_B, lane, h1, e0, e1);
                const int cb = mat == 0 ? ((blk >> 2) * 2 + (blk & 1)) : (blk & 15);
                const int c0 = mat == 0 ? ((blk >> 1) & 1) + 4 * kk : 2 * (blk >> 4) + 4 * kk, c1 = c0 + (mat == 0 ? 2 : 1);
                const bool own = c0 == rank || c1 == rank;
                if (own) {
                    const unsigned dst = (unsigned)mat * STG_B + (unsigned)(2 * cb * 64 + t) * HTB + (unsigned)rank * 2048u;
#pragma unroll
                    for (int m = 0; m < 4; ++m) st128<AUX_SYS>(v[m], rs, lane * 16, dst + (m >> 1) * (64 * HTB) + (m & 1) * 1024);
                }
                char* const w0 = p.ws[c0 & 7];
                char* const w1 = p.ws[c1 & 7];
                WAIT_V(0);
                if (lane < 2) {
                    const int c = lane == 0 ? c0 : c1;
                    if (c != rank) gst128_sys((lane == 0 ? w0 : w1) + OFF_IH + (size_t)par * IH_B + (size_t)((((mat * 16 + cb) * NR + rank) * 64 + t) * 16), (u32x4){call, h1, e0, e1});
                    else __hip_atomic_store(rdyw + ((mat * 16 + cb) * 64 + t) * NR + rank, call, __ATOMIC_RELAXED, __HIP_MEMORY_SCOPE_SYSTEM);
                }
            }
            BAR;
        }
        STAMP(dslot); ++dslot;
        const int pm = 2 * (wg & 7) + ((wg >> 3) & 1), pn = wg >> 4;
        const rsrc_t ra = mk_rsrc(myws + OFF_STG, STG_B), rb = mk_rsrc(myws + OFF_STG + STG_B, STG_B);
        const unsigned voff = (unsigned)(wid * 1024 + lane * 16);
        constexpr unsigned S_K = HTB, S_H = NT * HTB, S_T = 2 * NT * HTB;
        const unsigned ldsw = (unsigned)wid * 1024u;
        const int aoff = fq * 2048 + (wr * 64 + fr) * 16;
        unsigned boff[2];
        { const int q = (lane & 15) >> 2, pp = lane & 3, g = fq;
#pragma unroll
          for (int n = 0; n < 2; ++n) boff[n] = (unsigned)(size_t)lds + 4 * HTB + 256 * (8 * g + q) + 16 * (((wc ^ q) << 2) | ((n ^ (g & 1)) << 1) | (pp >> 1)) + 8 * (pp & 1); }
        const unsigned* rdyf = (const unsigned*)(myws + OFF_RDY);
#define SA(b, h) (((b) * 2 + (h)) * HTB)
#define SB(b, h) ((4 + (b) * 2 + (h)) * HTB)
#define STAGE(rs, bufoff, soff) do { _Pragma("unroll") for (int _i = 0; _i < 2; ++_i) \
        __builtin_amdgcn_raw_ptr_buffer_load_lds(rs, (LAS void*)(lds + (bufoff) + ldsw + _i * 8192), 16, voff, (soff) + _i * 8192, 0, GAUX); } while (0)
#define LDA(dst, b, h) do { _Pragma("unroll") for (int m = 0; m < 4; ++m) _Pragma("unroll") for (int k = 0; k < 2; ++k) dst[m][k] = *(const LAS bf16x8*)(lds + SA(b, h) + aoff + m * 256 + k * 8192); } while (0)
#define LDB(dst, b, h) do { _Pragma("unroll") for (int n = 0; n < 2; ++n) _Pragma("unroll") for (int k = 0; k < 2; ++k) { \
        TRD(dst[n][k][0], boff[n], ((b) * 2 + (h)) * HTB + k * 8192); TRD(dst[n][k][1], boff[n], ((b) * 2 + (h)) * HTB + k * 8192 + 1024); } } while (0)
#define MMA(ai, bj, At, Bt) do { __builtin_amdgcn_s_setprio(1); _Pragma("unroll") for (int m = 0; m < 4; ++m) _Pragma("unroll") for (int n = 0; n < 2; ++n) _Pragma("unroll") for (int k = 0; k < 2; ++k) \
        acc[ai][bj][m][n] = __builtin_amdgcn_mfma_f32_16x16x32_bf16(PKB(Bt[n][k]), At[m][k], acc[ai][bj][m][n], 0, 0, 0); __builtin_amdgcn_s_setprio(0); } while (0)
#define WAIT_B(X, Y) asm volatile("s_waitcnt lgkmcnt(0)" : "+v"(X[0][0][0]), "+v"(X[0][0][1]), "+v"(X[0][1][0]), "+v"(X[0][1][1]), "+v"(X[1][0][0]), "+v"(X[1][0][1]), "+v"(X[1][1][0]), "+v"(X[1][1][1]), \
        "+v"(Y[0][0][0]), "+v"(Y[0][0][1]), "+v"(Y[0][1][0]), "+v"(Y[0][1][1]), "+v"(Y[1][0][0]), "+v"(Y[1][0][1]), "+v"(Y[1][1][0]), "+v"(Y[1][1][1]) :: "memory")
#define WAIT_B1(X) asm volatile("s_waitcnt lgkmcnt(0)" : "+v"(X[0][0][0]), "+v"(X[0][0][1]), "+v"(X[0][1][0]), "+v"(X[0][1][1]), "+v"(X[1][0][0]), "+v"(X[1][0][1]), "+v"(X[1][1][0]), "+v"(X[1][1][1]) :: "memory")
        const rsrc_t rr = mk_rsrc(myws + OFF_RDY, 2 * 16 * 64 * 32);
        const unsigned pvo = (unsigned)(((lane >> 5) * 16 + (lane < 32 ? pm : pn)) * 2048 + (lane & 1) * 16);
        const unsigned pla = ldsa + LDS_AUX + wid * 1024 + lane * 16;
#define NOGATE 0
        int rdy = NOGATE ? NT : 0, pbase = 0;
        bool pol = false;
        u32x4 fv;
#define COUNT_RDY(v) do { const bool _ok = (int)((v)[0] - call) >= 0 && (int)((v)[1] - call) >= 0 && (int)((v)[2] - call) >= 0 && (int)((v)[3] - call) >= 0; \
            const unsigned long long _b = __ballot(_ok); \
            unsigned _m = (unsigned)_b & (unsigned)(_b >> 32); _m = ~(_m & (_m >> 1)) & 0x55555555u; \
            rdy = min(pbase + (_m ? (int)(__builtin_ctz(_m) >> 1) : 16), NT); } while (0)
#define FLAGREAD do { if (pol) DSR128(fv, pla, 0); } while (0)
#define GATE(tt) do { \
            if (pol) { asm volatile("s_waitcnt lgkmcnt(0)" : "+v"(fv) :: "memory"); COUNT_RDY(fv); } \
            if ((int)(tt) >= rdy) { \
                const unsigned long long _t0 = __builtin_amdgcn_s_memrealtime(); \
                DIAG_BLK0; \
                for (;;) { \
                    pbase = rdy; \
                    const char* _p = (const char*)rdyf + pvo + min(rdy + ((lane >> 1) & 15), NT - 1) * 32; \
                    u32x4 _v; \
                    asm volatile("global_load_dwordx4 %0, %1, off sc0 sc1\n\ts_waitcnt vmcnt(0)" : "=&v"(_v) : "v"(_p) : "memory"); \
                    COUNT_RDY(_v); \
                    if ((int)(tt) < rdy) break; \
                    if (timed_out(tmo, call, _t0)) { *badf = 1; rdy = NT; break; } \
                    __builtin_amdgcn_s_sleep(8); \
                } DIAG_BLK1(_t0); } \
            pol = rdy < min((int)(tt) + LOOK, NT); \
            if (pol) { pbase = rdy; DMA16(rr, LDS_AUX + wid * 1024, pvo + min(rdy + ((lane >> 1) & 15), NT - 1) * 32, 0); } } while (0)
#define WAIT_VP do { if (pol) WAIT_V(9); else WAIT_V(8); } while (0)
        const int owner = pm >> 1;
        char* chd = p.ws[owner] + OFF_CH;
        f32x4 acc[2][2][4][2];
#pragma unroll
        for (int a = 0; a < 2; ++a)
#pragma unroll
            for (int b = 0; b < 2; ++b)
#pragma unroll
                for (int m = 0; m < 4; ++m)
#pragma unroll
                    for (int n = 0; n < 2; ++n) acc[a][b][m][n] = (f32x4){0.f, 0.f, 0.f, 0.f};
        bf16x8 At[4][2]; s16x4 B0[2][2][2], B1[2][2][2];
        const unsigned cA = (unsigned)pm * S_T, cB = (unsigned)pn * S_T;
        if (!NOGEMM) { GATE(1); }
        STAGE(rb, SB(0, 0), cB); STAGE(rb, SB(0, 1), cB + S_H); STAGE(ra, SA(0, 0), cA); STAGE(ra, SA(0, 1), cA + S_H);
        if (wr == 1) BAR;
        WAIT_V(2); BAR;
        STAGE(rb, SB(1, 0), cB + S_K); STAGE(ra, SA(1, 0), cA + S_K); STAGE(rb, SB(1, 1), cB + S_H + S_K);
        WAIT_V(6); BAR;
        STAMP(dslot); ++dslot;
        for (int t = 0; t < (NOGEMM ? 0 : NT); t += 2) {
            const int t2 = min(t + 2, NT - 1), t3 = min(t + 3, NT - 1);
            const unsigned a1 = cA + (unsigned)(t + 1) * S_K;
            const unsigned a2 = cA + (unsigned)t2 * S_K, b2 = cB + (unsigned)t2 * S_K, a3 = cA + (unsigned)t3 * S_K, b3 = cB + (unsigned)t3 * S_K;
            LDB(B0, 0, 0); LDB(B1, 0, 1); SCHED; LDA(At, 0, 0); STAGE(ra, SA(1, 1), a1 + S_H);
            WAIT_V(8); WAIT_B(B0, B1); FLAGREAD; BAR; MMA(0, 0, At, B0); MMA(0, 1, At, B1); BAR; SCHED;
            GATE(t2);
            LDA(At, 0, 1); STAGE(rb, SB(0, 0), b2); STAGE(rb, SB(0, 1), b2 + S_H); STAGE(ra, SA(0, 0), a2);
            WAIT_VP; WAIT_L(0); BAR; MMA(1, 0, At, B0); MMA(1, 1, At, B1); BAR; SCHED;
            LDB(B0, 1, 0); LDB(B1, 1, 1); SCHED; LDA(At, 1, 0); STAGE(ra, SA(0, 1), a2 + S_H);
            WAIT_V(8); WAIT_B(B0, B1); FLAGREAD; BAR; MMA(0, 0, At, B0); MMA(0, 1, At, B1); BAR; SCHED;
            GATE(t3);
            LDA(At, 1, 1); STAGE(rb, SB(1, 0), b3); STAGE(rb, SB(1, 1), b3 + S_H); STAGE(ra, SA(1, 0), a3);
            WAIT_VP; WAIT_L(0); BAR; MMA(1, 0, At, B0); MMA(1, 1, At, B1); BAR; SCHED;
        }
        if (wr == 0) BAR;
        STAMP(dslot); ++dslot;
#define EPI_GROUP(A, pnx, g, H1, E0, E1) do { unsigned _w[4][4]; \
            _Pragma("unroll") for (int m = 0; m < 4; ++m) { \
                _w[m][0] = cvtpk(A[m][0][0], A[m][0][1]); _w[m][1] = cvtpk(A[m][0][2], A[m][0][3]); \
                _w[m][2] = cvtpk(A[m][1][0], A[m][1][1]); _w[m][3] = cvtpk(A[m][1][2], A[m][1][3]); } \
            encode_unit(_w, myws + OFF_CP + (unsigned)(((pm * 16 + (pnx)) * 8 + wid) * 4 + (g)) * UNIT_B, lane, H1, E0, E1); } while (0)
#define CH_ADDR(pnx, g) (chd + (size_t)(((((bi * 2 + (pm & 1)) * 32 + bj_ * 16 + (pnx)) * 8 + wid) * 4 + (g)) * 2 + bk) * 16)
        {
            unsigned bad, h[4][3];
            DSR32(bad, ldsa + LDS_BAD);
            asm volatile("s_waitcnt lgkmcnt(0)" : "+v"(bad) :: "memory");
            const unsigned pois = rfl(bad) ? POISON << 8 : 0;
#define EPI_HDR(g) do { if (lane == 0) gst128_sys(CH_ADDR(pn, g), (u32x4){call, pois ? pois : h[g][0], h[g][1], h[g][2]}); } while (0)
            EPI_GROUP(acc[0][0], pn, 0, h[0][0], h[0][1], h[0][2]);
            EPI_GROUP(acc[0][1], pn, 1, h[1][0], h[1][1], h[1][2]);
            WAIT_V(3); EPI_HDR(0);
            EPI_GROUP(acc[1][0], pn, 2, h[2][0], h[2][1], h[2][2]);
            EPI_GROUP(acc[1][1], pn, 3, h[3][0], h[3][1], h[3][2]);
            WAIT_V(7); EPI_HDR(1);
            WAIT_V(4); EPI_HDR(2);
            WAIT_V(2); EPI_HDR(3);
        }
        STAMP(dslot); ++dslot;
        for (int e = pn; e < 4 * PN2; e += PN1) {
            const int ai = e & 1, bj = (e >> 1) & 1, pn2 = PN1 + (e >> 2);
            if (!NOGEMM) {
                const unsigned long long t0 = __builtin_amdgcn_s_memrealtime();
                const u32x4* f = (const u32x4*)(rdyf + ((16 + pn2) * 64) * NR) + lane;
                for (;;) {
                    u32x4 v0, v1;
                    asm volatile("global_load_dwordx4 %0, %2, off sc0 sc1\n\tglobal_load_dwordx4 %1, %2, off offset:1024 sc0 sc1\n\ts_waitcnt vmcnt(0)"
                                 : "=&v"(v0), "=&v"(v1) : "v"(f) : "memory");
                    bool ok = true;
#pragma unroll
                    for (int q = 0; q < 4; ++q) ok = ok && (int)(v0[q] - call) >= 0 && (int)(v1[q] - call) >= 0;
                    if (__all(ok)) break;
                    if (timed_out(tmo, call, t0)) { *badf = 1; break; }
                    __builtin_amdgcn_s_sleep(16);
                }
            }
            BAR;
            STAMP(dslot); ++dslot;
            const unsigned gA = (unsigned)(2 * pm + ai) * S_H, gB = (unsigned)(2 * pn2 + bj) * S_H;
#pragma unroll
            for (int m = 0; m < 4; ++m)
#pragma unroll
                for (int n = 0; n < 2; ++n) acc[0][0][m][n] = (f32x4){0.f, 0.f, 0.f, 0.f};
#pragma unroll
            for (int q = 0; q < 3; ++q) { STAGE(ra, q * HTB, gA + q * S_K); STAGE(rb, (4 + q) * HTB, gB + q * S_K); }
            for (int t = 0; t < (NOGEMM ? 0 : NT); t += 4) {
#pragma unroll
                for (int q = 0; q < 4; ++q) {
                    const unsigned tn = (unsigned)min(t + q + 3, NT - 1) * S_K;
                    WAIT_V(8); BAR;
                    STAGE(ra, ((q + 3) & 3) * HTB, gA + tn); STAGE(rb, (4 + ((q + 3) & 3)) * HTB, gB + tn);
                    LDB(B0, q >> 1, q & 1); SCHED; LDA(At, q >> 1, q & 1);
                    WAIT_B1(B0);
                    MMA(0, 0, At, B0);
                    SCHED;
                }
            }
            WAIT_V(0);
            unsigned h1, e0, e1;
            EPI_GROUP(acc[0][0], pn2, ai * 2 + bj, h1, e0, e1);
            unsigned bad;
            DSR32(bad, ldsa + LDS_BAD);
            asm volatile("s_waitcnt lgkmcnt(0)" : "+v"(bad) :: "memory");
            WAIT_V(0);
            if (rfl(bad)) h1 = POISON << 8;
            if (lane == 0) gst128_sys(CH_ADDR(pn2, ai * 2 + bj), (u32x4){call, h1, e0, e1});
            STAMP(dslot); ++dslot;
        }
#undef SA
#undef SB
    } else {
        const unsigned call = rfl(gld32_sys(cntp)) + 1;
        if (lane == 0) __hip_atomic_store(cntp, call, __ATOMIC_RELAXED, __HIP_MEMORY_SCOPE_AGENT);
        const unsigned par = call & 1;
        unsigned* const tkt = ctl + C_TKT / 4 + par * 1024;
        if (wg == NGEMM && wid == 7 && lane < 16) __hip_atomic_store(ctl + C_TKT / 4 + (par ^ 1) * 1024 + lane * 64, 0u, __ATOMIC_RELAXED, __HIP_MEMORY_SCOPE_SYSTEM);
        const rsrc_t rh = mk_rsrc(myws, (unsigned)WS_NEED);
        const unsigned lw = (unsigned)wid * 16384u, lh = LDS_AUX + (unsigned)wid * 1024u;
        const unsigned vl16 = (unsigned)lane * 16u;
#define PULL_UNIT(rp, slot, so, h1) do { DMA16(rp, slot, vl16, so); DMA16(rp, (slot) + 1024, vl16, (so) + 1024); DMA16(rp, (slot) + 2048, vl16, (so) + 2048); \
            if ((h1) >> 8 == RAWMODE) DMA16(rp, (slot) + 3072, vl16, (so) + 3072); } while (0)
#define READ_UNITS(d, slot) do { const unsigned _a = ldsa + (slot) + vl16; \
            DSR128(d[0][0], _a, 0); DSR128(d[0][1], _a, 1024); DSR128(d[0][2], _a, 2048); DSR128(d[0][3], _a, 3072); \
            DSR128(d[1][0], _a, 4096); DSR128(d[1][1], _a, 5120); DSR128(d[1][2], _a, 6144); DSR128(d[1][3], _a, 7168); } while (0)
#define WAIT_UNITS(d, hv) asm volatile("s_waitcnt lgkmcnt(0)" : "+v"(d[0][0]), "+v"(d[0][1]), "+v"(d[0][2]), "+v"(d[0][3]), \
            "+v"(d[1][0]), "+v"(d[1][1]), "+v"(d[1][2]), "+v"(d[1][3]), "+v"(hv) :: "memory")
        if (const int cw = (wg - NGEMM) * CW + wid; wid < CW) {
            const int si = cw % (NR - 1), sq = cw / (NR - 1), s = (rank + 1 + si) & 7;
            constexpr int w_in = W_IN;
            const int oc = sq < NCW / 8 ? (s & 3) : (rank & 3);
            const int ti = oc & 1, tj = oc >> 1;
            const rsrc_t rp = mk_rsrc(p.ws[s] + OFF_PACK + (size_t)par * PACK_B, PACK_B);
            const rsrc_t rc0 = mk_rsrc(p.ws[oc] + OFF_CP, CP_B), rc1 = mk_rsrc(p.ws[oc + 4] + OFF_CP, CP_B);
            const rsrc_t ro = mk_rsrc(p.out, (MM / NR) * NN * 2);
            unsigned* rdyf = (unsigned*)(myws + OFF_RDY);
            auto jblk = [&](int j) { return JC(j) >> 6; };
            auto jstep = [&](int j) { return j & 63; };
            auto hdr = [&](int j) {
                unsigned b;
                if (JC(j) < JD_OUT) b = (unsigned)OFF_IH + par * IH_B + (unsigned)(s * 1024 + jblk(j) * 8192 + jstep(j) * 16);
                else { const int o = min(JC(j), JD_END - 1) - JD_OUT, y = out_tile(o);
                       b = (unsigned)OFF_CH + (unsigned)((((ti * 2 + (y & 1)) * 32 + tj * 16 + (y >> 1)) * 32 + (o & 31)) * 32); }
                return lane < 8 ? b + (unsigned)lane * 4u : (unsigned)CTRL_OFF + C_TMO;
            };
            auto ready = [&](unsigned hv) { return __all((lane & 3) != 0 || lane >= 8 || (int)(hv - call) >= 0) != 0; };
            auto pull = [&](int j, unsigned hv, unsigned slot) {
                const unsigned ha = (unsigned)__builtin_amdgcn_readlane((int)hv, 1), hb = (unsigned)__builtin_amdgcn_readlane((int)hv, 5);
                if (JC(j) < JD_OUT) {
                    const int e = jblk(j), mat = e >> 4, blk = e & 15;
                    const int gblk = mat == 0 ? (blk >> 1) * 4 + bi * 2 + (blk & 1) : bj_ * 16 + blk;
                    const unsigned so = (unsigned)(((mat * 2 + bk) * 32 + gblk) * 64 + jstep(j)) * UNIT_B;
                    PULL_UNIT(rp, slot, so, ha);
                    PULL_UNIT(rp, slot + 4096, so + UNIT_B, hb);
                } else {
                    const int o = JC(j) - JD_OUT, y = out_tile(o);
                    const unsigned so = (unsigned)(((rank * 2 + (y & 1)) * 16 + (y >> 1)) * 32 + (o & 31)) * UNIT_B;
                    PULL_UNIT(rc0, slot, so, ha);
                    PULL_UNIT(rc1, slot + 4096, so, hb);
                }
            };
            auto publish = [&](int j) {
                if (lane < 2) __hip_atomic_store(rdyf + (jblk(j) * 64 + jstep(j) + lane) * NR + s, call, __ATOMIC_RELAXED, __HIP_MEMORY_SCOPE_SYSTEM);
            };
            auto out_off = [&](int j) {
                const int o = JC(j) - JD_OUT, gg = o & 3, w8 = (o >> 2) & 7, y = out_tile(o);
                const unsigned orow = (ti * 2 + (y & 1)) * 256 + (gg >> 1) * 128 + (w8 >> 2) * 64 + fr;
                const unsigned ocol = (tj * 16 + (y >> 1)) * 256 + (gg & 1) * 128 + (w8 & 3) * 32 + ((fq & 1) * 16 + (fq >> 1) * 8);
                return ((orow & 1023u) * NN + ocol) * 2;
            };
            auto nan_fill = [&](int j) {
                const unsigned ooff = out_off(j);
#pragma unroll
                for (int m = 0; m < 4; ++m)
                    st128<0>((u32x4){0x7fc07fc0u, 0x7fc07fc0u, 0x7fc07fc0u, 0x7fc07fc0u}, ro, ooff, m * 16 * NN * 2);
            };
            const unsigned* const tki = tkt + si * 64;
            const unsigned* const tko = tkt + (8 + oc) * 64;
            int o1;
            { unsigned tk; TICKET(tk, tko); WAIT_V(0); o1 = JD_OUT + (int)ticket_value(tk); }
            auto next = [&](int jprev, unsigned tk) {
                if (JC(jprev) >= JD_OUT) return (int)min((unsigned)JD_OUT + tk, (unsigned)JD_END);
                const int j = 3 * w_in + (int)tk;
                return j < NJ_IN ? JCODE(j) : o1;
            };
            int jj = JCODE(sq), jB = JCODE(sq + w_in), jC = JCODE(sq + 2 * w_in);
            if (sq >= w_in) {
                unsigned tb, tc;
                TICKET(tb, tko); TICKET(tc, tko); WAIT_V(0);
                jj = o1; jB = next(JD_OUT, ticket_value(tb)); jC = next(JD_OUT, ticket_value(tc));
            }
            bool dead = false;
            while (JC(jj) < JD_END) {
                unsigned hvA, hvB;
                gld32x2_sys(myws + hdr(jj), myws + hdr(jB), hvA, hvB);
                if (!ready(hvA)) {
                    const unsigned long long t0 = __builtin_amdgcn_s_memrealtime();
                    for (;;) {
                        __builtin_amdgcn_s_sleep(4);
                        gld32x2_sys(myws + hdr(jj), myws + hdr(jB), hvA, hvB);
                        if (ready(hvA)) break;
                        if (timed_out(tmo, call, t0)) { dead = true; break; }
                    }
                    if (dead) break;
                }
                unsigned sl = lw, hs = lh;
                pull(jj, hvA, sl);
                DMA4(rh, hs, hdr(jC));
                int pend = -1;
                for (;;) {
                    const int nx = jB;
                    const bool more = JC(nx) < JD_END && ready(hvB);
                    unsigned tk;
                    TICKET(tk, JC(jC) < JD_OUT ? tki : tko);
                    if (more) { pull(nx, hvB, sl ^ 8192); WAIT_V(6); } else WAIT_V(0);
                    if (pend >= 0) publish(pend);
                    const int jD = next(jC, ticket_value(tk));
                    unsigned hvC;
                    u32x4 d[2][4];
                    DSR32(hvC, ldsa + hs + lane * 4);
                    READ_UNITS(d, sl);
                    WAIT_UNITS(d, hvC);
                    hs ^= 256;
                    if (more) DMA4(rh, hs, hdr(jD));
                    if (JC(jj) < JD_OUT) {
                        const int e = jblk(jj);
                        const char* dst = myws + OFF_STG + (size_t)((unsigned)(e >> 4) * STG_B + (unsigned)(2 * (e & 15) * 64 + jstep(jj)) * HTB + (unsigned)s * 2048u);
#pragma unroll
                        for (int u = 0; u < 2; ++u) {
                            const unsigned h1 = __builtin_amdgcn_readlane((int)hvA, u * 4 + 1);
                            u32x4 o[4];
                            if (h1 >> 8 != RAWMODE) {
                                unsigned H[8];
                                decode_high(d[u][2], h1, __builtin_amdgcn_readlane((int)hvA, u * 4 + 2), __builtin_amdgcn_readlane((int)hvA, u * 4 + 3), lane, H);
#pragma unroll
                                for (int m = 0; m < 4; ++m) {
                                    const unsigned l0 = d[u][m >> 1][(m & 1) * 2], l1 = d[u][m >> 1][(m & 1) * 2 + 1];
                                    o[m] = (u32x4){vperm(H[2 * m], l0, 0x05010400u), vperm(H[2 * m], l0, 0x07030602u),
                                                   vperm(H[2 * m + 1], l1, 0x05010400u), vperm(H[2 * m + 1], l1, 0x07030602u)};
                                }
                            } else {
#pragma unroll
                                for (int m = 0; m < 4; ++m) o[m] = d[u][m];
                            }
                            const char* d0 = dst + u * HTB;
                            gst128s_sys<0>(d0, vl16, o[0]); gst128s_sys<1024>(d0, vl16, o[1]);
                            gst128s_sys<0>(d0 + 64 * HTB, vl16, o[2]); gst128s_sys<1024>(d0 + 64 * HTB, vl16, o[3]);
                        }
                        pend = jj;
                    } else {
                        const bool bad = (unsigned)__builtin_amdgcn_readlane((int)hvA, 8) == call
                                         || (unsigned)__builtin_amdgcn_readlane((int)hvA, 1) >> 8 == POISON
                                         || (unsigned)__builtin_amdgcn_readlane((int)hvA, 5) >> 8 == POISON;
                        if (bad) nan_fill(jj);
                        else {
                            const unsigned ooff = out_off(jj);
                            float a[4][8];
#pragma unroll
                            for (int k = 0; k < 2; ++k) {
                                const unsigned h1 = __builtin_amdgcn_readlane((int)hvA, k * 4 + 1);
                                if (h1 >> 8 == RAWMODE) {
#pragma unroll
                                    for (int m = 0; m < 4; ++m)
#pragma unroll
                                        for (int j = 0; j < 4; ++j) {
                                            const unsigned v = d[k][m][j];
                                            const float lo = __uint_as_float(v << 16), hi = __uint_as_float(v & 0xffff0000u);
                                            a[m][2 * j] = k == 0 ? lo : a[m][2 * j] + lo; a[m][2 * j + 1] = k == 0 ? hi : a[m][2 * j + 1] + hi;
                                        }
                                } else {
                                    unsigned H[8];
                                    decode_high(d[k][2], h1, __builtin_amdgcn_readlane((int)hvA, k * 4 + 2), __builtin_amdgcn_readlane((int)hvA, k * 4 + 3), lane, H);
#pragma unroll
                                    for (int m = 0; m < 4; ++m)
#pragma unroll
                                        for (int j = 0; j < 8; ++j) {
                                            const unsigned Lw = d[k][m >> 1][(m & 1) * 2 + (j >> 2)];
                                            const float v = __uint_as_float(vperm(H[2 * m + (j >> 2)], Lw, (4u + (j & 3)) << 24 | (unsigned)(j & 3) << 16 | 0x0c0cu));
                                            a[m][j] = k == 0 ? v : a[m][j] + v;
                                        }
                                }
                            }
#pragma unroll
                            for (int m = 0; m < 4; ++m) {
                                unsigned r0 = cvtpk(a[m][0], a[m][1]), r1 = cvtpk(a[m][2], a[m][3]), r2 = cvtpk(a[m][4], a[m][5]), r3 = cvtpk(a[m][6], a[m][7]);
                                auto s02 = __builtin_amdgcn_permlane16_swap(r0, r2, false, false);
                                auto s13 = __builtin_amdgcn_permlane16_swap(r1, r3, false, false);
                                u32x4 o; o.x = s02[0]; o.y = s13[0]; o.z = s02[1]; o.w = s13[1];
                                st128<OAUX>(o, ro, ooff, m * 16 * NN * 2);
                            }
                        }
                        STAMPJ(8 + oc, jj - JD_OUT);
                        if (jj == o1) STAMPW(32);
                        pend = -1;
                    }
                    jj = nx; jB = jC; jC = jD; hvA = hvB; hvB = hvC; sl ^= 8192;
                    if (!more) break;
                }
                if (pend >= 0) { WAIT_V(0); publish(pend); }
            }
            if (dead) {
                while (JC(jj) < JD_END) {
                    if (JC(jj) >= JD_OUT) nan_fill(jj);
                    unsigned tk;
                    TICKET(tk, JC(jC) < JD_OUT ? tki : tko);
                    WAIT_V(0);
                    const int jD = next(jC, ticket_value(tk));
                    jj = jB; jB = jC; jC = jD;
                }
            }
        }
        STAMPW(40);
    }
}

}

extern "C" void kernel_launch(int rank, int nranks, ncclComm_t comm,
                              hipStream_t stream,
                              void* const* d_in, const int* in_sizes, int n_in,
                              void* d_out, int out_size,
                              const int* in_dtypes, int out_dtype,
                              void* const* d_out_all,
                              void* const* d_ws_all, size_t ws_bytes) {
    (void)comm; (void)in_dtypes; (void)out_dtype; (void)d_out_all;
    if (nranks != NR || n_in < 2 || in_sizes[0] != MM * KL || in_sizes[1] != KL * NN || out_size != (MM / NR) * NN
        || ws_bytes < WS_NEED || (unsigned)rank >= (unsigned)NR) {
        fprintf(stderr, "kernel_launch: unexpected shapes\n");
        return;
    }
    static bool inited[NR] = {};
    if (!inited[rank]) {
        HIP_CHECK(hipFuncSetAttribute((const void*)gemm_rs_kernel, hipFuncAttributeMaxDynamicSharedMemorySize, LDS_BYTES));
        inited[rank] = true;
    }
    Params p;
    p.x = d_in[0]; p.w = d_in[1]; p.out = d_out; p.rank = rank;
    for (int i = 0; i < NR; ++i) p.ws[i] = (char*)d_ws_all[i];
    hipLaunchKernelGGL(gemm_rs_kernel, dim3(NWG), dim3(NTHR), LDS_BYTES, stream, p);
    HIP_CHECK(hipGetLastError());
}
```

```cpp
#include <hip/hip_runtime.h>
#include <rccl/rccl.h>
#include <cstdio>
#include <cstdint>

#define HIP_CHECK(expr) do { hipError_t _e = (expr); if (_e != hipSuccess) { \
    fprintf(stderr, "HIP error %s at %s:%d\n", hipGetErrorString(_e), __FILE__, __LINE__); \
    return; } } while (0)

#define LAS __attribute__((address_space(3)))
#define UNLIKELY(x) __builtin_expect(!!(x), 0)
typedef short bf16x8 __attribute__((ext_vector_type(8)));
typedef short s16x4 __attribute__((ext_vector_type(4)));
typedef float f32x4 __attribute__((ext_vector_type(4)));
typedef float f32x2 __attribute__((ext_vector_type(2)));
typedef unsigned u32x4 __attribute__((ext_vector_type(4)));
typedef unsigned short u16x2 __attribute__((ext_vector_type(2)));
typedef __bf16 bf16x2_t __attribute__((ext_vector_type(2)));
typedef __amdgpu_buffer_rsrc_t rsrc_t;

namespace {

constexpr int NR = 8;
constexpr int MM = 8192, NN = 8192, KL = 1024;
constexpr int NWG = 256, NTHR = 512;
#define PN1 13
#define LOOK 8
#define CW 2
#define PKSLEEP 100
#define FORCE_RAW 0
#define NOGEMM 0
#define GAUX 17
#define GAUX2 GAUX
#define G2X 0
#define NOGEMM2 0
#define OAUX 0
constexpr int NGEMM = 16 * PN1, NCOMM = NWG - NGEMM, NCW = NCOMM * CW;
constexpr int PN2 = 16 - PN1;
constexpr int HTB = 16384, STAGE_BYTES = 8 * HTB;
constexpr int NT = 64;
constexpr unsigned UNIT_B = 4096;
constexpr size_t MIB = 1u << 20;
constexpr size_t OFF_PACK = 0;
constexpr unsigned PACK_B = 32u << 20;
constexpr size_t OFF_STG = 64 * MIB;
constexpr unsigned STG_B = 32u << 20;
constexpr size_t OFF_CP = 128 * MIB;
constexpr unsigned CP_B = 32u << 20;
constexpr size_t OFF_IH = 160 * MIB;
constexpr unsigned IH_B = 2 * 16 * 8 * 64 * 16;
constexpr size_t OFF_RDY = 161 * MIB;
constexpr size_t OFF_CH = 162 * MIB;
constexpr size_t CTRL_OFF = 256 * MIB;
constexpr size_t WS_NEED = CTRL_OFF + MIB;
constexpr unsigned C_TMO = 256, C_CNT = 4096;
constexpr unsigned C_TKT = 16384;
constexpr unsigned MAXEXC = 2, RAWMODE = 255, POISON = 254;
constexpr unsigned long long DEADLINE = 30000000ull;
constexpr int LDS_AUX = STAGE_BYTES;
constexpr int LDS_BAD = STAGE_BYTES + 8192;
constexpr int LDS_BYTES = LDS_BAD + 256;
constexpr int AUX_SYS = 17;
constexpr int E1 = 16 + PN1;
constexpr int NJ1 = (NT / 2) * E1, NJ_IN = (NT / 2) * 32;
#define NFB 4
#define SKD 2
constexpr int EF = 8 + NFB, ES = E1 - EF;
constexpr int Q_OUT = NJ_IN, Q_END = Q_OUT + 1024;
constexpr int LDS_TAB = 65536;
__host__ __device__ constexpr int in_job(int j) {
    int i = 0, r = 0;
    if (j < SKD * EF) { i = j % EF; r = j / EF; }
    else if (j < SKD * EF + (NT / 2 - SKD) * E1) { const int q = j - SKD * EF; i = q % E1; r = SKD + q / E1 - (i < EF ? 0 : SKD); }
    else if (j < NJ1) { const int q = j - (SKD * EF + (NT / 2 - SKD) * E1); i = EF + q % ES; r = NT / 2 - SKD + q / ES; }
    else { const int q = j - NJ1; i = E1 + q % PN2; r = q / PN2; }
    const int e = i < 8 ? 2 * i : i < EF ? 8 + i : i < 8 + EF ? 2 * (i - EF) + 1 : i;
    return e * 64 + 2 * r;
}
__host__ __device__ constexpr int out_tile(int o) { const int y = o >> 5; return y < NFB ? 2 * y : y < 2 * NFB ? 2 * (y - NFB) + 1 : y; }
#define W_IN (NCW / (NR - 1))

#define STAMP(slot) do {} while (0)
#define STAMPW(slot) do {} while (0)
#define STAMPJ(row, j) do {} while (0)
#define DIAG_BLK0 do {} while (0)
#define DIAG_BLK1(t0) do {} while (0)
struct Params {
    const void* x; const void* w; void* out;
    char* ws[NR];
    int rank;
};

__device__ __forceinline__ unsigned cvtpk(float lo, float hi) { f32x2 v = {lo, hi}; bf16x2_t b = __builtin_convertvector(v, bf16x2_t); return __builtin_bit_cast(unsigned, b); }
__device__ __forceinline__ rsrc_t mk_rsrc(const void* p, unsigned bytes) { return __builtin_amdgcn_make_buffer_rsrc((void*)p, 0, bytes, 0x00020000); }

template <int AUX> __device__ __forceinline__ void st128(u32x4 v, rsrc_t r, unsigned voff, unsigned soff) {
    __builtin_amdgcn_raw_buffer_store_b128(v, r, voff, soff, AUX);
    asm volatile("s_nop 1" :: "v"(v) : "memory");
}
__device__ __forceinline__ void gst128_sys(void* ptr, u32x4 v) {
    asm volatile("global_store_dwordx4 %0, %1, off sc0 sc1\n\ts_nop 1" :: "v"(ptr), "v"(v) : "memory");
}
__device__ __forceinline__ unsigned gld32_sys(const void* ptr) {
    unsigned v;
    asm volatile("global_load_dword %0, %1, off sc0 sc1\n\ts_waitcnt vmcnt(0)" : "=&v"(v) : "v"(ptr) : "memory");
    return v;
}
__device__ __forceinline__ void gld32x2_sys(const void* pa, const void* pb, unsigned& a, unsigned& b) {
    asm volatile("global_load_dword %0, %2, off sc0 sc1\n\tglobal_load_dword %1, %3, off sc0 sc1\n\ts_waitcnt vmcnt(0)"
                 : "=&v"(a), "=&v"(b) : "v"(pa), "v"(pb) : "memory");
}
template <int OFF> __device__ __forceinline__ void gst128s_sys(const void* base, unsigned voff, u32x4 v) {
    asm volatile("global_store_dwordx4 %0, %1, %2 offset:%c3 sc0 sc1\n\ts_nop 1" :: "v"(voff), "v"(v), "s"(base), "i"(OFF) : "memory");
}
__device__ __forceinline__ unsigned bfi(unsigned m, unsigned a, unsigned b) { return (a & m) | (b & ~m); }
__device__ __forceinline__ unsigned vperm(unsigned hi, unsigned lo, unsigned sel) { return __builtin_amdgcn_perm(hi, lo, sel); }
__device__ __forceinline__ unsigned rfl(unsigned v) { return __builtin_amdgcn_readfirstlane(v); }
__device__ __forceinline__ unsigned wave_max_u(int x) {
    x = max(x, __builtin_amdgcn_update_dpp(0, x, 0xB1, 0xf, 0xf, true));
    x = max(x, __builtin_amdgcn_update_dpp(0, x, 0x4E, 0xf, 0xf, true));
    x = max(x, __builtin_amdgcn_update_dpp(0, x, 0x141, 0xf, 0xf, true));
    x = max(x, __builtin_amdgcn_update_dpp(0, x, 0x140, 0xf, 0xf, true));
    const int a = __builtin_amdgcn_readlane(x, 0), b = __builtin_amdgcn_readlane(x, 16), c = __builtin_amdgcn_readlane(x, 32), d = __builtin_amdgcn_readlane(x, 48);
    return (unsigned)max(max(a, b), max(c, d));
}
__device__ __forceinline__ bool timed_out(unsigned* tmo, unsigned call, unsigned long long t0) {
    if (__hip_atomic_load(tmo, __ATOMIC_RELAXED, __HIP_MEMORY_SCOPE_AGENT) == call) return true;
    if (__builtin_amdgcn_s_memrealtime() - t0 > DEADLINE) { __hip_atomic_store(tmo, call, __ATOMIC_RELAXED, __HIP_MEMORY_SCOPE_AGENT); return true; }
    return false;
}

__device__ __forceinline__ void encode_unit(const unsigned (&w)[4][4], const char* dst, int lane, unsigned& h1, unsigned& e0, unsigned& e1) {
    u16x2 mx = {0, 0};
#pragma unroll
    for (int m = 0; m < 4; ++m)
#pragma unroll
        for (int q = 0; q < 4; ++q) mx = __builtin_elementwise_max(mx, __builtin_bit_cast(u16x2, w[m][q] & 0x7f007f00u));
    const unsigned base = wave_max_u((int)max(mx[0], mx[1])) >> 8;
    const unsigned bq = base * 0x01010101u | 0x80808080u;
    unsigned L[8], P[4], ex = 0;
#pragma unroll
    for (int m = 0; m < 4; ++m) {
        L[2 * m] = vperm(w[m][1], w[m][0], 0x06040200u); L[2 * m + 1] = vperm(w[m][3], w[m][2], 0x06040200u);
        const unsigned h01 = vperm(w[m][1], w[m][0], 0x07050301u), h23 = vperm(w[m][3], w[m][2], 0x07050301u);
        const unsigned d01 = bq - (h01 & 0x7f7f7f7fu), d23 = bq - (h23 & 0x7f7f7f7fu);
        ex |= d01 | d23;
        P[m] = bfi(0x0f0f0f0fu, bfi(0x07070707u, d01, h01 >> 4), bfi(0x70707070u, d23 << 4, h23));
    }
    unsigned cnt = FORCE_RAW ? RAWMODE : 0;
    e0 = 0; e1 = 0;
#define NL3 64
#define NOEXC 0
    unsigned long long bal = NL3 == 64 && !NOEXC ? __ballot((ex & 0x78787878u) != 0) : 0;
    if (UNLIKELY(bal != 0)) while (bal && cnt <= MAXEXC) {
        const unsigned l = (unsigned)__builtin_ctzll(bal);
        bal &= bal - 1;
#pragma unroll
        for (int i = 0; i < 8; ++i) {
            const unsigned hb = (unsigned)__builtin_amdgcn_readlane((int)vperm(w[i >> 1][(i & 1) * 2 + 1], w[i >> 1][(i & 1) * 2], 0x07050301u), (int)l);
            unsigned f = (bq - (hb & 0x7f7f7f7fu)) & 0x78787878u;
            while (f) {
                const unsigned sh = (unsigned)__builtin_ctz(f) & 24u;
                const unsigned en = (unsigned)((i >> 1) << 17 | (i & 1) << 10) | sh << 5 | l << 11 | (hb >> sh & 0xffu);
                if (cnt == 0) e0 = en; else e1 = en;
                ++cnt; f &= ~(0xffu << sh);
            }
        }
    }
#define NORAW 0
    if (cnt > MAXEXC) cnt = NORAW ? MAXEXC : RAWMODE;
    const unsigned vo = (unsigned)lane * 16u;
    if (UNLIKELY(cnt == RAWMODE)) {
        gst128s_sys<0>(dst, vo, (u32x4){w[0][0], w[0][1], w[0][2], w[0][3]});
        gst128s_sys<1024>(dst, vo, (u32x4){w[1][0], w[1][1], w[1][2], w[1][3]});
        gst128s_sys<2048>(dst, vo, (u32x4){w[2][0], w[2][1], w[2][2], w[2][3]});
        gst128s_sys<3072>(dst, vo, (u32x4){w[3][0], w[3][1], w[3][2], w[3][3]});
    } else {
        gst128s_sys<0>(dst, vo, (u32x4){L[0], L[1], L[2], L[3]});
        gst128s_sys<1024>(dst, vo, (u32x4){L[4], L[5], L[6], L[7]});
        gst128s_sys<2048>(dst, vo, (u32x4){P[0], P[1], P[2], P[3]});
    }
    h1 = base | cnt << 8;
}
__device__ __forceinline__ void patch_high(unsigned (&H)[8], unsigned en, int lane) {
    const unsigned sh = en >> 5 & 24u, val = (en & 0xffu) << sh;
    const unsigned mk = (unsigned)lane == (en >> 11 & 63) ? 0xffu << sh : 0u;
    asm volatile("s_bitcmp1_b32 %10, 18\n\ts_cbranch_scc1 4f\n\ts_bitcmp1_b32 %10, 17\n\ts_cbranch_scc1 2f\n\ts_bitcmp1_b32 %10, 10\n\ts_cbranch_scc1 1f\n\t"
                 "v_bfi_b32 %0, %8, %9, %0\n\ts_branch 8f\n"
                 "1:\n\tv_bfi_b32 %1, %8, %9, %1\n\ts_branch 8f\n"
                 "2:\n\ts_bitcmp1_b32 %10, 10\n\ts_cbranch_scc1 3f\n\tv_bfi_b32 %2, %8, %9, %2\n\ts_branch 8f\n"
                 "3:\n\tv_bfi_b32 %3, %8, %9, %3\n\ts_branch 8f\n"
                 "4:\n\ts_bitcmp1_b32 %10, 17\n\ts_cbranch_scc1 6f\n\ts_bitcmp1_b32 %10, 10\n\ts_cbranch_scc1 5f\n\tv_bfi_b32 %4, %8, %9, %4\n\ts_branch 8f\n"
                 "5:\n\tv_bfi_b32 %5, %8, %9, %5\n\ts_branch 8f\n"
                 "6:\n\ts_bitcmp1_b32 %10, 10\n\ts_cbranch_scc1 7f\n\tv_bfi_b32 %6, %8, %9, %6\n\ts_branch 8f\n"
                 "7:\n\tv_bfi_b32 %7, %8, %9, %7\n"
                 "8:"
                 : "+v"(H[0]), "+v"(H[1]), "+v"(H[2]), "+v"(H[3]), "+v"(H[4]), "+v"(H[5]), "+v"(H[6]), "+v"(H[7]) : "v"(mk), "s"(val), "s"(en) : "scc");
}
__device__ __forceinline__ void decode_high(u32x4 P, unsigned h1, unsigned e0, unsigned e1, int lane, unsigned (&H)[8]) {
    const unsigned bq = (h1 & 0x7fu) * 0x01010101u | 0x80808080u;
#pragma unroll
    for (int m = 0; m < 4; ++m) {
        H[2 * m] = bfi(0x7f7f7f7fu, bq - (P[m] & 0x07070707u), P[m] << 4);
        H[2 * m + 1] = bfi(0x7f7f7f7fu, bq - ((P[m] >> 4) & 0x07070707u), P[m]);
    }
    const unsigned cnt = h1 >> 8;
#define NODEC 0
    if (!NODEC && UNLIKELY(cnt != 0)) {
        patch_high(H, e0, lane);
        if (cnt > 1) patch_high(H, e1, lane);
    }
}

#define TRD(dst, addr, off) asm volatile("ds_read_b64_tr_b16 %0, %1 offset:%c2" : "=&v"(dst) : "v"(addr), "i"(off) : "memory")
#define PKB(x) (bf16x8){(x)[0][0], (x)[0][1], (x)[0][2], (x)[0][3], (x)[1][0], (x)[1][1], (x)[1][2], (x)[1][3]}
#define WAIT_V(n) asm volatile("s_waitcnt vmcnt(" #n ")" ::: "memory")
#define WAIT_L(n) asm volatile("s_waitcnt lgkmcnt(" #n ")" ::: "memory")
#define BAR __builtin_amdgcn_s_barrier()
#define SCHED __builtin_amdgcn_sched_barrier(0)
#define DSR128(dst, addr, off) asm volatile("ds_read_b128 %0, %1 offset:%c2" : "=&v"(dst) : "v"(addr), "i"(off) : "memory")
#define DSR32(dst, addr) asm volatile("ds_read_b32 %0, %1" : "=&v"(dst) : "v"(addr) : "memory")
#define DMA16(rs, ldsoff, voff, soff) __builtin_amdgcn_raw_ptr_buffer_load_lds(rs, (LAS void*)(lds + (ldsoff)), 16, voff, soff, 0, AUX_SYS)
#define DMA4(rs, ldsoff, voff) __builtin_amdgcn_raw_ptr_buffer_load_lds(rs, (LAS void*)(lds + (ldsoff)), 4, voff, 0, 0, AUX_SYS)
#define TICKET(tk, ptr) asm volatile("s_mov_b64 exec, 1\n\tglobal_atomic_add %0, %1, %2, %3 sc0 sc1\n\ts_mov_b64 exec, -1" : "=&v"(tk) : "v"(0u), "v"(1u), "s"(ptr) : "memory")
__device__ __forceinline__ unsigned ticket_value(unsigned tk) {
    unsigned r;
    asm volatile("v_readfirstlane_b32 %0, %1" : "=s"(r) : "v"(tk) : "memory");
    return r;
}

__global__ __launch_bounds__(NTHR, 2) void gemm_rs_kernel(Params p) {
    extern __shared__ __attribute__((aligned(16))) unsigned char smem[];
    LAS unsigned char* lds = (LAS unsigned char*)smem;
    const int tid = threadIdx.x, wid = __builtin_amdgcn_readfirstlane(tid >> 6), lane = tid & 63;
    const int wr = wid >> 2, wc = wid & 3, fr = lane & 15, fq = lane >> 4;
    const int wg = blockIdx.x, rank = p.rank;
    const int bi = rank & 1, bj_ = (rank >> 1) & 1, bk = rank >> 2;
    char* myws = p.ws[rank];
    unsigned* ctl = (unsigned*)(myws + CTRL_OFF);
    unsigned* tmo = ctl + C_TMO / 4;

    int dslot = 0;
    STAMP(dslot); ++dslot;
    const unsigned ldsa = (unsigned)(size_t)lds;
    unsigned* cntp = ctl + C_CNT / 4 + wg * 8 + wid;

    if (wg < NGEMM) {
        const rsrc_t rx = mk_rsrc(p.x, MM * KL * 2), rw = mk_rsrc(p.w, KL * NN * 2);
        const unsigned vx = (unsigned)((tid >> 3) * (KL * 2) + (((tid & 7) ^ ((tid >> 4) & 7)) << 4));
        const unsigned vw = (unsigned)(fq * (NN * 2) + ((fr ^ ((fq << 2) | ((rank & 1) << 1))) << 4));
        auto pk_issue = [&](int idx, int buf) {
            const int g8 = idx >> 7, mat = (idx >> 6) & 1, kk = (idx >> 5) & 1, blk = idx & 31;
            if (mat == 0) {
#pragma unroll
                for (int i = 0; i < 4; ++i)
                    __builtin_amdgcn_raw_ptr_buffer_load_lds(rx, (LAS void*)(lds + buf + i * 8192 + wid * 1024), 16, vx,
                                                             (unsigned)((blk * 256 + i * 64) * (KL * 2) + kk * 1024 + g8 * 128), 0, 0);
            } else {
#pragma unroll
                for (int c = 0; c < 4; ++c)
                    __builtin_amdgcn_raw_ptr_buffer_load_lds(rw, (LAS void*)(lds + buf + wid * 4096 + c * 1024), 16, vw,
                                                             (unsigned)((kk * 512 + (g8 * 8 + wid) * 8 + (c & 1) * 4) * (NN * 2) + blk * 512 + (c >> 1) * 256), 0, 0);
            }
        };
        pk_issue(wg, 0);
        volatile LAS unsigned* const badf = (volatile LAS unsigned*)(lds + LDS_BAD);
        if (tid == 0) asm volatile("ds_write_b32 %0, %1" :: "v"(ldsa + LDS_BAD), "v"(0u) : "memory");
        const unsigned call = rfl(gld32_sys(cntp)) + 1;
        if (lane == 0) __hip_atomic_store(cntp, call, __ATOMIC_RELAXED, __HIP_MEMORY_SCOPE_AGENT);
        const unsigned par = call & 1;
        unsigned* rdyw = (unsigned*)(myws + OFF_RDY);
        {
            const char* const pk = myws + OFF_PACK + (size_t)par * PACK_B;
            const rsrc_t rs = mk_rsrc(myws + OFF_STG, 2 * STG_B);
            int buf = 0;
            for (int idx = wg; idx < 1024; idx += NGEMM, buf ^= 32768) {
                const int g8 = idx >> 7, mat = (idx >> 6) & 1, kk = (idx >> 5) & 1, blk = idx & 31, t = g8 * 8 + wid;
                BAR;
                if (idx >= NGEMM) __builtin_amdgcn_s_sleep(PKSLEEP);
                if (idx + NGEMM < 1024) pk_issue(idx + NGEMM, buf ^ 32768);
                u32x4 v[4];
                if (mat == 0) {
                    const unsigned a = ldsa + buf + lane * 128 + ((wid ^ ((lane >> 1) & 7)) << 4);
                    DSR128(v[0], a, 0); DSR128(v[1], a, 8192); DSR128(v[2], a, 16384); DSR128(v[3], a, 24576);
                } else {
                    const unsigned a = ldsa + buf + wid * 4096 + lane * 16;
                    DSR128(v[0], a, 0); DSR128(v[1], a, 1024); DSR128(v[2], a, 2048); DSR128(v[3], a, 3072);
                }
                asm volatile("s_waitcnt lgkmcnt(0)" : "+v"(v[0]), "+v"(v[1]), "+v"(v[2]), "+v"(v[3]) :: "memory");
                unsigned w[4][4];
#pragma unroll
                for (int c = 0; c < 4; ++c) { w[c][0] = v[c][0]; w[c][1] = v[c][1]; w[c][2] = v[c][2]; w[c][3] = v[c][3]; }
                unsigned h1, e0, e1;
                encode_unit(w, pk + (unsigned)(((mat * 2 + kk) * 32 + blk) * 64 + t) * UNIT_B, lane, h1, e0, e1);
                const int cb = mat == 0 ? ((blk >> 2) * 2 + (blk & 1)) : (blk & 15);
                const int c0 = mat == 0 ? ((blk >> 1) & 1) + 4 * kk : 2 * (blk >> 4) + 4 * kk, c1 = c0 + (mat == 0 ? 2 : 1);
                const bool own = c0 == rank || c1 == rank;
                if (own) {
                    const unsigned dst = (unsigned)mat * STG_B + (unsigned)(2 * cb * 64 + t) * HTB + (unsigned)rank * 2048u;
#pragma unroll
                    for (int m = 0; m < 4; ++m) st128<AUX_SYS>(v[m], rs, lane * 16, dst + (m >> 1) * (64 * HTB) + (m & 1) * 1024);
                }
                char* const w0 = p.ws[c0 & 7];
                char* const w1 = p.ws[c1 & 7];
                WAIT_V(0);
                if (lane < 2) {
                    const int c = lane == 0 ? c0 : c1;
                    if (c != rank) gst128_sys((lane == 0 ? w0 : w1) + OFF_IH + (size_t)par * IH_B + (size_t)((((mat * 16 + cb) * NR + rank) * 64 + t) * 16), (u32x4){call, h1, e0, e1});
                    else __hip_atomic_store(rdyw + ((mat * 16 + cb) * 64 + t) * NR + rank, call, __ATOMIC_RELAXED, __HIP_MEMORY_SCOPE_SYSTEM);
                }
            }
            BAR;
        }
        STAMP(dslot); ++dslot;
        const int pm = 2 * (wg & 7) + ((wg >> 3) & 1), pn = wg >> 4;
        const rsrc_t ra = mk_rsrc(myws + OFF_STG, STG_B), rb = mk_rsrc(myws + OFF_STG + STG_B, STG_B);
        const unsigned voff = (unsigned)(wid * 1024 + lane * 16);
        constexpr unsigned S_K = HTB, S_H = NT * HTB, S_T = 2 * NT * HTB;
        const unsigned ldsw = (unsigned)wid * 1024u;
        const int aoff = fq * 2048 + (wr * 64 + fr) * 16;
        unsigned boff[2];
        { const int q = (lane & 15) >> 2, pp = lane & 3, g = fq;
#pragma unroll
          for (int n = 0; n < 2; ++n) boff[n] = (unsigned)(size_t)lds + 4 * HTB + 256 * (8 * g + q) + 16 * (((wc ^ q) << 2) | ((n ^ (g & 1)) << 1) | (pp >> 1)) + 8 * (pp & 1); }
        const unsigned* rdyf = (const unsigned*)(myws + OFF_RDY);
#define SA(b, h) (((b) * 2 + (h)) * HTB)
#define SB(b, h) ((4 + (b) * 2 + (h)) * HTB)
#define STAGEX(rs, bufoff, soff, aux) do { _Pragma("unroll") for (int _i = 0; _i < 2; ++_i) \
        __builtin_amdgcn_raw_ptr_buffer_load_lds(rs, (LAS void*)(lds + (bufoff) + ldsw + _i * 8192), 16, voff, (soff) + _i * 8192, 0, aux); } while (0)
#define STAGE(rs, bufoff, soff) STAGEX(rs, bufoff, soff, GAUX)
#define STAGE2(rs, bufoff, soff) STAGEX(rs, bufoff, soff, GAUX2)
#define LDA(dst, b, h) do { _Pragma("unroll") for (int m = 0; m < 4; ++m) _Pragma("unroll") for (int k = 0; k < 2; ++k) dst[m][k] = *(const LAS bf16x8*)(lds + SA(b, h) + aoff + m * 256 + k * 8192); } while (0)
#define LDB(dst, b, h) do { _Pragma("unroll") for (int n = 0; n < 2; ++n) _Pragma("unroll") for (int k = 0; k < 2; ++k) { \
        TRD(dst[n][k][0], boff[n], ((b) * 2 + (h)) * HTB + k * 8192); TRD(dst[n][k][1], boff[n], ((b) * 2 + (h)) * HTB + k * 8192 + 1024); } } while (0)
#define MMA(ai, bj, At, Bt) do { __builtin_amdgcn_s_setprio(1); _Pragma("unroll") for (int m = 0; m < 4; ++m) _Pragma("unroll") for (int n = 0; n < 2; ++n) _Pragma("unroll") for (int k = 0; k < 2; ++k) \
        acc[ai][bj][m][n] = __builtin_amdgcn_mfma_f32_16x16x32_bf16(PKB(Bt[n][k]), At[m][k], acc[ai][bj][m][n], 0, 0, 0); __builtin_amdgcn_s_setprio(0); } while (0)
#define WAIT_B(X, Y) asm volatile("s_waitcnt lgkmcnt(0)" : "+v"(X[0][0][0]), "+v"(X[0][0][1]), "+v"(X[0][1][0]), "+v"(X[0][1][1]), "+v"(X[1][0][0]), "+v"(X[1][0][1]), "+v"(X[1][1][0]), "+v"(X[1][1][1]), \
        "+v"(Y[0][0][0]), "+v"(Y[0][0][1]), "+v"(Y[0][1][0]), "+v"(Y[0][1][1]), "+v"(Y[1][0][0]), "+v"(Y[1][0][1]), "+v"(Y[1][1][0]), "+v"(Y[1][1][1]) :: "memory")
#define WAIT_B1(X) asm volatile("s_waitcnt lgkmcnt(0)" : "+v"(X[0][0][0]), "+v"(X[0][0][1]), "+v"(X[0][1][0]), "+v"(X[0][1][1]), "+v"(X[1][0][0]), "+v"(X[1][0][1]), "+v"(X[1][1][0]), "+v"(X[1][1][1]) :: "memory")
        const rsrc_t rr = mk_rsrc(myws + OFF_RDY, 2 * 16 * 64 * 32);
        const unsigned pvo = (unsigned)(((lane >> 5) * 16 + (lane < 32 ? pm : pn)) * 2048 + (lane & 1) * 16);
        const unsigned pla = ldsa + LDS_AUX + wid * 1024 + lane * 16;
#define NOGATE 0
        int rdy = NOGATE ? NT : 0, pbase = 0;
        bool pol = false;
        u32x4 fv;
#define COUNT_RDY(v) do { const bool _ok = (int)((v)[0] - call) >= 0 && (int)((v)[1] - call) >= 0 && (int)((v)[2] - call) >= 0 && (int)((v)[3] - call) >= 0; \
            const unsigned long long _b = __ballot(_ok); \
            unsigned _m = (unsigned)_b & (unsigned)(_b >> 32); _m = ~(_m & (_m >> 1)) & 0x55555555u; \
            rdy = min(pbase + (_m ? (int)(__builtin_ctz(_m) >> 1) : 16), NT); } while (0)
#define FLAGREAD do { if (pol) DSR128(fv, pla, 0); } while (0)
#define GATE(tt) do { \
            if (pol) { asm volatile("s_waitcnt lgkmcnt(0)" : "+v"(fv) :: "memory"); COUNT_RDY(fv); } \
            if (UNLIKELY((int)(tt) >= rdy)) { \
                const unsigned long long _t0 = __builtin_amdgcn_s_memrealtime(); \
                DIAG_BLK0; \
                for (;;) { \
                    pbase = rdy; \
                    const char* _p = (const char*)rdyf + pvo + min(rdy + ((lane >> 1) & 15), NT - 1) * 32; \
                    u32x4 _v; \
                    asm volatile("global_load_dwordx4 %0, %1, off sc0 sc1\n\ts_waitcnt vmcnt(0)" : "=&v"(_v) : "v"(_p) : "memory"); \
                    COUNT_RDY(_v); \
                    if ((int)(tt) < rdy) break; \
                    if (timed_out(tmo, call, _t0)) { *badf = 1; rdy = NT; break; } \
                    __builtin_amdgcn_s_sleep(8); \
                } DIAG_BLK1(_t0); } \
            pol = rdy < min((int)(tt) + LOOK, NT); \
            if (pol) { pbase = rdy; DMA16(rr, LDS_AUX + wid * 1024, pvo + min(rdy + ((lane >> 1) & 15), NT - 1) * 32, 0); } } while (0)
#define WAIT_VP do { if (pol) WAIT_V(9); else WAIT_V(8); } while (0)
        const int owner = pm >> 1;
        char* chd = p.ws[owner] + OFF_CH;
        f32x4 acc[2][2][4][2];
#pragma unroll
        for (int a = 0; a < 2; ++a)
#pragma unroll
            for (int b = 0; b < 2; ++b)
#pragma unroll
                for (int m = 0; m < 4; ++m)
#pragma unroll
                    for (int n = 0; n < 2; ++n) acc[a][b][m][n] = (f32x4){0.f, 0.f, 0.f, 0.f};
        bf16x8 At[4][2]; s16x4 B0[2][2][2], B1[2][2][2];
        const unsigned cA = (unsigned)pm * S_T, cB = (unsigned)pn * S_T;
        if (!NOGEMM) { GATE(1); }
        STAGE(rb, SB(0, 0), cB); STAGE(rb, SB(0, 1), cB + S_H); STAGE(ra, SA(0, 0), cA); STAGE(ra, SA(0, 1), cA + S_H);
        if (wr == 1) BAR;
        WAIT_V(2); BAR;
        STAGE(rb, SB(1, 0), cB + S_K); STAGE(ra, SA(1, 0), cA + S_K); STAGE(rb, SB(1, 1), cB + S_H + S_K);
        WAIT_V(6); BAR;
        STAMP(dslot); ++dslot;
        for (int t = 0; t < (NOGEMM ? 0 : NT); t += 2) {
            const int t2 = min(t + 2, NT - 1), t3 = min(t + 3, NT - 1);
            const unsigned a1 = cA + (unsigned)(t + 1) * S_K;
            const unsigned a2 = cA + (unsigned)t2 * S_K, b2 = cB + (unsigned)t2 * S_K, a3 = cA + (unsigned)t3 * S_K, b3 = cB + (unsigned)t3 * S_K;
            LDB(B0, 0, 0); LDB(B1, 0, 1); SCHED; LDA(At, 0, 0); STAGE(ra, SA(1, 1), a1 + S_H);
            WAIT_V(8); WAIT_B(B0, B1); FLAGREAD; BAR; MMA(0, 0, At, B0); MMA(0, 1, At, B1); BAR; SCHED;
            GATE(t2);
            LDA(At, 0, 1); STAGE(rb, SB(0, 0), b2); STAGE(rb, SB(0, 1), b2 + S_H); STAGE(ra, SA(0, 0), a2);
            WAIT_VP; WAIT_L(0); BAR; MMA(1, 0, At, B0); MMA(1, 1, At, B1); BAR; SCHED;
            LDB(B0, 1, 0); LDB(B1, 1, 1); SCHED; LDA(At, 1, 0); STAGE(ra, SA(0, 1), a2 + S_H);
            WAIT_V(8); WAIT_B(B0, B1); FLAGREAD; BAR; MMA(0, 0, At, B0); MMA(0, 1, At, B1); BAR; SCHED;
            GATE(t3);
            LDA(At, 1, 1); STAGE(rb, SB(1, 0), b3); STAGE(rb, SB(1, 1), b3 + S_H); STAGE(ra, SA(1, 0), a3);
            WAIT_VP; WAIT_L(0); BAR; MMA(1, 0, At, B0); MMA(1, 1, At, B1); BAR; SCHED;
        }
        if (wr == 0) BAR;
        STAMP(dslot); ++dslot;
#define EPI_GROUP(A, pnx, g, H1, E0, E1) do { unsigned _w[4][4]; \
            _Pragma("unroll") for (int m = 0; m < 4; ++m) { \
                _w[m][0] = cvtpk(A[m][0][0], A[m][0][1]); _w[m][1] = cvtpk(A[m][0][2], A[m][0][3]); \
                _w[m][2] = cvtpk(A[m][1][0], A[m][1][1]); _w[m][3] = cvtpk(A[m][1][2], A[m][1][3]); } \
            encode_unit(_w, myws + OFF_CP + (unsigned)(((pm * 16 + (pnx)) * 8 + wid) * 4 + (g)) * UNIT_B, lane, H1, E0, E1); } while (0)
#define CH_ADDR(pnx, g) (chd + (size_t)(((((bi * 2 + (pm & 1)) * 32 + bj_ * 16 + (pnx)) * 8 + wid) * 4 + (g)) * 2 + bk) * 16)
        {
            unsigned bad, h[4][3];
            DSR32(bad, ldsa + LDS_BAD);
            asm volatile("s_waitcnt lgkmcnt(0)" : "+v"(bad) :: "memory");
            const unsigned pois = rfl(bad) ? POISON << 8 : 0;
#define EPI_HDR(g) do { if (lane == 0) gst128_sys(CH_ADDR(pn, g), (u32x4){call, pois ? pois : h[g][0], h[g][1], h[g][2]}); } while (0)
            EPI_GROUP(acc[0][0], pn, 0, h[0][0], h[0][1], h[0][2]);
            EPI_GROUP(acc[0][1], pn, 1, h[1][0], h[1][1], h[1][2]);
            WAIT_V(3); EPI_HDR(0);
            EPI_GROUP(acc[1][0], pn, 2, h[2][0], h[2][1], h[2][2]);
            EPI_GROUP(acc[1][1], pn, 3, h[3][0], h[3][1], h[3][2]);
            WAIT_V(7); EPI_HDR(1);
            WAIT_V(4); EPI_HDR(2);
            WAIT_V(2); EPI_HDR(3);
        }
        STAMP(dslot); ++dslot;
        for (int e = pn; e < 4 * PN2; e += PN1) {
            const int ai = e & 1, bj = (e >> 1) & 1, pn2 = PN1 + (e >> 2);
            if (!NOGEMM) {
                const unsigned long long t0 = __builtin_amdgcn_s_memrealtime();
                const u32x4* f = (const u32x4*)(rdyf + ((16 + pn2) * 64) * NR) + lane;
                for (;;) {
                    u32x4 v0, v1;
                    asm volatile("global_load_dwordx4 %0, %2, off sc0 sc1\n\tglobal_load_dwordx4 %1, %2, off offset:1024 sc0 sc1\n\ts_waitcnt vmcnt(0)"
                                 : "=&v"(v0), "=&v"(v1) : "v"(f) : "memory");
                    bool ok = true;
#pragma unroll
                    for (int q = 0; q < 4; ++q) ok = ok && (int)(v0[q] - call) >= 0 && (int)(v1[q] - call) >= 0;
                    if (__all(ok)) break;
                    if (timed_out(tmo, call, t0)) { *badf = 1; break; }
                    __builtin_amdgcn_s_sleep(16);
                }
            }
            BAR;
            STAMP(dslot); ++dslot;
            const unsigned gA = (unsigned)(2 * pm + ai) * S_H, gB = (unsigned)(2 * pn2 + bj) * S_H;
#pragma unroll
            for (int m = 0; m < 4; ++m)
#pragma unroll
                for (int n = 0; n < 2; ++n) acc[0][0][m][n] = (f32x4){0.f, 0.f, 0.f, 0.f};
#pragma unroll
            for (int q = 0; q < 3; ++q) { STAGE2(ra, q * HTB, gA + q * S_K); STAGE2(rb, (4 + q) * HTB, gB + q * S_K); }
            for (int t = 0; t < (NOGEMM || NOGEMM2 ? 0 : NT); t += 4) {
#pragma unroll
                for (int q = 0; q < 4; ++q) {
                    const unsigned tn = (unsigned)min(t + q + 3, NT - 1) * S_K;
                    WAIT_V(8); BAR;
                    if (G2X != 1) { STAGE2(ra, ((q + 3) & 3) * HTB, gA + tn); STAGE2(rb, (4 + ((q + 3) & 3)) * HTB, gB + tn); }
                    LDB(B0, q >> 1, q & 1); SCHED; LDA(At, q >> 1, q & 1);
                    WAIT_B1(B0);
                    if (G2X != 2) MMA(0, 0, At, B0); else __builtin_amdgcn_s_sleep(6);
                    SCHED;
                }
            }
            WAIT_V(0);
            unsigned h1, e0, e1;
            EPI_GROUP(acc[0][0], pn2, ai * 2 + bj, h1, e0, e1);
            unsigned bad;
            DSR32(bad, ldsa + LDS_BAD);
            asm volatile("s_waitcnt lgkmcnt(0)" : "+v"(bad) :: "memory");
            WAIT_V(0);
            if (rfl(bad)) h1 = POISON << 8;
            if (lane == 0) gst128_sys(CH_ADDR(pn2, ai * 2 + bj), (u32x4){call, h1, e0, e1});
            STAMP(dslot); ++dslot;
        }
#undef SA
#undef SB
    } else {
        for (int q = tid; q <= Q_END; q += NTHR) {
            u32x4 en = {0u, 0u, 0u, 0u};
            if (q < Q_OUT) {
                const int c = in_job(q), e = c >> 6, st = c & 63, mat = e >> 4, blk = e & 15;
                const int gblk = mat == 0 ? (blk >> 1) * 4 + bi * 2 + (blk & 1) : bj_ * 16 + blk;
                en = (u32x4){(unsigned)(e * 8192 + st * 16), (unsigned)(((mat * 2 + bk) * 32 + gblk) * 64 + st) * UNIT_B,
                             (unsigned)mat * STG_B + (unsigned)(2 * blk * 64 + st) * HTB, (unsigned)(c * NR * 4)};
            } else if (q < Q_END) {
                const int o = q - Q_OUT, y = out_tile(o), gg = o & 3, w8 = (o >> 2) & 7;
                en = (u32x4){(unsigned)((((y & 1) * 32 + (y >> 1)) * 32 + (o & 31)) * 32), (unsigned)(((rank * 2 + (y & 1)) * 16 + (y >> 1)) * 32 + (o & 31)) * UNIT_B,
                             (unsigned)((((y & 1) * 256 + (gg >> 1) * 128 + (w8 >> 2) * 64) * NN + (y >> 1) * 256 + (gg & 1) * 128 + (w8 & 3) * 32) * 2), 0u};
            }
            asm volatile("ds_write_b128 %0, %1" :: "v"(ldsa + LDS_TAB + (unsigned)q * 16u), "v"(en) : "memory");
        }
        const unsigned call = rfl(gld32_sys(cntp)) + 1;
        if (lane == 0) __hip_atomic_store(cntp, call, __ATOMIC_RELAXED, __HIP_MEMORY_SCOPE_AGENT);
        const unsigned par = call & 1;
        unsigned* const tkt = ctl + C_TKT / 4 + par * 1024;
        if (wg == NGEMM && wid == 7 && lane < 16) __hip_atomic_store(ctl + C_TKT / 4 + (par ^ 1) * 1024 + lane * 64, 0u, __ATOMIC_RELAXED, __HIP_MEMORY_SCOPE_SYSTEM);
        WAIT_L(0);
        BAR;
        const rsrc_t rh = mk_rsrc(myws, (unsigned)WS_NEED);
        const unsigned lw = (unsigned)wid * 16384u, lh = LDS_AUX + (unsigned)wid * 1024u;
        const unsigned vl16 = (unsigned)lane * 16u;
#define PULL_UNIT(rp, slot, so, h1) do { DMA16(rp, slot, vl16, so); DMA16(rp, (slot) + 1024, vl16, (so) + 1024); if (NL3 == 64 || lane < NL3) DMA16(rp, (slot) + 2048, vl16, (so) + 2048); \
            if (UNLIKELY((h1) >> 8 == RAWMODE)) DMA16(rp, (slot) + 3072, vl16, (so) + 3072); } while (0)
#define READ_UNITS(d, slot) do { const unsigned _a = ldsa + (slot) + vl16; \
            DSR128(d[0][0], _a, 0); DSR128(d[0][1], _a, 1024); DSR128(d[0][2], _a, 2048); DSR128(d[0][3], _a, 3072); \
            DSR128(d[1][0], _a, 4096); DSR128(d[1][1], _a, 5120); DSR128(d[1][2], _a, 6144); DSR128(d[1][3], _a, 7168); } while (0)
#define WAIT_UNITS(d, hv, en) asm volatile("s_waitcnt lgkmcnt(0)" : "+v"(d[0][0]), "+v"(d[0][1]), "+v"(d[0][2]), "+v"(d[0][3]), \
            "+v"(d[1][0]), "+v"(d[1][1]), "+v"(d[1][2]), "+v"(d[1][3]), "+v"(hv), "+v"(en) :: "memory")
        if (const int cw = (wg - NGEMM) * CW + wid; wid < CW) {
            const int si = cw % (NR - 1), sq = cw / (NR - 1), s = (rank + 1 + si) & 7;
            constexpr int w_in = W_IN;
            const int oc = sq < NCW / 8 ? (s & 3) : (rank & 3);
            const int ti = oc & 1, tj = oc >> 1;
            const rsrc_t rp = mk_rsrc(p.ws[s] + OFF_PACK + (size_t)par * PACK_B, PACK_B);
            const rsrc_t rc0 = mk_rsrc(p.ws[oc] + OFF_CP, CP_B), rc1 = mk_rsrc(p.ws[oc + 4] + OFF_CP, CP_B);
            const rsrc_t ro = mk_rsrc(p.out, (MM / NR) * NN * 2);
            const unsigned hb_in = (unsigned)OFF_IH + par * IH_B + (unsigned)s * 1024u, hb_out = (unsigned)OFF_CH + (unsigned)((ti * 64 + tj * 16) * 1024);
            const char* const stg = myws + OFF_STG + (unsigned)s * 2048u;
            const char* const rdys = myws + OFF_RDY + (unsigned)s * 4u;
            const unsigned olane = (unsigned)(((ti * 512 + fr) * NN + tj * 4096 + (fq & 1) * 16 + (fq >> 1) * 8) * 2);
            auto hdr = [&](int q, unsigned x) {
                return lane < 8 ? (q < Q_OUT ? hb_in : hb_out) + x + (unsigned)lane * 4u : (unsigned)CTRL_OFF + C_TMO;
            };
            auto entry = [&](int q) {
                u32x4 en;
                asm volatile("ds_read_b128 %0, %1\n\ts_waitcnt lgkmcnt(0)" : "=&v"(en) : "v"(ldsa + LDS_TAB + (unsigned)q * 16u) : "memory");
                return (u32x4){rfl(en[0]), rfl(en[1]), rfl(en[2]), rfl(en[3])};
            };
            auto ready = [&](unsigned hv) { return ((unsigned)__ballot((int)(hv - call) >= 0) & 0x11u) == 0x11u; };
            auto pull = [&](int q, unsigned so, unsigned hv, unsigned slot) {
                const unsigned ha = (unsigned)__builtin_amdgcn_readlane((int)hv, 1), hb = (unsigned)__builtin_amdgcn_readlane((int)hv, 5);
                if (q < Q_OUT) {
                    PULL_UNIT(rp, slot, so, ha);
                    PULL_UNIT(rp, slot + 4096, so + UNIT_B, hb);
                } else {
                    PULL_UNIT(rc0, slot, so, ha);
                    PULL_UNIT(rc1, slot + 4096, so, hb);
                }
            };
            auto publish = [&](unsigned w) {
                if (lane < 2) asm volatile("global_store_dword %0, %1, %2 sc0 sc1" :: "v"(w + (unsigned)lane * 32u), "v"(call), "s"(rdys) : "memory");
            };
            auto nan_fill = [&](unsigned z) {
#pragma unroll
                for (int m = 0; m < 4; ++m)
                    st128<0>((u32x4){0x7fc07fc0u, 0x7fc07fc0u, 0x7fc07fc0u, 0x7fc07fc0u}, ro, olane + z, m * 16 * NN * 2);
            };
            const unsigned* const tki = tkt + si * 64;
            const unsigned* const tko = tkt + (8 + oc) * 64;
            int o1;
            unsigned tk;
            TICKET(tk, tko); WAIT_V(0);
            o1 = (int)min((unsigned)Q_OUT + ticket_value(tk), (unsigned)Q_END);
            auto next = [&](int qprev, unsigned t) {
                if (qprev >= Q_OUT) return (int)min((unsigned)Q_OUT + t, (unsigned)Q_END);
                const int j = 4 * w_in + (int)t;
                return j < Q_OUT ? j : o1;
            };
            int qA = sq, qB = sq + w_in, qC = sq + 2 * w_in;
            if (sq >= w_in) {
                unsigned tb, tc;
                TICKET(tb, tko); TICKET(tc, tko); TICKET(tk, tko); WAIT_V(0);
                qA = o1; qB = next(Q_OUT, ticket_value(tb)); qC = next(Q_OUT, ticket_value(tc));
            } else {
                asm volatile("v_mov_b32 %0, %1" : "=v"(tk) : "v"((unsigned)(sq - w_in)));
            }
            u32x4 eA = entry(qA), eB = entry(qB), eC = entry(qC);
            bool dead = false;
            while (qA < Q_END) {
                unsigned hvA, hvB;
                gld32x2_sys(myws + hdr(qA, eA[0]), myws + hdr(qB, eB[0]), hvA, hvB);
                if (!ready(hvA)) {
                    const unsigned long long t0 = __builtin_amdgcn_s_memrealtime();
                    for (;;) {
                        __builtin_amdgcn_s_sleep(4);
                        gld32x2_sys(myws + hdr(qA, eA[0]), myws + hdr(qB, eB[0]), hvA, hvB);
                        if (ready(hvA)) break;
                        if (timed_out(tmo, call, t0)) { dead = true; break; }
                    }
                    if (dead) break;
                }
                unsigned sl = lw, hs = lh;
                pull(qA, eA[1], hvA, sl);
                DMA4(rh, hs, hdr(qC, eC[0]));
                int nst = 0;
                unsigned pend = 0;
                for (;;) {
                    const bool more = qB < Q_END && ready(hvB);
                    if (more) {
                        pull(qB, eB[1], hvB, sl ^ 8192);
                        if (nst == 8) WAIT_V(14); else if (nst == 4) WAIT_V(10); else WAIT_V(6);
                    } else {
                        WAIT_V(0);
                        if (nst == 8) { publish(pend); nst = 0; }
                    }
                    const int qD = next(qC, ticket_value(tk));
                    unsigned hvC;
                    u32x4 d[2][4], eDv;
                    DSR128(eDv, ldsa + LDS_TAB + (unsigned)qD * 16u, 0);
                    DSR32(hvC, ldsa + hs + lane * 4);
                    READ_UNITS(d, sl);
                    WAIT_UNITS(d, hvC, eDv);
                    const u32x4 eD = {rfl(eDv[0]), rfl(eDv[1]), rfl(eDv[2]), rfl(eDv[3])};
                    TICKET(tk, qD < Q_OUT ? tki : tko);
                    hs ^= 256;
                    if (more) DMA4(rh, hs, hdr(qD, eD[0]));
                    if (qA < Q_OUT) {
                        const char* dst = stg + eA[2];
#pragma unroll
                        for (int u = 0; u < 2; ++u) {
                            const unsigned h1 = __builtin_amdgcn_readlane((int)hvA, u * 4 + 1);
                            u32x4 o[4];
                            if (!UNLIKELY(h1 >> 8 == RAWMODE)) {
                                unsigned H[8];
                                decode_high(d[u][2], h1, __builtin_amdgcn_readlane((int)hvA, u * 4 + 2), __builtin_amdgcn_readlane((int)hvA, u * 4 + 3), lane, H);
#pragma unroll
                                for (int m = 0; m < 4; ++m) {
                                    const unsigned l0 = d[u][m >> 1][(m & 1) * 2], l1 = d[u][m >> 1][(m & 1) * 2 + 1];
                                    o[m] = (u32x4){vperm(H[2 * m], l0, 0x05010400u), vperm(H[2 * m], l0, 0x07030602u),
                                                   vperm(H[2 * m + 1], l1, 0x05010400u), vperm(H[2 * m + 1], l1, 0x07030602u)};
                                }
                            } else {
#pragma unroll
                                for (int m = 0; m < 4; ++m) o[m] = d[u][m];
                            }
                            if (u == 0 && nst == 8) {
                                asm volatile("s_waitcnt vmcnt(8)" : "+v"(o[0]), "+v"(o[1]), "+v"(o[2]), "+v"(o[3]) :: "memory");
                                publish(pend);
                            }
                            const char* d0 = dst + u * HTB;
                            gst128s_sys<0>(d0, vl16, o[0]); gst128s_sys<1024>(d0, vl16, o[1]);
                            gst128s_sys<0>(d0 + 64 * HTB, vl16, o[2]); gst128s_sys<1024>(d0 + 64 * HTB, vl16, o[3]);
                        }
                        pend = eA[3]; nst = 8;
                    } else {
                        const bool bad = (unsigned)__builtin_amdgcn_readlane((int)hvA, 8) == call
                                         || (unsigned)__builtin_amdgcn_readlane((int)hvA, 1) >> 8 == POISON
                                         || (unsigned)__builtin_amdgcn_readlane((int)hvA, 5) >> 8 == POISON;
                        if (nst == 8) { WAIT_V(8); publish(pend); }
                        if (UNLIKELY(bad)) nan_fill(eA[2]);
                        else {
                            float a[4][8];
#pragma unroll
                            for (int k = 0; k < 2; ++k) {
                                const unsigned h1 = __builtin_amdgcn_readlane((int)hvA, k * 4 + 1);
                                if (UNLIKELY(h1 >> 8 == RAWMODE)) {
#pragma unroll
                                    for (int m = 0; m < 4; ++m)
#pragma unroll
                                        for (int j = 0; j < 4; ++j) {
                                            const unsigned v = d[k][m][j];
                                            const float lo = __uint_as_float(v << 16), hi = __uint_as_float(v & 0xffff0000u);
                                            a[m][2 * j] = k == 0 ? lo : a[m][2 * j] + lo; a[m][2 * j + 1] = k == 0 ? hi : a[m][2 * j + 1] + hi;
                                        }
                                } else {
                                    unsigned H[8];
                                    decode_high(d[k][2], h1, __builtin_amdgcn_readlane((int)hvA, k * 4 + 2), __builtin_amdgcn_readlane((int)hvA, k * 4 + 3), lane, H);
#pragma unroll
                                    for (int m = 0; m < 4; ++m)
#pragma unroll
                                        for (int j = 0; j < 8; ++j) {
                                            const unsigned Lw = d[k][m >> 1][(m & 1) * 2 + (j >> 2)];
                                            const float v = __uint_as_float(vperm(H[2 * m + (j >> 2)], Lw, (4u + (j & 3)) << 24 | (unsigned)(j & 3) << 16 | 0x0c0cu));
                                            a[m][j] = k == 0 ? v : a[m][j] + v;
                                        }
                                }
                            }
#pragma unroll
                            for (int m = 0; m < 4; ++m) {
                                unsigned r0 = cvtpk(a[m][0], a[m][1]), r1 = cvtpk(a[m][2], a[m][3]), r2 = cvtpk(a[m][4], a[m][5]), r3 = cvtpk(a[m][6], a[m][7]);
                                auto s02 = __builtin_amdgcn_permlane16_swap(r0, r2, false, false);
                                auto s13 = __builtin_amdgcn_permlane16_swap(r1, r3, false, false);
                                u32x4 o; o.x = s02[0]; o.y = s13[0]; o.z = s02[1]; o.w = s13[1];
                                st128<OAUX>(o, ro, olane + eA[2], m * 16 * NN * 2);
                            }
                        }
                        STAMPJ(8 + oc, qA - Q_OUT);
                        if (qA == o1) STAMPW(32);
                        nst = 4;
                    }
                    qA = qB; qB = qC; qC = qD; eA = eB; eB = eC; eC = eD; hvA = hvB; hvB = hvC; sl ^= 8192;
                    if (!more) break;
                }
                if (nst == 8) { WAIT_V(0); publish(pend); }
            }
            if (dead) {
                while (qA < Q_END) {
                    if (qA >= Q_OUT) nan_fill(eA[2]);
                    WAIT_V(0);
                    const int qD = next(qC, ticket_value(tk));
                    TICKET(tk, qD < Q_OUT ? tki : tko);
                    qA = qB; qB = qC; qC = qD; eA = eB; eB = eC; eC = entry(qD);
                }
            }
            WAIT_V(0);
        }
        STAMPW(40);
    }
}

}

extern "C" void kernel_launch(int rank, int nranks, ncclComm_t comm,
                              hipStream_t stream,
                              void* const* d_in, const int* in_sizes, int n_in,
                              void* d_out, int out_size,
                              const int* in_dtypes, int out_dtype,
                              void* const* d_out_all,
                              void* const* d_ws_all, size_t ws_bytes) {
    (void)comm; (void)in_dtypes; (void)out_dtype; (void)d_out_all;
    if (nranks != NR || n_in < 2 || in_sizes[0] != MM * KL || in_sizes[1] != KL * NN || out_size != (MM / NR) * NN
        || ws_bytes < WS_NEED || (unsigned)rank >= (unsigned)NR) {
        fprintf(stderr, "kernel_launch: unexpected shapes\n");
        return;
    }
    static bool inited[NR] = {};
    if (!inited[rank]) {
        HIP_CHECK(hipFuncSetAttribute((const void*)gemm_rs_kernel, hipFuncAttributeMaxDynamicSharedMemorySize, LDS_BYTES));
        inited[rank] = true;
    }
    Params p;
    p.x = d_in[0]; p.w = d_in[1]; p.out = d_out; p.rank = rank;
    for (int i = 0; i < NR; ++i) p.ws[i] = (char*)d_ws_all[i];
    hipLaunchKernelGGL(gemm_rs_kernel, dim3(NWG), dim3(NTHR), LDS_BYTES, stream, p);
    HIP_CHECK(hipGetLastError());
}
```

```cpp
#include <hip/hip_runtime.h>
#include <rccl/rccl.h>
#include <cstdio>
#include <cstdint>

#define HIP_CHECK(expr) do { hipError_t _e = (expr); if (_e != hipSuccess) { \
    fprintf(stderr, "HIP error %s at %s:%d\n", hipGetErrorString(_e), __FILE__, __LINE__); \
    return; } } while (0)

#define LAS __attribute__((address_space(3)))
#define UNLIKELY(x) __builtin_expect(!!(x), 0)
typedef short bf16x8 __attribute__((ext_vector_type(8)));
typedef short s16x4 __attribute__((ext_vector_type(4)));
typedef float f32x4 __attribute__((ext_vector_type(4)));
typedef float f32x2 __attribute__((ext_vector_type(2)));
typedef unsigned u32x4 __attribute__((ext_vector_type(4)));
typedef unsigned short u16x2 __attribute__((ext_vector_type(2)));
typedef __bf16 bf16x2_t __attribute__((ext_vector_type(2)));
typedef __amdgpu_buffer_rsrc_t rsrc_t;

namespace {

constexpr int NR = 8;
constexpr int MM = 8192, NN = 8192, KL = 1024;
constexpr int NWG = 256, NTHR = 512;
#define PN1 13
#define LOOK 2
#define CW 2
#define PKSLEEP 0
#define FORCE_RAW 0
#define NOGEMM 0
#define GAUX 17
#define GAUX2 GAUX
#define G2X 0
#define NOGEMM2 0
#define OAUX 0
constexpr int NGEMM = 16 * PN1, NCOMM = NWG - NGEMM, NCW = NCOMM * CW;
constexpr int PN2 = 16 - PN1;
constexpr int HTB = 16384, STAGE_BYTES = 8 * HTB;
constexpr int NT = 64;
constexpr unsigned UNIT_B = 4096;
constexpr size_t MIB = 1u << 20;
constexpr size_t OFF_PACK = 0;
constexpr unsigned PACK_B = 32u << 20;
constexpr size_t OFF_STG = 64 * MIB;
constexpr unsigned STG_B = 32u << 20;
constexpr size_t OFF_CP = 128 * MIB;
constexpr unsigned CP_B = 32u << 20;
constexpr size_t OFF_IH = 160 * MIB;
constexpr unsigned IH_B = 2 * 16 * 8 * 64 * 16;
constexpr size_t OFF_RDY = 161 * MIB;
constexpr size_t OFF_CH = 162 * MIB;
constexpr size_t CTRL_OFF = 256 * MIB;
constexpr size_t WS_NEED = CTRL_OFF + MIB;
constexpr unsigned C_TMO = 256, C_CNT = 4096;
constexpr unsigned C_TKT = 16384;
constexpr unsigned MAXEXC = 2, RAWMODE = 255, POISON = 254;
constexpr unsigned long long DEADLINE = 30000000ull;
constexpr int LDS_AUX = STAGE_BYTES;
constexpr int LDS_BAD = STAGE_BYTES + 8192;
constexpr int LDS_BYTES = LDS_BAD + 256;
constexpr int AUX_SYS = 17;
constexpr int E1 = 16 + PN1;
constexpr int NJ1 = (NT / 2) * E1, NJ_IN = (NT / 2) * 32;
#define NFB 4
#define SKD 2
constexpr int EF = 8 + NFB, ES = E1 - EF;
constexpr int Q_OUT = NJ_IN, Q_END = Q_OUT + 1024;
constexpr int LDS_TAB = 65536;
__host__ __device__ constexpr int in_job(int j) {
    int i = 0, r = 0;
    if (j < SKD * EF) { i = j % EF; r = j / EF; }
    else if (j < SKD * EF + (NT / 2 - SKD) * E1) { const int q = j - SKD * EF; i = q % E1; r = SKD + q / E1 - (i < EF ? 0 : SKD); }
    else if (j < NJ1) { const int q = j - (SKD * EF + (NT / 2 - SKD) * E1); i = EF + q % ES; r = NT / 2 - SKD + q / ES; }
    else { const int q = j - NJ1; i = E1 + q % PN2; r = q / PN2; }
    const int e = i < 8 ? 2 * i : i < EF ? 8 + i : i < 8 + EF ? 2 * (i - EF) + 1 : i;
    return e * 64 + 2 * r;
}
__host__ __device__ constexpr int out_tile(int o) { const int y = o >> 5; return y < NFB ? 2 * y : y < 2 * NFB ? 2 * (y - NFB) + 1 : y; }
#define W_IN (NCW / (NR - 1) - 1)

#define STAMP(slot) do {} while (0)
#define STAMPW(slot) do {} while (0)
#define STAMPJ(row, j) do {} while (0)
#define DIAG_BLK0 do {} while (0)
#define DIAG_BLK1(t0) do {} while (0)
struct Params {
    const void* x; const void* w; void* out;
    char* ws[NR];
    int rank;
};

__device__ __forceinline__ unsigned cvtpk(float lo, float hi) { f32x2 v = {lo, hi}; bf16x2_t b = __builtin_convertvector(v, bf16x2_t); return __builtin_bit_cast(unsigned, b); }
__device__ __forceinline__ rsrc_t mk_rsrc(const void* p, unsigned bytes) { return __builtin_amdgcn_make_buffer_rsrc((void*)p, 0, bytes, 0x00020000); }

template <int AUX> __device__ __forceinline__ void st128(u32x4 v, rsrc_t r, unsigned voff, unsigned soff) {
    __builtin_amdgcn_raw_buffer_store_b128(v, r, voff, soff, AUX);
    asm volatile("s_nop 1" :: "v"(v) : "memory");
}
__device__ __forceinline__ void gst128_sys(void* ptr, u32x4 v) {
    asm volatile("global_store_dwordx4 %0, %1, off sc0 sc1\n\ts_nop 1" :: "v"(ptr), "v"(v) : "memory");
}
__device__ __forceinline__ unsigned gld32_sys(const void* ptr) {
    unsigned v;
    asm volatile("global_load_dword %0, %1, off sc0 sc1\n\ts_waitcnt vmcnt(0)" : "=&v"(v) : "v"(ptr) : "memory");
    return v;
}
__device__ __forceinline__ void gld32x2_sys(const void* pa, const void* pb, unsigned& a, unsigned& b) {
    asm volatile("global_load_dword %0, %2, off sc0 sc1\n\tglobal_load_dword %1, %3, off sc0 sc1\n\ts_waitcnt vmcnt(0)"
                 : "=&v"(a), "=&v"(b) : "v"(pa), "v"(pb) : "memory");
}
template <int OFF> __device__ __forceinline__ void gst128s_sys(const void* base, unsigned voff, u32x4 v) {
    asm volatile("global_store_dwordx4 %0, %1, %2 offset:%c3 sc0 sc1\n\ts_nop 1" :: "v"(voff), "v"(v), "s"(base), "i"(OFF) : "memory");
}
__device__ __forceinline__ unsigned bfi(unsigned m, unsigned a, unsigned b) { return (a & m) | (b & ~m); }
__device__ __forceinline__ unsigned vperm(unsigned hi, unsigned lo, unsigned sel) { return __builtin_amdgcn_perm(hi, lo, sel); }
__device__ __forceinline__ unsigned rfl(unsigned v) { return __builtin_amdgcn_readfirstlane(v); }
__device__ __forceinline__ unsigned wave_max_u(int x) {
    x = max(x, __builtin_amdgcn_update_dpp(0, x, 0xB1, 0xf, 0xf, true));
    x = max(x, __builtin_amdgcn_update_dpp(0, x, 0x4E, 0xf, 0xf, true));
    x = max(x, __builtin_amdgcn_update_dpp(0, x, 0x141, 0xf, 0xf, true));
    x = max(x, __builtin_amdgcn_update_dpp(0, x, 0x140, 0xf, 0xf, true));
    const int a = __builtin_amdgcn_readlane(x, 0), b = __builtin_amdgcn_readlane(x, 16), c = __builtin_amdgcn_readlane(x, 32), d = __builtin_amdgcn_readlane(x, 48);
    return (unsigned)max(max(a, b), max(c, d));
}
__device__ __forceinline__ bool timed_out(unsigned* tmo, unsigned call, unsigned long long t0) {
    if (__hip_atomic_load(tmo, __ATOMIC_RELAXED, __HIP_MEMORY_SCOPE_AGENT) == call) return true;
    if (__builtin_amdgcn_s_memrealtime() - t0 > DEADLINE) { __hip_atomic_store(tmo, call, __ATOMIC_RELAXED, __HIP_MEMORY_SCOPE_AGENT); return true; }
    return false;
}

__device__ __forceinline__ void encode_unit(const unsigned (&w)[4][4], const char* dst, int lane, unsigned& h1, unsigned& e0, unsigned& e1) {
    u16x2 mx = {0, 0};
#pragma unroll
    for (int m = 0; m < 4; ++m)
#pragma unroll
        for (int q = 0; q < 4; ++q) mx = __builtin_elementwise_max(mx, __builtin_bit_cast(u16x2, w[m][q] & 0x7f007f00u));
    const unsigned base = wave_max_u((int)max(mx[0], mx[1])) >> 8;
    const unsigned bq = base * 0x01010101u | 0x80808080u;
    unsigned L[8], P[4], ex = 0;
#pragma unroll
    for (int m = 0; m < 4; ++m) {
        L[2 * m] = vperm(w[m][1], w[m][0], 0x06040200u); L[2 * m + 1] = vperm(w[m][3], w[m][2], 0x06040200u);
        const unsigned h01 = vperm(w[m][1], w[m][0], 0x07050301u), h23 = vperm(w[m][3], w[m][2], 0x07050301u);
        const unsigned d01 = bq - (h01 & 0x7f7f7f7fu), d23 = bq - (h23 & 0x7f7f7f7fu);
        ex |= d01 | d23;
        P[m] = bfi(0x0f0f0f0fu, bfi(0x07070707u, d01, h01 >> 4), bfi(0x70707070u, d23 << 4, h23));
    }
    unsigned cnt = FORCE_RAW ? RAWMODE : 0;
    e0 = 0; e1 = 0;
#define NL3 64
#define NOEXC 0
    unsigned long long bal = NL3 == 64 && !NOEXC ? __ballot((ex & 0x78787878u) != 0) : 0;
    if (UNLIKELY(bal != 0)) while (bal && cnt <= MAXEXC) {
        const unsigned l = (unsigned)__builtin_ctzll(bal);
        bal &= bal - 1;
#pragma unroll
        for (int i = 0; i < 8; ++i) {
            const unsigned hb = (unsigned)__builtin_amdgcn_readlane((int)vperm(w[i >> 1][(i & 1) * 2 + 1], w[i >> 1][(i & 1) * 2], 0x07050301u), (int)l);
            unsigned f = (bq - (hb & 0x7f7f7f7fu)) & 0x78787878u;
            while (f) {
                const unsigned sh = (unsigned)__builtin_ctz(f) & 24u;
                const unsigned en = (unsigned)((i >> 1) << 17 | (i & 1) << 10) | sh << 5 | l << 11 | (hb >> sh & 0xffu);
                if (cnt == 0) e0 = en; else e1 = en;
                ++cnt; f &= ~(0xffu << sh);
            }
        }
    }
#define NORAW 0
    if (cnt > MAXEXC) cnt = NORAW ? MAXEXC : RAWMODE;
    const unsigned vo = (unsigned)lane * 16u;
    if (UNLIKELY(cnt == RAWMODE)) {
        gst128s_sys<0>(dst, vo, (u32x4){w[0][0], w[0][1], w[0][2], w[0][3]});
        gst128s_sys<1024>(dst, vo, (u32x4){w[1][0], w[1][1], w[1][2], w[1][3]});
        gst128s_sys<2048>(dst, vo, (u32x4){w[2][0], w[2][1], w[2][2], w[2][3]});
        gst128s_sys<3072>(dst, vo, (u32x4){w[3][0], w[3][1], w[3][2], w[3][3]});
    } else {
        gst128s_sys<0>(dst, vo, (u32x4){L[0], L[1], L[2], L[3]});
        gst128s_sys<1024>(dst, vo, (u32x4){L[4], L[5], L[6], L[7]});
        gst128s_sys<2048>(dst, vo, (u32x4){P[0], P[1], P[2], P[3]});
    }
    h1 = base | cnt << 8;
}
__device__ __forceinline__ void patch_high(unsigned (&H)[8], unsigned en, int lane) {
    const unsigned sh = en >> 5 & 24u, val = (en & 0xffu) << sh;
    const unsigned mk = (unsigned)lane == (en >> 11 & 63) ? 0xffu << sh : 0u;
    asm volatile("s_bitcmp1_b32 %10, 18\n\ts_cbranch_scc1 4f\n\ts_bitcmp1_b32 %10, 17\n\ts_cbranch_scc1 2f\n\ts_bitcmp1_b32 %10, 10\n\ts_cbranch_scc1 1f\n\t"
                 "v_bfi_b32 %0, %8, %9, %0\n\ts_branch 8f\n"
                 "1:\n\tv_bfi_b32 %1, %8, %9, %1\n\ts_branch 8f\n"
                 "2:\n\ts_bitcmp1_b32 %10, 10\n\ts_cbranch_scc1 3f\n\tv_bfi_b32 %2, %8, %9, %2\n\ts_branch 8f\n"
                 "3:\n\tv_bfi_b32 %3, %8, %9, %3\n\ts_branch 8f\n"
                 "4:\n\ts_bitcmp1_b32 %10, 17\n\ts_cbranch_scc1 6f\n\ts_bitcmp1_b32 %10, 10\n\ts_cbranch_scc1 5f\n\tv_bfi_b32 %4, %8, %9, %4\n\ts_branch 8f\n"
                 "5:\n\tv_bfi_b32 %5, %8, %9, %5\n\ts_branch 8f\n"
                 "6:\n\ts_bitcmp1_b32 %10, 10\n\ts_cbranch_scc1 7f\n\tv_bfi_b32 %6, %8, %9, %6\n\ts_branch 8f\n"
                 "7:\n\tv_bfi_b32 %7, %8, %9, %7\n"
                 "8:"
                 : "+v"(H[0]), "+v"(H[1]), "+v"(H[2]), "+v"(H[3]), "+v"(H[4]), "+v"(H[5]), "+v"(H[6]), "+v"(H[7]) : "v"(mk), "s"(val), "s"(en) : "scc");
}
__device__ __forceinline__ void decode_high(u32x4 P, unsigned h1, unsigned e0, unsigned e1, int lane, unsigned (&H)[8]) {
    const unsigned bq = (h1 & 0x7fu) * 0x01010101u | 0x80808080u;
#pragma unroll
    for (int m = 0; m < 4; ++m) {
        H[2 * m] = bfi(0x7f7f7f7fu, bq - (P[m] & 0x07070707u), P[m] << 4);
        H[2 * m + 1] = bfi(0x7f7f7f7fu, bq - ((P[m] >> 4) & 0x07070707u), P[m]);
    }
    const unsigned cnt = h1 >> 8;
#define NODEC 0
    if (!NODEC && UNLIKELY(cnt != 0)) {
        patch_high(H, e0, lane);
        if (cnt > 1) patch_high(H, e1, lane);
    }
}

#define TRD(dst, addr, off) asm volatile("ds_read_b64_tr_b16 %0, %1 offset:%c2" : "=&v"(dst) : "v"(addr), "i"(off) : "memory")
#define PKB(x) (bf16x8){(x)[0][0], (x)[0][1], (x)[0][2], (x)[0][3], (x)[1][0], (x)[1][1], (x)[1][2], (x)[1][3]}
#define WAIT_V(n) asm volatile("s_waitcnt vmcnt(" #n ")" ::: "memory")
#define WAIT_L(n) asm volatile("s_waitcnt lgkmcnt(" #n ")" ::: "memory")
#define BAR __builtin_amdgcn_s_barrier()
#define SCHED __builtin_amdgcn_sched_barrier(0)
#define DSR128(dst, addr, off) asm volatile("ds_read_b128 %0, %1 offset:%c2" : "=&v"(dst) : "v"(addr), "i"(off) : "memory")
#define DSR32(dst, addr) asm volatile("ds_read_b32 %0, %1" : "=&v"(dst) : "v"(addr) : "memory")
#define DMA16(rs, ldsoff, voff, soff) __builtin_amdgcn_raw_ptr_buffer_load_lds(rs, (LAS void*)(lds + (ldsoff)), 16, voff, soff, 0, AUX_SYS)
#define DMA4(rs, ldsoff, voff) __builtin_amdgcn_raw_ptr_buffer_load_lds(rs, (LAS void*)(lds + (ldsoff)), 4, voff, 0, 0, AUX_SYS)
#define TICKET(tk, ptr) asm volatile("s_mov_b64 exec, 1\n\tglobal_atomic_add %0, %1, %2, %3 sc0 sc1\n\ts_mov_b64 exec, -1" : "=&v"(tk) : "v"(0u), "v"(1u), "s"(ptr) : "memory")
__device__ __forceinline__ unsigned ticket_value(unsigned tk) {
    unsigned r;
    asm volatile("v_readfirstlane_b32 %0, %1" : "=s"(r) : "v"(tk) : "memory");
    return r;
}

__global__ __launch_bounds__(NTHR, 2) void gemm_rs_kernel(Params p) {
    extern __shared__ __attribute__((aligned(16))) unsigned char smem[];
    LAS unsigned char* lds = (LAS unsigned char*)smem;
    const int tid = threadIdx.x, wid = __builtin_amdgcn_readfirstlane(tid >> 6), lane = tid & 63;
    const int wr = wid >> 2, wc = wid & 3, fr = lane & 15, fq = lane >> 4;
    const int wg = blockIdx.x, rank = p.rank;
    const int bi = rank & 1, bj_ = (rank >> 1) & 1, bk = rank >> 2;
    char* myws = p.ws[rank];
    unsigned* ctl = (unsigned*)(myws + CTRL_OFF);
    unsigned* tmo = ctl + C_TMO / 4;

    int dslot = 0;
    STAMP(dslot); ++dslot;
    const unsigned ldsa = (unsigned)(size_t)lds;
    unsigned* cntp = ctl + C_CNT / 4 + wg * 8 + wid;

    if (wg < NGEMM) {
        const rsrc_t rx = mk_rsrc(p.x, MM * KL * 2), rw = mk_rsrc(p.w, KL * NN * 2);
        const unsigned vx = (unsigned)((tid >> 3) * (KL * 2) + (((tid & 7) ^ ((tid >> 4) & 7)) << 4));
        const unsigned vw = (unsigned)(fq * (NN * 2) + ((fr ^ ((fq << 2) | ((rank & 1) << 1))) << 4));
        auto pk_issue = [&](int idx, int buf) {
            const int g8 = idx >> 7, mat = (idx >> 6) & 1, kk = (idx >> 5) & 1, blk = idx & 31;
            if (mat == 0) {
#pragma unroll
                for (int i = 0; i < 4; ++i)
                    __builtin_amdgcn_raw_ptr_buffer_load_lds(rx, (LAS void*)(lds + buf + i * 8192 + wid * 1024), 16, vx,
                                                             (unsigned)((blk * 256 + i * 64) * (KL * 2) + kk * 1024 + g8 * 128), 0, 0);
            } else {
#pragma unroll
                for (int c = 0; c < 4; ++c)
                    __builtin_amdgcn_raw_ptr_buffer_load_lds(rw, (LAS void*)(lds + buf + wid * 4096 + c * 1024), 16, vw,
                                                             (unsigned)((kk * 512 + (g8 * 8 + wid) * 8 + (c & 1) * 4) * (NN * 2) + blk * 512 + (c >> 1) * 256), 0, 0);
            }
        };
        pk_issue(wg, 0);
        volatile LAS unsigned* const badf = (volatile LAS unsigned*)(lds + LDS_BAD);
        if (tid == 0) asm volatile("ds_write_b32 %0, %1" :: "v"(ldsa + LDS_BAD), "v"(0u) : "memory");
        const unsigned call = rfl(gld32_sys(cntp)) + 1;
        if (lane == 0) __hip_atomic_store(cntp, call, __ATOMIC_RELAXED, __HIP_MEMORY_SCOPE_AGENT);
        const unsigned par = call & 1;
        unsigned* rdyw = (unsigned*)(myws + OFF_RDY);
        {
            const char* const pk = myws + OFF_PACK + (size_t)par * PACK_B;
            const rsrc_t rs = mk_rsrc(myws + OFF_STG, 2 * STG_B);
            int buf = 0;
            for (int idx = wg; idx < 1024; idx += NGEMM, buf ^= 32768) {
                const int g8 = idx >> 7, mat = (idx >> 6) & 1, kk = (idx >> 5) & 1, blk = idx & 31, t = g8 * 8 + wid;
                BAR;
                if (idx + NGEMM < 1024) pk_issue(idx + NGEMM, buf ^ 32768);
                u32x4 v[4];
                if (mat == 0) {
                    const unsigned a = ldsa + buf + lane * 128 + ((wid ^ ((lane >> 1) & 7)) << 4);
                    DSR128(v[0], a, 0); DSR128(v[1], a, 8192); DSR128(v[2], a, 16384); DSR128(v[3], a, 24576);
                } else {
                    const unsigned a = ldsa + buf + wid * 4096 + lane * 16;
                    DSR128(v[0], a, 0); DSR128(v[1], a, 1024); DSR128(v[2], a, 2048); DSR128(v[3], a, 3072);
                }
                asm volatile("s_waitcnt lgkmcnt(0)" : "+v"(v[0]), "+v"(v[1]), "+v"(v[2]), "+v"(v[3]) :: "memory");
                unsigned w[4][4];
#pragma unroll
                for (int c = 0; c < 4; ++c) { w[c][0] = v[c][0]; w[c][1] = v[c][1]; w[c][2] = v[c][2]; w[c][3] = v[c][3]; }
                unsigned h1, e0, e1;
                encode_unit(w, pk + (unsigned)(((mat * 2 + kk) * 32 + blk) * 64 + t) * UNIT_B, lane, h1, e0, e1);
                const int cb = mat == 0 ? ((blk >> 2) * 2 + (blk & 1)) : (blk & 15);
                const int c0 = mat == 0 ? ((blk >> 1) & 1) + 4 * kk : 2 * (blk >> 4) + 4 * kk, c1 = c0 + (mat == 0 ? 2 : 1);
                const bool own = c0 == rank || c1 == rank;
                if (own) {
                    const unsigned dst = (unsigned)mat * STG_B + (unsigned)(2 * cb * 64 + t) * HTB + (unsigned)rank * 2048u;
#pragma unroll
                    for (int m = 0; m < 4; ++m) st128<AUX_SYS>(v[m], rs, lane * 16, dst + (m >> 1) * (64 * HTB) + (m & 1) * 1024);
                }
                char* const w0 = p.ws[c0 & 7];
                char* const w1 = p.ws[c1 & 7];
                WAIT_V(0);
                if (lane < 2) {
                    const int c = lane == 0 ? c0 : c1;
                    if (c != rank) gst128_sys((lane == 0 ? w0 : w1) + OFF_IH + (size_t)par * IH_B + (size_t)((((mat * 16 + cb) * NR + rank) * 64 + t) * 16), (u32x4){call, h1, e0, e1});
                    else __hip_atomic_store(rdyw + ((mat * 16 + cb) * 64 + t) * NR + rank, call, __ATOMIC_RELAXED, __HIP_MEMORY_SCOPE_SYSTEM);
                }
            }
            BAR;
        }
        STAMP(dslot); ++dslot;
        const int pm = 2 * (wg & 7) + ((wg >> 3) & 1), pn = wg >> 4;
        const rsrc_t ra = mk_rsrc(myws + OFF_STG, STG_B), rb = mk_rsrc(myws + OFF_STG + STG_B, STG_B);
        const unsigned voff = (unsigned)(wid * 1024 + lane * 16);
        constexpr unsigned S_K = HTB, S_H = NT * HTB, S_T = 2 * NT * HTB;
        const unsigned ldsw = (unsigned)wid * 1024u;
        const int aoff = fq * 2048 + (wr * 64 + fr) * 16;
        unsigned boff[2];
        { const int q = (lane & 15) >> 2, pp = lane & 3, g = fq;
#pragma unroll
          for (int n = 0; n < 2; ++n) boff[n] = (unsigned)(size_t)lds + 4 * HTB + 256 * (8 * g + q) + 16 * (((wc ^ q) << 2) | ((n ^ (g & 1)) << 1) | (pp >> 1)) + 8 * (pp & 1); }
        const unsigned* rdyf = (const unsigned*)(myws + OFF_RDY);
#define SA(b, h) (((b) * 2 + (h)) * HTB)
#define SB(b, h) ((4 + (b) * 2 + (h)) * HTB)
#define STAGEX(rs, bufoff, soff, aux) do { _Pragma("unroll") for (int _i = 0; _i < 2; ++_i) \
        __builtin_amdgcn_raw_ptr_buffer_load_lds(rs, (LAS void*)(lds + (bufoff) + ldsw + _i * 8192), 16, voff, (soff) + _i * 8192, 0, aux); } while (0)
#define STAGE(rs, bufoff, soff) STAGEX(rs, bufoff, soff, GAUX)
#define STAGE2(rs, bufoff, soff) STAGEX(rs, bufoff, soff, GAUX2)
#define LDA(dst, b, h) do { _Pragma("unroll") for (int m = 0; m < 4; ++m) _Pragma("unroll") for (int k = 0; k < 2; ++k) dst[m][k] = *(const LAS bf16x8*)(lds + SA(b, h) + aoff + m * 256 + k * 8192); } while (0)
#define LDB(dst, b, h) do { _Pragma("unroll") for (int n = 0; n < 2; ++n) _Pragma("unroll") for (int k = 0; k < 2; ++k) { \
        TRD(dst[n][k][0], boff[n], ((b) * 2 + (h)) * HTB + k * 8192); TRD(dst[n][k][1], boff[n], ((b) * 2 + (h)) * HTB + k * 8192 + 1024); } } while (0)
#define MMA(ai, bj, At, Bt) do { __builtin_amdgcn_s_setprio(1); _Pragma("unroll") for (int m = 0; m < 4; ++m) _Pragma("unroll") for (int n = 0; n < 2; ++n) _Pragma("unroll") for (int k = 0; k < 2; ++k) \
        acc[ai][bj][m][n] = __builtin_amdgcn_mfma_f32_16x16x32_bf16(PKB(Bt[n][k]), At[m][k], acc[ai][bj][m][n], 0, 0, 0); __builtin_amdgcn_s_setprio(0); } while (0)
#define WAIT_B(X, Y) asm volatile("s_waitcnt lgkmcnt(0)" : "+v"(X[0][0][0]), "+v"(X[0][0][1]), "+v"(X[0][1][0]), "+v"(X[0][1][1]), "+v"(X[1][0][0]), "+v"(X[1][0][1]), "+v"(X[1][1][0]), "+v"(X[1][1][1]), \
        "+v"(Y[0][0][0]), "+v"(Y[0][0][1]), "+v"(Y[0][1][0]), "+v"(Y[0][1][1]), "+v"(Y[1][0][0]), "+v"(Y[1][0][1]), "+v"(Y[1][1][0]), "+v"(Y[1][1][1]) :: "memory")
#define WAIT_B1(X) asm volatile("s_waitcnt lgkmcnt(0)" : "+v"(X[0][0][0]), "+v"(X[0][0][1]), "+v"(X[0][1][0]), "+v"(X[0][1][1]), "+v"(X[1][0][0]), "+v"(X[1][0][1]), "+v"(X[1][1][0]), "+v"(X[1][1][1]) :: "memory")
        const rsrc_t rr = mk_rsrc(myws + OFF_RDY, 2 * 16 * 64 * 32);
        const unsigned pvo = (unsigned)(((lane >> 5) * 16 + (lane < 32 ? pm : pn)) * 2048 + (lane & 1) * 16);
        const unsigned pla = ldsa + LDS_AUX + wid * 1024 + lane * 16;
#define NOGATE 0
        int rdy = NOGATE ? NT : 0, pbase = 0;
        bool pol = false;
        u32x4 fv;
#define COUNT_RDY(v) do { const bool _ok = (int)((v)[0] - call) >= 0 && (int)((v)[1] - call) >= 0 && (int)((v)[2] - call) >= 0 && (int)((v)[3] - call) >= 0; \
            const unsigned long long _b = __ballot(_ok); \
            unsigned _m = (unsigned)_b & (unsigned)(_b >> 32); _m = ~(_m & (_m >> 1)) & 0x55555555u; \
            rdy = min(pbase + (_m ? (int)(__builtin_ctz(_m) >> 1) : 16), NT); } while (0)
#define FLAGREAD do { if (pol) DSR128(fv, pla, 0); } while (0)
#define GATE(tt) do { \
            if (pol) { asm volatile("s_waitcnt lgkmcnt(0)" : "+v"(fv) :: "memory"); COUNT_RDY(fv); } \
            if (UNLIKELY((int)(tt) >= rdy)) { \
                const unsigned long long _t0 = __builtin_amdgcn_s_memrealtime(); \
                DIAG_BLK0; \
                for (;;) { \
                    pbase = rdy; \
                    const char* _p = (const char*)rdyf + pvo + min(rdy + ((lane >> 1) & 15), NT - 1) * 32; \
                    u32x4 _v; \
                    asm volatile("global_load_dwordx4 %0, %1, off sc0 sc1\n\ts_waitcnt vmcnt(0)" : "=&v"(_v) : "v"(_p) : "memory"); \
                    COUNT_RDY(_v); \
                    if ((int)(tt) < rdy) break; \
                    if (timed_out(tmo, call, _t0)) { *badf = 1; rdy = NT; break; } \
                    __builtin_amdgcn_s_sleep(8); \
                } DIAG_BLK1(_t0); } \
            pol = rdy < min((int)(tt) + LOOK, NT); \
            if (pol) { pbase = rdy; DMA16(rr, LDS_AUX + wid * 1024, pvo + min(rdy + ((lane >> 1) & 15), NT - 1) * 32, 0); } } while (0)
#define WAIT_VP do { if (pol) WAIT_V(9); else WAIT_V(8); } while (0)
        const int owner = pm >> 1;
        char* chd = p.ws[owner] + OFF_CH;
        f32x4 acc[2][2][4][2];
#pragma unroll
        for (int a = 0; a < 2; ++a)
#pragma unroll
            for (int b = 0; b < 2; ++b)
#pragma unroll
                for (int m = 0; m < 4; ++m)
#pragma unroll
                    for (int n = 0; n < 2; ++n) acc[a][b][m][n] = (f32x4){0.f, 0.f, 0.f, 0.f};
        bf16x8 At[4][2]; s16x4 B0[2][2][2], B1[2][2][2];
        const unsigned cA = (unsigned)pm * S_T, cB = (unsigned)pn * S_T;
        if (!NOGEMM) { GATE(1); }
        STAGE(rb, SB(0, 0), cB); STAGE(rb, SB(0, 1), cB + S_H); STAGE(ra, SA(0, 0), cA); STAGE(ra, SA(0, 1), cA + S_H);
        if (wr == 1) BAR;
        WAIT_V(2); BAR;
        STAGE(rb, SB(1, 0), cB + S_K); STAGE(ra, SA(1, 0), cA + S_K); STAGE(rb, SB(1, 1), cB + S_H + S_K);
        WAIT_V(6); BAR;
        STAMP(dslot); ++dslot;
        for (int t = 0; t < (NOGEMM ? 0 : NT); t += 2) {
            const int t2 = min(t + 2, NT - 1), t3 = min(t + 3, NT - 1);
            const unsigned a1 = cA + (unsigned)(t + 1) * S_K;
            const unsigned a2 = cA + (unsigned)t2 * S_K, b2 = cB + (unsigned)t2 * S_K, a3 = cA + (unsigned)t3 * S_K, b3 = cB + (unsigned)t3 * S_K;
            LDB(B0, 0, 0); LDB(B1, 0, 1); SCHED; LDA(At, 0, 0); STAGE(ra, SA(1, 1), a1 + S_H);
            WAIT_V(8); WAIT_B(B0, B1); FLAGREAD; BAR; MMA(0, 0, At, B0); MMA(0, 1, At, B1); BAR; SCHED;
            GATE(t2);
            LDA(At, 0, 1); STAGE(rb, SB(0, 0), b2); STAGE(rb, SB(0, 1), b2 + S_H); STAGE(ra, SA(0, 0), a2);
            WAIT_VP; WAIT_L(0); BAR; MMA(1, 0, At, B0); MMA(1, 1, At, B1); BAR; SCHED;
            LDB(B0, 1, 0); LDB(B1, 1, 1); SCHED; LDA(At, 1, 0); STAGE(ra, SA(0, 1), a2 + S_H);
            WAIT_V(8); WAIT_B(B0, B1); FLAGREAD; BAR; MMA(0, 0, At, B0); MMA(0, 1, At, B1); BAR; SCHED;
            GATE(t3);
            LDA(At, 1, 1); STAGE(rb, SB(1, 0), b3); STAGE(rb, SB(1, 1), b3 + S_H); STAGE(ra, SA(1, 0), a3);
            WAIT_VP; WAIT_L(0); BAR; MMA(1, 0, At, B0); MMA(1, 1, At, B1); BAR; SCHED;
        }
        if (wr == 0) BAR;
        STAMP(dslot); ++dslot;
#define EPI_GROUP(A, pnx, g, H1, E0, E1) do { unsigned _w[4][4]; \
            _Pragma("unroll") for (int m = 0; m < 4; ++m) { \
                _w[m][0] = cvtpk(A[m][0][0], A[m][0][1]); _w[m][1] = cvtpk(A[m][0][2], A[m][0][3]); \
                _w[m][2] = cvtpk(A[m][1][0], A[m][1][1]); _w[m][3] = cvtpk(A[m][1][2], A[m][1][3]); } \
            encode_unit(_w, myws + OFF_CP + (unsigned)(((pm * 16 + (pnx)) * 8 + wid) * 4 + (g)) * UNIT_B, lane, H1, E0, E1); } while (0)
#define CH_ADDR(pnx, g) (chd + (size_t)(((((bi * 2 + (pm & 1)) * 32 + bj_ * 16 + (pnx)) * 8 + wid) * 4 + (g)) * 2 + bk) * 16)
        {
            unsigned bad, h[4][3];
            DSR32(bad, ldsa + LDS_BAD);
            asm volatile("s_waitcnt lgkmcnt(0)" : "+v"(bad) :: "memory");
            const unsigned pois = rfl(bad) ? POISON << 8 : 0;
#define EPI_HDR(g) do { if (lane == 0) gst128_sys(CH_ADDR(pn, g), (u32x4){call, pois ? pois : h[g][0], h[g][1], h[g][2]}); } while (0)
            EPI_GROUP(acc[0][0], pn, 0, h[0][0], h[0][1], h[0][2]);
            EPI_GROUP(acc[0][1], pn, 1, h[1][0], h[1][1], h[1][2]);
            WAIT_V(3); EPI_HDR(0);
            EPI_GROUP(acc[1][0], pn, 2, h[2][0], h[2][1], h[2][2]);
            EPI_GROUP(acc[1][1], pn, 3, h[3][0], h[3][1], h[3][2]);
            WAIT_V(7); EPI_HDR(1);
            WAIT_V(4); EPI_HDR(2);
            WAIT_V(2); EPI_HDR(3);
        }
        STAMP(dslot); ++dslot;
        for (int e = pn; e < 4 * PN2; e += PN1) {
            const int ai = e & 1, bj = (e >> 1) & 1, pn2 = PN1 + (e >> 2);
            if (!NOGEMM && !NOGATE) {
                const unsigned long long t0 = __builtin_amdgcn_s_memrealtime();
                const u32x4* f = (const u32x4*)(rdyf + ((16 + pn2) * 64) * NR) + lane;
                for (;;) {
                    u32x4 v0, v1;
                    asm volatile("global_load_dwordx4 %0, %2, off sc0 sc1\n\tglobal_load_dwordx4 %1, %2, off offset:1024 sc0 sc1\n\ts_waitcnt vmcnt(0)"
                                 : "=&v"(v0), "=&v"(v1) : "v"(f) : "memory");
                    bool ok = true;
#pragma unroll
                    for (int q = 0; q < 4; ++q) ok = ok && (int)(v0[q] - call) >= 0 && (int)(v1[q] - call) >= 0;
                    if (__all(ok)) break;
                    if (timed_out(tmo, call, t0)) { *badf = 1; break; }
                    __builtin_amdgcn_s_sleep(16);
                }
            }
            BAR;
            STAMP(dslot); ++dslot;
            const unsigned gA = (unsigned)(2 * pm + ai) * S_H, gB = (unsigned)(2 * pn2 + bj) * S_H;
#pragma unroll
            for (int m = 0; m < 4; ++m)
#pragma unroll
                for (int n = 0; n < 2; ++n) acc[0][0][m][n] = (f32x4){0.f, 0.f, 0.f, 0.f};
#pragma unroll
            for (int q = 0; q < 3; ++q) { STAGE2(ra, q * HTB, gA + q * S_K); STAGE2(rb, (4 + q) * HTB, gB + q * S_K); }
            for (int t = 0; t < (NOGEMM || NOGEMM2 ? 0 : NT); t += 4) {
#pragma unroll
                for (int q = 0; q < 4; ++q) {
                    const unsigned tn = (unsigned)min(t + q + 3, NT - 1) * S_K;
                    WAIT_V(8); BAR;
                    if (G2X != 1) { STAGE2(ra, ((q + 3) & 3) * HTB, gA + tn); STAGE2(rb, (4 + ((q + 3) & 3)) * HTB, gB + tn); }
                    LDB(B0, q >> 1, q & 1); SCHED; LDA(At, q >> 1, q & 1);
                    WAIT_B1(B0);
                    if (G2X != 2) MMA(0, 0, At, B0); else __builtin_amdgcn_s_sleep(6);
                    SCHED;
                }
            }
            WAIT_V(0);
            unsigned h1, e0, e1;
            EPI_GROUP(acc[0][0], pn2, ai * 2 + bj, h1, e0, e1);
            unsigned bad;
            DSR32(bad, ldsa + LDS_BAD);
            asm volatile("s_waitcnt lgkmcnt(0)" : "+v"(bad) :: "memory");
            WAIT_V(0);
            if (rfl(bad)) h1 = POISON << 8;
            if (lane == 0) gst128_sys(CH_ADDR(pn2, ai * 2 + bj), (u32x4){call, h1, e0, e1});
            STAMP(dslot); ++dslot;
        }
#undef SA
#undef SB
    } else {
        for (int q = tid; q <= Q_END; q += NTHR) {
            u32x4 en = {0u, 0u, 0u, 0u};
            if (q < Q_OUT) {
                const int c = in_job(q), e = c >> 6, st = c & 63, mat = e >> 4, blk = e & 15;
                const int gblk = mat == 0 ? (blk >> 1) * 4 + bi * 2 + (blk & 1) : bj_ * 16 + blk;
                en = (u32x4){(unsigned)(e * 8192 + st * 16), (unsigned)(((mat * 2 + bk) * 32 + gblk) * 64 + st) * UNIT_B,
                             (unsigned)mat * STG_B + (unsigned)(2 * blk * 64 + st) * HTB, (unsigned)(c * NR * 4)};
            } else if (q < Q_END) {
                const int o = q - Q_OUT, y = out_tile(o), gg = o & 3, w8 = (o >> 2) & 7;
                en = (u32x4){(unsigned)((((y & 1) * 32 + (y >> 1)) * 32 + (o & 31)) * 32), (unsigned)(((rank * 2 + (y & 1)) * 16 + (y >> 1)) * 32 + (o & 31)) * UNIT_B,
                             (unsigned)((((y & 1) * 256 + (gg >> 1) * 128 + (w8 >> 2) * 64) * NN + (y >> 1) * 256 + (gg & 1) * 128 + (w8 & 3) * 32) * 2), 0u};
            }
            asm volatile("ds_write_b128 %0, %1" :: "v"(ldsa + LDS_TAB + (unsigned)q * 16u), "v"(en) : "memory");
        }
        const unsigned call = rfl(gld32_sys(cntp)) + 1;
        if (lane == 0) __hip_atomic_store(cntp, call, __ATOMIC_RELAXED, __HIP_MEMORY_SCOPE_AGENT);
        const unsigned par = call & 1;
        unsigned* const tkt = ctl + C_TKT / 4 + par * 1024;
        if (wg == NGEMM && wid == 7 && lane < 16) __hip_atomic_store(ctl + C_TKT / 4 + (par ^ 1) * 1024 + lane * 64, 0u, __ATOMIC_RELAXED, __HIP_MEMORY_SCOPE_SYSTEM);
        WAIT_L(0);
        BAR;
        const rsrc_t rh = mk_rsrc(myws, (unsigned)WS_NEED);
        const unsigned lw = (unsigned)wid * 16384u, lh = LDS_AUX + (unsigned)wid * 1024u;
        const unsigned vl16 = (unsigned)lane * 16u;
#define PULL_UNIT(rp, slot, so, h1) do { DMA16(rp, slot, vl16, so); DMA16(rp, (slot) + 1024, vl16, (so) + 1024); if (NL3 == 64 || lane < NL3) DMA16(rp, (slot) + 2048, vl16, (so) + 2048); \
            if (UNLIKELY((h1) >> 8 == RAWMODE)) DMA16(rp, (slot) + 3072, vl16, (so) + 3072); } while (0)
#define READ_UNITS(d, slot) do { const unsigned _a = ldsa + (slot) + vl16; \
            DSR128(d[0][0], _a, 0); DSR128(d[0][1], _a, 1024); DSR128(d[0][2], _a, 2048); DSR128(d[0][3], _a, 3072); \
            DSR128(d[1][0], _a, 4096); DSR128(d[1][1], _a, 5120); DSR128(d[1][2], _a, 6144); DSR128(d[1][3], _a, 7168); } while (0)
#define WAIT_UNITS(d, hv, en) asm volatile("s_waitcnt lgkmcnt(0)" : "+v"(d[0][0]), "+v"(d[0][1]), "+v"(d[0][2]), "+v"(d[0][3]), \
            "+v"(d[1][0]), "+v"(d[1][1]), "+v"(d[1][2]), "+v"(d[1][3]), "+v"(hv), "+v"(en) :: "memory")
#define NOCOMM 0
        if (const int cw = (wg - NGEMM) * CW + wid; wid < CW && !NOCOMM) {
            const int si = cw % (NR - 1), sq = cw / (NR - 1), s = (rank + 1 + si) & 7;
            constexpr int w_in = W_IN;
            const int oc = sq < NCW / 8 ? (s & 3) : (rank & 3);
            const int ti = oc & 1, tj = oc >> 1;
            const rsrc_t rp = mk_rsrc(p.ws[s] + OFF_PACK + (size_t)par * PACK_B, PACK_B);
            const rsrc_t rc0 = mk_rsrc(p.ws[oc] + OFF_CP, CP_B), rc1 = mk_rsrc(p.ws[oc + 4] + OFF_CP, CP_B);
            const rsrc_t ro = mk_rsrc(p.out, (MM / NR) * NN * 2);
            const unsigned hb_in = (unsigned)OFF_IH + par * IH_B + (unsigned)s * 1024u, hb_out = (unsigned)OFF_CH + (unsigned)((ti * 64 + tj * 16) * 1024);
            const char* const stg = myws + OFF_STG + (unsigned)s * 2048u;
            const char* const rdys = myws + OFF_RDY + (unsigned)s * 4u;
            const unsigned olane = (unsigned)(((ti * 512 + fr) * NN + tj * 4096 + (fq & 1) * 16 + (fq >> 1) * 8) * 2);
            auto hdr = [&](int q, unsigned x) {
                return lane < 8 ? (q < Q_OUT ? hb_in : hb_out) + x + (unsigned)lane * 4u : (unsigned)CTRL_OFF + C_TMO;
            };
            auto entry = [&](int q) {
                u32x4 en;
                asm volatile("ds_read_b128 %0, %1\n\ts_waitcnt lgkmcnt(0)" : "=&v"(en) : "v"(ldsa + LDS_TAB + (unsigned)q * 16u) : "memory");
                return (u32x4){rfl(en[0]), rfl(en[1]), rfl(en[2]), rfl(en[3])};
            };
            auto ready = [&](unsigned hv) { return ((unsigned)__ballot((int)(hv - call) >= 0) & 0x11u) == 0x11u; };
            auto pull = [&](int q, unsigned so, unsigned hv, unsigned slot) {
                const unsigned ha = (unsigned)__builtin_amdgcn_readlane((int)hv, 1), hb = (unsigned)__builtin_amdgcn_readlane((int)hv, 5);
                if (q < Q_OUT) {
                    PULL_UNIT(rp, slot, so, ha);
                    PULL_UNIT(rp, slot + 4096, so + UNIT_B, hb);
                } else {
                    PULL_UNIT(rc0, slot, so, ha);
                    PULL_UNIT(rc1, slot + 4096, so, hb);
                }
            };
            auto publish = [&](unsigned w) {
                if (lane < 2) asm volatile("global_store_dword %0, %1, %2 sc0 sc1" :: "v"(w + (unsigned)lane * 32u), "v"(call), "s"(rdys) : "memory");
            };
            auto nan_fill = [&](unsigned z) {
#pragma unroll
                for (int m = 0; m < 4; ++m)
                    st128<0>((u32x4){0x7fc07fc0u, 0x7fc07fc0u, 0x7fc07fc0u, 0x7fc07fc0u}, ro, olane + z, m * 16 * NN * 2);
            };
            const unsigned* const tki = tkt + si * 64;
            const unsigned* const tko = tkt + (8 + oc) * 64;
            int o1;
            unsigned tk;
            TICKET(tk, tko); WAIT_V(0);
            o1 = (int)min((unsigned)Q_OUT + ticket_value(tk), (unsigned)Q_END);
            auto next = [&](int qprev, unsigned t) {
                if (qprev >= Q_OUT) return (int)min((unsigned)Q_OUT + t, (unsigned)Q_END);
                const int j = 4 * w_in + (int)t;
                return j < Q_OUT ? j : o1;
            };
            int qA = sq, qB = sq + w_in, qC = sq + 2 * w_in;
            if (sq >= w_in) {
                unsigned tb, tc;
                TICKET(tb, tko); TICKET(tc, tko); TICKET(tk, tko); WAIT_V(0);
                qA = o1; qB = next(Q_OUT, ticket_value(tb)); qC = next(Q_OUT, ticket_value(tc));
            } else {
                asm volatile("v_mov_b32 %0, %1" : "=v"(tk) : "v"((unsigned)(sq - w_in)));
            }
            u32x4 eA = entry(qA), eB = entry(qB), eC = entry(qC);
            bool dead = false;
            while (qA < Q_END) {
                unsigned hvA, hvB;
                gld32x2_sys(myws + hdr(qA, eA[0]), myws + hdr(qB, eB[0]), hvA, hvB);
                if (!ready(hvA)) {
                    const unsigned long long t0 = __builtin_amdgcn_s_memrealtime();
                    for (;;) {
                        __builtin_amdgcn_s_sleep(4);
                        gld32x2_sys(myws + hdr(qA, eA[0]), myws + hdr(qB, eB[0]), hvA, hvB);
                        if (ready(hvA)) break;
                        if (timed_out(tmo, call, t0)) { dead = true; break; }
                    }
                    if (dead) break;
                }
                unsigned sl = lw, hs = lh;
                pull(qA, eA[1], hvA, sl);
                DMA4(rh, hs, hdr(qC, eC[0]));
                int nst = 0;
                unsigned pend = 0;
                for (;;) {
                    const bool more = qB < Q_END && ready(hvB);
                    if (more) {
                        pull(qB, eB[1], hvB, sl ^ 8192);
                        if (nst == 8) WAIT_V(14); else if (nst == 4) WAIT_V(10); else WAIT_V(6);
                    } else {
                        WAIT_V(0);
                        if (nst == 8) { publish(pend); nst = 0; }
                    }
                    const int qD = next(qC, ticket_value(tk));
                    unsigned hvC;
                    u32x4 d[2][4], eDv;
                    DSR128(eDv, ldsa + LDS_TAB + (unsigned)qD * 16u, 0);
                    DSR32(hvC, ldsa + hs + lane * 4);
                    READ_UNITS(d, sl);
                    WAIT_UNITS(d, hvC, eDv);
                    const u32x4 eD = {rfl(eDv[0]), rfl(eDv[1]), rfl(eDv[2]), rfl(eDv[3])};
                    TICKET(tk, qD < Q_OUT ? tki : tko);
                    hs ^= 256;
                    if (more) DMA4(rh, hs, hdr(qD, eD[0]));
                    if (qA < Q_OUT) {
                        const char* dst = stg + eA[2];
#pragma unroll
                        for (int u = 0; u < 2; ++u) {
                            const unsigned h1 = __builtin_amdgcn_readlane((int)hvA, u * 4 + 1);
                            u32x4 o[4];
                            if (!UNLIKELY(h1 >> 8 == RAWMODE)) {
                                unsigned H[8];
                                decode_high(d[u][2], h1, __builtin_amdgcn_readlane((int)hvA, u * 4 + 2), __builtin_amdgcn_readlane((int)hvA, u * 4 + 3), lane, H);
#pragma unroll
                                for (int m = 0; m < 4; ++m) {
                                    const unsigned l0 = d[u][m >> 1][(m & 1) * 2], l1 = d[u][m >> 1][(m & 1) * 2 + 1];
                                    o[m] = (u32x4){vperm(H[2 * m], l0, 0x05010400u), vperm(H[2 * m], l0, 0x07030602u),
                                                   vperm(H[2 * m + 1], l1, 0x05010400u), vperm(H[2 * m + 1], l1, 0x07030602u)};
                                }
                            } else {
#pragma unroll
                                for (int m = 0; m < 4; ++m) o[m] = d[u][m];
                            }
                            if (u == 0 && nst == 8) {
                                asm volatile("s_waitcnt vmcnt(8)" : "+v"(o[0]), "+v"(o[1]), "+v"(o[2]), "+v"(o[3]) :: "memory");
                                publish(pend);
                            }
                            const char* d0 = dst + u * HTB;
                            gst128s_sys<0>(d0, vl16, o[0]); gst128s_sys<1024>(d0, vl16, o[1]);
                            gst128s_sys<0>(d0 + 64 * HTB, vl16, o[2]); gst128s_sys<1024>(d0 + 64 * HTB, vl16, o[3]);
                        }
                        pend = eA[3]; nst = 8;
                    } else {
                        const bool bad = (unsigned)__builtin_amdgcn_readlane((int)hvA, 8) == call
                                         || (unsigned)__builtin_amdgcn_readlane((int)hvA, 1) >> 8 == POISON
                                         || (unsigned)__builtin_amdgcn_readlane((int)hvA, 5) >> 8 == POISON;
                        if (nst == 8) { WAIT_V(8); publish(pend); }
                        if (UNLIKELY(bad)) nan_fill(eA[2]);
                        else {
                            float a[4][8];
#pragma unroll
                            for (int k = 0; k < 2; ++k) {
                                const unsigned h1 = __builtin_amdgcn_readlane((int)hvA, k * 4 + 1);
                                if (UNLIKELY(h1 >> 8 == RAWMODE)) {
#pragma unroll
                                    for (int m = 0; m < 4; ++m)
#pragma unroll
                                        for (int j = 0; j < 4; ++j) {
                                            const unsigned v = d[k][m][j];
                                            const float lo = __uint_as_float(v << 16), hi = __uint_as_float(v & 0xffff0000u);
                                            a[m][2 * j] = k == 0 ? lo : a[m][2 * j] + lo; a[m][2 * j + 1] = k == 0 ? hi : a[m][2 * j + 1] + hi;
                                        }
                                } else {
                                    unsigned H[8];
                                    decode_high(d[k][2], h1, __builtin_amdgcn_readlane((int)hvA, k * 4 + 2), __builtin_amdgcn_readlane((int)hvA, k * 4 + 3), lane, H);
#pragma unroll
                                    for (int m = 0; m < 4; ++m)
#pragma unroll
                                        for (int j = 0; j < 8; ++j) {
                                            const unsigned Lw = d[k][m >> 1][(m & 1) * 2 + (j >> 2)];
                                            const float v = __uint_as_float(vperm(H[2 * m + (j >> 2)], Lw, (4u + (j & 3)) << 24 | (unsigned)(j & 3) << 16 | 0x0c0cu));
                                            a[m][j] = k == 0 ? v : a[m][j] + v;
                                        }
                                }
                            }
#pragma unroll
                            for (int m = 0; m < 4; ++m) {
                                unsigned r0 = cvtpk(a[m][0], a[m][1]), r1 = cvtpk(a[m][2], a[m][3]), r2 = cvtpk(a[m][4], a[m][5]), r3 = cvtpk(a[m][6], a[m][7]);
                                auto s02 = __builtin_amdgcn_permlane16_swap(r0, r2, false, false);
                                auto s13 = __builtin_amdgcn_permlane16_swap(r1, r3, false, false);
                                u32x4 o; o.x = s02[0]; o.y = s13[0]; o.z = s02[1]; o.w = s13[1];
                                st128<OAUX>(o, ro, olane + eA[2], m * 16 * NN * 2);
                            }
                        }
                        STAMPJ(8 + oc, qA - Q_OUT);
                        if (qA == o1) STAMPW(32);
                        nst = 4;
                    }
                    qA = qB; qB = qC; qC = qD; eA = eB; eB = eC; eC = eD; hvA = hvB; hvB = hvC; sl ^= 8192;
                    if (!more) break;
                }
                if (nst == 8) { WAIT_V(0); publish(pend); }
            }
            if (dead) {
                while (qA < Q_END) {
                    if (qA >= Q_OUT) nan_fill(eA[2]);
                    WAIT_V(0);
                    const int qD = next(qC, ticket_value(tk));
                    TICKET(tk, qD < Q_OUT ? tki : tko);
                    qA = qB; qB = qC; qC = qD; eA = eB; eB = eC; eC = entry(qD);
                }
            }
            WAIT_V(0);
        }
        STAMPW(40);
    }
}

}

extern "C" void kernel_launch(int rank, int nranks, ncclComm_t comm,
                              hipStream_t stream,
                              void* const* d_in, const int* in_sizes, int n_in,
                              void* d_out, int out_size,
                              const int* in_dtypes, int out_dtype,
                              void* const* d_out_all,
                              void* const* d_ws_all, size_t ws_bytes) {
    (void)comm; (void)in_dtypes; (void)out_dtype; (void)d_out_all;
    if (nranks != NR || n_in < 2 || in_sizes[0] != MM * KL || in_sizes[1] != KL * NN || out_size != (MM / NR) * NN
        || ws_bytes < WS_NEED || (unsigned)rank >= (unsigned)NR) {
        fprintf(stderr, "kernel_launch: unexpected shapes\n");
        return;
    }
    static bool inited[NR] = {};
    if (!inited[rank]) {
        HIP_CHECK(hipFuncSetAttribute((const void*)gemm_rs_kernel, hipFuncAttributeMaxDynamicSharedMemorySize, LDS_BYTES));
        inited[rank] = true;
    }
    Params p;
    p.x = d_in[0]; p.w = d_in[1]; p.out = d_out; p.rank = rank;
    for (int i = 0; i < NR; ++i) p.ws[i] = (char*)d_ws_all[i];
    hipLaunchKernelGGL(gemm_rs_kernel, dim3(NWG), dim3(NTHR), LDS_BYTES, stream, p);
    HIP_CHECK(hipGetLastError());
}
```

```cpp
#include <hip/hip_runtime.h>
#include <rccl/rccl.h>
#include <cstdio>
#include <cstdint>

#define HIP_CHECK(expr) do { hipError_t _e = (expr); if (_e != hipSuccess) { \
    fprintf(stderr, "HIP error %s at %s:%d\n", hipGetErrorString(_e), __FILE__, __LINE__); \
    return; } } while (0)

#define LAS __attribute__((address_space(3)))
#define UNLIKELY(x) __builtin_expect(!!(x), 0)
typedef short bf16x8 __attribute__((ext_vector_type(8)));
typedef short s16x4 __attribute__((ext_vector_type(4)));
typedef float f32x4 __attribute__((ext_vector_type(4)));
typedef float f32x2 __attribute__((ext_vector_type(2)));
typedef unsigned u32x4 __attribute__((ext_vector_type(4)));
typedef unsigned u32x3 __attribute__((ext_vector_type(3)));
typedef unsigned u32x2 __attribute__((ext_vector_type(2)));
typedef unsigned short u16x2 __attribute__((ext_vector_type(2)));
typedef __bf16 bf16x2_t __attribute__((ext_vector_type(2)));
typedef __amdgpu_buffer_rsrc_t rsrc_t;

namespace {

constexpr int NR = 8;
constexpr int MM = 8192, NN = 8192, KL = 1024;
constexpr int NWG = 256, NTHR = 512;
#define PN1 13
#define LOOK 2
#define CW 3
#define LATE 1
#define PKSLEEP 0
#define FORCE_RAW 0
#define NOGEMM 0
#define GAUX 17
#define GAUX2 GAUX
#define G2X 0
#define NOGEMM2 0
#define OAUX 0
constexpr int NGEMM = 16 * PN1, NCOMM = NWG - NGEMM, NCW = NCOMM * CW;
constexpr int PN2 = 16 - PN1;
constexpr int HTB = 16384, STAGE_BYTES = 8 * HTB;
constexpr int NT = 64;
constexpr unsigned UNIT_B = 4096;
constexpr size_t MIB = 1u << 20;
constexpr size_t OFF_PACK = 0;
constexpr unsigned PACK_B = 32u << 20;
constexpr size_t OFF_STG = 64 * MIB;
constexpr unsigned STG_B = 32u << 20;
constexpr size_t OFF_CP = 128 * MIB;
constexpr unsigned CP_B = 32u << 20;
constexpr size_t OFF_IH = 160 * MIB;
constexpr unsigned IH_B = 2 * 16 * 8 * 64 * 16;
constexpr size_t OFF_RDY = 161 * MIB;
constexpr size_t OFF_CH = 162 * MIB;
constexpr size_t CTRL_OFF = 256 * MIB;
constexpr size_t WS_NEED = CTRL_OFF + MIB;
constexpr unsigned C_TMO = 256, C_CNT = 4096;
constexpr unsigned C_TKT = 16384;
constexpr unsigned MAXEXC = 64, RAWMODE = 255, POISON = 254;
constexpr int OFF_CODE = 2048, OFF_LIST = 2816;
constexpr unsigned long long DEADLINE = 30000000ull;
constexpr int LDS_AUX = STAGE_BYTES;
constexpr int LDS_BAD = STAGE_BYTES + 8192;
constexpr int LDS_BYTES = LDS_BAD + 256;
constexpr int AUX_SYS = 17;
constexpr int E1 = 16 + PN1;
constexpr int NJ1 = (NT / 2) * E1, NJ_IN = (NT / 2) * 32;
#define NFB 4
#define SKD 3
constexpr int EF = 8 + NFB, ES = E1 - EF;
constexpr int Q_OUT = NJ_IN, Q_END = Q_OUT + 1024;
constexpr int LDS_PAT = 98304 + 2048;
constexpr int LDS_TAB = 65536;
__host__ __device__ constexpr int in_job(int j) {
    int i = 0, r = 0;
    if (j < SKD * EF) { i = j % EF; r = j / EF; }
    else if (j < SKD * EF + (NT / 2 - SKD) * E1) { const int q = j - SKD * EF; i = q % E1; r = SKD + q / E1 - (i < EF ? 0 : SKD); }
    else if (j < NJ1) { const int q = j - (SKD * EF + (NT / 2 - SKD) * E1); i = EF + q % ES; r = NT / 2 - SKD + q / ES; }
    else { const int q = j - NJ1; i = E1 + q % PN2; r = q / PN2; }
    const int e = i < 8 ? 2 * i : i < EF ? 8 + i : i < 8 + EF ? 2 * (i - EF) + 1 : i;
    return e * 64 + 2 * r;
}
__host__ __device__ constexpr int out_tile(int o) { const int y = o >> 5; return y < NFB ? 2 * y : y < 2 * NFB ? 2 * (y - NFB) + 1 : y; }
#define W_IN 14
#define NACT 112

#define STAMP(slot) do {} while (0)
#define STAMPW(slot) do {} while (0)
#define STAMPJ(row, j) do {} while (0)
#define DIAG_BLK0 do {} while (0)
#define DIAG_BLK1(t0) do {} while (0)
struct Params {
    const void* x; const void* w; void* out;
    char* ws[NR];
    int rank;
};

__device__ __forceinline__ unsigned cvtpk(float lo, float hi) { f32x2 v = {lo, hi}; bf16x2_t b = __builtin_convertvector(v, bf16x2_t); return __builtin_bit_cast(unsigned, b); }
__device__ __forceinline__ rsrc_t mk_rsrc(const void* p, unsigned bytes) { return __builtin_amdgcn_make_buffer_rsrc((void*)p, 0, bytes, 0x00020000); }

template <int AUX> __device__ __forceinline__ void st128(u32x4 v, rsrc_t r, unsigned voff, unsigned soff) {
    __builtin_amdgcn_raw_buffer_store_b128(v, r, voff, soff, AUX);
    asm volatile("s_nop 1" :: "v"(v) : "memory");
}
__device__ __forceinline__ void gst128_sys(void* ptr, u32x4 v) {
    asm volatile("global_store_dwordx4 %0, %1, off sc0 sc1\n\ts_nop 1" :: "v"(ptr), "v"(v) : "memory");
}
__device__ __forceinline__ unsigned gld32_sys(const void* ptr) {
    unsigned v;
    asm volatile("global_load_dword %0, %1, off sc0 sc1\n\ts_waitcnt vmcnt(0)" : "=&v"(v) : "v"(ptr) : "memory");
    return v;
}
__device__ __forceinline__ void gld32x2_sys(const void* pa, const void* pb, unsigned& a, unsigned& b) {
    asm volatile("global_load_dword %0, %2, off sc0 sc1\n\tglobal_load_dword %1, %3, off sc0 sc1\n\ts_waitcnt vmcnt(0)"
                 : "=&v"(a), "=&v"(b) : "v"(pa), "v"(pb) : "memory");
}
template <int OFF> __device__ __forceinline__ void gst128s_sys(const void* base, unsigned voff, u32x4 v) {
    asm volatile("global_store_dwordx4 %0, %1, %2 offset:%c3 sc0 sc1\n\ts_nop 1" :: "v"(voff), "v"(v), "s"(base), "i"(OFF) : "memory");
}
__device__ __forceinline__ unsigned bfi(unsigned m, unsigned a, unsigned b) { return (a & m) | (b & ~m); }
__device__ __forceinline__ unsigned vperm(unsigned hi, unsigned lo, unsigned sel) { return __builtin_amdgcn_perm(hi, lo, sel); }
__device__ __forceinline__ unsigned rfl(unsigned v) { return __builtin_amdgcn_readfirstlane(v); }
__device__ __forceinline__ bool timed_out(unsigned* tmo, unsigned call, unsigned long long t0) {
    if (__hip_atomic_load(tmo, __ATOMIC_RELAXED, __HIP_MEMORY_SCOPE_AGENT) == call) return true;
    if (__builtin_amdgcn_s_memrealtime() - t0 > DEADLINE) { __hip_atomic_store(tmo, call, __ATOMIC_RELAXED, __HIP_MEMORY_SCOPE_AGENT); return true; }
    return false;
}

__device__ __forceinline__ unsigned wave_sum_u(unsigned x) {
    x += (unsigned)__builtin_amdgcn_update_dpp(0, (int)x, 0xB1, 0xf, 0xf, true);
    x += (unsigned)__builtin_amdgcn_update_dpp(0, (int)x, 0x4E, 0xf, 0xf, true);
    x += (unsigned)__builtin_amdgcn_update_dpp(0, (int)x, 0x141, 0xf, 0xf, true);
    x += (unsigned)__builtin_amdgcn_update_dpp(0, (int)x, 0x140, 0xf, 0xf, true);
    return (unsigned)(__builtin_amdgcn_readlane((int)x, 0) + __builtin_amdgcn_readlane((int)x, 16) + __builtin_amdgcn_readlane((int)x, 32) + __builtin_amdgcn_readlane((int)x, 48));
}
__device__ __forceinline__ unsigned pk_add16(unsigned a, unsigned b) { return __builtin_bit_cast(unsigned, __builtin_bit_cast(u16x2, a) + __builtin_bit_cast(u16x2, b)); }
__device__ __forceinline__ unsigned mbcnt(unsigned long long m) { return __builtin_amdgcn_mbcnt_hi((unsigned)(m >> 32), __builtin_amdgcn_mbcnt_lo((unsigned)m, 0u)); }
__device__ __forceinline__ void encode_unit(unsigned (&w)[4][4], const char* dst, unsigned lst, int lane, unsigned& h1) {
    unsigned sum = 0;
#pragma unroll
    for (int m = 0; m < 4; ++m) sum = __builtin_amdgcn_udot2(__builtin_bit_cast(u16x2, w[m][0] & 0x7f807f80u), (u16x2){1, 1}, sum, false);
    const unsigned T = (wave_sum_u(sum) + 185467u) >> 16;
    const unsigned par = ~T & 1u, top = (T + par) >> 1;
    if (par) {
#pragma unroll
        for (int m = 0; m < 4; ++m)
#pragma unroll
            for (int q = 0; q < 4; ++q) w[m][q] = pk_add16(w[m][q], 0x00800080u);
    }
    const unsigned bq = (top | 0x80u) * 0x01010101u;
    unsigned L[8], d[8], c[8], fl[4];
#pragma unroll
    for (int g = 0; g < 8; ++g) {
        const unsigned lo = w[g >> 1][(g & 1) * 2], hi = w[g >> 1][(g & 1) * 2 + 1];
        L[g] = vperm(hi, lo, 0x06040200u);
        const unsigned h = vperm(hi, lo, 0x07050301u);
        d[g] = bq - (h & 0x7f7f7f7fu);
        c[g] = (h >> 5 & 0x04040404u) | (d[g] & 0x03030303u);
        const unsigned f = (((d[g] >> 2 & 0x3f3f3f3fu) ^ 0x20202020u) + 0x7f7f7f7fu) & 0x80808080u;
        fl[g >> 1] = __builtin_amdgcn_udot4(f, g & 1 ? 0x80402010u : 0x08040201u, g & 1 ? fl[g >> 1] : 0u, false);
    }
    unsigned ex = fl[0] >> 7 | fl[1] << 1 | fl[2] << 9 | fl[3] << 17;
    const unsigned n = (unsigned)__builtin_popcount(ex);
    const unsigned long long b0 = __ballot(n & 1), b1 = __ballot(n & 2), b2 = __ballot(n & 4);
    unsigned cnt = (unsigned)(__builtin_popcountll(b0) + 2 * __builtin_popcountll(b1) + 4 * __builtin_popcountll(b2));
    if (FORCE_RAW || top > 123 || cnt > MAXEXC || __ballot(n >= 8) != 0) cnt = RAWMODE;
    if (!UNLIKELY(cnt == RAWMODE)) {
        unsigned la = lst + 2 * (mbcnt(b0) + 2 * mbcnt(b1) + 4 * mbcnt(b2)), inv = 0;
        while (__ballot(ex != 0) != 0) {
            const unsigned j = (unsigned)__builtin_ctz(ex | 0x80000000u) , sel = (j & 7) | 0x0c0c0c00u;
            const unsigned r0 = vperm(d[1], d[0], sel), r1 = vperm(d[3], d[2], sel), r2 = vperm(d[5], d[4], sel), r3 = vperm(d[7], d[6], sel);
            const unsigned ra = j & 8 ? r1 : r0, rb = j & 8 ? r3 : r2, db = j & 16 ? rb : ra;
            const unsigned q = ((db >> 2) + 1) ^ 0x20u;
            inv |= q;
            const unsigned en = (j & 16) << 6 | (unsigned)lane << 4 | (j & 15) | q << 11;
            if (ex) asm volatile("ds_write_b16 %0, %1" :: "v"(la), "v"(en) : "memory");
            la += 2; ex &= ex - 1;
        }
        if (__ballot(inv > 31) != 0) cnt = RAWMODE;
    }
    const unsigned vo = (unsigned)lane * 16u;
    if (UNLIKELY(cnt == RAWMODE)) {
        const unsigned back = par ? 0xff80ff80u : 0u;
#pragma unroll
        for (int m = 0; m < 4; ++m)
#pragma unroll
            for (int q = 0; q < 4; ++q) w[m][q] = pk_add16(w[m][q], back);
        gst128s_sys<0>(dst, vo, (u32x4){w[0][0], w[0][1], w[0][2], w[0][3]});
        gst128s_sys<1024>(dst, vo, (u32x4){w[1][0], w[1][1], w[1][2], w[1][3]});
        gst128s_sys<2048>(dst, vo, (u32x4){w[2][0], w[2][1], w[2][2], w[2][3]});
        gst128s_sys<3072>(dst, vo, (u32x4){w[3][0], w[3][1], w[3][2], w[3][3]});
        h1 = RAWMODE << 8;
    } else {
        u32x4 lv;
        asm volatile("ds_read_b128 %0, %1" : "=&v"(lv) : "v"(lst + ((unsigned)lane & 7u) * 16u) : "memory");
        gst128s_sys<0>(dst, vo, (u32x4){L[0], L[1], L[2], L[3]});
        gst128s_sys<1024>(dst, vo, (u32x4){L[4], L[5], L[6], L[7]});
        const u32x3 cd = {c[0] | c[1] << 3 | (c[6] << 6 & 0xc0c0c0c0u), c[2] | c[3] << 3 | (c[7] << 6 & 0xc0c0c0c0u),
                          c[4] | c[5] << 3 | (c[6] << 4 & 0x40404040u) | (c[7] << 5 & 0x80808080u)};
        asm volatile("global_store_dwordx3 %0, %1, %2 offset:%c3 sc0 sc1\n\ts_nop 1" :: "v"((unsigned)lane * 12u), "v"(cd), "s"(dst), "i"(OFF_CODE) : "memory");
        asm volatile("s_waitcnt lgkmcnt(0)" : "+v"(lv) :: "memory");
        if (lane < 8) gst128s_sys<OFF_LIST>(dst, vo, lv);
        h1 = top | par << 7 | cnt << 8;
    }
}
__device__ __forceinline__ void decode_high(unsigned d0, unsigned d1, unsigned d2, unsigned h1, unsigned (&H)[8]) {
    const unsigned lutl = ((h1 & 0x7fu) + 4) * 0x01010101u - 0x03020100u, luth = lutl | 0x80808080u;
    const unsigned c[8] = {d0 & 0x07070707u, d0 >> 3 & 0x07070707u, d1 & 0x07070707u, d1 >> 3 & 0x07070707u, d2 & 0x07070707u, d2 >> 3 & 0x07070707u,
                           (d0 >> 6 & 0x03030303u) | (d2 >> 4 & 0x04040404u), (d1 >> 6 & 0x03030303u) | (d2 >> 5 & 0x04040404u)};
#pragma unroll
    for (int g = 0; g < 8; ++g) H[g] = vperm(luth, lutl, c[g]) - H[g];
}

#define TRD(dst, addr, off) asm volatile("ds_read_b64_tr_b16 %0, %1 offset:%c2" : "=&v"(dst) : "v"(addr), "i"(off) : "memory")
#define PKB(x) (bf16x8){(x)[0][0], (x)[0][1], (x)[0][2], (x)[0][3], (x)[1][0], (x)[1][1], (x)[1][2], (x)[1][3]}
#define WAIT_V(n) asm volatile("s_waitcnt vmcnt(" #n ")" ::: "memory")
#define WAIT_L(n) asm volatile("s_waitcnt lgkmcnt(" #n ")" ::: "memory")
#define BAR __builtin_amdgcn_s_barrier()
#define SCHED __builtin_amdgcn_sched_barrier(0)
#define DSR128(dst, addr, off) asm volatile("ds_read_b128 %0, %1 offset:%c2" : "=&v"(dst) : "v"(addr), "i"(off) : "memory")
#define DSR32(dst, addr) asm volatile("ds_read_b32 %0, %1" : "=&v"(dst) : "v"(addr) : "memory")
#define DMA16(rs, ldsoff, voff, soff) __builtin_amdgcn_raw_ptr_buffer_load_lds(rs, (LAS void*)(lds + (ldsoff)), 16, voff, soff, 0, AUX_SYS)
#define DMA4(rs, ldsoff, voff) __builtin_amdgcn_raw_ptr_buffer_load_lds(rs, (LAS void*)(lds + (ldsoff)), 4, voff, 0, 0, AUX_SYS)
#define TICKET(tk, ptr) asm volatile("s_mov_b64 exec, 1\n\tglobal_atomic_add %0, %1, %2, %3 sc0 sc1\n\ts_mov_b64 exec, -1" : "=&v"(tk) : "v"(0u), "v"(1u), "s"(ptr) : "memory")
__device__ __forceinline__ unsigned ticket_value(unsigned tk) {
    unsigned r;
    asm volatile("v_readfirstlane_b32 %0, %1" : "=s"(r) : "v"(tk) : "memory");
    return r;
}

__global__ __launch_bounds__(NTHR, 2) void gemm_rs_kernel(Params p) {
    extern __shared__ __attribute__((aligned(16))) unsigned char smem[];
    LAS unsigned char* lds = (LAS unsigned char*)smem;
    const int tid = threadIdx.x, wid = __builtin_amdgcn_readfirstlane(tid >> 6), lane = tid & 63;
    const int wr = wid >> 2, wc = wid & 3, fr = lane & 15, fq = lane >> 4;
    const int wg = blockIdx.x, rank = p.rank;
    const int bi = rank & 1, bj_ = (rank >> 1) & 1, bk = rank >> 2;
    char* myws = p.ws[rank];
    unsigned* ctl = (unsigned*)(myws + CTRL_OFF);
    unsigned* tmo = ctl + C_TMO / 4;

    int dslot = 0;
    STAMP(dslot); ++dslot;
    const unsigned ldsa = (unsigned)(size_t)lds;
    unsigned* cntp = ctl + C_CNT / 4 + wg * 8 + wid;

    if (wg < NGEMM) {
        const rsrc_t rx = mk_rsrc(p.x, MM * KL * 2), rw = mk_rsrc(p.w, KL * NN * 2);
        const unsigned vx = (unsigned)((tid >> 3) * (KL * 2) + (((tid & 7) ^ ((tid >> 4) & 7)) << 4));
        const unsigned vw = (unsigned)(fq * (NN * 2) + ((fr ^ ((fq << 2) | ((rank & 1) << 1))) << 4));
        auto pk_issue = [&](int idx, int buf) {
            const int g8 = idx >> 7, mat = (idx >> 6) & 1, kk = (idx >> 5) & 1, blk = idx & 31;
            if (mat == 0) {
#pragma unroll
                for (int i = 0; i < 4; ++i)
                    __builtin_amdgcn_raw_ptr_buffer_load_lds(rx, (LAS void*)(lds + buf + i * 8192 + wid * 1024), 16, vx,
                                                             (unsigned)((blk * 256 + i * 64) * (KL * 2) + kk * 1024 + g8 * 128), 0, 0);
            } else {
#pragma unroll
                for (int c = 0; c < 4; ++c)
                    __builtin_amdgcn_raw_ptr_buffer_load_lds(rw, (LAS void*)(lds + buf + wid * 4096 + c * 1024), 16, vw,
                                                             (unsigned)((kk * 512 + (g8 * 8 + wid) * 8 + (c & 1) * 4) * (NN * 2) + blk * 512 + (c >> 1) * 256), 0, 0);
            }
        };
        pk_issue(wg, 0);
        volatile LAS unsigned* const badf = (volatile LAS unsigned*)(lds + LDS_BAD);
        if (tid == 0) asm volatile("ds_write_b32 %0, %1" :: "v"(ldsa + LDS_BAD), "v"(0u) : "memory");
        const unsigned call = rfl(gld32_sys(cntp)) + 1;
        if (lane == 0) __hip_atomic_store(cntp, call, __ATOMIC_RELAXED, __HIP_MEMORY_SCOPE_AGENT);
        const unsigned par = call & 1;
        unsigned* rdyw = (unsigned*)(myws + OFF_RDY);
        {
            const char* const pk = myws + OFF_PACK + (size_t)par * PACK_B;
            const rsrc_t rs = mk_rsrc(myws + OFF_STG, 2 * STG_B);
            int buf = 0;
            for (int idx = wg; idx < 1024; idx += NGEMM, buf ^= 32768) {
                const int g8 = idx >> 7, mat = (idx >> 6) & 1, kk = (idx >> 5) & 1, blk = idx & 31, t = g8 * 8 + wid;
                BAR;
                if (idx + NGEMM < 1024) pk_issue(idx + NGEMM, buf ^ 32768);
                u32x4 v[4];
                if (mat == 0) {
                    const unsigned a = ldsa + buf + lane * 128 + ((wid ^ ((lane >> 1) & 7)) << 4);
                    DSR128(v[0], a, 0); DSR128(v[1], a, 8192); DSR128(v[2], a, 16384); DSR128(v[3], a, 24576);
                } else {
                    const unsigned a = ldsa + buf + wid * 4096 + lane * 16;
                    DSR128(v[0], a, 0); DSR128(v[1], a, 1024); DSR128(v[2], a, 2048); DSR128(v[3], a, 3072);
                }
                asm volatile("s_waitcnt lgkmcnt(0)" : "+v"(v[0]), "+v"(v[1]), "+v"(v[2]), "+v"(v[3]) :: "memory");
                unsigned w[4][4];
#pragma unroll
                for (int c = 0; c < 4; ++c) { w[c][0] = v[c][0]; w[c][1] = v[c][1]; w[c][2] = v[c][2]; w[c][3] = v[c][3]; }
                unsigned h1;
                encode_unit(w, pk + (unsigned)(((mat * 2 + kk) * 32 + blk) * 64 + t) * UNIT_B, ldsa + LDS_AUX + wid * 1024, lane, h1);
                const int cb = mat == 0 ? ((blk >> 2) * 2 + (blk & 1)) : (blk & 15);
                const int c0 = mat == 0 ? ((blk >> 1) & 1) + 4 * kk : 2 * (blk >> 4) + 4 * kk, c1 = c0 + (mat == 0 ? 2 : 1);
                const bool own = c0 == rank || c1 == rank;
                if (own) {
                    const unsigned dst = (unsigned)mat * STG_B + (unsigned)(2 * cb * 64 + t) * HTB + (unsigned)rank * 2048u;
#pragma unroll
                    for (int m = 0; m < 4; ++m) st128<AUX_SYS>(v[m], rs, lane * 16, dst + (m >> 1) * (64 * HTB) + (m & 1) * 1024);
                }
                char* const w0 = p.ws[c0 & 7];
                char* const w1 = p.ws[c1 & 7];
                WAIT_V(0);
                if (lane < 2) {
                    const int c = lane == 0 ? c0 : c1;
                    if (c != rank) gst128_sys((lane == 0 ? w0 : w1) + OFF_IH + (size_t)par * IH_B + (size_t)((((mat * 16 + cb) * NR + rank) * 64 + t) * 16), (u32x4){call, h1, 0u, 0u});
                    else __hip_atomic_store(rdyw + ((mat * 16 + cb) * 64 + t) * NR + rank, call, __ATOMIC_RELAXED, __HIP_MEMORY_SCOPE_SYSTEM);
                }
            }
            BAR;
        }
        STAMP(dslot); ++dslot;
        const int pm = 2 * (wg & 7) + ((wg >> 3) & 1), pn = wg >> 4;
        const rsrc_t ra = mk_rsrc(myws + OFF_STG, STG_B), rb = mk_rsrc(myws + OFF_STG + STG_B, STG_B);
        const unsigned voff = (unsigned)(wid * 1024 + lane * 16);
        constexpr unsigned S_K = HTB, S_H = NT * HTB, S_T = 2 * NT * HTB;
        const unsigned ldsw = (unsigned)wid * 1024u;
        const int aoff = fq * 2048 + (wr * 64 + fr) * 16;
        unsigned boff[2];
        { const int q = (lane & 15) >> 2, pp = lane & 3, g = fq;
#pragma unroll
          for (int n = 0; n < 2; ++n) boff[n] = (unsigned)(size_t)lds + 4 * HTB + 256 * (8 * g + q) + 16 * (((wc ^ q) << 2) | ((n ^ (g & 1)) << 1) | (pp >> 1)) + 8 * (pp & 1); }
        const unsigned* rdyf = (const unsigned*)(myws + OFF_RDY);
#define SA(b, h) (((b) * 2 + (h)) * HTB)
#define SB(b, h) ((4 + (b) * 2 + (h)) * HTB)
#define STAGEX(rs, bufoff, soff, aux) do { _Pragma("unroll") for (int _i = 0; _i < 2; ++_i) \
        __builtin_amdgcn_raw_ptr_buffer_load_lds(rs, (LAS void*)(lds + (bufoff) + ldsw + _i * 8192), 16, voff, (soff) + _i * 8192, 0, aux); } while (0)
#define STAGE(rs, bufoff, soff) STAGEX(rs, bufoff, soff, GAUX)
#define STAGE2(rs, bufoff, soff) STAGEX(rs, bufoff, soff, GAUX2)
#define LDA(dst, b, h) do { _Pragma("unroll") for (int m = 0; m < 4; ++m) _Pragma("unroll") for (int k = 0; k < 2; ++k) dst[m][k] = *(const LAS bf16x8*)(lds + SA(b, h) + aoff + m * 256 + k * 8192); } while (0)
#define LDB(dst, b, h) do { _Pragma("unroll") for (int n = 0; n < 2; ++n) _Pragma("unroll") for (int k = 0; k < 2; ++k) { \
        TRD(dst[n][k][0], boff[n], ((b) * 2 + (h)) * HTB + k * 8192); TRD(dst[n][k][1], boff[n], ((b) * 2 + (h)) * HTB + k * 8192 + 1024); } } while (0)
#define MMA(ai, bj, At, Bt) do { __builtin_amdgcn_s_setprio(1); _Pragma("unroll") for (int m = 0; m < 4; ++m) _Pragma("unroll") for (int n = 0; n < 2; ++n) _Pragma("unroll") for (int k = 0; k < 2; ++k) \
        acc[ai][bj][m][n] = __builtin_amdgcn_mfma_f32_16x16x32_bf16(PKB(Bt[n][k]), At[m][k], acc[ai][bj][m][n], 0, 0, 0); __builtin_amdgcn_s_setprio(0); } while (0)
#define WAIT_B(X, Y) asm volatile("s_waitcnt lgkmcnt(0)" : "+v"(X[0][0][0]), "+v"(X[0][0][1]), "+v"(X[0][1][0]), "+v"(X[0][1][1]), "+v"(X[1][0][0]), "+v"(X[1][0][1]), "+v"(X[1][1][0]), "+v"(X[1][1][1]), \
        "+v"(Y[0][0][0]), "+v"(Y[0][0][1]), "+v"(Y[0][1][0]), "+v"(Y[0][1][1]), "+v"(Y[1][0][0]), "+v"(Y[1][0][1]), "+v"(Y[1][1][0]), "+v"(Y[1][1][1]) :: "memory")
#define WAIT_B1(X) asm volatile("s_waitcnt lgkmcnt(0)" : "+v"(X[0][0][0]), "+v"(X[0][0][1]), "+v"(X[0][1][0]), "+v"(X[0][1][1]), "+v"(X[1][0][0]), "+v"(X[1][0][1]), "+v"(X[1][1][0]), "+v"(X[1][1][1]) :: "memory")
        const rsrc_t rr = mk_rsrc(myws + OFF_RDY, 2 * 16 * 64 * 32);
        const unsigned pvo = (unsigned)(((lane >> 5) * 16 + (lane < 32 ? pm : pn)) * 2048 + (lane & 1) * 16);
        const unsigned pla = ldsa + LDS_AUX + wid * 1024 + lane * 16;
#define NOGATE 0
        int rdy = NOGATE ? NT : 0, pbase = 0;
        bool pol = false;
        u32x4 fv;
#define COUNT_RDY(v) do { const bool _ok = (int)((v)[0] - call) >= 0 && (int)((v)[1] - call) >= 0 && (int)((v)[2] - call) >= 0 && (int)((v)[3] - call) >= 0; \
            const unsigned long long _b = __ballot(_ok); \
            unsigned _m = (unsigned)_b & (unsigned)(_b >> 32); _m = ~(_m & (_m >> 1)) & 0x55555555u; \
            rdy = min(pbase + (_m ? (int)(__builtin_ctz(_m) >> 1) : 16), NT); } while (0)
#define FLAGREAD do { if (pol) DSR128(fv, pla, 0); } while (0)
#define GATE(tt) do { \
            if (pol) { asm volatile("s_waitcnt lgkmcnt(0)" : "+v"(fv) :: "memory"); COUNT_RDY(fv); } \
            if (UNLIKELY((int)(tt) >= rdy)) { \
                const unsigned long long _t0 = __builtin_amdgcn_s_memrealtime(); \
                DIAG_BLK0; \
                for (;;) { \
                    pbase = rdy; \
                    const char* _p = (const char*)rdyf + pvo + min(rdy + ((lane >> 1) & 15), NT - 1) * 32; \
                    u32x4 _v; \
                    asm volatile("global_load_dwordx4 %0, %1, off sc0 sc1\n\ts_waitcnt vmcnt(0)" : "=&v"(_v) : "v"(_p) : "memory"); \
                    COUNT_RDY(_v); \
                    if ((int)(tt) < rdy) break; \
                    if (timed_out(tmo, call, _t0)) { *badf = 1; rdy = NT; break; } \
                    __builtin_amdgcn_s_sleep(8); \
                } DIAG_BLK1(_t0); } \
            pol = rdy < min((int)(tt) + LOOK, NT); \
            if (pol) { pbase = rdy; DMA16(rr, LDS_AUX + wid * 1024, pvo + min(rdy + ((lane >> 1) & 15), NT - 1) * 32, 0); } } while (0)
#define WAIT_VP do { if (pol) WAIT_V(9); else WAIT_V(8); } while (0)
        const int owner = pm >> 1;
        char* chd = p.ws[owner] + OFF_CH;
        f32x4 acc[2][2][4][2];
#pragma unroll
        for (int a = 0; a < 2; ++a)
#pragma unroll
            for (int b = 0; b < 2; ++b)
#pragma unroll
                for (int m = 0; m < 4; ++m)
#pragma unroll
                    for (int n = 0; n < 2; ++n) acc[a][b][m][n] = (f32x4){0.f, 0.f, 0.f, 0.f};
        bf16x8 At[4][2]; s16x4 B0[2][2][2], B1[2][2][2];
        const unsigned cA = (unsigned)pm * S_T, cB = (unsigned)pn * S_T;
        if (!NOGEMM) { GATE(1); }
        STAGE(rb, SB(0, 0), cB); STAGE(rb, SB(0, 1), cB + S_H); STAGE(ra, SA(0, 0), cA); STAGE(ra, SA(0, 1), cA + S_H);
        if (wr == 1) BAR;
        WAIT_V(2); BAR;
        STAGE(rb, SB(1, 0), cB + S_K); STAGE(ra, SA(1, 0), cA + S_K); STAGE(rb, SB(1, 1), cB + S_H + S_K);
        WAIT_V(6); BAR;
        STAMP(dslot); ++dslot;
        for (int t = 0; t < (NOGEMM ? 0 : NT); t += 2) {
            const int t2 = min(t + 2, NT - 1), t3 = min(t + 3, NT - 1);
            const unsigned a1 = cA + (unsigned)(t + 1) * S_K;
            const unsigned a2 = cA + (unsigned)t2 * S_K, b2 = cB + (unsigned)t2 * S_K, a3 = cA + (unsigned)t3 * S_K, b3 = cB + (unsigned)t3 * S_K;
            LDB(B0, 0, 0); LDB(B1, 0, 1); SCHED; LDA(At, 0, 0); STAGE(ra, SA(1, 1), a1 + S_H);
            WAIT_V(8); WAIT_B(B0, B1); FLAGREAD; BAR; MMA(0, 0, At, B0); MMA(0, 1, At, B1); BAR; SCHED;
            GATE(t2);
            LDA(At, 0, 1); STAGE(rb, SB(0, 0), b2); STAGE(rb, SB(0, 1), b2 + S_H); STAGE(ra, SA(0, 0), a2);
            WAIT_VP; WAIT_L(0); BAR; MMA(1, 0, At, B0); MMA(1, 1, At, B1); BAR; SCHED;
            LDB(B0, 1, 0); LDB(B1, 1, 1); SCHED; LDA(At, 1, 0); STAGE(ra, SA(0, 1), a2 + S_H);
            WAIT_V(8); WAIT_B(B0, B1); FLAGREAD; BAR; MMA(0, 0, At, B0); MMA(0, 1, At, B1); BAR; SCHED;
            GATE(t3);
            LDA(At, 1, 1); STAGE(rb, SB(1, 0), b3); STAGE(rb, SB(1, 1), b3 + S_H); STAGE(ra, SA(1, 0), a3);
            WAIT_VP; WAIT_L(0); BAR; MMA(1, 0, At, B0); MMA(1, 1, At, B1); BAR; SCHED;
        }
        if (wr == 0) BAR;
        STAMP(dslot); ++dslot;
#define EPI_GROUP(A, pnx, g, H1) do { unsigned _w[4][4]; \
            _Pragma("unroll") for (int m = 0; m < 4; ++m) { \
                _w[m][0] = cvtpk(A[m][0][0], A[m][0][1]); _w[m][1] = cvtpk(A[m][0][2], A[m][0][3]); \
                _w[m][2] = cvtpk(A[m][1][0], A[m][1][1]); _w[m][3] = cvtpk(A[m][1][2], A[m][1][3]); } \
            encode_unit(_w, myws + OFF_CP + (unsigned)(((pm * 16 + (pnx)) * 8 + wid) * 4 + (g)) * UNIT_B, ldsa + LDS_AUX + wid * 1024, lane, H1); } while (0)
#define CH_ADDR(pnx, g) (chd + (size_t)(((((bi * 2 + (pm & 1)) * 32 + bj_ * 16 + (pnx)) * 8 + wid) * 4 + (g)) * 2 + bk) * 16)
        {
            unsigned bad, h[4];
            DSR32(bad, ldsa + LDS_BAD);
            asm volatile("s_waitcnt lgkmcnt(0)" : "+v"(bad) :: "memory");
            const unsigned pois = rfl(bad) ? POISON << 8 : 0;
#define EPI_HDR(g) do { if (lane == 0) gst128_sys(CH_ADDR(pn, g), (u32x4){call, pois ? pois : h[g], 0u, 0u}); } while (0)
            EPI_GROUP(acc[0][0], pn, 0, h[0]);
            EPI_GROUP(acc[0][1], pn, 1, h[1]);
            WAIT_V(4); EPI_HDR(0);
            EPI_GROUP(acc[1][0], pn, 2, h[2]);
            EPI_GROUP(acc[1][1], pn, 3, h[3]);
            WAIT_V(9); EPI_HDR(1);
            WAIT_V(5); EPI_HDR(2);
            WAIT_V(2); EPI_HDR(3);
        }
        STAMP(dslot); ++dslot;
        for (int e = pn; e < 4 * PN2; e += PN1) {
            const int ai = e & 1, bj = (e >> 1) & 1, pn2 = PN1 + (e >> 2);
            if (!NOGEMM && !NOGATE) {
                const unsigned long long t0 = __builtin_amdgcn_s_memrealtime();
                const u32x4* f = (const u32x4*)(rdyf + ((16 + pn2) * 64) * NR) + lane;
                for (;;) {
                    u32x4 v0, v1;
                    asm volatile("global_load_dwordx4 %0, %2, off sc0 sc1\n\tglobal_load_dwordx4 %1, %2, off offset:1024 sc0 sc1\n\ts_waitcnt vmcnt(0)"
                                 : "=&v"(v0), "=&v"(v1) : "v"(f) : "memory");
                    bool ok = true;
#pragma unroll
                    for (int q = 0; q < 4; ++q) ok = ok && (int)(v0[q] - call) >= 0 && (int)(v1[q] - call) >= 0;
                    if (__all(ok)) break;
                    if (timed_out(tmo, call, t0)) { *badf = 1; break; }
                    __builtin_amdgcn_s_sleep(16);
                }
            }
            BAR;
            STAMP(dslot); ++dslot;
            const unsigned gA = (unsigned)(2 * pm + ai) * S_H, gB = (unsigned)(2 * pn2 + bj) * S_H;
#pragma unroll
            for (int m = 0; m < 4; ++m)
#pragma unroll
                for (int n = 0; n < 2; ++n) acc[0][0][m][n] = (f32x4){0.f, 0.f, 0.f, 0.f};
#pragma unroll
            for (int q = 0; q < 3; ++q) { STAGE2(ra, q * HTB, gA + q * S_K); STAGE2(rb, (4 + q) * HTB, gB + q * S_K); }
            for (int t = 0; t < (NOGEMM || NOGEMM2 ? 0 : NT); t += 4) {
#pragma unroll
                for (int q = 0; q < 4; ++q) {
                    const unsigned tn = (unsigned)min(t + q + 3, NT - 1) * S_K;
                    WAIT_V(8); BAR;
                    if (G2X != 1) { STAGE2(ra, ((q + 3) & 3) * HTB, gA + tn); STAGE2(rb, (4 + ((q + 3) & 3)) * HTB, gB + tn); }
                    LDB(B0, q >> 1, q & 1); SCHED; LDA(At, q >> 1, q & 1);
                    WAIT_B1(B0);
                    if (G2X != 2) MMA(0, 0, At, B0); else __builtin_amdgcn_s_sleep(6);
                    SCHED;
                }
            }
            WAIT_V(0);
            unsigned h1;
            EPI_GROUP(acc[0][0], pn2, ai * 2 + bj, h1);
            unsigned bad;
            DSR32(bad, ldsa + LDS_BAD);
            asm volatile("s_waitcnt lgkmcnt(0)" : "+v"(bad) :: "memory");
            WAIT_V(0);
            if (rfl(bad)) h1 = POISON << 8;
            if (lane == 0) gst128_sys(CH_ADDR(pn2, ai * 2 + bj), (u32x4){call, h1, 0u, 0u});
            STAMP(dslot); ++dslot;
        }
#undef SA
#undef SB
    } else {
        for (int q = tid; q <= Q_END; q += NTHR) {
            u32x4 en = {0u, 0u, 0u, 0u};
            if (q < Q_OUT) {
                const int c = in_job(q), e = c >> 6, st = c & 63, mat = e >> 4, blk = e & 15;
                const int gblk = mat == 0 ? (blk >> 1) * 4 + bi * 2 + (blk & 1) : bj_ * 16 + blk;
                en = (u32x4){(unsigned)(e * 8192 + st * 16), (unsigned)(((mat * 2 + bk) * 32 + gblk) * 64 + st) * UNIT_B,
                             (unsigned)mat * STG_B + (unsigned)(2 * blk * 64 + st) * HTB, (unsigned)(c * NR * 4)};
            } else if (q < Q_END) {
                const int o = q - Q_OUT, y = out_tile(o), gg = o & 3, w8 = (o >> 2) & 7;
                en = (u32x4){(unsigned)((((y & 1) * 32 + (y >> 1)) * 32 + (o & 31)) * 32), (unsigned)(((rank * 2 + (y & 1)) * 16 + (y >> 1)) * 32 + (o & 31)) * UNIT_B,
                             (unsigned)((((y & 1) * 256 + (gg >> 1) * 128 + (w8 >> 2) * 64) * NN + (y >> 1) * 256 + (gg & 1) * 128 + (w8 & 3) * 32) * 2), 0u};
            }
            asm volatile("ds_write_b128 %0, %1" :: "v"(ldsa + LDS_TAB + (unsigned)q * 16u), "v"(en) : "memory");
        }
        const unsigned call = rfl(gld32_sys(cntp)) + 1;
        if (lane == 0) __hip_atomic_store(cntp, call, __ATOMIC_RELAXED, __HIP_MEMORY_SCOPE_AGENT);
        const unsigned par = call & 1;
        unsigned* const tkt = ctl + C_TKT / 4 + par * 1024;
        if (wg == NGEMM && wid == 7 && lane < 16) __hip_atomic_store(ctl + C_TKT / 4 + (par ^ 1) * 1024 + lane * 64, 0u, __ATOMIC_RELAXED, __HIP_MEMORY_SCOPE_SYSTEM);
        WAIT_L(0);
        BAR;
        const rsrc_t rh = mk_rsrc(myws, (unsigned)WS_NEED);
        const unsigned lw = (unsigned)wid * 16384u, lh = LDS_AUX + (unsigned)wid * 1024u;
        const unsigned pat = ldsa + LDS_PAT + (unsigned)wid * 4096u;
        const unsigned vl16 = (unsigned)lane * 16u;
#define PULL_UNIT(rp, slot, so, h1) do { DMA16(rp, slot, vl16, so); DMA16(rp, (slot) + 1024, vl16, (so) + 1024); \
            if ((unsigned)lane < ((h1) >> 8 > 32 ? ((h1) >> 8 > MAXEXC ? 64u : 56u) : 52u)) DMA16(rp, (slot) + 2048, vl16, (so) + 2048); \
            if (UNLIKELY((h1) >> 8 == RAWMODE)) DMA16(rp, (slot) + 3072, vl16, (so) + 3072); } while (0)
#define READ_ENTRY(en, slot, h1) asm volatile("ds_read_u16 %0, %1 offset:%c2" : "=&v"(en) \
            : "v"(ldsa + (slot) + 2 * min((unsigned)lane, min(max((h1) >> 8, 1u), MAXEXC) - 1)), "i"(OFF_LIST) : "memory")
#define READ_UNIT(d, cd, ce, slot, h1) asm volatile("ds_read_b128 %0, %6\n\tds_read_b128 %1, %6 offset:1024\n\t" \
            "s_cmp_eq_u32 %8, 255\n\ts_cbranch_scc1 1f\n\t" \
            "ds_read2_b32 %4, %7 offset1:1\n\tds_read_b32 %5, %7 offset:8\n\ts_branch 2f\n" \
            "1:\n\tds_read_b128 %2, %6 offset:2048\n\tds_read_b128 %3, %6 offset:3072\n" \
            "2:" : "=&v"(d[0]), "=&v"(d[1]), "=&v"(d[2]), "=&v"(d[3]), "=&v"(cd), "=&v"(ce) \
            : "v"(ldsa + (slot) + vl16), "v"(ldsa + (slot) + OFF_CODE + (unsigned)lane * 12u), "s"((h1) >> 8) : "scc", "memory")
#define PATCH_UNIT(d, en, u, h1) do { const unsigned _en = (h1) >> 8 == 0 ? 0x800u : (en), _ad = pat + (u) * 2048 + (_en & 0x7ffu); \
            asm volatile("s_cmp_eq_u32 %6, 255\n\ts_cbranch_scc1 1f\n\t" \
                         "ds_write_b8 %2, %3\n\tds_read_b128 %0, %4\n\tds_read_b128 %1, %4 offset:1024\n\tds_write_b8 %2, %5\n1:" \
                         : "+v"(d[2]), "+v"(d[3]) : "v"(_ad), "v"(_en >> 9 & 0x7cu), "v"(pat + (u) * 2048 + vl16), "v"(4u), "s"((h1) >> 8) : "scc", "memory"); } while (0)
#define WAIT_UNITS(d, cd, ce, hv, nx) asm volatile("s_waitcnt lgkmcnt(0)" : "+v"(d[0][0]), "+v"(d[0][1]), "+v"(d[0][2]), "+v"(d[0][3]), \
            "+v"(d[1][0]), "+v"(d[1][1]), "+v"(d[1][2]), "+v"(d[1][3]), "+v"(cd[0]), "+v"(cd[1]), "+v"(ce[0]), "+v"(ce[1]), \
            "+v"(hv), "+v"(nx) :: "memory")
#define NOCOMM 0
        if (const int cw = NACT == NCW ? (wg - NGEMM) * CW + wid : wid * NCOMM + (wg - NGEMM); wid < CW && cw < NACT && !NOCOMM) {
#pragma unroll
            for (int i = 0; i < 4; ++i) asm volatile("ds_write_b128 %0, %1" :: "v"(pat + i * 1024 + vl16), "v"((u32x4){0x04040404u, 0x04040404u, 0x04040404u, 0x04040404u}) : "memory");
            const int si = cw % (NR - 1), sq = cw / (NR - 1), s = (rank + 1 + si) & 7;
            constexpr int w_in = W_IN;
            const int oc = sq < NACT / 8 ? (s & 3) : (rank & 3);
            const int ti = oc & 1, tj = oc >> 1;
            const rsrc_t rp = mk_rsrc(p.ws[s] + OFF_PACK + (size_t)par * PACK_B, PACK_B);
            const rsrc_t rc0 = mk_rsrc(p.ws[oc] + OFF_CP, CP_B), rc1 = mk_rsrc(p.ws[oc + 4] + OFF_CP, CP_B);
            const rsrc_t ro = mk_rsrc(p.out, (MM / NR) * NN * 2);
            const unsigned hb_in = (unsigned)OFF_IH + par * IH_B + (unsigned)s * 1024u, hb_out = (unsigned)OFF_CH + (unsigned)((ti * 64 + tj * 16) * 1024);
            const char* const stg = myws + OFF_STG + (unsigned)s * 2048u;
            const char* const rdys = myws + OFF_RDY + (unsigned)s * 4u;
            const unsigned olane = (unsigned)(((ti * 512 + fr) * NN + tj * 4096 + (fq & 1) * 16 + (fq >> 1) * 8) * 2);
            auto hdr = [&](int q, unsigned x) {
                return lane < 8 ? (q < Q_OUT ? hb_in : hb_out) + x + (unsigned)lane * 4u : (unsigned)CTRL_OFF + C_TMO;
            };
            auto entry = [&](int q) {
                u32x4 en;
                asm volatile("ds_read_b128 %0, %1\n\ts_waitcnt lgkmcnt(0)" : "=&v"(en) : "v"(ldsa + LDS_TAB + (unsigned)q * 16u) : "memory");
                return (u32x4){rfl(en[0]), rfl(en[1]), rfl(en[2]), rfl(en[3])};
            };
            auto ready = [&](unsigned hv) { return ((unsigned)__ballot((int)(hv - call) >= 0) & 0x11u) == 0x11u; };
            auto pull = [&](int q, unsigned so, unsigned hv, unsigned slot) {
                const unsigned ha = (unsigned)__builtin_amdgcn_readlane((int)hv, 1), hb = (unsigned)__builtin_amdgcn_readlane((int)hv, 5);
                if (q < Q_OUT) {
                    PULL_UNIT(rp, slot, so, ha);
                    PULL_UNIT(rp, slot + 4096, so + UNIT_B, hb);
                } else {
                    PULL_UNIT(rc0, slot, so, ha);
                    PULL_UNIT(rc1, slot + 4096, so, hb);
                }
            };
            auto publish = [&](unsigned w) {
                if (lane < 2) asm volatile("global_store_dword %0, %1, %2 sc0 sc1" :: "v"(w + (unsigned)lane * 32u), "v"(call), "s"(rdys) : "memory");
            };
            auto nan_fill = [&](unsigned z) {
#pragma unroll
                for (int m = 0; m < 4; ++m)
                    st128<0>((u32x4){0x7fc07fc0u, 0x7fc07fc0u, 0x7fc07fc0u, 0x7fc07fc0u}, ro, olane + z, m * 16 * NN * 2);
            };
            const unsigned* const tki = tkt + si * 64;
            const unsigned* const tko = tkt + (8 + oc) * 64;
            int o1;
            unsigned tk;
            TICKET(tk, tko); WAIT_V(0);
            o1 = (int)min((unsigned)Q_OUT + ticket_value(tk), (unsigned)Q_END);
            auto next = [&](int qprev, unsigned t) {
                if (qprev >= Q_OUT) return (int)min((unsigned)Q_OUT + t, (unsigned)Q_END);
                const int j = 4 * w_in + (int)t;
                return j < Q_OUT ? j : o1;
            };
            int qA = sq, qB = sq + w_in, qC = sq + 2 * w_in;
            if (sq >= w_in) {
                unsigned tb, tc;
                TICKET(tb, tko); TICKET(tc, tko); TICKET(tk, tko); WAIT_V(0);
                qA = o1; qB = next(Q_OUT, ticket_value(tb)); qC = next(Q_OUT, ticket_value(tc));
            } else {
                asm volatile("v_mov_b32 %0, %1" : "=v"(tk) : "v"((unsigned)(sq - w_in)));
            }
            u32x4 eA = entry(qA), eB = entry(qB), eC = entry(qC);
            bool dead = false;
            while (qA < Q_END) {
                unsigned hvA, hvB;
                gld32x2_sys(myws + hdr(qA, eA[0]), myws + hdr(qB, eB[0]), hvA, hvB);
                if (!ready(hvA)) {
                    const unsigned long long t0 = __builtin_amdgcn_s_memrealtime();
                    for (;;) {
                        __builtin_amdgcn_s_sleep(4);
                        gld32x2_sys(myws + hdr(qA, eA[0]), myws + hdr(qB, eB[0]), hvA, hvB);
                        if (ready(hvA)) break;
                        if (timed_out(tmo, call, t0)) { dead = true; break; }
                    }
                    if (dead) break;
                }
                unsigned sl = lw, hs = lh;
                pull(qA, eA[1], hvA, sl);
                DMA4(rh, hs, hdr(qC, eC[0]));
                int nst = 0;
                unsigned pend = 0;
                for (;;) {
                    const bool more = qB < Q_END && ready(hvB);
                    if (more) {
                        if (nst == 8) WAIT_V(8); else if (nst == 4) WAIT_V(4); else WAIT_V(0);
                        pull(qB, eB[1], hvB, sl ^ 8192);
                    } else {
                        WAIT_V(0);
                        if (nst == 8) { publish(pend); nst = 0; }
                    }
                    const int qD = next(qC, ticket_value(tk));
                    unsigned hvC;
                    u32x4 d[2][4], eDv;
                    u32x2 cd[2];
                    unsigned ce[2], en0, en1;
                    const unsigned hu[2] = {(unsigned)__builtin_amdgcn_readlane((int)hvA, 1), (unsigned)__builtin_amdgcn_readlane((int)hvA, 5)};
                    READ_ENTRY(en0, sl, hu[0]);
                    READ_ENTRY(en1, sl + 4096, hu[1]);
                    DSR128(eDv, ldsa + LDS_TAB + (unsigned)qD * 16u, 0);
                    DSR32(hvC, ldsa + hs + lane * 4);
                    READ_UNIT(d[0], cd[0], ce[0], sl, hu[0]);
                    READ_UNIT(d[1], cd[1], ce[1], sl + 4096, hu[1]);
                    asm volatile("s_waitcnt lgkmcnt(10)" : "+v"(en0), "+v"(en1) :: "memory");
                    PATCH_UNIT(d[0], en0, 0, hu[0]);
                    PATCH_UNIT(d[1], en1, 1, hu[1]);
                    WAIT_UNITS(d, cd, ce, hvC, eDv);
                    const u32x4 eD = {rfl(eDv[0]), rfl(eDv[1]), rfl(eDv[2]), rfl(eDv[3])};
                    TICKET(tk, qD < Q_OUT ? tki : tko);
                    hs ^= 256;
                    if (more) DMA4(rh, hs, hdr(qD, eD[0]));
                    if (qA < Q_OUT) {
                        const char* dst = stg + eA[2];
#pragma unroll
                        for (int u = 0; u < 2; ++u) {
                            const unsigned h1 = hu[u];
                            u32x4 o[4];
                            if (!UNLIKELY(h1 >> 8 == RAWMODE)) {
                                unsigned H[8] = {d[u][2][0], d[u][2][1], d[u][2][2], d[u][2][3], d[u][3][0], d[u][3][1], d[u][3][2], d[u][3][3]};
                                decode_high(cd[u][0], cd[u][1], ce[u], h1, H);
#pragma unroll
                                for (int m = 0; m < 4; ++m) {
                                    const unsigned l0 = d[u][m >> 1][(m & 1) * 2], l1 = d[u][m >> 1][(m & 1) * 2 + 1];
                                    o[m] = (u32x4){vperm(H[2 * m], l0, 0x05010400u), vperm(H[2 * m], l0, 0x07030602u),
                                                   vperm(H[2 * m + 1], l1, 0x05010400u), vperm(H[2 * m + 1], l1, 0x07030602u)};
                                }
                                if (h1 & 0x80u) {
#pragma unroll
                                    for (int m = 0; m < 4; ++m)
#pragma unroll
                                        for (int j = 0; j < 4; ++j) o[m][j] = pk_add16(o[m][j], 0xff80ff80u);
                                }
                            } else {
#pragma unroll
                                for (int m = 0; m < 4; ++m) o[m] = d[u][m];
                            }
                            if (u == 0 && nst == 8) {
                                asm volatile("s_waitcnt vmcnt(8)" : "+v"(o[0]), "+v"(o[1]), "+v"(o[2]), "+v"(o[3]) :: "memory");
                                publish(pend);
                            }
                            const char* d0 = dst + u * HTB;
                            gst128s_sys<0>(d0, vl16, o[0]); gst128s_sys<1024>(d0, vl16, o[1]);
                            gst128s_sys<0>(d0 + 64 * HTB, vl16, o[2]); gst128s_sys<1024>(d0 + 64 * HTB, vl16, o[3]);
                        }
                        pend = eA[3]; nst = 8;
                    } else {
                        const bool bad = (unsigned)__builtin_amdgcn_readlane((int)hvA, 8) == call
                                         || (unsigned)__builtin_amdgcn_readlane((int)hvA, 1) >> 8 == POISON
                                         || (unsigned)__builtin_amdgcn_readlane((int)hvA, 5) >> 8 == POISON;
                        if (nst == 8) { WAIT_V(8); publish(pend); }
                        if (UNLIKELY(bad)) nan_fill(eA[2]);
                        else {
                            float a[4][8];
#pragma unroll
                            for (int k = 0; k < 2; ++k) {
                                const unsigned h1 = hu[k];
                                if (UNLIKELY(h1 >> 8 == RAWMODE)) {
#pragma unroll
                                    for (int m = 0; m < 4; ++m)
#pragma unroll
                                        for (int j = 0; j < 4; ++j) {
                                            const unsigned v = d[k][m][j];
                                            const float lo = __uint_as_float(v << 16), hi = __uint_as_float(v & 0xffff0000u);
                                            a[m][2 * j] = k == 0 ? lo : a[m][2 * j] + lo; a[m][2 * j + 1] = k == 0 ? hi : a[m][2 * j + 1] + hi;
                                        }
                                } else {
                                    unsigned H[8] = {d[k][2][0], d[k][2][1], d[k][2][2], d[k][2][3], d[k][3][0], d[k][3][1], d[k][3][2], d[k][3][3]};
                                    decode_high(cd[k][0], cd[k][1], ce[k], h1, H);
                                    unsigned vb[4][8];
#pragma unroll
                                    for (int m = 0; m < 4; ++m)
#pragma unroll
                                        for (int j = 0; j < 8; ++j) {
                                            const unsigned Lw = d[k][m >> 1][(m & 1) * 2 + (j >> 2)];
                                            vb[m][j] = vperm(H[2 * m + (j >> 2)], Lw, (4u + (j & 3)) << 24 | (unsigned)(j & 3) << 16 | 0x0c0cu);
                                        }
                                    if (h1 & 0x80u) {
#pragma unroll
                                        for (int m = 0; m < 4; ++m)
#pragma unroll
                                            for (int j = 0; j < 8; ++j) vb[m][j] -= 0x00800000u;
                                    }
#pragma unroll
                                    for (int m = 0; m < 4; ++m)
#pragma unroll
                                        for (int j = 0; j < 8; ++j) a[m][j] = k == 0 ? __uint_as_float(vb[m][j]) : a[m][j] + __uint_as_float(vb[m][j]);
                                }
                            }
#pragma unroll
                            for (int m = 0; m < 4; ++m) {
                                unsigned r0 = cvtpk(a[m][0], a[m][1]), r1 = cvtpk(a[m][2], a[m][3]), r2 = cvtpk(a[m][4], a[m][5]), r3 = cvtpk(a[m][6], a[m][7]);
                                auto s02 = __builtin_amdgcn_permlane16_swap(r0, r2, false, false);
                                auto s13 = __builtin_amdgcn_permlane16_swap(r1, r3, false, false);
                                u32x4 o; o.x = s02[0]; o.y = s13[0]; o.z = s02[1]; o.w = s13[1];
                                st128<OAUX>(o, ro, olane + eA[2], m * 16 * NN * 2);
                            }
                        }
                        STAMPJ(8 + oc, qA - Q_OUT);
                        if (qA == o1) STAMPW(32);
                        nst = 4;
                    }
                    qA = qB; qB = qC; qC = qD; eA = eB; eB = eC; eC = eD; hvA = hvB; hvB = hvC; sl ^= 8192;
                    if (!more) break;
                }
                if (nst == 8) { WAIT_V(0); publish(pend); }
            }
            if (dead) {
                while (qA < Q_END) {
                    if (qA >= Q_OUT) nan_fill(eA[2]);
                    WAIT_V(0);
                    const int qD = next(qC, ticket_value(tk));
                    TICKET(tk, qD < Q_OUT ? tki : tko);
                    qA = qB; qB = qC; qC = qD; eA = eB; eB = eC; eC = entry(qD);
                }
            }
            WAIT_V(0);
        }
        STAMPW(40);
    }
}

}

extern "C" void kernel_launch(int rank, int nranks, ncclComm_t comm,
                              hipStream_t stream,
                              void* const* d_in, const int* in_sizes, int n_in,
                              void* d_out, int out_size,
                              const int* in_dtypes, int out_dtype,
                              void* const* d_out_all,
                              void* const* d_ws_all, size_t ws_bytes) {
    (void)comm; (void)in_dtypes; (void)out_dtype; (void)d_out_all;
    if (nranks != NR || n_in < 2 || in_sizes[0] != MM * KL || in_sizes[1] != KL * NN || out_size != (MM / NR) * NN
        || ws_bytes < WS_NEED || (unsigned)rank >= (unsigned)NR) {
        fprintf(stderr, "kernel_launch: unexpected shapes\n");
        return;
    }
    static bool inited[NR] = {};
    if (!inited[rank]) {
        HIP_CHECK(hipFuncSetAttribute((const void*)gemm_rs_kernel, hipFuncAttributeMaxDynamicSharedMemorySize, LDS_BYTES));
        inited[rank] = true;
    }
    Params p;
    p.x = d_in[0]; p.w = d_in[1]; p.out = d_out; p.rank = rank;
    for (int i = 0; i < NR; ++i) p.ws[i] = (char*)d_ws_all[i];
    hipLaunchKernelGGL(gemm_rs_kernel, dim3(NWG), dim3(NTHR), LDS_BYTES, stream, p);
    HIP_CHECK(hipGetLastError());
}
```

```cpp
#include <hip/hip_runtime.h>
#include <rccl/rccl.h>
#include <cstdio>
#include <cstdint>

#define HIP_CHECK(expr) do { hipError_t _e = (expr); if (_e != hipSuccess) { \
    fprintf(stderr, "HIP error %s at %s:%d\n", hipGetErrorString(_e), __FILE__, __LINE__); \
    return; } } while (0)

#define LAS __attribute__((address_space(3)))
#define UNLIKELY(x) __builtin_expect(!!(x), 0)
typedef short bf16x8 __attribute__((ext_vector_type(8)));
typedef short s16x4 __attribute__((ext_vector_type(4)));
typedef float f32x4 __attribute__((ext_vector_type(4)));
typedef float f32x2 __attribute__((ext_vector_type(2)));
typedef unsigned u32x4 __attribute__((ext_vector_type(4)));
typedef unsigned u32x3 __attribute__((ext_vector_type(3)));
typedef unsigned u32x2 __attribute__((ext_vector_type(2)));
typedef unsigned short u16x2 __attribute__((ext_vector_type(2)));
typedef __bf16 bf16x2_t __attribute__((ext_vector_type(2)));
typedef __amdgpu_buffer_rsrc_t rsrc_t;

namespace {

constexpr int NR = 8;
constexpr int MM = 8192, NN = 8192, KL = 1024;
constexpr int NWG = 256, NTHR = 512;
#define PN1 13
#define LOOK 2
#define CW 3
#define LATE 1
#define PUB2 1
#define LEAD 1
#define GSLEEP 2
#define CSLEEP 2
#define HBATCH 1
#define OBATCH 1
#define FASTP 1
#define FASTC 1
#define URG_LO 0
#define URG_HI 4
#define PKSLEEP 0
#define FORCE_RAW 0
#define NOGEMM 0
#define GAUX 17
#define GAUX2 GAUX
#define G2X 0
#define NOGEMM2 0
#define OAUX 0
constexpr int NGEMM = 16 * PN1, NCOMM = NWG - NGEMM, NCW = NCOMM * CW;
constexpr int PN2 = 16 - PN1;
constexpr int HTB = 16384, STAGE_BYTES = 8 * HTB;
constexpr int NT = 64;
constexpr unsigned UNIT_B = 4096;
constexpr size_t MIB = 1u << 20;
constexpr size_t OFF_PACK = 0;
constexpr unsigned PACK_B = 32u << 20;
constexpr size_t OFF_STG = 64 * MIB;
constexpr unsigned STG_B = 32u << 20;
constexpr size_t OFF_CP = 128 * MIB;
constexpr unsigned CP_B = 32u << 20;
constexpr size_t OFF_IH = 160 * MIB;
constexpr unsigned IH_B = 2 * 16 * 8 * 64 * 16;
constexpr size_t OFF_RDY = 161 * MIB;
constexpr size_t OFF_CH = 162 * MIB;
constexpr size_t CTRL_OFF = 256 * MIB;
constexpr size_t WS_NEED = CTRL_OFF + MIB;
constexpr unsigned C_TMO = 256, C_CNT = 4096;
constexpr unsigned C_TKT = 16384;
constexpr unsigned MAXEXC = 64, RAWMODE = 255, POISON = 254;
constexpr int OFF_CODE = 2048, OFF_LIST = 2816;
constexpr unsigned long long DEADLINE = 30000000ull;
constexpr int LDS_AUX = STAGE_BYTES;
constexpr int LDS_BAD = STAGE_BYTES + 8192;
constexpr int LDS_BYTES = LDS_BAD + 512;
constexpr int AUX_SYS = 17;
constexpr int E1 = 16 + PN1;
constexpr int NJ1 = (NT / 2) * E1, NJ_IN = (NT / 2) * 32;
#define NFB 4
#define SKD 3
constexpr int EF = 8 + NFB, ES = E1 - EF;
constexpr int Q_OUT = NJ_IN, Q_END = Q_OUT + 1024;
constexpr int LDS_PAT = 98304 + 2048;
constexpr int LDS_TAB = 65536;
__host__ __device__ constexpr int in_job(int j) {
    constexpr int R1 = SKD, R2 = NT / 2 - SKD;
    int i = 0, r = 0;
    if (j < R1 * EF) { i = j % EF; r = j / EF; }
    else if (j < R1 * EF + R2 * E1) { const int q = j - R1 * EF; i = q % E1; r = q / E1 + (i < EF ? SKD : 0); }
    else if (j < NJ1) { const int q = j - (R1 * EF + R2 * E1); i = EF + q % ES; r = NT / 2 - SKD + q / ES; }
    else { const int q = j - NJ1; i = E1 + q % PN2; r = q / PN2; }
    const int e = i < 8 ? 2 * i : i < EF ? 8 + i : i < 8 + EF ? 2 * (i - EF) + 1 : i;
    return e * 64 + 2 * r;
}
__host__ __device__ constexpr int out_tile(int o) { const int y = o >> 5; return y < NFB ? 2 * y : y < 2 * NFB ? 2 * (y - NFB) + 1 : y; }
#define W_IN 13
#define NACT 112

#define STAMP(slot) do {} while (0)
#define STAMPW(slot) do {} while (0)
#define STAMPJ(row, j) do {} while (0)
#define DIAG_BLK0 do {} while (0)
#define DIAG_BLK1(t0) do {} while (0)
struct Params {
    const void* x; const void* w; void* out;
    char* ws[NR];
    int rank;
};

__device__ __forceinline__ unsigned cvtpk(float lo, float hi) { f32x2 v = {lo, hi}; bf16x2_t b = __builtin_convertvector(v, bf16x2_t); return __builtin_bit_cast(unsigned, b); }
__device__ __forceinline__ rsrc_t mk_rsrc(const void* p, unsigned bytes) { return __builtin_amdgcn_make_buffer_rsrc((void*)p, 0, bytes, 0x00020000); }

template <int AUX> __device__ __forceinline__ void st128(u32x4 v, rsrc_t r, unsigned voff, unsigned soff) {
    __builtin_amdgcn_raw_buffer_store_b128(v, r, voff, soff, AUX);
    asm volatile("s_nop 1" :: "v"(v) : "memory");
}
__device__ __forceinline__ void gst128_sys(void* ptr, u32x4 v) {
    asm volatile("global_store_dwordx4 %0, %1, off sc0 sc1\n\ts_nop 1" :: "v"(ptr), "v"(v) : "memory");
}
__device__ __forceinline__ unsigned gld32_sys(const void* ptr) {
    unsigned v;
    asm volatile("global_load_dword %0, %1, off sc0 sc1\n\ts_waitcnt vmcnt(0)" : "=&v"(v) : "v"(ptr) : "memory");
    return v;
}
__device__ __forceinline__ void gld32x2_sys(const void* pa, const void* pb, unsigned& a, unsigned& b) {
    asm volatile("global_load_dword %0, %2, off sc0 sc1\n\tglobal_load_dword %1, %3, off sc0 sc1\n\ts_waitcnt vmcnt(0)"
                 : "=&v"(a), "=&v"(b) : "v"(pa), "v"(pb) : "memory");
}
template <int OFF> __device__ __forceinline__ void gst128s_sys(const void* base, unsigned voff, u32x4 v) {
    asm volatile("global_store_dwordx4 %0, %1, %2 offset:%c3 sc0 sc1\n\ts_nop 1" :: "v"(voff), "v"(v), "s"(base), "i"(OFF) : "memory");
}
__device__ __forceinline__ unsigned bfi(unsigned m, unsigned a, unsigned b) { return (a & m) | (b & ~m); }
__device__ __forceinline__ unsigned vperm(unsigned hi, unsigned lo, unsigned sel) { return __builtin_amdgcn_perm(hi, lo, sel); }
__device__ __forceinline__ unsigned rfl(unsigned v) { return __builtin_amdgcn_readfirstlane(v); }
__device__ __forceinline__ bool timed_out(unsigned* tmo, unsigned call, unsigned long long t0) {
    if (__hip_atomic_load(tmo, __ATOMIC_RELAXED, __HIP_MEMORY_SCOPE_AGENT) == call) return true;
    if (__builtin_amdgcn_s_memrealtime() - t0 > DEADLINE) { __hip_atomic_store(tmo, call, __ATOMIC_RELAXED, __HIP_MEMORY_SCOPE_AGENT); return true; }
    return false;
}

__device__ __forceinline__ unsigned wave_sum_u(unsigned x) {
    x += (unsigned)__builtin_amdgcn_update_dpp(0, (int)x, 0xB1, 0xf, 0xf, true);
    x += (unsigned)__builtin_amdgcn_update_dpp(0, (int)x, 0x4E, 0xf, 0xf, true);
    x += (unsigned)__builtin_amdgcn_update_dpp(0, (int)x, 0x141, 0xf, 0xf, true);
    x += (unsigned)__builtin_amdgcn_update_dpp(0, (int)x, 0x140, 0xf, 0xf, true);
    return (unsigned)(__builtin_amdgcn_readlane((int)x, 0) + __builtin_amdgcn_readlane((int)x, 16) + __builtin_amdgcn_readlane((int)x, 32) + __builtin_amdgcn_readlane((int)x, 48));
}
__device__ __forceinline__ unsigned pk_add16(unsigned a, unsigned b) { return __builtin_bit_cast(unsigned, __builtin_bit_cast(u16x2, a) + __builtin_bit_cast(u16x2, b)); }
__device__ __forceinline__ unsigned mbcnt(unsigned long long m) { return __builtin_amdgcn_mbcnt_hi((unsigned)(m >> 32), __builtin_amdgcn_mbcnt_lo((unsigned)m, 0u)); }
__device__ __forceinline__ void encode_unit(unsigned (&w)[4][4], const char* dst, unsigned lst, int lane, unsigned& h1) {
    unsigned sum = 0;
#pragma unroll
    for (int m = 0; m < 4; ++m) sum = __builtin_amdgcn_udot2(__builtin_bit_cast(u16x2, w[m][0] & 0x7f807f80u), (u16x2){1, 1}, sum, false);
    const unsigned T = (wave_sum_u(sum) + 185467u) >> 16;
    const unsigned par = ~T & 1u, top = (T + par) >> 1;
    if (par) {
#pragma unroll
        for (int m = 0; m < 4; ++m)
#pragma unroll
            for (int q = 0; q < 4; ++q) w[m][q] = pk_add16(w[m][q], 0x00800080u);
    }
    const unsigned bq = (top | 0x80u) * 0x01010101u;
    unsigned L[8], d[8], c[8], fl[4];
#pragma unroll
    for (int g = 0; g < 8; ++g) {
        const unsigned lo = w[g >> 1][(g & 1) * 2], hi = w[g >> 1][(g & 1) * 2 + 1];
        L[g] = vperm(hi, lo, 0x06040200u);
        const unsigned h = vperm(hi, lo, 0x07050301u);
        d[g] = bq - (h & 0x7f7f7f7fu);
        c[g] = (h >> 5 & 0x04040404u) | (d[g] & 0x03030303u);
        const unsigned f = (((d[g] >> 2 & 0x3f3f3f3fu) ^ 0x20202020u) + 0x7f7f7f7fu) & 0x80808080u;
        fl[g >> 1] = __builtin_amdgcn_udot4(f, g & 1 ? 0x80402010u : 0x08040201u, g & 1 ? fl[g >> 1] : 0u, false);
    }
    unsigned ex = fl[0] >> 7 | fl[1] << 1 | fl[2] << 9 | fl[3] << 17;
    const unsigned n = (unsigned)__builtin_popcount(ex);
    const unsigned long long b0 = __ballot(n & 1), b1 = __ballot(n & 2), b2 = __ballot(n & 4);
    unsigned cnt = (unsigned)(__builtin_popcountll(b0) + 2 * __builtin_popcountll(b1) + 4 * __builtin_popcountll(b2));
    if (FORCE_RAW || top > 123 || cnt > MAXEXC || __ballot(n >= 8) != 0) cnt = RAWMODE;
    if (!UNLIKELY(cnt == RAWMODE)) {
        unsigned la = lst + 2 * (mbcnt(b0) + 2 * mbcnt(b1) + 4 * mbcnt(b2)), inv = 0;
        while (__ballot(ex != 0) != 0) {
            const unsigned j = (unsigned)__builtin_ctz(ex | 0x80000000u) , sel = (j & 7) | 0x0c0c0c00u;
            const unsigned r0 = vperm(d[1], d[0], sel), r1 = vperm(d[3], d[2], sel), r2 = vperm(d[5], d[4], sel), r3 = vperm(d[7], d[6], sel);
            const unsigned ra = j & 8 ? r1 : r0, rb = j & 8 ? r3 : r2, db = j & 16 ? rb : ra;
            const unsigned q = ((db >> 2) + 1) ^ 0x20u;
            inv |= q;
            const unsigned en = (j & 16) << 6 | (unsigned)lane << 4 | (j & 15) | q << 11;
            if (ex) asm volatile("ds_write_b16 %0, %1" :: "v"(la), "v"(en) : "memory");
            la += 2; ex &= ex - 1;
        }
        if (__ballot(inv > 31) != 0) cnt = RAWMODE;
    }
    const unsigned vo = (unsigned)lane * 16u;
    if (UNLIKELY(cnt == RAWMODE)) {
        const unsigned back = par ? 0xff80ff80u : 0u;
#pragma unroll
        for (int m = 0; m < 4; ++m)
#pragma unroll
            for (int q = 0; q < 4; ++q) w[m][q] = pk_add16(w[m][q], back);
        gst128s_sys<0>(dst, vo, (u32x4){w[0][0], w[0][1], w[0][2], w[0][3]});
        gst128s_sys<1024>(dst, vo, (u32x4){w[1][0], w[1][1], w[1][2], w[1][3]});
        gst128s_sys<2048>(dst, vo, (u32x4){w[2][0], w[2][1], w[2][2], w[2][3]});
        gst128s_sys<3072>(dst, vo, (u32x4){w[3][0], w[3][1], w[3][2], w[3][3]});
        h1 = RAWMODE << 8;
    } else {
        u32x4 lv;
        asm volatile("ds_read_b128 %0, %1" : "=&v"(lv) : "v"(lst + ((unsigned)lane & 7u) * 16u) : "memory");
        gst128s_sys<0>(dst, vo, (u32x4){L[0], L[1], L[2], L[3]});
        gst128s_sys<1024>(dst, vo, (u32x4){L[4], L[5], L[6], L[7]});
        const u32x3 cd = {c[0] | c[1] << 3 | (c[6] << 6 & 0xc0c0c0c0u), c[2] | c[3] << 3 | (c[7] << 6 & 0xc0c0c0c0u),
                          c[4] | c[5] << 3 | (c[6] << 4 & 0x40404040u) | (c[7] << 5 & 0x80808080u)};
        asm volatile("global_store_dwordx3 %0, %1, %2 offset:%c3 sc0 sc1\n\ts_nop 1" :: "v"((unsigned)lane * 12u), "v"(cd), "s"(dst), "i"(OFF_CODE) : "memory");
        asm volatile("s_waitcnt lgkmcnt(0)" : "+v"(lv) :: "memory");
        if (lane < 8) gst128s_sys<OFF_LIST>(dst, vo, lv);
        h1 = top | par << 7 | cnt << 8;
    }
}
__device__ __forceinline__ void decode_high(unsigned d0, unsigned d1, unsigned d2, unsigned h1, unsigned (&H)[8]) {
    const unsigned lutl = ((h1 & 0x7fu) + 4) * 0x01010101u - 0x03020100u, luth = lutl | 0x80808080u;
    const unsigned c[8] = {d0 & 0x07070707u, d0 >> 3 & 0x07070707u, d1 & 0x07070707u, d1 >> 3 & 0x07070707u, d2 & 0x07070707u, d2 >> 3 & 0x07070707u,
                           (d0 >> 6 & 0x03030303u) | (d2 >> 4 & 0x04040404u), (d1 >> 6 & 0x03030303u) | (d2 >> 5 & 0x04040404u)};
#pragma unroll
    for (int g = 0; g < 8; ++g) H[g] = vperm(luth, lutl, c[g]) - H[g];
}

#define TRD(dst, addr, off) asm volatile("ds_read_b64_tr_b16 %0, %1 offset:%c2" : "=&v"(dst) : "v"(addr), "i"(off) : "memory")
#define PKB(x) (bf16x8){(x)[0][0], (x)[0][1], (x)[0][2], (x)[0][3], (x)[1][0], (x)[1][1], (x)[1][2], (x)[1][3]}
#define WAIT_V(n) asm volatile("s_waitcnt vmcnt(" #n ")" ::: "memory")
#define WAIT_L(n) asm volatile("s_waitcnt lgkmcnt(" #n ")" ::: "memory")
#define BAR __builtin_amdgcn_s_barrier()
#define SCHED __builtin_amdgcn_sched_barrier(0)
#define DSR128(dst, addr, off) asm volatile("ds_read_b128 %0, %1 offset:%c2" : "=&v"(dst) : "v"(addr), "i"(off) : "memory")
#define DSR32(dst, addr) asm volatile("ds_read_b32 %0, %1" : "=&v"(dst) : "v"(addr) : "memory")
#define DMA16(rs, ldsoff, voff, soff) __builtin_amdgcn_raw_ptr_buffer_load_lds(rs, (LAS void*)(lds + (ldsoff)), 16, voff, soff, 0, AUX_SYS)
#define DMA4(rs, ldsoff, voff) __builtin_amdgcn_raw_ptr_buffer_load_lds(rs, (LAS void*)(lds + (ldsoff)), 4, voff, 0, 0, AUX_SYS)
#define TICKET(tk, ptr) asm volatile("s_mov_b64 exec, 1\n\tglobal_atomic_add %0, %1, %2, %3 sc0 sc1\n\ts_mov_b64 exec, -1" : "=&v"(tk) : "v"(0u), "v"(1u), "s"(ptr) : "memory")
__device__ __forceinline__ unsigned ticket_value(unsigned tk) {
    unsigned r;
    asm volatile("v_readfirstlane_b32 %0, %1" : "=s"(r) : "v"(tk) : "memory");
    return r;
}

__global__ __launch_bounds__(NTHR, 2) void gemm_rs_kernel(Params p) {
    extern __shared__ __attribute__((aligned(16))) unsigned char smem[];
    LAS unsigned char* lds = (LAS unsigned char*)smem;
    const int tid = threadIdx.x, wid = __builtin_amdgcn_readfirstlane(tid >> 6), lane = tid & 63;
    const int wr = wid >> 2, wc = wid & 3, fr = lane & 15, fq = lane >> 4;
    const int wg = blockIdx.x, rank = p.rank;
    const int bi = rank & 1, bj_ = (rank >> 1) & 1, bk = rank >> 2;
    char* myws = p.ws[rank];
    unsigned* ctl = (unsigned*)(myws + CTRL_OFF);
    unsigned* tmo = ctl + C_TMO / 4;

    int dslot = 0;
    STAMP(dslot); ++dslot;
    const unsigned ldsa = (unsigned)(size_t)lds;
    unsigned* cntp = ctl + C_CNT / 4 + wg * 8 + wid;

    if (wg < NGEMM) {
        const rsrc_t rx = mk_rsrc(p.x, MM * KL * 2), rw = mk_rsrc(p.w, KL * NN * 2);
        const unsigned vx = (unsigned)((tid >> 3) * (KL * 2) + (((tid & 7) ^ ((tid >> 4) & 7)) << 4));
        const unsigned vw = (unsigned)(fq * (NN * 2) + ((fr ^ ((fq << 2) | ((rank & 1) << 1))) << 4));
        auto pk_issue = [&](int idx, int buf) {
            const int g8 = idx >> 7, mat = (idx >> 6) & 1, kk = (idx >> 5) & 1, blk = idx & 31;
            if (mat == 0) {
#pragma unroll
                for (int i = 0; i < 4; ++i)
                    __builtin_amdgcn_raw_ptr_buffer_load_lds(rx, (LAS void*)(lds + buf + i * 8192 + wid * 1024), 16, vx,
                                                             (unsigned)((blk * 256 + i * 64) * (KL * 2) + kk * 1024 + g8 * 128), 0, 0);
            } else {
#pragma unroll
                for (int c = 0; c < 4; ++c)
                    __builtin_amdgcn_raw_ptr_buffer_load_lds(rw, (LAS void*)(lds + buf + wid * 4096 + c * 1024), 16, vw,
                                                             (unsigned)((kk * 512 + (g8 * 8 + wid) * 8 + (c & 1) * 4) * (NN * 2) + blk * 512 + (c >> 1) * 256), 0, 0);
            }
        };
        pk_issue(wg, 0);
        volatile LAS unsigned* const badf = (volatile LAS unsigned*)(lds + LDS_BAD);
        if (tid < 16) asm volatile("ds_write_b32 %0, %1" :: "v"(ldsa + LDS_BAD + tid * 4), "v"(0u) : "memory");
        const unsigned call = rfl(gld32_sys(cntp)) + 1;
        if (lane == 0) __hip_atomic_store(cntp, call, __ATOMIC_RELAXED, __HIP_MEMORY_SCOPE_AGENT);
        const unsigned par = call & 1;
        {
            const char* const pk = myws + OFF_PACK + (size_t)par * PACK_B;
            const rsrc_t rs = mk_rsrc(myws + OFF_STG, 2 * STG_B);
            int buf = 0;
            const unsigned hsl = ldsa + LDS_BAD + 64;
            auto pk_headers = [&](int idx, int b) {
                const int g8 = idx >> 7, mat = (idx >> 6) & 1, kk = (idx >> 5) & 1, blk = idx & 31;
                const int cb = mat == 0 ? ((blk >> 2) * 2 + (blk & 1)) : (blk & 15);
                const int c0 = mat == 0 ? ((blk >> 1) & 1) + 4 * kk : 2 * (blk >> 4) + 4 * kk, c1 = c0 + (mat == 0 ? 2 : 1);
                char* const w0 = p.ws[c0 & 7];
                char* const w1 = p.ws[c1 & 7];
                unsigned h;
                asm volatile("ds_read_b32 %0, %1\n\ts_waitcnt lgkmcnt(0)" : "=&v"(h) : "v"(hsl + (b >> 10) + (lane & 7) * 4) : "memory");
                if (lane < 16 && (lane < 8 ? c0 : c1) != rank)
                    gst128_sys((lane < 8 ? w0 : w1) + OFF_IH + (size_t)par * IH_B + (size_t)((((mat * 16 + cb) * NR + rank) * 64 + g8 * 8 + (lane & 7)) * 16), (u32x4){call, h, 0u, 0u});
            };
            for (int idx = wg; idx < 1024; idx += NGEMM, buf ^= 32768) {
                const int g8 = idx >> 7, mat = (idx >> 6) & 1, kk = (idx >> 5) & 1, blk = idx & 31, t = g8 * 8 + wid;
                BAR;
                if (HBATCH && wid == 0 && idx != wg) pk_headers(idx - NGEMM, buf ^ 32768);
                if (idx + NGEMM < 1024) pk_issue(idx + NGEMM, buf ^ 32768);
                const bool urgent = FASTP && idx < 128 && wid >= URG_LO && wid < URG_HI;
                if (urgent) __builtin_amdgcn_s_setprio(3);
                u32x4 v[4];
                if (mat == 0) {
                    const unsigned a = ldsa + buf + lane * 128 + ((wid ^ ((lane >> 1) & 7)) << 4);
                    DSR128(v[0], a, 0); DSR128(v[1], a, 8192); DSR128(v[2], a, 16384); DSR128(v[3], a, 24576);
                } else {
                    const unsigned a = ldsa + buf + wid * 4096 + lane * 16;
                    DSR128(v[0], a, 0); DSR128(v[1], a, 1024); DSR128(v[2], a, 2048); DSR128(v[3], a, 3072);
                }
                asm volatile("s_waitcnt lgkmcnt(0)" : "+v"(v[0]), "+v"(v[1]), "+v"(v[2]), "+v"(v[3]) :: "memory");
                const int cb = mat == 0 ? ((blk >> 2) * 2 + (blk & 1)) : (blk & 15);
                const int c0 = mat == 0 ? ((blk >> 1) & 1) + 4 * kk : 2 * (blk >> 4) + 4 * kk, c1 = c0 + (mat == 0 ? 2 : 1);
                char* const w0 = p.ws[c0 & 7];
                char* const w1 = p.ws[c1 & 7];
                char* const wl = lane == 0 ? w0 : w1;
                const unsigned fo = (unsigned)(OFF_RDY + (((mat * 16 + cb) * 64 + t) * NR + rank) * 4);
                unsigned w[4][4];
#pragma unroll
                for (int c = 0; c < 4; ++c) { w[c][0] = v[c][0]; w[c][1] = v[c][1]; w[c][2] = v[c][2]; w[c][3] = v[c][3]; }
                unsigned h1;
                encode_unit(w, pk + (unsigned)(((mat * 2 + kk) * 32 + blk) * 64 + t) * UNIT_B, ldsa + LDS_AUX + wid * 1024, lane, h1);
                const bool own = c0 == rank || c1 == rank;
                if (own) {
                    const unsigned dst = (unsigned)mat * STG_B + (unsigned)(2 * cb * 64 + t) * HTB + (unsigned)rank * 2048u;
#pragma unroll
                    for (int m = 0; m < 4; ++m) st128<AUX_SYS>(v[m], rs, lane * 16, dst + (m >> 1) * (64 * HTB) + (m & 1) * 1024);
                }
                if (HBATCH) asm volatile("ds_write_b32 %0, %1" :: "v"(hsl + (buf >> 10) + wid * 4), "v"(h1) : "memory");
                WAIT_V(0);
                if (lane < 2) {
                    const int c = lane == 0 ? c0 : c1;
                    if (c != rank) { if (!HBATCH || urgent) gst128_sys(wl + OFF_IH + (size_t)par * IH_B + (size_t)((((mat * 16 + cb) * NR + rank) * 64 + t) * 16), (u32x4){call, h1, 0u, 0u}); }
                    else __hip_atomic_store((unsigned*)(wl + fo), call, __ATOMIC_RELAXED, __HIP_MEMORY_SCOPE_SYSTEM);
                }
                if (urgent) __builtin_amdgcn_s_setprio(0);
                if (HBATCH) WAIT_L(0);
            }
            BAR;
            if (HBATCH && wid == 0) pk_headers(wg + (1023 - wg) / NGEMM * NGEMM, buf ^ 32768);
        }
        STAMP(dslot); ++dslot;
        const int pm = 2 * (wg & 7) + ((wg >> 3) & 1), pn = wg >> 4;
        const rsrc_t ra = mk_rsrc(myws + OFF_STG, STG_B), rb = mk_rsrc(myws + OFF_STG + STG_B, STG_B);
        const unsigned voff = (unsigned)(wid * 1024 + lane * 16);
        constexpr unsigned S_K = HTB, S_H = NT * HTB, S_T = 2 * NT * HTB;
        const unsigned ldsw = (unsigned)wid * 1024u;
        const int aoff = fq * 2048 + (wr * 64 + fr) * 16;
        unsigned boff[2];
        { const int q = (lane & 15) >> 2, pp = lane & 3, g = fq;
#pragma unroll
          for (int n = 0; n < 2; ++n) boff[n] = (unsigned)(size_t)lds + 4 * HTB + 256 * (8 * g + q) + 16 * (((wc ^ q) << 2) | ((n ^ (g & 1)) << 1) | (pp >> 1)) + 8 * (pp & 1); }
        const unsigned* rdyf = (const unsigned*)(myws + OFF_RDY);
#define SA(b, h) (((b) * 2 + (h)) * HTB)
#define SB(b, h) ((4 + (b) * 2 + (h)) * HTB)
#define STAGEX(rs, bufoff, soff, aux) do { _Pragma("unroll") for (int _i = 0; _i < 2; ++_i) \
        __builtin_amdgcn_raw_ptr_buffer_load_lds(rs, (LAS void*)(lds + (bufoff) + ldsw + _i * 8192), 16, voff, (soff) + _i * 8192, 0, aux); } while (0)
#define STAGE(rs, bufoff, soff) STAGEX(rs, bufoff, soff, GAUX)
#define STAGE2(rs, bufoff, soff) STAGEX(rs, bufoff, soff, GAUX2)
#define LDA(dst, b, h) do { _Pragma("unroll") for (int m = 0; m < 4; ++m) _Pragma("unroll") for (int k = 0; k < 2; ++k) dst[m][k] = *(const LAS bf16x8*)(lds + SA(b, h) + aoff + m * 256 + k * 8192); } while (0)
#define LDB(dst, b, h) do { _Pragma("unroll") for (int n = 0; n < 2; ++n) _Pragma("unroll") for (int k = 0; k < 2; ++k) { \
        TRD(dst[n][k][0], boff[n], ((b) * 2 + (h)) * HTB + k * 8192); TRD(dst[n][k][1], boff[n], ((b) * 2 + (h)) * HTB + k * 8192 + 1024); } } while (0)
#define MMA(ai, bj, At, Bt) do { __builtin_amdgcn_s_setprio(1); _Pragma("unroll") for (int m = 0; m < 4; ++m) _Pragma("unroll") for (int n = 0; n < 2; ++n) _Pragma("unroll") for (int k = 0; k < 2; ++k) \
        acc[ai][bj][m][n] = __builtin_amdgcn_mfma_f32_16x16x32_bf16(PKB(Bt[n][k]), At[m][k], acc[ai][bj][m][n], 0, 0, 0); __builtin_amdgcn_s_setprio(0); } while (0)
#define WAIT_B(X, Y) asm volatile("s_waitcnt lgkmcnt(0)" : "+v"(X[0][0][0]), "+v"(X[0][0][1]), "+v"(X[0][1][0]), "+v"(X[0][1][1]), "+v"(X[1][0][0]), "+v"(X[1][0][1]), "+v"(X[1][1][0]), "+v"(X[1][1][1]), \
        "+v"(Y[0][0][0]), "+v"(Y[0][0][1]), "+v"(Y[0][1][0]), "+v"(Y[0][1][1]), "+v"(Y[1][0][0]), "+v"(Y[1][0][1]), "+v"(Y[1][1][0]), "+v"(Y[1][1][1]) :: "memory")
#define WAIT_B1(X) asm volatile("s_waitcnt lgkmcnt(0)" : "+v"(X[0][0][0]), "+v"(X[0][0][1]), "+v"(X[0][1][0]), "+v"(X[0][1][1]), "+v"(X[1][0][0]), "+v"(X[1][0][1]), "+v"(X[1][1][0]), "+v"(X[1][1][1]) :: "memory")
        const rsrc_t rr = mk_rsrc(myws + OFF_RDY, 2 * 16 * 64 * 32);
        const unsigned pvo = (unsigned)(((lane >> 5) * 16 + (lane < 32 ? pm : pn)) * 2048 + (lane & 1) * 16);
        const unsigned pla = ldsa + LDS_AUX + wid * 1024 + lane * 16;
#define NOGATE 0
        int rdy = NOGATE ? NT : 0, pbase = 0;
        bool pol = false;
        u32x4 fv;
#define COUNT_RDY(v) do { const bool _ok = (int)((v)[0] - call) >= 0 && (int)((v)[1] - call) >= 0 && (int)((v)[2] - call) >= 0 && (int)((v)[3] - call) >= 0; \
            const unsigned long long _b = __ballot(_ok); \
            unsigned _m = (unsigned)_b & (unsigned)(_b >> 32); _m = ~(_m & (_m >> 1)) & 0x55555555u; \
            rdy = min(pbase + (_m ? (int)(__builtin_ctz(_m) >> 1) : 16), NT); } while (0)
#define GATE(tt) do { if (wid == 4) { \
            if (pol) { DSR128(fv, pla, 0); asm volatile("s_waitcnt lgkmcnt(0)" : "+v"(fv) :: "memory"); COUNT_RDY(fv); } \
            if (UNLIKELY((int)(tt) >= rdy)) { \
                const unsigned long long _t0 = __builtin_amdgcn_s_memrealtime(); \
                DIAG_BLK0; \
                for (;;) { \
                    pbase = rdy; \
                    const char* _p = (const char*)rdyf + pvo + min(rdy + ((lane >> 1) & 15), NT - 1) * 32; \
                    u32x4 _v; unsigned _tm; \
                    asm volatile("global_load_dwordx4 %0, %2, off sc0 sc1\n\tglobal_load_dword %1, %3, off sc0 sc1\n\ts_waitcnt vmcnt(0)" \
                                 : "=&v"(_v), "=&v"(_tm) : "v"(_p), "v"(tmo) : "memory"); \
                    COUNT_RDY(_v); \
                    if ((int)(tt) < rdy) break; \
                    if (rfl(_tm) == call || __builtin_amdgcn_s_memrealtime() - _t0 > DEADLINE) { \
                        __hip_atomic_store(tmo, call, __ATOMIC_RELAXED, __HIP_MEMORY_SCOPE_AGENT); *badf = 1; rdy = NT; break; } \
                    __builtin_amdgcn_s_sleep(GSLEEP); \
                } DIAG_BLK1(_t0); } \
            pol = rdy < min((int)(tt) + LOOK, NT); \
            if (pol) { pbase = rdy; DMA16(rr, LDS_AUX + wid * 1024, pvo + min(rdy + ((lane >> 1) & 15), NT - 1) * 32, 0); } } } while (0)
#define FLAGREAD do {} while (0)
#define GATE1(tt) GATE(tt)
#define GATE2(tt) do {} while (0)
#define WAIT_VP do { if (pol) WAIT_V(9); else WAIT_V(8); } while (0)
        const int owner = pm >> 1;
        char* chd = p.ws[owner] + OFF_CH;
        f32x4 acc[2][2][4][2];
#pragma unroll
        for (int a = 0; a < 2; ++a)
#pragma unroll
            for (int b = 0; b < 2; ++b)
#pragma unroll
                for (int m = 0; m < 4; ++m)
#pragma unroll
                    for (int n = 0; n < 2; ++n) acc[a][b][m][n] = (f32x4){0.f, 0.f, 0.f, 0.f};
        bf16x8 At[4][2]; s16x4 B0[2][2][2], B1[2][2][2];
        const unsigned cA = (unsigned)pm * S_T, cB = (unsigned)pn * S_T;
        if (!NOGEMM) { GATE(1); }
        BAR;
        STAGE(rb, SB(0, 0), cB); STAGE(rb, SB(0, 1), cB + S_H); STAGE(ra, SA(0, 0), cA); STAGE(ra, SA(0, 1), cA + S_H);
        if (wr == 1) BAR;
        WAIT_V(2); BAR;
        STAGE(rb, SB(1, 0), cB + S_K); STAGE(ra, SA(1, 0), cA + S_K); STAGE(rb, SB(1, 1), cB + S_H + S_K);
        WAIT_V(6); BAR;
        STAMP(dslot); ++dslot;
        for (int t = 0; t < (NOGEMM ? 0 : NT); t += 2) {
            const int t2 = min(t + 2, NT - 1), t3 = min(t + 3, NT - 1);
            const unsigned a1 = cA + (unsigned)(t + 1) * S_K;
            const unsigned a2 = cA + (unsigned)t2 * S_K, b2 = cB + (unsigned)t2 * S_K, a3 = cA + (unsigned)t3 * S_K, b3 = cB + (unsigned)t3 * S_K;
            LDB(B0, 0, 0); LDB(B1, 0, 1); SCHED; LDA(At, 0, 0); STAGE(ra, SA(1, 1), a1 + S_H);
            WAIT_V(8); WAIT_B(B0, B1); FLAGREAD; GATE1(t2); BAR; MMA(0, 0, At, B0); MMA(0, 1, At, B1); BAR; SCHED;
            GATE2(t2);
            LDA(At, 0, 1); STAGE(rb, SB(0, 0), b2); STAGE(rb, SB(0, 1), b2 + S_H); STAGE(ra, SA(0, 0), a2);
            WAIT_VP; WAIT_L(0); BAR; MMA(1, 0, At, B0); MMA(1, 1, At, B1); BAR; SCHED;
            LDB(B0, 1, 0); LDB(B1, 1, 1); SCHED; LDA(At, 1, 0); STAGE(ra, SA(0, 1), a2 + S_H);
            WAIT_V(8); WAIT_B(B0, B1); FLAGREAD; GATE1(t3); BAR; MMA(0, 0, At, B0); MMA(0, 1, At, B1); BAR; SCHED;
            GATE2(t3);
            LDA(At, 1, 1); STAGE(rb, SB(1, 0), b3); STAGE(rb, SB(1, 1), b3 + S_H); STAGE(ra, SA(1, 0), a3);
            WAIT_VP; WAIT_L(0); BAR; MMA(1, 0, At, B0); MMA(1, 1, At, B1); BAR; SCHED;
        }
        if (wr == 0) BAR;
        STAMP(dslot); ++dslot;
#define EPI_GROUP(A, pnx, g, H1) do { unsigned _w[4][4]; \
            _Pragma("unroll") for (int m = 0; m < 4; ++m) { \
                _w[m][0] = cvtpk(A[m][0][0], A[m][0][1]); _w[m][1] = cvtpk(A[m][0][2], A[m][0][3]); \
                _w[m][2] = cvtpk(A[m][1][0], A[m][1][1]); _w[m][3] = cvtpk(A[m][1][2], A[m][1][3]); } \
            encode_unit(_w, myws + OFF_CP + (unsigned)(((pm * 16 + (pnx)) * 8 + wid) * 4 + (g)) * UNIT_B, ldsa + LDS_AUX + wid * 1024, lane, H1); } while (0)
        auto epi_hdr = [&](int pnx, int g, unsigned h, int slot) {
            unsigned old, hv, bad;
            asm volatile("ds_write_b32 %1, %2\n\ts_mov_b64 exec, 1\n\tds_add_rtn_u32 %0, %3, %4\n\ts_mov_b64 exec, -1\n\ts_waitcnt lgkmcnt(0)"
                         : "=&v"(old) : "v"(ldsa + LDS_BAD + 128 + slot * 32 + wid * 4), "v"(h), "v"(ldsa + LDS_BAD + 16 + slot * 4), "v"(1u) : "memory");
            if (rfl(old) != 7) return false;
            asm volatile("ds_read_b32 %0, %2\n\tds_read_b32 %1, %3\n\ts_waitcnt lgkmcnt(0)"
                         : "=&v"(hv), "=&v"(bad) : "v"(ldsa + LDS_BAD + 128 + slot * 32 + (lane & 7) * 4), "v"(ldsa + LDS_BAD) : "memory");
            if (lane < 8) gst128_sys(chd + (size_t)((((((bi * 2 + (pm & 1)) * 32 + bj_ * 16 + pnx) * 4 + g) * 2 + bk) * 8 + lane) * 16),
                                     (u32x4){call, bad ? POISON << 8 : hv, 0u, 0u});
            return true;
        };
        {
            unsigned h[4];
            EPI_GROUP(acc[0][0], pn, 0, h[0]);
            EPI_GROUP(acc[0][1], pn, 1, h[1]);
            WAIT_V(4);
            const bool s0 = epi_hdr(pn, 0, h[0], 0);
            EPI_GROUP(acc[1][0], pn, 2, h[2]);
            EPI_GROUP(acc[1][1], pn, 3, h[3]);
            if (s0) WAIT_V(9); else WAIT_V(8);
            const bool s1 = epi_hdr(pn, 1, h[1], 1);
            if (s1) WAIT_V(5); else WAIT_V(4);
            const bool s2 = epi_hdr(pn, 2, h[2], 2);
            if (s1 && s2) WAIT_V(2); else if (s1 || s2) WAIT_V(1); else WAIT_V(0);
            epi_hdr(pn, 3, h[3], 3);
        }
        STAMP(dslot); ++dslot;
        for (int e = pn; e < 4 * PN2; e += PN1) {
            const int ai = e & 1, bj = (e >> 1) & 1, pn2 = PN1 + (e >> 2);
            if (!NOGEMM && !NOGATE) {
                const unsigned long long t0 = __builtin_amdgcn_s_memrealtime();
                const u32x4* f = (const u32x4*)(rdyf + ((16 + pn2) * 64) * NR) + lane;
                for (;;) {
                    u32x4 v0, v1;
                    asm volatile("global_load_dwordx4 %0, %2, off sc0 sc1\n\tglobal_load_dwordx4 %1, %2, off offset:1024 sc0 sc1\n\ts_waitcnt vmcnt(0)"
                                 : "=&v"(v0), "=&v"(v1) : "v"(f) : "memory");
                    bool ok = true;
#pragma unroll
                    for (int q = 0; q < 4; ++q) ok = ok && (int)(v0[q] - call) >= 0 && (int)(v1[q] - call) >= 0;
                    if (__all(ok)) break;
                    if (timed_out(tmo, call, t0)) { *badf = 1; break; }
                    __builtin_amdgcn_s_sleep(16);
                }
            }
            BAR;
            STAMP(dslot); ++dslot;
            const unsigned gA = (unsigned)(2 * pm + ai) * S_H, gB = (unsigned)(2 * pn2 + bj) * S_H;
#pragma unroll
            for (int m = 0; m < 4; ++m)
#pragma unroll
                for (int n = 0; n < 2; ++n) acc[0][0][m][n] = (f32x4){0.f, 0.f, 0.f, 0.f};
#pragma unroll
            for (int q = 0; q < 3; ++q) { STAGE2(ra, q * HTB, gA + q * S_K); STAGE2(rb, (4 + q) * HTB, gB + q * S_K); }
            for (int t = 0; t < (NOGEMM || NOGEMM2 ? 0 : NT); t += 4) {
#pragma unroll
                for (int q = 0; q < 4; ++q) {
                    const unsigned tn = (unsigned)min(t + q + 3, NT - 1) * S_K;
                    WAIT_V(8); BAR;
                    if (G2X != 1) { STAGE2(ra, ((q + 3) & 3) * HTB, gA + tn); STAGE2(rb, (4 + ((q + 3) & 3)) * HTB, gB + tn); }
                    LDB(B0, q >> 1, q & 1); SCHED; LDA(At, q >> 1, q & 1);
                    WAIT_B1(B0);
                    if (G2X != 2) MMA(0, 0, At, B0); else __builtin_amdgcn_s_sleep(6);
                    SCHED;
                }
            }
            WAIT_V(0);
            unsigned h1;
            EPI_GROUP(acc[0][0], pn2, ai * 2 + bj, h1);
            WAIT_V(0);
            epi_hdr(pn2, ai * 2 + bj, h1, 4 + (e - pn) / PN1);
            STAMP(dslot); ++dslot;
        }
#undef SA
#undef SB
    } else {
        unsigned cnt0;
        if (FASTC) asm volatile("global_load_dword %0, %1, off sc0 sc1" : "=&v"(cnt0) : "v"(cntp) : "memory");
        for (int q = tid; q <= Q_END; q += NTHR) {
            u32x4 en = {0u, 0u, 0u, 0u};
            if (q < Q_OUT) {
                const int c = in_job(q), e = c >> 6, st = c & 63, mat = e >> 4, blk = e & 15;
                const int gblk = mat == 0 ? (blk >> 1) * 4 + bi * 2 + (blk & 1) : bj_ * 16 + blk;
                en = (u32x4){(unsigned)(e * 8192 + st * 16), (unsigned)(((mat * 2 + bk) * 32 + gblk) * 64 + st) * UNIT_B,
                             (unsigned)mat * STG_B + (unsigned)(2 * blk * 64 + st) * HTB, (unsigned)(c * NR * 4)};
            } else if (q < Q_END) {
                const int o = q - Q_OUT, y = out_tile(o), gg = OBATCH ? (o >> 3) & 3 : o & 3, w8 = OBATCH ? o & 7 : (o >> 2) & 7;
                en = (u32x4){(unsigned)(((y & 1) * 32 + (y >> 1)) * 1024 + (OBATCH == 1 ? gg * 256 + w8 * 16 : (w8 * 4 + gg) * 32)),
                             (unsigned)(((rank * 2 + (y & 1)) * 16 + (y >> 1)) * 32 + w8 * 4 + gg) * UNIT_B,
                             (unsigned)((((y & 1) * 256 + (gg >> 1) * 128 + (w8 >> 2) * 64) * NN + (y >> 1) * 256 + (gg & 1) * 128 + (w8 & 3) * 32) * 2), 0u};
            }
            asm volatile("ds_write_b128 %0, %1" :: "v"(ldsa + LDS_TAB + (unsigned)q * 16u), "v"(en) : "memory");
        }
        if (FASTC) asm volatile("s_waitcnt vmcnt(0)" : "+v"(cnt0) :: "memory"); else cnt0 = gld32_sys(cntp);
        const unsigned call = rfl(cnt0) + 1;
        if (lane == 0) __hip_atomic_store(cntp, call, __ATOMIC_RELAXED, __HIP_MEMORY_SCOPE_AGENT);
        const unsigned par = call & 1;
        unsigned* const tkt = ctl + C_TKT / 4 + par * 1024;
        if (wg == NGEMM && wid == 7 && lane < 16) __hip_atomic_store(ctl + C_TKT / 4 + (par ^ 1) * 1024 + lane * 64, 0u, __ATOMIC_RELAXED, __HIP_MEMORY_SCOPE_SYSTEM);
        WAIT_L(0);
        BAR;
        const rsrc_t rh = mk_rsrc(myws, (unsigned)WS_NEED);
        const unsigned lw = (unsigned)wid * 16384u, lh = LDS_AUX + (unsigned)wid * 1024u;
        const unsigned pat = ldsa + LDS_PAT + (unsigned)wid * 4096u;
        const unsigned vl16 = (unsigned)lane * 16u;
#define PULL_UNIT(rp, slot, so, h1) do { DMA16(rp, slot, vl16, so); DMA16(rp, (slot) + 1024, vl16, (so) + 1024); \
            if ((unsigned)lane < ((h1) >> 8 > 32 ? ((h1) >> 8 > MAXEXC ? 64u : 56u) : 52u)) DMA16(rp, (slot) + 2048, vl16, (so) + 2048); \
            if (UNLIKELY((h1) >> 8 == RAWMODE)) DMA16(rp, (slot) + 3072, vl16, (so) + 3072); } while (0)
#define READ_ENTRY(en, slot, h1) asm volatile("ds_read_u16 %0, %1 offset:%c2" : "=&v"(en) \
            : "v"(ldsa + (slot) + 2 * min((unsigned)lane, min(max((h1) >> 8, 1u), MAXEXC) - 1)), "i"(OFF_LIST) : "memory")
#define READ_UNIT(d, cd, ce, slot, h1) asm volatile("ds_read_b128 %0, %6\n\tds_read_b128 %1, %6 offset:1024\n\t" \
            "s_cmp_eq_u32 %8, 255\n\ts_cbranch_scc1 1f\n\t" \
            "ds_read2_b32 %4, %7 offset1:1\n\tds_read_b32 %5, %7 offset:8\n\ts_branch 2f\n" \
            "1:\n\tds_read_b128 %2, %6 offset:2048\n\tds_read_b128 %3, %6 offset:3072\n" \
            "2:" : "=&v"(d[0]), "=&v"(d[1]), "=&v"(d[2]), "=&v"(d[3]), "=&v"(cd), "=&v"(ce) \
            : "v"(ldsa + (slot) + vl16), "v"(ldsa + (slot) + OFF_CODE + (unsigned)lane * 12u), "s"((h1) >> 8) : "scc", "memory")
#define PATCH_UNIT(d, en, u, h1) do { const unsigned _en = (h1) >> 8 == 0 ? 0x800u : (en), _ad = pat + (u) * 2048 + (_en & 0x7ffu); \
            asm volatile("s_cmp_eq_u32 %6, 255\n\ts_cbranch_scc1 1f\n\t" \
                         "ds_write_b8 %2, %3\n\tds_read_b128 %0, %4\n\tds_read_b128 %1, %4 offset:1024\n\tds_write_b8 %2, %5\n1:" \
                         : "+v"(d[2]), "+v"(d[3]) : "v"(_ad), "v"(_en >> 9 & 0x7cu), "v"(pat + (u) * 2048 + vl16), "v"(4u), "s"((h1) >> 8) : "scc", "memory"); } while (0)
#define WAIT_UNITS(d, cd, ce, hv, nx) asm volatile("s_waitcnt lgkmcnt(0)" : "+v"(d[0][0]), "+v"(d[0][1]), "+v"(d[0][2]), "+v"(d[0][3]), \
            "+v"(d[1][0]), "+v"(d[1][1]), "+v"(d[1][2]), "+v"(d[1][3]), "+v"(cd[0]), "+v"(cd[1]), "+v"(ce[0]), "+v"(ce[1]), \
            "+v"(hv), "+v"(nx) :: "memory")
#define NOCOMM 0
        if (const int cw = NACT == NCW ? (wg - NGEMM) * CW + wid : wid * NCOMM + (wg - NGEMM); wid < CW && cw < NACT && !NOCOMM) {
#pragma unroll
            for (int i = 0; i < 4; ++i) asm volatile("ds_write_b128 %0, %1" :: "v"(pat + i * 1024 + vl16), "v"((u32x4){0x04040404u, 0x04040404u, 0x04040404u, 0x04040404u}) : "memory");
            const int si = cw % (NR - 1), sq = cw / (NR - 1), s = (rank + 1 + si) & 7;
            constexpr int w_in = W_IN;
            const int oc = sq < NACT / 8 ? (s & 3) : (rank & 3);
            const int ti = oc & 1, tj = oc >> 1;
            const rsrc_t rp = mk_rsrc(p.ws[s] + OFF_PACK + (size_t)par * PACK_B, PACK_B);
            const rsrc_t rc0 = mk_rsrc(p.ws[oc] + OFF_CP, CP_B), rc1 = mk_rsrc(p.ws[oc + 4] + OFF_CP, CP_B);
            const rsrc_t ro = mk_rsrc(p.out, (MM / NR) * NN * 2);
            const unsigned hb_in = (unsigned)OFF_IH + par * IH_B + (unsigned)s * 1024u, hb_out = (unsigned)OFF_CH + (unsigned)((ti * 64 + tj * 16) * 1024);
            const char* const stg = myws + OFF_STG + (unsigned)s * 2048u;
            const char* const rdys = myws + OFF_RDY + (unsigned)s * 4u;
            const unsigned olane = (unsigned)(((ti * 512 + fr) * NN + tj * 4096 + (fq & 1) * 16 + (fq >> 1) * 8) * 2);
            auto hdr = [&](int q, unsigned x) {
                return lane < 8 ? (q < Q_OUT ? hb_in + (unsigned)lane * 4u : hb_out + (OBATCH == 1 ? ((unsigned)lane >> 2) * 128u + ((unsigned)lane & 3u) * 4u : (unsigned)lane * 4u)) + x
                                : (unsigned)CTRL_OFF + C_TMO;
            };
            auto entry = [&](int q) {
                u32x4 en;
                asm volatile("ds_read_b128 %0, %1\n\ts_waitcnt lgkmcnt(0)" : "=&v"(en) : "v"(ldsa + LDS_TAB + (unsigned)q * 16u) : "memory");
                return (u32x4){rfl(en[0]), rfl(en[1]), rfl(en[2]), rfl(en[3])};
            };
            auto ready = [&](unsigned hv) { return ((unsigned)__ballot((int)(hv - call) >= 0) & 0x11u) == 0x11u; };
            auto pull = [&](int q, unsigned so, unsigned hv, unsigned slot) {
                const unsigned ha = (unsigned)__builtin_amdgcn_readlane((int)hv, 1), hb = (unsigned)__builtin_amdgcn_readlane((int)hv, 5);
                if (q < Q_OUT) {
                    PULL_UNIT(rp, slot, so, ha);
                    PULL_UNIT(rp, slot + 4096, so + UNIT_B, hb);
                } else {
                    PULL_UNIT(rc0, slot, so, ha);
                    PULL_UNIT(rc1, slot + 4096, so, hb);
                }
            };
            auto publish = [&](unsigned w) {
                if (lane < 2) asm volatile("global_store_dword %0, %1, %2 sc0 sc1" :: "v"(w + (unsigned)lane * 32u), "v"(call), "s"(rdys) : "memory");
            };
            auto nan_fill = [&](unsigned z) {
#pragma unroll
                for (int m = 0; m < 4; ++m)
                    st128<0>((u32x4){0x7fc07fc0u, 0x7fc07fc0u, 0x7fc07fc0u, 0x7fc07fc0u}, ro, olane + z, m * 16 * NN * 2);
            };
            const unsigned* const tki = tkt + si * 64;
            const unsigned* const tko = tkt + (8 + oc) * 64;
            int o1 = -1;
            unsigned tk, tk0;
            TICKET(tk0, tko);
            if (!FASTC || sq >= w_in) { WAIT_V(0); o1 = (int)min((unsigned)Q_OUT + ticket_value(tk0), (unsigned)Q_END); }
            auto next = [&](int qprev, unsigned t) {
                if (qprev >= Q_OUT) return (int)min((unsigned)Q_OUT + t, (unsigned)Q_END);
                const int j = 4 * w_in + (int)t;
                return j < Q_OUT ? j : o1;
            };
            int qA = sq, qB = sq + w_in, qC = sq + 2 * w_in;
            if (sq >= w_in) {
                unsigned tb, tc;
                TICKET(tb, tko); TICKET(tc, tko); TICKET(tk, tko); WAIT_V(0);
                qA = o1; qB = next(Q_OUT, ticket_value(tb)); qC = next(Q_OUT, ticket_value(tc));
            } else {
                asm volatile("v_mov_b32 %0, %1" : "=v"(tk) : "v"((unsigned)(sq - w_in)));
            }
            u32x4 eA = entry(qA), eB = entry(qB), eC = entry(qC);
            bool dead = false;
            while (qA < Q_END) {
                unsigned hvA, hvB;
                gld32x2_sys(myws + hdr(qA, eA[0]), myws + hdr(qB, eB[0]), hvA, hvB);
                if (FASTC && o1 < 0) o1 = (int)min((unsigned)Q_OUT + ticket_value(tk0), (unsigned)Q_END);
                if (!ready(hvA)) {
                    const unsigned long long t0 = __builtin_amdgcn_s_memrealtime();
                    for (;;) {
                        __builtin_amdgcn_s_sleep(CSLEEP);
                        gld32x2_sys(myws + hdr(qA, eA[0]), myws + hdr(qB, eB[0]), hvA, hvB);
                        if (ready(hvA)) break;
                        if ((unsigned)__builtin_amdgcn_readlane((int)hvA, 8) == call || __builtin_amdgcn_s_memrealtime() - t0 > DEADLINE) {
                            __hip_atomic_store(tmo, call, __ATOMIC_RELAXED, __HIP_MEMORY_SCOPE_AGENT); dead = true; break;
                        }
                    }
                    if (dead) break;
                }
                unsigned sl = lw, hs = lh;
                pull(qA, eA[1], hvA, sl);
                DMA4(rh, hs, hdr(qC, eC[0]));
                int nst = 0;
                unsigned pend = 0;
                unsigned pend2 = 0;
                bool have2 = false;
                for (;;) {
                    const bool more = qB < Q_END && ready(hvB);
                    if (more) {
                        if (nst == 8) WAIT_V(8); else if (nst == 4) WAIT_V(4); else WAIT_V(0);
                        pull(qB, eB[1], hvB, sl ^ 8192);
                        if (have2) publish(pend2);
                        have2 = nst == 8; pend2 = pend;
                    } else {
                        WAIT_V(0);
                        if (have2) publish(pend2);
                        have2 = false;
                        if (nst == 8) { publish(pend); nst = 0; }
                    }
                    const int qD = next(qC, ticket_value(tk));
                    unsigned hvC;
                    u32x4 d[2][4], eDv;
                    u32x2 cd[2];
                    unsigned ce[2], en0, en1;
                    const unsigned hu[2] = {(unsigned)__builtin_amdgcn_readlane((int)hvA, 1), (unsigned)__builtin_amdgcn_readlane((int)hvA, 5)};
                    READ_ENTRY(en0, sl, hu[0]);
                    READ_ENTRY(en1, sl + 4096, hu[1]);
                    DSR128(eDv, ldsa + LDS_TAB + (unsigned)qD * 16u, 0);
                    DSR32(hvC, ldsa + hs + lane * 4);
                    READ_UNIT(d[0], cd[0], ce[0], sl, hu[0]);
                    READ_UNIT(d[1], cd[1], ce[1], sl + 4096, hu[1]);
                    asm volatile("s_waitcnt lgkmcnt(10)" : "+v"(en0), "+v"(en1) :: "memory");
                    PATCH_UNIT(d[0], en0, 0, hu[0]);
                    PATCH_UNIT(d[1], en1, 1, hu[1]);
                    WAIT_UNITS(d, cd, ce, hvC, eDv);
                    const u32x4 eD = {rfl(eDv[0]), rfl(eDv[1]), rfl(eDv[2]), rfl(eDv[3])};
                    TICKET(tk, qD < Q_OUT ? tki : tko);
                    hs ^= 256;
                    if (more) DMA4(rh, hs, hdr(qD, eD[0]));
                    if (qA < Q_OUT) {
                        const char* dst = stg + eA[2];
#pragma unroll
                        for (int u = 0; u < 2; ++u) {
                            const unsigned h1 = hu[u];
                            u32x4 o[4];
                            if (!UNLIKELY(h1 >> 8 == RAWMODE)) {
                                unsigned H[8] = {d[u][2][0], d[u][2][1], d[u][2][2], d[u][2][3], d[u][3][0], d[u][3][1], d[u][3][2], d[u][3][3]};
                                decode_high(cd[u][0], cd[u][1], ce[u], h1, H);
#pragma unroll
                                for (int m = 0; m < 4; ++m) {
                                    const unsigned l0 = d[u][m >> 1][(m & 1) * 2], l1 = d[u][m >> 1][(m & 1) * 2 + 1];
                                    o[m] = (u32x4){vperm(H[2 * m], l0, 0x05010400u), vperm(H[2 * m], l0, 0x07030602u),
                                                   vperm(H[2 * m + 1], l1, 0x05010400u), vperm(H[2 * m + 1], l1, 0x07030602u)};
                                }
                                if (h1 & 0x80u) {
#pragma unroll
                                    for (int m = 0; m < 4; ++m)
#pragma unroll
                                        for (int j = 0; j < 4; ++j) o[m][j] = pk_add16(o[m][j], 0xff80ff80u);
                                }
                            } else {
#pragma unroll
                                for (int m = 0; m < 4; ++m) o[m] = d[u][m];
                            }
                            if (!PUB2 && u == 0 && nst == 8) {
                                asm volatile("s_waitcnt vmcnt(8)" : "+v"(o[0]), "+v"(o[1]), "+v"(o[2]), "+v"(o[3]) :: "memory");
                                publish(pend);
                            }
                            const char* d0 = dst + u * HTB;
                            gst128s_sys<0>(d0, vl16, o[0]); gst128s_sys<1024>(d0, vl16, o[1]);
                            gst128s_sys<0>(d0 + 64 * HTB, vl16, o[2]); gst128s_sys<1024>(d0 + 64 * HTB, vl16, o[3]);
                        }
                        pend = eA[3]; nst = 8;
                    } else {
                        const bool bad = (unsigned)__builtin_amdgcn_readlane((int)hvA, 8) == call
                                         || (unsigned)__builtin_amdgcn_readlane((int)hvA, 1) >> 8 == POISON
                                         || (unsigned)__builtin_amdgcn_readlane((int)hvA, 5) >> 8 == POISON;
                        if (!PUB2 && nst == 8) { WAIT_V(8); publish(pend); }
                        if (UNLIKELY(bad)) nan_fill(eA[2]);
                        else {
                            float a[4][8];
#pragma unroll
                            for (int k = 0; k < 2; ++k) {
                                const unsigned h1 = hu[k];
                                if (UNLIKELY(h1 >> 8 == RAWMODE)) {
#pragma unroll
                                    for (int m = 0; m < 4; ++m)
#pragma unroll
                                        for (int j = 0; j < 4; ++j) {
                                            const unsigned v = d[k][m][j];
                                            const float lo = __uint_as_float(v << 16), hi = __uint_as_float(v & 0xffff0000u);
                                            a[m][2 * j] = k == 0 ? lo : a[m][2 * j] + lo; a[m][2 * j + 1] = k == 0 ? hi : a[m][2 * j + 1] + hi;
                                        }
                                } else {
                                    unsigned H[8] = {d[k][2][0], d[k][2][1], d[k][2][2], d[k][2][3], d[k][3][0], d[k][3][1], d[k][3][2], d[k][3][3]};
                                    decode_high(cd[k][0], cd[k][1], ce[k], h1, H);
                                    unsigned vb[4][8];
#pragma unroll
                                    for (int m = 0; m < 4; ++m)
#pragma unroll
                                        for (int j = 0; j < 8; ++j) {
                                            const unsigned Lw = d[k][m >> 1][(m & 1) * 2 + (j >> 2)];
                                            vb[m][j] = vperm(H[2 * m + (j >> 2)], Lw, (4u + (j & 3)) << 24 | (unsigned)(j & 3) << 16 | 0x0c0cu);
                                        }
                                    if (h1 & 0x80u) {
#pragma unroll
                                        for (int m = 0; m < 4; ++m)
#pragma unroll
                                            for (int j = 0; j < 8; ++j) vb[m][j] -= 0x00800000u;
                                    }
#pragma unroll
                                    for (int m = 0; m < 4; ++m)
#pragma unroll
                                        for (int j = 0; j < 8; ++j) a[m][j] = k == 0 ? __uint_as_float(vb[m][j]) : a[m][j] + __uint_as_float(vb[m][j]);
                                }
                            }
#pragma unroll
                            for (int m = 0; m < 4; ++m) {
                                unsigned r0 = cvtpk(a[m][0], a[m][1]), r1 = cvtpk(a[m][2], a[m][3]), r2 = cvtpk(a[m][4], a[m][5]), r3 = cvtpk(a[m][6], a[m][7]);
                                auto s02 = __builtin_amdgcn_permlane16_swap(r0, r2, false, false);
                                auto s13 = __builtin_amdgcn_permlane16_swap(r1, r3, false, false);
                                u32x4 o; o.x = s02[0]; o.y = s13[0]; o.z = s02[1]; o.w = s13[1];
                                st128<OAUX>(o, ro, olane + eA[2], m * 16 * NN * 2);
                            }
                        }
                        STAMPJ(8 + oc, qA - Q_OUT);
                        if (qA == o1) STAMPW(32);
                        nst = 4;
                    }
                    qA = qB; qB = qC; qC = qD; eA = eB; eB = eC; eC = eD; hvA = hvB; hvB = hvC; sl ^= 8192;
                    if (!more) break;
                }
                if (nst == 8) { WAIT_V(0); publish(pend); }
            }
            if (dead) {
                while (qA < Q_END) {
                    if (qA >= Q_OUT) nan_fill(eA[2]);
                    WAIT_V(0);
                    const int qD = next(qC, ticket_value(tk));
                    TICKET(tk, qD < Q_OUT ? tki : tko);
                    qA = qB; qB = qC; qC = qD; eA = eB; eB = eC; eC = entry(qD);
                }
            }
            WAIT_V(0);
        }
        STAMPW(40);
    }
}

}

extern "C" void kernel_launch(int rank, int nranks, ncclComm_t comm,
                              hipStream_t stream,
                              void* const* d_in, const int* in_sizes, int n_in,
                              void* d_out, int out_size,
                              const int* in_dtypes, int out_dtype,
                              void* const* d_out_all,
                              void* const* d_ws_all, size_t ws_bytes) {
    (void)comm; (void)in_dtypes; (void)out_dtype; (void)d_out_all;
    if (nranks != NR || n_in < 2 || in_sizes[0] != MM * KL || in_sizes[1] != KL * NN || out_size != (MM / NR) * NN
        || ws_bytes < WS_NEED || (unsigned)rank >= (unsigned)NR) {
        fprintf(stderr, "kernel_launch: unexpected shapes\n");
        return;
    }
    static bool inited[NR] = {};
    if (!inited[rank]) {
        HIP_CHECK(hipFuncSetAttribute((const void*)gemm_rs_kernel, hipFuncAttributeMaxDynamicSharedMemorySize, LDS_BYTES));
        inited[rank] = true;
    }
    Params p;
    p.x = d_in[0]; p.w = d_in[1]; p.out = d_out; p.rank = rank;
    for (int i = 0; i < NR; ++i) p.ws[i] = (char*)d_ws_all[i];
    hipLaunchKernelGGL(gemm_rs_kernel, dim3(NWG), dim3(NTHR), LDS_BYTES, stream, p);
    HIP_CHECK(hipGetLastError());
}
```

```cpp
#include <hip/hip_runtime.h>
#include <rccl/rccl.h>
#include <cstdio>
#include <cstdint>

#define HIP_CHECK(expr) do { hipError_t _e = (expr); if (_e != hipSuccess) { \
    fprintf(stderr, "HIP error %s at %s:%d\n", hipGetErrorString(_e), __FILE__, __LINE__); \
    return; } } while (0)

#define LAS __attribute__((address_space(3)))
#define UNLIKELY(x) __builtin_expect(!!(x), 0)
typedef short bf16x8 __attribute__((ext_vector_type(8)));
typedef short s16x4 __attribute__((ext_vector_type(4)));
typedef float f32x4 __attribute__((ext_vector_type(4)));
typedef float f32x2 __attribute__((ext_vector_type(2)));
typedef unsigned u32x4 __attribute__((ext_vector_type(4)));
typedef unsigned u32x3 __attribute__((ext_vector_type(3)));
typedef unsigned u32x2 __attribute__((ext_vector_type(2)));
typedef unsigned short u16x2 __attribute__((ext_vector_type(2)));
typedef __bf16 bf16x2_t __attribute__((ext_vector_type(2)));
typedef __amdgpu_buffer_rsrc_t rsrc_t;

namespace {

constexpr int NR = 8;
constexpr int MM = 8192, NN = 8192, KL = 1024;
constexpr int NWG = 256, NTHR = 512;
#define PN1 13
#define LOOK 2
#define CW 3
#define LATE 1
#define PUB2 1
#define LEAD 1
#define GSLEEP 2
#define CSLEEP 2
#define FASTP 1
#define FASTC 1
#define URG_LO 0
#define URG_HI 4
#define PKSLEEP 0
#define FORCE_RAW 0
#define NOGEMM 0
#define GAUX 17
#define GAUX2 GAUX
#define G2X 0
#define NOGEMM2 0
#define OAUX 0
constexpr int NGEMM = 16 * PN1, NCOMM = NWG - NGEMM, NCW = NCOMM * CW;
constexpr int PN2 = 16 - PN1;
constexpr int HTB = 16384, STAGE_BYTES = 8 * HTB;
constexpr int NT = 64;
constexpr unsigned UNIT_B = 4096;
constexpr size_t MIB = 1u << 20;
constexpr size_t OFF_PACK = 0;
constexpr unsigned PACK_B = 32u << 20;
constexpr size_t OFF_STG = 64 * MIB;
constexpr unsigned STG_B = 32u << 20;
constexpr size_t OFF_CP = 128 * MIB;
constexpr unsigned CP_B = 32u << 20;
constexpr unsigned REC_B = 96;
constexpr size_t OFF_IH = 164 * MIB;
constexpr unsigned IH_B = 2 * 16 * 8 * 64 * REC_B;
constexpr size_t OFF_RDY = 161 * MIB;
constexpr size_t OFF_CH = 162 * MIB;
constexpr size_t CTRL_OFF = 256 * MIB;
constexpr size_t WS_NEED = CTRL_OFF + MIB;
constexpr unsigned C_TMO = 256, C_CNT = 4096;
constexpr unsigned C_TKT = 16384;
constexpr unsigned MAXEXC = 64, RECN = 41, RAWMODE = 255, POISON = 254;
constexpr int OFF_CODE = 2048, OFF_LIST = 2816;
constexpr unsigned long long DEADLINE = 30000000ull;
constexpr int LDS_AUX = STAGE_BYTES;
constexpr int LDS_BAD = STAGE_BYTES + 8192;
constexpr int LDS_REC = LDS_BAD + 128;
constexpr int LDS_BYTES = LDS_REC + 6 * 8 * REC_B;
constexpr int AUX_SYS = 17;
constexpr int E1 = 16 + PN1;
constexpr int NJ1 = (NT / 2) * E1, NJ_IN = (NT / 2) * 32;
#define NFB 4
#define SKD 3
constexpr int EF = 8 + NFB, ES = E1 - EF;
constexpr int Q_OUT = NJ_IN, Q_END = Q_OUT + 1024;
constexpr int LDS_PAT = 98304 + 2048;
constexpr int LDS_TAB = 65536;
__host__ __device__ constexpr int in_job(int j) {
    constexpr int R1 = SKD, R2 = NT / 2 - SKD;
    int i = 0, r = 0;
    if (j < R1 * EF) { i = j % EF; r = j / EF; }
    else if (j < R1 * EF + R2 * E1) { const int q = j - R1 * EF; i = q % E1; r = q / E1 + (i < EF ? SKD : 0); }
    else if (j < NJ1) { const int q = j - (R1 * EF + R2 * E1); i = EF + q % ES; r = NT / 2 - SKD + q / ES; }
    else { const int q = j - NJ1; i = E1 + q % PN2; r = q / PN2; }
    const int e = i < 8 ? 2 * i : i < EF ? 8 + i : i < 8 + EF ? 2 * (i - EF) + 1 : i;
    return e * 64 + 2 * r;
}
__host__ __device__ constexpr int out_tile(int o) { const int y = o >> 5; return y < NFB ? 2 * y : y < 2 * NFB ? 2 * (y - NFB) + 1 : y; }
#define W_IN 14
#define NACT 120

#define STAMP(slot) do {} while (0)
#define STAMPW(slot) do {} while (0)
#define STAMPJ(row, j) do {} while (0)
#define DIAG_BLK0 do {} while (0)
#define DIAG_BLK1(t0) do {} while (0)
struct Params {
    const void* x; const void* w; void* out;
    char* ws[NR];
    int rank;
};

__device__ __forceinline__ unsigned cvtpk(float lo, float hi) { f32x2 v = {lo, hi}; bf16x2_t b = __builtin_convertvector(v, bf16x2_t); return __builtin_bit_cast(unsigned, b); }
__device__ __forceinline__ rsrc_t mk_rsrc(const void* p, unsigned bytes) { return __builtin_amdgcn_make_buffer_rsrc((void*)p, 0, bytes, 0x00020000); }

template <int AUX> __device__ __forceinline__ void st128(u32x4 v, rsrc_t r, unsigned voff, unsigned soff) {
    __builtin_amdgcn_raw_buffer_store_b128(v, r, voff, soff, AUX);
    asm volatile("s_nop 1" :: "v"(v) : "memory");
}
__device__ __forceinline__ void gst128_sys(void* ptr, u32x4 v) {
    asm volatile("global_store_dwordx4 %0, %1, off sc0 sc1\n\ts_nop 1" :: "v"(ptr), "v"(v) : "memory");
}
__device__ __forceinline__ unsigned gld32_sys(const void* ptr) {
    unsigned v;
    asm volatile("global_load_dword %0, %1, off sc0 sc1\n\ts_waitcnt vmcnt(0)" : "=&v"(v) : "v"(ptr) : "memory");
    return v;
}
__device__ __forceinline__ void gld32x2_sys(const void* pa, const void* pb, unsigned& a, unsigned& b) {
    asm volatile("global_load_dword %0, %2, off sc0 sc1\n\tglobal_load_dword %1, %3, off sc0 sc1\n\ts_waitcnt vmcnt(0)"
                 : "=&v"(a), "=&v"(b) : "v"(pa), "v"(pb) : "memory");
}
template <int OFF> __device__ __forceinline__ void gst128s_sys(const void* base, unsigned voff, u32x4 v) {
    asm volatile("global_store_dwordx4 %0, %1, %2 offset:%c3 sc0 sc1\n\ts_nop 1" :: "v"(voff), "v"(v), "s"(base), "i"(OFF) : "memory");
}
__device__ __forceinline__ unsigned bfi(unsigned m, unsigned a, unsigned b) { return (a & m) | (b & ~m); }
__device__ __forceinline__ unsigned vperm(unsigned hi, unsigned lo, unsigned sel) { return __builtin_amdgcn_perm(hi, lo, sel); }
__device__ __forceinline__ unsigned rfl(unsigned v) { return __builtin_amdgcn_readfirstlane(v); }
__device__ __forceinline__ bool timed_out(unsigned* tmo, unsigned call, unsigned long long t0) {
    if (__hip_atomic_load(tmo, __ATOMIC_RELAXED, __HIP_MEMORY_SCOPE_AGENT) == call) return true;
    if (__builtin_amdgcn_s_memrealtime() - t0 > DEADLINE) { __hip_atomic_store(tmo, call, __ATOMIC_RELAXED, __HIP_MEMORY_SCOPE_AGENT); return true; }
    return false;
}

__device__ __forceinline__ unsigned wave_sum_u(unsigned x) {
    x += (unsigned)__builtin_amdgcn_update_dpp(0, (int)x, 0xB1, 0xf, 0xf, true);
    x += (unsigned)__builtin_amdgcn_update_dpp(0, (int)x, 0x4E, 0xf, 0xf, true);
    x += (unsigned)__builtin_amdgcn_update_dpp(0, (int)x, 0x141, 0xf, 0xf, true);
    x += (unsigned)__builtin_amdgcn_update_dpp(0, (int)x, 0x140, 0xf, 0xf, true);
    return (unsigned)(__builtin_amdgcn_readlane((int)x, 0) + __builtin_amdgcn_readlane((int)x, 16) + __builtin_amdgcn_readlane((int)x, 32) + __builtin_amdgcn_readlane((int)x, 48));
}
__device__ __forceinline__ unsigned pk_add16(unsigned a, unsigned b) { return __builtin_bit_cast(unsigned, __builtin_bit_cast(u16x2, a) + __builtin_bit_cast(u16x2, b)); }
__device__ __forceinline__ unsigned mbcnt(unsigned long long m) { return __builtin_amdgcn_mbcnt_hi((unsigned)(m >> 32), __builtin_amdgcn_mbcnt_lo((unsigned)m, 0u)); }
__device__ __forceinline__ unsigned list_pos(unsigned i, bool rec) {
    return i + (rec ? 2u : 0u) + (__umul24(i + 1, rec ? 37u : 0u) >> 8);
}
__device__ __forceinline__ void encode_unit(unsigned (&w)[4][4], const char* dst, unsigned lst, unsigned rec, unsigned tag, int lane) {
    unsigned h1;
    unsigned sum = 0;
#pragma unroll
    for (int m = 0; m < 4; ++m) sum = __builtin_amdgcn_udot2(__builtin_bit_cast(u16x2, w[m][0] & 0x7f807f80u), (u16x2){1, 1}, sum, false);
    const unsigned T = (wave_sum_u(sum) + 185467u) >> 16;
    const unsigned par = ~T & 1u, top = (T + par) >> 1;
    if (par) {
#pragma unroll
        for (int m = 0; m < 4; ++m)
#pragma unroll
            for (int q = 0; q < 4; ++q) w[m][q] = pk_add16(w[m][q], 0x00800080u);
    }
    const unsigned bq = (top | 0x80u) * 0x01010101u;
    unsigned L[8], d[8], c[8], fl[4];
#pragma unroll
    for (int g = 0; g < 8; ++g) {
        const unsigned lo = w[g >> 1][(g & 1) * 2], hi = w[g >> 1][(g & 1) * 2 + 1];
        L[g] = vperm(hi, lo, 0x06040200u);
        const unsigned h = vperm(hi, lo, 0x07050301u);
        d[g] = bq - (h & 0x7f7f7f7fu);
        c[g] = (h >> 5 & 0x04040404u) | (d[g] & 0x03030303u);
        const unsigned f = (((d[g] >> 2 & 0x3f3f3f3fu) ^ 0x20202020u) + 0x7f7f7f7fu) & 0x80808080u;
        fl[g >> 1] = __builtin_amdgcn_udot4(f, g & 1 ? 0x80402010u : 0x08040201u, g & 1 ? fl[g >> 1] : 0u, false);
    }
    unsigned ex = fl[0] >> 7 | fl[1] << 1 | fl[2] << 9 | fl[3] << 17;
    const unsigned n = (unsigned)__builtin_popcount(ex);
    const unsigned long long b0 = __ballot(n & 1), b1 = __ballot(n & 2), b2 = __ballot(n & 4);
    unsigned cnt = (unsigned)(__builtin_popcountll(b0) + 2 * __builtin_popcountll(b1) + 4 * __builtin_popcountll(b2));
    if (FORCE_RAW || top > 123 || cnt > MAXEXC || __ballot(n >= 8) != 0) cnt = RAWMODE;
    if (!UNLIKELY(cnt == RAWMODE)) {
        const bool inrec = cnt <= RECN;
        const unsigned lb = inrec ? rec : lst;
        unsigned li = mbcnt(b0) + 2 * mbcnt(b1) + 4 * mbcnt(b2), inv = 0;
        while (__ballot(ex != 0) != 0) {
            const unsigned j = (unsigned)__builtin_ctz(ex | 0x80000000u) , sel = (j & 7) | 0x0c0c0c00u;
            const unsigned r0 = vperm(d[1], d[0], sel), r1 = vperm(d[3], d[2], sel), r2 = vperm(d[5], d[4], sel), r3 = vperm(d[7], d[6], sel);
            const unsigned ra = j & 8 ? r1 : r0, rb = j & 8 ? r3 : r2, db = j & 16 ? rb : ra;
            const unsigned q = ((db >> 2) + 1) ^ 0x20u;
            inv |= q;
            const unsigned en = (j & 16) << 6 | (unsigned)lane << 4 | (j & 15) | q << 11;
            if (ex) asm volatile("ds_write_b16 %0, %1" :: "v"(lb + 2 * list_pos(li, inrec)), "v"(en) : "memory");
            li += 1; ex &= ex - 1;
        }
        if (__ballot(inv > 31) != 0) cnt = RAWMODE;
    }
    const unsigned vo = (unsigned)lane * 16u;
    if (UNLIKELY(cnt == RAWMODE)) {
        const unsigned back = par ? 0xff80ff80u : 0u;
#pragma unroll
        for (int m = 0; m < 4; ++m)
#pragma unroll
            for (int q = 0; q < 4; ++q) w[m][q] = pk_add16(w[m][q], back);
        gst128s_sys<0>(dst, vo, (u32x4){w[0][0], w[0][1], w[0][2], w[0][3]});
        gst128s_sys<1024>(dst, vo, (u32x4){w[1][0], w[1][1], w[1][2], w[1][3]});
        gst128s_sys<2048>(dst, vo, (u32x4){w[2][0], w[2][1], w[2][2], w[2][3]});
        gst128s_sys<3072>(dst, vo, (u32x4){w[3][0], w[3][1], w[3][2], w[3][3]});
        h1 = RAWMODE << 8;
    } else {
        u32x4 lv;
        asm volatile("ds_read_b128 %0, %1" : "=&v"(lv) : "v"(lst + ((unsigned)lane & 7u) * 16u) : "memory");
        gst128s_sys<0>(dst, vo, (u32x4){L[0], L[1], L[2], L[3]});
        gst128s_sys<1024>(dst, vo, (u32x4){L[4], L[5], L[6], L[7]});
        const u32x3 cd = {c[0] | c[1] << 3 | (c[6] << 6 & 0xc0c0c0c0u), c[2] | c[3] << 3 | (c[7] << 6 & 0xc0c0c0c0u),
                          c[4] | c[5] << 3 | (c[6] << 4 & 0x40404040u) | (c[7] << 5 & 0x80808080u)};
        asm volatile("global_store_dwordx3 %0, %1, %2 offset:%c3 sc0 sc1\n\ts_nop 1" :: "v"((unsigned)lane * 12u), "v"(cd), "s"(dst), "i"(OFF_CODE) : "memory");
        asm volatile("s_waitcnt lgkmcnt(0)" : "+v"(lv) :: "memory");
        if (lane < 8) gst128s_sys<OFF_LIST>(dst, vo, lv);
        h1 = top | par << 7 | cnt << 8;
    }
    if (lane < 7) asm volatile("ds_write_b16 %0, %1" :: "v"(rec + (lane < 6 ? (unsigned)lane * 16u : 2u)), "v"(lane < 6 ? tag : h1) : "memory");
}
__device__ __forceinline__ void decode_high(unsigned d0, unsigned d1, unsigned d2, unsigned h1, unsigned (&H)[8]) {
    const unsigned lutl = ((h1 & 0x7fu) + 4) * 0x01010101u - 0x03020100u, luth = lutl | 0x80808080u;
    const unsigned c[8] = {d0 & 0x07070707u, d0 >> 3 & 0x07070707u, d1 & 0x07070707u, d1 >> 3 & 0x07070707u, d2 & 0x07070707u, d2 >> 3 & 0x07070707u,
                           (d0 >> 6 & 0x03030303u) | (d2 >> 4 & 0x04040404u), (d1 >> 6 & 0x03030303u) | (d2 >> 5 & 0x04040404u)};
#pragma unroll
    for (int g = 0; g < 8; ++g) H[g] = vperm(luth, lutl, c[g]) - H[g];
}

#define TRD(dst, addr, off) asm volatile("ds_read_b64_tr_b16 %0, %1 offset:%c2" : "=&v"(dst) : "v"(addr), "i"(off) : "memory")
#define PKB(x) (bf16x8){(x)[0][0], (x)[0][1], (x)[0][2], (x)[0][3], (x)[1][0], (x)[1][1], (x)[1][2], (x)[1][3]}
#define WAIT_V(n) asm volatile("s_waitcnt vmcnt(" #n ")" ::: "memory")
#define WAIT_L(n) asm volatile("s_waitcnt lgkmcnt(" #n ")" ::: "memory")
#define BAR __builtin_amdgcn_s_barrier()
#define SCHED __builtin_amdgcn_sched_barrier(0)
#define DSR128(dst, addr, off) asm volatile("ds_read_b128 %0, %1 offset:%c2" : "=&v"(dst) : "v"(addr), "i"(off) : "memory")
#define DSR32(dst, addr) asm volatile("ds_read_b32 %0, %1" : "=&v"(dst) : "v"(addr) : "memory")
#define DMA16(rs, ldsoff, voff, soff) __builtin_amdgcn_raw_ptr_buffer_load_lds(rs, (LAS void*)(lds + (ldsoff)), 16, voff, soff, 0, AUX_SYS)
#define TICKET(tk, ptr) asm volatile("s_mov_b64 exec, 1\n\tglobal_atomic_add %0, %1, %2, %3 sc0 sc1\n\ts_mov_b64 exec, -1" : "=&v"(tk) : "v"(0u), "v"(1u), "s"(ptr) : "memory")
__device__ __forceinline__ unsigned ticket_value(unsigned tk) {
    unsigned r;
    asm volatile("v_readfirstlane_b32 %0, %1" : "=s"(r) : "v"(tk) : "memory");
    return r;
}

__global__ __launch_bounds__(NTHR, 2) void gemm_rs_kernel(Params p) {
    extern __shared__ __attribute__((aligned(16))) unsigned char smem[];
    LAS unsigned char* lds = (LAS unsigned char*)smem;
    const int tid = threadIdx.x, wid = __builtin_amdgcn_readfirstlane(tid >> 6), lane = tid & 63;
    const int wr = wid >> 2, wc = wid & 3, fr = lane & 15, fq = lane >> 4;
    const int wg = blockIdx.x, rank = p.rank;
    const int bi = rank & 1, bj_ = (rank >> 1) & 1, bk = rank >> 2;
    char* myws = p.ws[rank];
    unsigned* ctl = (unsigned*)(myws + CTRL_OFF);
    unsigned* tmo = ctl + C_TMO / 4;

    int dslot = 0;
    STAMP(dslot); ++dslot;
    const unsigned ldsa = (unsigned)(size_t)lds;
    unsigned* cntp = ctl + C_CNT / 4 + wg * 8 + wid;

    if (wg < NGEMM) {
        const rsrc_t rx = mk_rsrc(p.x, MM * KL * 2), rw = mk_rsrc(p.w, KL * NN * 2);
        const unsigned vx = (unsigned)((tid >> 3) * (KL * 2) + (((tid & 7) ^ ((tid >> 4) & 7)) << 4));
        const unsigned vw = (unsigned)(fq * (NN * 2) + ((fr ^ ((fq << 2) | ((rank & 1) << 1))) << 4));
        auto pk_issue = [&](int idx, int buf) {
            const int g8 = idx >> 7, mat = (idx >> 6) & 1, kk = (idx >> 5) & 1, blk = idx & 31;
            if (mat == 0) {
#pragma unroll
                for (int i = 0; i < 4; ++i)
                    __builtin_amdgcn_raw_ptr_buffer_load_lds(rx, (LAS void*)(lds + buf + i * 8192 + wid * 1024), 16, vx,
                                                             (unsigned)((blk * 256 + i * 64) * (KL * 2) + kk * 1024 + g8 * 128), 0, 0);
            } else {
#pragma unroll
                for (int c = 0; c < 4; ++c)
                    __builtin_amdgcn_raw_ptr_buffer_load_lds(rw, (LAS void*)(lds + buf + wid * 4096 + c * 1024), 16, vw,
                                                             (unsigned)((kk * 512 + (g8 * 8 + wid) * 8 + (c & 1) * 4) * (NN * 2) + blk * 512 + (c >> 1) * 256), 0, 0);
            }
        };
        pk_issue(wg, 0);
        volatile LAS unsigned* const badf = (volatile LAS unsigned*)(lds + LDS_BAD);
        if (tid < 16) asm volatile("ds_write_b32 %0, %1" :: "v"(ldsa + LDS_BAD + tid * 4), "v"(0u) : "memory");
        const unsigned call = rfl(gld32_sys(cntp)) + 1;
        if (lane == 0) __hip_atomic_store(cntp, call, __ATOMIC_RELAXED, __HIP_MEMORY_SCOPE_AGENT);
        const unsigned par = call & 1;
        {
            const char* const pk = myws + OFF_PACK + (size_t)par * PACK_B;
            const rsrc_t rs = mk_rsrc(myws + OFF_STG, 2 * STG_B);
            int buf = 0;
            const unsigned tag = call & 0xffffu;
            auto pk_headers = [&](int idx, int b) {
                const int g8 = idx >> 7, mat = (idx >> 6) & 1, kk = (idx >> 5) & 1, blk = idx & 31;
                const int cb = mat == 0 ? ((blk >> 2) * 2 + (blk & 1)) : (blk & 15);
                const int c0 = mat == 0 ? ((blk >> 1) & 1) + 4 * kk : 2 * (blk >> 4) + 4 * kk, c1 = c0 + (mat == 0 ? 2 : 1);
                char* const w0 = p.ws[c0 & 7];
                char* const w1 = p.ws[c1 & 7];
                if (lane < 48) {
                    u32x4 h;
                    asm volatile("ds_read_b128 %0, %1\n\ts_waitcnt lgkmcnt(0)" : "=&v"(h) : "v"(ldsa + LDS_REC + (b >> 15) * (8 * REC_B) + lane * 16) : "memory");
                    const size_t o = OFF_IH + (size_t)par * IH_B + (size_t)((((mat * 16 + cb) * NR + rank) * 64 + g8 * 8) * REC_B);
                    if (c0 != rank) gst128s_sys<0>(w0 + o, lane * 16, h);
                    if (c1 != rank) gst128s_sys<0>(w1 + o, lane * 16, h);
                }
            };
            for (int idx = wg; idx < 1024; idx += NGEMM, buf ^= 32768) {
                const int g8 = idx >> 7, mat = (idx >> 6) & 1, kk = (idx >> 5) & 1, blk = idx & 31, t = g8 * 8 + wid;
                BAR;
                if (wid == 0 && idx != wg) pk_headers(idx - NGEMM, buf ^ 32768);
                if (idx + NGEMM < 1024) pk_issue(idx + NGEMM, buf ^ 32768);
                const bool urgent = FASTP && idx < 128 && wid >= URG_LO && wid < URG_HI;
                if (urgent) __builtin_amdgcn_s_setprio(3);
                u32x4 v[4];
                if (mat == 0) {
                    const unsigned a = ldsa + buf + lane * 128 + ((wid ^ ((lane >> 1) & 7)) << 4);
                    DSR128(v[0], a, 0); DSR128(v[1], a, 8192); DSR128(v[2], a, 16384); DSR128(v[3], a, 24576);
                } else {
                    const unsigned a = ldsa + buf + wid * 4096 + lane * 16;
                    DSR128(v[0], a, 0); DSR128(v[1], a, 1024); DSR128(v[2], a, 2048); DSR128(v[3], a, 3072);
                }
                asm volatile("s_waitcnt lgkmcnt(0)" : "+v"(v[0]), "+v"(v[1]), "+v"(v[2]), "+v"(v[3]) :: "memory");
                const int cb = mat == 0 ? ((blk >> 2) * 2 + (blk & 1)) : (blk & 15);
                const int c0 = mat == 0 ? ((blk >> 1) & 1) + 4 * kk : 2 * (blk >> 4) + 4 * kk, c1 = c0 + (mat == 0 ? 2 : 1);
                char* const w0 = p.ws[c0 & 7];
                char* const w1 = p.ws[c1 & 7];
                const unsigned fo = (unsigned)(OFF_RDY + (((mat * 16 + cb) * 64 + t) * NR + rank) * 4);
                unsigned w[4][4];
#pragma unroll
                for (int c = 0; c < 4; ++c) { w[c][0] = v[c][0]; w[c][1] = v[c][1]; w[c][2] = v[c][2]; w[c][3] = v[c][3]; }
                const unsigned rec = ldsa + LDS_REC + (buf >> 15) * (8 * REC_B) + wid * REC_B;
                encode_unit(w, pk + (unsigned)(((mat * 2 + kk) * 32 + blk) * 64 + t) * UNIT_B, ldsa + LDS_AUX + wid * 1024, rec, tag, lane);
                const bool own = c0 == rank || c1 == rank;
                if (own) {
                    const unsigned dst = (unsigned)mat * STG_B + (unsigned)(2 * cb * 64 + t) * HTB + (unsigned)rank * 2048u;
#pragma unroll
                    for (int m = 0; m < 4; ++m) st128<AUX_SYS>(v[m], rs, lane * 16, dst + (m >> 1) * (64 * HTB) + (m & 1) * 1024);
                }
                WAIT_V(0);
                if (own) { if (lane == 0) __hip_atomic_store((unsigned*)(myws + fo), call, __ATOMIC_RELAXED, __HIP_MEMORY_SCOPE_SYSTEM); }
                if (urgent && lane < 12 && (lane < 6 ? c0 : c1) != rank) {
                    u32x4 h;
                    asm volatile("ds_read_b128 %0, %1\n\ts_waitcnt lgkmcnt(0)" : "=&v"(h) : "v"(rec + (lane < 6 ? lane : lane - 6) * 16) : "memory");
                    gst128_sys((lane < 6 ? w0 : w1) + OFF_IH + (size_t)par * IH_B + (size_t)((((mat * 16 + cb) * NR + rank) * 64 + t) * REC_B + (lane < 6 ? lane : lane - 6) * 16), h);
                }
                if (urgent) __builtin_amdgcn_s_setprio(0);
                WAIT_L(0);
            }
            BAR;
            if (wid == 0) pk_headers(wg + (1023 - wg) / NGEMM * NGEMM, buf ^ 32768);
        }
        STAMP(dslot); ++dslot;
        const int pm = 2 * (wg & 7) + ((wg >> 3) & 1), pn = wg >> 4;
        const rsrc_t ra = mk_rsrc(myws + OFF_STG, STG_B), rb = mk_rsrc(myws + OFF_STG + STG_B, STG_B);
        const unsigned voff = (unsigned)(wid * 1024 + lane * 16);
        constexpr unsigned S_K = HTB, S_H = NT * HTB, S_T = 2 * NT * HTB;
        const unsigned ldsw = (unsigned)wid * 1024u;
        const int aoff = fq * 2048 + (wr * 64 + fr) * 16;
        unsigned boff[2];
        { const int q = (lane & 15) >> 2, pp = lane & 3, g = fq;
#pragma unroll
          for (int n = 0; n < 2; ++n) boff[n] = (unsigned)(size_t)lds + 4 * HTB + 256 * (8 * g + q) + 16 * (((wc ^ q) << 2) | ((n ^ (g & 1)) << 1) | (pp >> 1)) + 8 * (pp & 1); }
        const unsigned* rdyf = (const unsigned*)(myws + OFF_RDY);
#define SA(b, h) (((b) * 2 + (h)) * HTB)
#define SB(b, h) ((4 + (b) * 2 + (h)) * HTB)
#define STAGEX(rs, bufoff, soff, aux) do { _Pragma("unroll") for (int _i = 0; _i < 2; ++_i) \
        __builtin_amdgcn_raw_ptr_buffer_load_lds(rs, (LAS void*)(lds + (bufoff) + ldsw + _i * 8192), 16, voff, (soff) + _i * 8192, 0, aux); } while (0)
#define STAGE(rs, bufoff, soff) STAGEX(rs, bufoff, soff, GAUX)
#define STAGE2(rs, bufoff, soff) STAGEX(rs, bufoff, soff, GAUX2)
#define LDA(dst, b, h) do { _Pragma("unroll") for (int m = 0; m < 4; ++m) _Pragma("unroll") for (int k = 0; k < 2; ++k) dst[m][k] = *(const LAS bf16x8*)(lds + SA(b, h) + aoff + m * 256 + k * 8192); } while (0)
#define LDB(dst, b, h) do { _Pragma("unroll") for (int n = 0; n < 2; ++n) _Pragma("unroll") for (int k = 0; k < 2; ++k) { \
        TRD(dst[n][k][0], boff[n], ((b) * 2 + (h)) * HTB + k * 8192); TRD(dst[n][k][1], boff[n], ((b) * 2 + (h)) * HTB + k * 8192 + 1024); } } while (0)
#define MMA(ai, bj, At, Bt) do { __builtin_amdgcn_s_setprio(1); _Pragma("unroll") for (int m = 0; m < 4; ++m) _Pragma("unroll") for (int n = 0; n < 2; ++n) _Pragma("unroll") for (int k = 0; k < 2; ++k) \
        acc[ai][bj][m][n] = __builtin_amdgcn_mfma_f32_16x16x32_bf16(PKB(Bt[n][k]), At[m][k], acc[ai][bj][m][n], 0, 0, 0); __builtin_amdgcn_s_setprio(0); } while (0)
#define WAIT_B(X, Y) asm volatile("s_waitcnt lgkmcnt(0)" : "+v"(X[0][0][0]), "+v"(X[0][0][1]), "+v"(X[0][1][0]), "+v"(X[0][1][1]), "+v"(X[1][0][0]), "+v"(X[1][0][1]), "+v"(X[1][1][0]), "+v"(X[1][1][1]), \
        "+v"(Y[0][0][0]), "+v"(Y[0][0][1]), "+v"(Y[0][1][0]), "+v"(Y[0][1][1]), "+v"(Y[1][0][0]), "+v"(Y[1][0][1]), "+v"(Y[1][1][0]), "+v"(Y[1][1][1]) :: "memory")
#define WAIT_B1(X) asm volatile("s_waitcnt lgkmcnt(0)" : "+v"(X[0][0][0]), "+v"(X[0][0][1]), "+v"(X[0][1][0]), "+v"(X[0][1][1]), "+v"(X[1][0][0]), "+v"(X[1][0][1]), "+v"(X[1][1][0]), "+v"(X[1][1][1]) :: "memory")
        const rsrc_t rr = mk_rsrc(myws + OFF_RDY, 2 * 16 * 64 * 32);
        const unsigned pvo = (unsigned)(((lane >> 5) * 16 + (lane < 32 ? pm : pn)) * 2048 + (lane & 1) * 16);
        const unsigned pla = ldsa + LDS_AUX + wid * 1024 + lane * 16;
#define NOGATE 0
        int rdy = NOGATE ? NT : 0, pbase = 0;
        bool pol = false;
        u32x4 fv;
#define COUNT_RDY(v) do { const bool _ok = (int)((v)[0] - call) >= 0 && (int)((v)[1] - call) >= 0 && (int)((v)[2] - call) >= 0 && (int)((v)[3] - call) >= 0; \
            const unsigned long long _b = __ballot(_ok); \
            unsigned _m = (unsigned)_b & (unsigned)(_b >> 32); _m = ~(_m & (_m >> 1)) & 0x55555555u; \
            rdy = min(pbase + (_m ? (int)(__builtin_ctz(_m) >> 1) : 16), NT); } while (0)
#define GATE(tt) do { if (wid == 4) { \
            if (pol) { DSR128(fv, pla, 0); asm volatile("s_waitcnt lgkmcnt(0)" : "+v"(fv) :: "memory"); COUNT_RDY(fv); } \
            if (UNLIKELY((int)(tt) >= rdy)) { \
                const unsigned long long _t0 = __builtin_amdgcn_s_memrealtime(); \
                DIAG_BLK0; \
                for (;;) { \
                    pbase = rdy; \
                    const char* _p = (const char*)rdyf + pvo + min(rdy + ((lane >> 1) & 15), NT - 1) * 32; \
                    u32x4 _v; unsigned _tm; \
                    asm volatile("global_load_dwordx4 %0, %2, off sc0 sc1\n\tglobal_load_dword %1, %3, off sc0 sc1\n\ts_waitcnt vmcnt(0)" \
                                 : "=&v"(_v), "=&v"(_tm) : "v"(_p), "v"(tmo) : "memory"); \
                    COUNT_RDY(_v); \
                    if ((int)(tt) < rdy) break; \
                    if (rfl(_tm) == call || __builtin_amdgcn_s_memrealtime() - _t0 > DEADLINE) { \
                        __hip_atomic_store(tmo, call, __ATOMIC_RELAXED, __HIP_MEMORY_SCOPE_AGENT); *badf = 1; rdy = NT; break; } \
                    __builtin_amdgcn_s_sleep(GSLEEP); \
                } DIAG_BLK1(_t0); } \
            pol = rdy < min((int)(tt) + LOOK, NT); \
            if (pol) { pbase = rdy; DMA16(rr, LDS_AUX + wid * 1024, pvo + min(rdy + ((lane >> 1) & 15), NT - 1) * 32, 0); } } } while (0)
#define FLAGREAD do {} while (0)
#define GATE1(tt) GATE(tt)
#define GATE2(tt) do {} while (0)
#define WAIT_VP do { if (pol) WAIT_V(9); else WAIT_V(8); } while (0)
        const int owner = pm >> 1;
        char* chd = p.ws[owner] + OFF_CH;
        f32x4 acc[2][2][4][2];
#pragma unroll
        for (int a = 0; a < 2; ++a)
#pragma unroll
            for (int b = 0; b < 2; ++b)
#pragma unroll
                for (int m = 0; m < 4; ++m)
#pragma unroll
                    for (int n = 0; n < 2; ++n) acc[a][b][m][n] = (f32x4){0.f, 0.f, 0.f, 0.f};
        bf16x8 At[4][2]; s16x4 B0[2][2][2], B1[2][2][2];
        const unsigned cA = (unsigned)pm * S_T, cB = (unsigned)pn * S_T;
        if (!NOGEMM) { GATE(1); }
        BAR;
        STAGE(rb, SB(0, 0), cB); STAGE(rb, SB(0, 1), cB + S_H); STAGE(ra, SA(0, 0), cA); STAGE(ra, SA(0, 1), cA + S_H);
        if (wr == 1) BAR;
        WAIT_V(2); BAR;
        STAGE(rb, SB(1, 0), cB + S_K); STAGE(ra, SA(1, 0), cA + S_K); STAGE(rb, SB(1, 1), cB + S_H + S_K);
        WAIT_V(6); BAR;
        STAMP(dslot); ++dslot;
        for (int t = 0; t < (NOGEMM ? 0 : NT); t += 2) {
            const int t2 = min(t + 2, NT - 1), t3 = min(t + 3, NT - 1);
            const unsigned a1 = cA + (unsigned)(t + 1) * S_K;
            const unsigned a2 = cA + (unsigned)t2 * S_K, b2 = cB + (unsigned)t2 * S_K, a3 = cA + (unsigned)t3 * S_K, b3 = cB + (unsigned)t3 * S_K;
            LDB(B0, 0, 0); LDB(B1, 0, 1); SCHED; LDA(At, 0, 0); STAGE(ra, SA(1, 1), a1 + S_H);
            WAIT_V(8); WAIT_B(B0, B1); FLAGREAD; GATE1(t2); BAR; MMA(0, 0, At, B0); MMA(0, 1, At, B1); BAR; SCHED;
            GATE2(t2);
            LDA(At, 0, 1); STAGE(rb, SB(0, 0), b2); STAGE(rb, SB(0, 1), b2 + S_H); STAGE(ra, SA(0, 0), a2);
            WAIT_VP; WAIT_L(0); BAR; MMA(1, 0, At, B0); MMA(1, 1, At, B1); BAR; SCHED;
            LDB(B0, 1, 0); LDB(B1, 1, 1); SCHED; LDA(At, 1, 0); STAGE(ra, SA(0, 1), a2 + S_H);
            WAIT_V(8); WAIT_B(B0, B1); FLAGREAD; GATE1(t3); BAR; MMA(0, 0, At, B0); MMA(0, 1, At, B1); BAR; SCHED;
            GATE2(t3);
            LDA(At, 1, 1); STAGE(rb, SB(1, 0), b3); STAGE(rb, SB(1, 1), b3 + S_H); STAGE(ra, SA(1, 0), a3);
            WAIT_VP; WAIT_L(0); BAR; MMA(1, 0, At, B0); MMA(1, 1, At, B1); BAR; SCHED;
        }
        if (wr == 0) BAR;
        STAMP(dslot); ++dslot;
#define EPI_GROUP(A, pnx, g, slot) do { unsigned _w[4][4]; \
            _Pragma("unroll") for (int m = 0; m < 4; ++m) { \
                _w[m][0] = cvtpk(A[m][0][0], A[m][0][1]); _w[m][1] = cvtpk(A[m][0][2], A[m][0][3]); \
                _w[m][2] = cvtpk(A[m][1][0], A[m][1][1]); _w[m][3] = cvtpk(A[m][1][2], A[m][1][3]); } \
            encode_unit(_w, myws + OFF_CP + (unsigned)(((pm * 16 + (pnx)) * 8 + wid) * 4 + (g)) * UNIT_B, ldsa + LDS_AUX + wid * 1024, \
                        ldsa + LDS_REC + (slot) * (8 * REC_B) + wid * REC_B, call & 0xffffu, lane); } while (0)
        auto epi_hdr = [&](int pnx, int g, int slot) {
            unsigned old, bad;
            asm volatile("s_mov_b64 exec, 1\n\tds_add_rtn_u32 %0, %1, %2\n\ts_mov_b64 exec, -1\n\ts_waitcnt lgkmcnt(0)"
                         : "=&v"(old) : "v"(ldsa + LDS_BAD + 16 + slot * 4), "v"(1u) : "memory");
            if (rfl(old) != 7) return false;
            u32x4 hv;
            asm volatile("ds_read_b128 %0, %2\n\tds_read_b32 %1, %3\n\ts_waitcnt lgkmcnt(0)"
                         : "=&v"(hv), "=&v"(bad) : "v"(ldsa + LDS_REC + slot * (8 * REC_B) + min(lane, 47) * 16), "v"(ldsa + LDS_BAD) : "memory");
            if (bad && lane % 6 == 0) hv[0] = (hv[0] & 0xffffu) | POISON << 24;
            if (lane < 48) gst128s_sys<0>(chd + (unsigned)((((((bi * 2 + (pm & 1)) * 32 + bj_ * 16 + pnx) * 4 + g) * 2 + bk) * 8) * REC_B), lane * 16, hv);
            return true;
        };
        {
            EPI_GROUP(acc[0][0], pn, 0, 0);
            EPI_GROUP(acc[0][1], pn, 1, 1);
            WAIT_V(4);
            const bool s0 = epi_hdr(pn, 0, 0);
            EPI_GROUP(acc[1][0], pn, 2, 2);
            EPI_GROUP(acc[1][1], pn, 3, 3);
            if (s0) WAIT_V(9); else WAIT_V(8);
            const bool s1 = epi_hdr(pn, 1, 1);
            if (s1) WAIT_V(5); else WAIT_V(4);
            const bool s2 = epi_hdr(pn, 2, 2);
            if (s1 && s2) WAIT_V(2); else if (s1 || s2) WAIT_V(1); else WAIT_V(0);
            epi_hdr(pn, 3, 3);
        }
        STAMP(dslot); ++dslot;
        for (int e = pn; e < 4 * PN2; e += PN1) {
            const int ai = e & 1, bj = (e >> 1) & 1, pn2 = PN1 + (e >> 2);
            if (!NOGEMM && !NOGATE) {
                const unsigned long long t0 = __builtin_amdgcn_s_memrealtime();
                const u32x4* f = (const u32x4*)(rdyf + ((16 + pn2) * 64) * NR) + lane;
                for (;;) {
                    u32x4 v0, v1;
                    asm volatile("global_load_dwordx4 %0, %2, off sc0 sc1\n\tglobal_load_dwordx4 %1, %2, off offset:1024 sc0 sc1\n\ts_waitcnt vmcnt(0)"
                                 : "=&v"(v0), "=&v"(v1) : "v"(f) : "memory");
                    bool ok = true;
#pragma unroll
                    for (int q = 0; q < 4; ++q) ok = ok && (int)(v0[q] - call) >= 0 && (int)(v1[q] - call) >= 0;
                    if (__all(ok)) break;
                    if (timed_out(tmo, call, t0)) { *badf = 1; break; }
                    __builtin_amdgcn_s_sleep(16);
                }
            }
            BAR;
            STAMP(dslot); ++dslot;
            const unsigned gA = (unsigned)(2 * pm + ai) * S_H, gB = (unsigned)(2 * pn2 + bj) * S_H;
#pragma unroll
            for (int m = 0; m < 4; ++m)
#pragma unroll
                for (int n = 0; n < 2; ++n) acc[0][0][m][n] = (f32x4){0.f, 0.f, 0.f, 0.f};
#pragma unroll
            for (int q = 0; q < 3; ++q) { STAGE2(ra, q * HTB, gA + q * S_K); STAGE2(rb, (4 + q) * HTB, gB + q * S_K); }
            for (int t = 0; t < (NOGEMM || NOGEMM2 ? 0 : NT); t += 4) {
#pragma unroll
                for (int q = 0; q < 4; ++q) {
                    const unsigned tn = (unsigned)min(t + q + 3, NT - 1) * S_K;
                    WAIT_V(8); BAR;
                    if (G2X != 1) { STAGE2(ra, ((q + 3) & 3) * HTB, gA + tn); STAGE2(rb, (4 + ((q + 3) & 3)) * HTB, gB + tn); }
                    LDB(B0, q >> 1, q & 1); SCHED; LDA(At, q >> 1, q & 1);
                    WAIT_B1(B0);
                    if (G2X != 2) MMA(0, 0, At, B0); else __builtin_amdgcn_s_sleep(6);
                    SCHED;
                }
            }
            WAIT_V(0);
            EPI_GROUP(acc[0][0], pn2, ai * 2 + bj, 4 + (e - pn) / PN1);
            WAIT_V(0);
            epi_hdr(pn2, ai * 2 + bj, 4 + (e - pn) / PN1);
            STAMP(dslot); ++dslot;
        }
#undef SA
#undef SB
    } else {
        unsigned cnt0;
        if (FASTC) asm volatile("global_load_dword %0, %1, off sc0 sc1" : "=&v"(cnt0) : "v"(cntp) : "memory");
        for (int q = tid; q <= Q_END; q += NTHR) {
            u32x4 en = {0u, 0u, 0u, 0u};
            if (q < Q_OUT) {
                const int c = in_job(q), e = c >> 6, st = c & 63, mat = e >> 4, blk = e & 15;
                const int gblk = mat == 0 ? (blk >> 1) * 4 + bi * 2 + (blk & 1) : bj_ * 16 + blk;
                en = (u32x4){(unsigned)(e * NR * 64 + st) * REC_B, (unsigned)(((mat * 2 + bk) * 32 + gblk) * 64 + st) * UNIT_B,
                             (unsigned)mat * STG_B + (unsigned)(2 * blk * 64 + st) * HTB, (unsigned)(c * NR * 4)};
            } else if (q < Q_END) {
                const int o = q - Q_OUT, y = out_tile(o), gg = (o >> 3) & 3, w8 = o & 7;
                en = (u32x4){(unsigned)((((y & 1) * 32 + (y >> 1)) * 4 + gg) * 16 + w8) * REC_B,
                             (unsigned)(((rank * 2 + (y & 1)) * 16 + (y >> 1)) * 32 + w8 * 4 + gg) * UNIT_B,
                             (unsigned)((((y & 1) * 256 + (gg >> 1) * 128 + (w8 >> 2) * 64) * NN + (y >> 1) * 256 + (gg & 1) * 128 + (w8 & 3) * 32) * 2), 0u};
            }
            asm volatile("ds_write_b128 %0, %1" :: "v"(ldsa + LDS_TAB + (unsigned)q * 16u), "v"(en) : "memory");
        }
        if (FASTC) asm volatile("s_waitcnt vmcnt(0)" : "+v"(cnt0) :: "memory"); else cnt0 = gld32_sys(cntp);
        const unsigned call = rfl(cnt0) + 1;
        if (lane == 0) __hip_atomic_store(cntp, call, __ATOMIC_RELAXED, __HIP_MEMORY_SCOPE_AGENT);
        const unsigned par = call & 1;
        unsigned* const tkt = ctl + C_TKT / 4 + par * 1024;
        if (wg == NGEMM && wid == 7 && lane < 16) __hip_atomic_store(ctl + C_TKT / 4 + (par ^ 1) * 1024 + lane * 64, 0u, __ATOMIC_RELAXED, __HIP_MEMORY_SCOPE_SYSTEM);
        WAIT_L(0);
        BAR;
        const rsrc_t rh = mk_rsrc(myws, (unsigned)WS_NEED);
        const unsigned lw = (unsigned)wid * 16384u, lh = LDS_AUX + (unsigned)wid * 1024u;
        const unsigned pat = ldsa + LDS_PAT + (unsigned)wid * 4096u;
        const unsigned vl16 = (unsigned)lane * 16u;
#define PULL_UNIT(rp, slot, so, h1) do { DMA16(rp, slot, vl16, so); DMA16(rp, (slot) + 1024, vl16, (so) + 1024); \
            if ((unsigned)lane < ((h1) >> 8 > RECN ? ((h1) >> 8 > MAXEXC ? 64u : 56u) : 48u)) DMA16(rp, (slot) + 2048, vl16, (so) + 2048); \
            if (UNLIKELY((h1) >> 8 == RAWMODE)) DMA16(rp, (slot) + 3072, vl16, (so) + 3072); } while (0)
#define READ_ENTRY(en, slot, rec, h1) asm volatile("ds_read_u16 %0, %1" : "=&v"(en) \
            : "v"(ldsa + ((h1) >> 8 <= RECN ? (rec) : (slot) + OFF_LIST) \
                  + 2 * list_pos(min((unsigned)lane, min(max((h1) >> 8, 1u), MAXEXC) - 1), (h1) >> 8 <= RECN)) : "memory")
#define READ_UNIT(d, cd, ce, slot, h1) asm volatile("ds_read_b128 %0, %6\n\tds_read_b128 %1, %6 offset:1024\n\t" \
            "s_cmp_eq_u32 %8, 255\n\ts_cbranch_scc1 1f\n\t" \
            "ds_read2_b32 %4, %7 offset1:1\n\tds_read_b32 %5, %7 offset:8\n\ts_branch 2f\n" \
            "1:\n\tds_read_b128 %2, %6 offset:2048\n\tds_read_b128 %3, %6 offset:3072\n" \
            "2:" : "=&v"(d[0]), "=&v"(d[1]), "=&v"(d[2]), "=&v"(d[3]), "=&v"(cd), "=&v"(ce) \
            : "v"(ldsa + (slot) + vl16), "v"(ldsa + (slot) + OFF_CODE + (unsigned)lane * 12u), "s"((h1) >> 8) : "scc", "memory")
#define PATCH_UNIT(d, en, u, h1) do { const unsigned _en = (h1) >> 8 == 0 ? 0x800u : (en), _ad = pat + (u) * 2048 + (_en & 0x7ffu); \
            asm volatile("s_cmp_eq_u32 %6, 255\n\ts_cbranch_scc1 1f\n\t" \
                         "ds_write_b8 %2, %3\n\tds_read_b128 %0, %4\n\tds_read_b128 %1, %4 offset:1024\n\tds_write_b8 %2, %5\n1:" \
                         : "+v"(d[2]), "+v"(d[3]) : "v"(_ad), "v"(_en >> 9 & 0x7cu), "v"(pat + (u) * 2048 + vl16), "v"(4u), "s"((h1) >> 8) : "scc", "memory"); } while (0)
#define WAIT_UNITS(d, cd, ce, hv, nx) asm volatile("s_waitcnt lgkmcnt(0)" : "+v"(d[0][0]), "+v"(d[0][1]), "+v"(d[0][2]), "+v"(d[0][3]), \
            "+v"(d[1][0]), "+v"(d[1][1]), "+v"(d[1][2]), "+v"(d[1][3]), "+v"(cd[0]), "+v"(cd[1]), "+v"(ce[0]), "+v"(ce[1]), \
            "+v"(hv), "+v"(nx) :: "memory")
#define NOCOMM 0
        if (const int cw = NACT == NCW ? (wg - NGEMM) * CW + wid : wid * NCOMM + (wg - NGEMM); wid < CW && cw < NACT && !NOCOMM) {
#pragma unroll
            for (int i = 0; i < 4; ++i) asm volatile("ds_write_b128 %0, %1" :: "v"(pat + i * 1024 + vl16), "v"((u32x4){0x04040404u, 0x04040404u, 0x04040404u, 0x04040404u}) : "memory");
            const int si = cw % (NR - 1), sq = cw / (NR - 1), s = (rank + 1 + si) & 7;
            constexpr int w_in = W_IN;
            const int oc = sq < NACT / 8 ? (s & 3) : (rank & 3);
            const int ti = oc & 1, tj = oc >> 1;
            const rsrc_t rp = mk_rsrc(p.ws[s] + OFF_PACK + (size_t)par * PACK_B, PACK_B);
            const rsrc_t rc0 = mk_rsrc(p.ws[oc] + OFF_CP, CP_B), rc1 = mk_rsrc(p.ws[oc + 4] + OFF_CP, CP_B);
            const rsrc_t ro = mk_rsrc(p.out, (MM / NR) * NN * 2);
            const unsigned hb_in = (unsigned)OFF_IH + par * IH_B + (unsigned)s * (64 * REC_B), hb_out = (unsigned)OFF_CH + (unsigned)((ti * 64 + tj * 16) * 64) * REC_B;
            const unsigned tag = call & 0xffffu;
            const char* const stg = myws + OFF_STG + (unsigned)s * 2048u;
            const char* const rdys = myws + OFF_RDY + (unsigned)s * 4u;
            const unsigned olane = (unsigned)(((ti * 512 + fr) * NN + tj * 4096 + (fq & 1) * 16 + (fq >> 1) * 8) * 2);
            auto hdr = [&](int q, unsigned x) {
                return lane < 12 ? (q < Q_OUT ? hb_in + (unsigned)lane * 16u : hb_out + (unsigned)lane * 16u + (lane < 6 ? 0u : 7 * REC_B)) + x
                                 : (unsigned)CTRL_OFF + C_TMO;
            };
            auto fetch_rec = [&](int q, unsigned x, unsigned slot) { if (lane < 13) DMA16(rh, slot, hdr(q, x), 0); };
            auto entry = [&](int q) {
                u32x4 en;
                asm volatile("ds_read_b128 %0, %1\n\ts_waitcnt lgkmcnt(0)" : "=&v"(en) : "v"(ldsa + LDS_TAB + (unsigned)q * 16u) : "memory");
                return (u32x4){rfl(en[0]), rfl(en[1]), rfl(en[2]), rfl(en[3])};
            };
            auto ready = [&](unsigned hv) { return ((unsigned)__ballot((hv & 0xffffu) == tag) & 0xfffu) == 0xfffu; };
            auto head = [&](unsigned hv, int u) { return (unsigned)__builtin_amdgcn_readlane((int)hv, 6 * u) >> 16; };
            auto gave_up = [&](unsigned hv) { return (unsigned)__builtin_amdgcn_readlane((int)hv, 12) == call; };
            auto pull = [&](int q, unsigned so, unsigned hv, unsigned slot) {
                const unsigned ha = head(hv, 0), hb = head(hv, 1);
                if (q < Q_OUT) {
                    PULL_UNIT(rp, slot, so, ha);
                    PULL_UNIT(rp, slot + 4096, so + UNIT_B, hb);
                } else {
                    PULL_UNIT(rc0, slot, so, ha);
                    PULL_UNIT(rc1, slot + 4096, so, hb);
                }
            };
            auto publish = [&](unsigned w) {
                if (lane < 2) asm volatile("global_store_dword %0, %1, %2 sc0 sc1" :: "v"(w + (unsigned)lane * 32u), "v"(call), "s"(rdys) : "memory");
            };
            auto nan_fill = [&](unsigned z) {
#pragma unroll
                for (int m = 0; m < 4; ++m)
                    st128<0>((u32x4){0x7fc07fc0u, 0x7fc07fc0u, 0x7fc07fc0u, 0x7fc07fc0u}, ro, olane + z, m * 16 * NN * 2);
            };
            const unsigned* const tki = tkt + si * 64;
            const unsigned* const tko = tkt + (8 + oc) * 64;
            int o1 = -1;
            unsigned tk, tk0;
            TICKET(tk0, tko);
            if (!FASTC || sq >= w_in) { WAIT_V(0); o1 = (int)min((unsigned)Q_OUT + ticket_value(tk0), (unsigned)Q_END); }
            auto next = [&](int qprev, unsigned t) {
                if (qprev >= Q_OUT) return (int)min((unsigned)Q_OUT + t, (unsigned)Q_END);
                const int j = 4 * w_in + (int)t;
                return j < Q_OUT ? j : o1;
            };
            int qA = sq, qB = sq + w_in, qC = sq + 2 * w_in;
            if (sq >= w_in) {
                unsigned tb, tc;
                TICKET(tb, tko); TICKET(tc, tko); TICKET(tk, tko); WAIT_V(0);
                qA = o1; qB = next(Q_OUT, ticket_value(tb)); qC = next(Q_OUT, ticket_value(tc));
            } else {
                asm volatile("v_mov_b32 %0, %1" : "=v"(tk) : "v"((unsigned)(sq - w_in)));
            }
            u32x4 eA = entry(qA), eB = entry(qB), eC = entry(qC);
            bool dead = false;
            while (qA < Q_END) {
                unsigned hvA, hvB;
                gld32x2_sys(myws + hdr(qA, eA[0]), myws + hdr(qB, eB[0]), hvA, hvB);
                if (FASTC && o1 < 0) o1 = (int)min((unsigned)Q_OUT + ticket_value(tk0), (unsigned)Q_END);
                if (!ready(hvA)) {
                    const unsigned long long t0 = __builtin_amdgcn_s_memrealtime();
                    for (;;) {
                        __builtin_amdgcn_s_sleep(CSLEEP);
                        gld32x2_sys(myws + hdr(qA, eA[0]), myws + hdr(qB, eB[0]), hvA, hvB);
                        if (ready(hvA)) break;
                        if (gave_up(hvA) || __builtin_amdgcn_s_memrealtime() - t0 > DEADLINE) {
                            __hip_atomic_store(tmo, call, __ATOMIC_RELAXED, __HIP_MEMORY_SCOPE_AGENT); dead = true; break;
                        }
                    }
                    if (dead) break;
                }
                unsigned sl = lw, hs = 0;
                pull(qA, eA[1], hvA, sl);
                fetch_rec(qA, eA[0], lh); fetch_rec(qB, eB[0], lh + 256); fetch_rec(qC, eC[0], lh + 512);
                int nst = 0;
                unsigned pend = 0;
                unsigned pend2 = 0;
                bool have2 = false;
                for (;;) {
                    const bool more = qB < Q_END && ready(hvB);
                    if (more) {
                        if (nst == 8) WAIT_V(8); else if (nst == 4) WAIT_V(4); else WAIT_V(0);
                        pull(qB, eB[1], hvB, sl ^ 8192);
                        if (have2) publish(pend2);
                        have2 = nst == 8; pend2 = pend;
                    } else {
                        WAIT_V(0);
                        if (have2) publish(pend2);
                        have2 = false;
                        if (nst == 8) { publish(pend); nst = 0; }
                    }
                    const int qD = next(qC, ticket_value(tk));
                    unsigned hvC;
                    u32x4 d[2][4], eDv;
                    u32x2 cd[2];
                    unsigned ce[2], en0, en1;
                    const unsigned hu[2] = {head(hvA, 0), head(hvA, 1)};
                    READ_ENTRY(en0, sl, lh + hs, hu[0]);
                    READ_ENTRY(en1, sl + 4096, lh + hs + REC_B, hu[1]);
                    DSR128(eDv, ldsa + LDS_TAB + (unsigned)qD * 16u, 0);
                    DSR32(hvC, ldsa + lh + ((hs + 512) & 768) + min((unsigned)lane, 12u) * 16u);
                    READ_UNIT(d[0], cd[0], ce[0], sl, hu[0]);
                    READ_UNIT(d[1], cd[1], ce[1], sl + 4096, hu[1]);
                    asm volatile("s_waitcnt lgkmcnt(10)" : "+v"(en0), "+v"(en1) :: "memory");
                    PATCH_UNIT(d[0], en0, 0, hu[0]);
                    PATCH_UNIT(d[1], en1, 1, hu[1]);
                    WAIT_UNITS(d, cd, ce, hvC, eDv);
                    const u32x4 eD = {rfl(eDv[0]), rfl(eDv[1]), rfl(eDv[2]), rfl(eDv[3])};
                    TICKET(tk, qD < Q_OUT ? tki : tko);
                    if (more) fetch_rec(qD, eD[0], lh + ((hs + 768) & 768));
                    hs = (hs + 256) & 768;
                    if (qA < Q_OUT) {
                        const char* dst = stg + eA[2];
#pragma unroll
                        for (int u = 0; u < 2; ++u) {
                            const unsigned h1 = hu[u];
                            u32x4 o[4];
                            if (!UNLIKELY(h1 >> 8 == RAWMODE)) {
                                unsigned H[8] = {d[u][2][0], d[u][2][1], d[u][2][2], d[u][2][3], d[u][3][0], d[u][3][1], d[u][3][2], d[u][3][3]};
                                decode_high(cd[u][0], cd[u][1], ce[u], h1, H);
#pragma unroll
                                for (int m = 0; m < 4; ++m) {
                                    const unsigned l0 = d[u][m >> 1][(m & 1) * 2], l1 = d[u][m >> 1][(m & 1) * 2 + 1];
                                    o[m] = (u32x4){vperm(H[2 * m], l0, 0x05010400u), vperm(H[2 * m], l0, 0x07030602u),
                                                   vperm(H[2 * m + 1], l1, 0x05010400u), vperm(H[2 * m + 1], l1, 0x07030602u)};
                                }
                                if (h1 & 0x80u) {
#pragma unroll
                                    for (int m = 0; m < 4; ++m)
#pragma unroll
                                        for (int j = 0; j < 4; ++j) o[m][j] = pk_add16(o[m][j], 0xff80ff80u);
                                }
                            } else {
#pragma unroll
                                for (int m = 0; m < 4; ++m) o[m] = d[u][m];
                            }
                            if (!PUB2 && u == 0 && nst == 8) {
                                asm volatile("s_waitcnt vmcnt(8)" : "+v"(o[0]), "+v"(o[1]), "+v"(o[2]), "+v"(o[3]) :: "memory");
                                publish(pend);
                            }
                            const char* d0 = dst + u * HTB;
                            gst128s_sys<0>(d0, vl16, o[0]); gst128s_sys<1024>(d0, vl16, o[1]);
                            gst128s_sys<0>(d0 + 64 * HTB, vl16, o[2]); gst128s_sys<1024>(d0 + 64 * HTB, vl16, o[3]);
                        }
                        pend = eA[3]; nst = 8;
                    } else {
                        const bool bad = gave_up(hvA) || hu[0] >> 8 == POISON || hu[1] >> 8 == POISON;
                        if (!PUB2 && nst == 8) { WAIT_V(8); publish(pend); }
                        if (UNLIKELY(bad)) nan_fill(eA[2]);
                        else {
                            float a[4][8];
#pragma unroll
                            for (int k = 0; k < 2; ++k) {
                                const unsigned h1 = hu[k];
                                if (UNLIKELY(h1 >> 8 == RAWMODE)) {
#pragma unroll
                                    for (int m = 0; m < 4; ++m)
#pragma unroll
                                        for (int j = 0; j < 4; ++j) {
                                            const unsigned v = d[k][m][j];
                                            const float lo = __uint_as_float(v << 16), hi = __uint_as_float(v & 0xffff0000u);
                                            a[m][2 * j] = k == 0 ? lo : a[m][2 * j] + lo; a[m][2 * j + 1] = k == 0 ? hi : a[m][2 * j + 1] + hi;
                                        }
                                } else {
                                    unsigned H[8] = {d[k][2][0], d[k][2][1], d[k][2][2], d[k][2][3], d[k][3][0], d[k][3][1], d[k][3][2], d[k][3][3]};
                                    decode_high(cd[k][0], cd[k][1], ce[k], h1, H);
                                    unsigned vb[4][8];
#pragma unroll
                                    for (int m = 0; m < 4; ++m)
#pragma unroll
                                        for (int j = 0; j < 8; ++j) {
                                            const unsigned Lw = d[k][m >> 1][(m & 1) * 2 + (j >> 2)];
                                            vb[m][j] = vperm(H[2 * m + (j >> 2)], Lw, (4u + (j & 3)) << 24 | (unsigned)(j & 3) << 16 | 0x0c0cu);
                                        }
                                    if (h1 & 0x80u) {
#pragma unroll
                                        for (int m = 0; m < 4; ++m)
#pragma unroll
                                            for (int j = 0; j < 8; ++j) vb[m][j] -= 0x00800000u;
                                    }
#pragma unroll
                                    for (int m = 0; m < 4; ++m)
#pragma unroll
                                        for (int j = 0; j < 8; ++j) a[m][j] = k == 0 ? __uint_as_float(vb[m][j]) : a[m][j] + __uint_as_float(vb[m][j]);
                                }
                            }
#pragma unroll
                            for (int m = 0; m < 4; ++m) {
                                unsigned r0 = cvtpk(a[m][0], a[m][1]), r1 = cvtpk(a[m][2], a[m][3]), r2 = cvtpk(a[m][4], a[m][5]), r3 = cvtpk(a[m][6], a[m][7]);
                                auto s02 = __builtin_amdgcn_permlane16_swap(r0, r2, false, false);
                                auto s13 = __builtin_amdgcn_permlane16_swap(r1, r3, false, false);
                                u32x4 o; o.x = s02[0]; o.y = s13[0]; o.z = s02[1]; o.w = s13[1];
                                st128<OAUX>(o, ro, olane + eA[2], m * 16 * NN * 2);
                            }
                        }
                        STAMPJ(8 + oc, qA - Q_OUT);
                        if (qA == o1) STAMPW(32);
                        nst = 4;
                    }
                    qA = qB; qB = qC; qC = qD; eA = eB; eB = eC; eC = eD; hvA = hvB; hvB = hvC; sl ^= 8192;
                    if (!more) break;
                }
                if (nst == 8) { WAIT_V(0); publish(pend); }
            }
            if (dead) {
                while (qA < Q_END) {
                    if (qA >= Q_OUT) nan_fill(eA[2]);
                    WAIT_V(0);
                    const int qD = next(qC, ticket_value(tk));
                    TICKET(tk, qD < Q_OUT ? tki : tko);
                    qA = qB; qB = qC; qC = qD; eA = eB; eB = eC; eC = entry(qD);
                }
            }
            WAIT_V(0);
        }
        STAMPW(40);
    }
}

}

extern "C" void kernel_launch(int rank, int nranks, ncclComm_t comm,
                              hipStream_t stream,
                              void* const* d_in, const int* in_sizes, int n_in,
                              void* d_out, int out_size,
                              const int* in_dtypes, int out_dtype,
                              void* const* d_out_all,
                              void* const* d_ws_all, size_t ws_bytes) {
    (void)comm; (void)in_dtypes; (void)out_dtype; (void)d_out_all;
    if (nranks != NR || n_in < 2 || in_sizes[0] != MM * KL || in_sizes[1] != KL * NN || out_size != (MM / NR) * NN
        || ws_bytes < WS_NEED || (unsigned)rank >= (unsigned)NR) {
        fprintf(stderr, "kernel_launch: unexpected shapes\n");
        return;
    }
    static bool inited[NR] = {};
    if (!inited[rank]) {
        HIP_CHECK(hipFuncSetAttribute((const void*)gemm_rs_kernel, hipFuncAttributeMaxDynamicSharedMemorySize, LDS_BYTES));
        inited[rank] = true;
    }
    Params p;
    p.x = d_in[0]; p.w = d_in[1]; p.out = d_out; p.rank = rank;
    for (int i = 0; i < NR; ++i) p.ws[i] = (char*)d_ws_all[i];
    hipLaunchKernelGGL(gemm_rs_kernel, dim3(NWG), dim3(NTHR), LDS_BYTES, stream, p);
    HIP_CHECK(hipGetLastError());
}
```

```cpp
#include <hip/hip_runtime.h>
#include <rccl/rccl.h>
#include <cstdio>
#include <cstdint>

#define HIP_CHECK(expr) do { hipError_t _e = (expr); if (_e != hipSuccess) { \
    fprintf(stderr, "HIP error %s at %s:%d\n", hipGetErrorString(_e), __FILE__, __LINE__); \
    return; } } while (0)

#define LAS __attribute__((address_space(3)))
#define UNLIKELY(x) __builtin_expect(!!(x), 0)
typedef short bf16x8 __attribute__((ext_vector_type(8)));
typedef short s16x4 __attribute__((ext_vector_type(4)));
typedef float f32x4 __attribute__((ext_vector_type(4)));
typedef float f32x2 __attribute__((ext_vector_type(2)));
typedef unsigned u32x4 __attribute__((ext_vector_type(4)));
typedef unsigned u32x3 __attribute__((ext_vector_type(3)));
typedef unsigned u32x2 __attribute__((ext_vector_type(2)));
typedef unsigned short u16x2 __attribute__((ext_vector_type(2)));
typedef __bf16 bf16x2_t __attribute__((ext_vector_type(2)));
typedef __amdgpu_buffer_rsrc_t rsrc_t;

namespace {

constexpr int NR = 8;
constexpr int MM = 8192, NN = 8192, KL = 1024;
constexpr int NWG = 256, NTHR = 512;
#define PN1 13
#define LOOK 2
#define CW 3
#define LATE 1
#define PUB2 1
#define LEAD 1
#define GSLEEP 2
#define CSLEEP 2
#define FASTP 1
#define FASTC 1
#define URG_LO 0
#define URG_HI 4
#define RAWT 0
#define PKSLEEP 0
#define FORCE_RAW 0
#define NOGEMM 0
#define GAUX 17
#define GAUX2 GAUX
#define G2X 0
#define NOGEMM2 0
#define XAUX 2
#define OAUX 2
constexpr int NGEMM = 16 * PN1, NCOMM = NWG - NGEMM, NCW = NCOMM * CW;
constexpr int PN2 = 16 - PN1;
constexpr int HTB = 16384, STAGE_BYTES = 8 * HTB;
constexpr int NT = 64;
constexpr unsigned UNIT_B = 4096;
constexpr size_t MIB = 1u << 20;
constexpr size_t OFF_PACK = 0;
constexpr unsigned PACK_B = 32u << 20;
constexpr size_t OFF_STG = 64 * MIB;
constexpr unsigned STG_B = 32u << 20;
constexpr size_t OFF_CP = 128 * MIB;
constexpr unsigned CP_B = 32u << 20;
constexpr unsigned REC_B = 96;
constexpr size_t OFF_IH = 164 * MIB;
constexpr unsigned IH_B = 2 * 16 * 8 * 64 * REC_B;
constexpr size_t OFF_RDY = 161 * MIB;
constexpr size_t OFF_CH = 162 * MIB;
constexpr size_t CTRL_OFF = 256 * MIB;
constexpr size_t WS_NEED = CTRL_OFF + MIB;
constexpr unsigned C_TMO = 256, C_CNT = 4096;
constexpr unsigned C_TKT = 16384;
constexpr unsigned MAXEXC = 64, RECN = 41, RAWMODE = 255, POISON = 254;
constexpr int OFF_CODE = 2048, OFF_LIST = 2816;
constexpr unsigned long long DEADLINE = 30000000ull;
constexpr int LDS_AUX = STAGE_BYTES;
constexpr int LDS_BAD = STAGE_BYTES + 8192;
constexpr int LDS_REC = LDS_BAD + 128;
constexpr int LDS_BYTES = LDS_REC + 6 * 8 * REC_B;
constexpr int AUX_SYS = 17;
constexpr int E1 = 16 + PN1;
constexpr int NJ1 = (NT / 2) * E1, NJ_IN = (NT / 2) * 32;
#define NFB 4
#define SKD 3
constexpr int EF = 8 + NFB, ES = E1 - EF;
constexpr int Q_OUT = NJ_IN, Q_END = Q_OUT + 1024;
constexpr int LDS_PAT = 98304 + 2048;
constexpr int LDS_TAB = 65536;
__host__ __device__ constexpr int in_job(int j) {
    constexpr int R1 = SKD, R2 = NT / 2 - SKD;
    int i = 0, r = 0;
    if (j < R1 * EF) { i = j % EF; r = j / EF; }
    else if (j < R1 * EF + R2 * E1) { const int q = j - R1 * EF; i = q % E1; r = q / E1 + (i < EF ? SKD : 0); }
    else if (j < NJ1) { const int q = j - (R1 * EF + R2 * E1); i = EF + q % ES; r = NT / 2 - SKD + q / ES; }
    else { const int q = j - NJ1; i = E1 + q % PN2; r = q / PN2; }
    const int e = i < 8 ? 2 * i : i < EF ? 8 + i : i < 8 + EF ? 2 * (i - EF) + 1 : i;
    return e * 64 + 2 * r;
}
#define ORD 3
__host__ __device__ constexpr int out_yg(int o) {
    constexpr int NL = NFB, NM = 2 * PN1 - NFB;
    const int u = o >> 3;
    if ((ORD & 1) && u >= 4 * NL && u < 4 * (NL + NM)) return (NL + (u - 4 * NL) % NM) * 4 + (u - 4 * NL) / NM;
    if ((ORD & 2) && u < 4 * NL) return (u % NL) * 4 + u / NL;
    return u;
}
__host__ __device__ constexpr int out_tile(int y) { return y < NFB ? 2 * y : y < 2 * NFB ? 2 * (y - NFB) + 1 : y; }
#define W_IN 14
#define JP0 NJ1
#define XE 0
#define JE 128
#define NACT 120

#define STAMP(slot) do {} while (0)
#define STAMPW(slot) do {} while (0)
#define STAMPJ(row, j) do {} while (0)
#define DIAG_BLK0 do {} while (0)
#define DIAG_BLK1(t0) do {} while (0)
struct Params {
    const void* x; const void* w; void* out;
    char* ws[NR];
    int rank;
};

__device__ __forceinline__ unsigned cvtpk(float lo, float hi) { f32x2 v = {lo, hi}; bf16x2_t b = __builtin_convertvector(v, bf16x2_t); return __builtin_bit_cast(unsigned, b); }
__device__ __forceinline__ rsrc_t mk_rsrc(const void* p, unsigned bytes) { return __builtin_amdgcn_make_buffer_rsrc((void*)p, 0, bytes, 0x00020000); }

template <int AUX> __device__ __forceinline__ void st128(u32x4 v, rsrc_t r, unsigned voff, unsigned soff) {
    __builtin_amdgcn_raw_buffer_store_b128(v, r, voff, soff, AUX);
    asm volatile("s_nop 1" :: "v"(v) : "memory");
}
__device__ __forceinline__ void gst128_sys(void* ptr, u32x4 v) {
    asm volatile("global_store_dwordx4 %0, %1, off sc0 sc1\n\ts_nop 1" :: "v"(ptr), "v"(v) : "memory");
}
__device__ __forceinline__ unsigned gld32_sys(const void* ptr) {
    unsigned v;
    asm volatile("global_load_dword %0, %1, off sc0 sc1\n\ts_waitcnt vmcnt(0)" : "=&v"(v) : "v"(ptr) : "memory");
    return v;
}
__device__ __forceinline__ void gld32x2_sys(const void* pa, const void* pb, unsigned& a, unsigned& b) {
    asm volatile("global_load_dword %0, %2, off sc0 sc1\n\tglobal_load_dword %1, %3, off sc0 sc1\n\ts_waitcnt vmcnt(0)"
                 : "=&v"(a), "=&v"(b) : "v"(pa), "v"(pb) : "memory");
}
template <int OFF> __device__ __forceinline__ void gst128s_sys(const void* base, unsigned voff, u32x4 v) {
    asm volatile("global_store_dwordx4 %0, %1, %2 offset:%c3 sc0 sc1\n\ts_nop 1" :: "v"(voff), "v"(v), "s"(base), "i"(OFF) : "memory");
}
#define SPOL "sc0 sc1"
template <int OFF> __device__ __forceinline__ void gst128s_stg(const void* base, unsigned voff, u32x4 v) {
    asm volatile("global_store_dwordx4 %0, %1, %2 offset:%c3 " SPOL "\n\ts_nop 1" :: "v"(voff), "v"(v), "s"(base), "i"(OFF) : "memory");
}
__device__ __forceinline__ unsigned bfi(unsigned m, unsigned a, unsigned b) { return (a & m) | (b & ~m); }
__device__ __forceinline__ unsigned vperm(unsigned hi, unsigned lo, unsigned sel) { return __builtin_amdgcn_perm(hi, lo, sel); }
__device__ __forceinline__ unsigned rfl(unsigned v) { return __builtin_amdgcn_readfirstlane(v); }
__device__ __forceinline__ bool timed_out(unsigned* tmo, unsigned call, unsigned long long t0) {
    if (__hip_atomic_load(tmo, __ATOMIC_RELAXED, __HIP_MEMORY_SCOPE_AGENT) == call) return true;
    if (__builtin_amdgcn_s_memrealtime() - t0 > DEADLINE) { __hip_atomic_store(tmo, call, __ATOMIC_RELAXED, __HIP_MEMORY_SCOPE_AGENT); return true; }
    return false;
}

__device__ __forceinline__ unsigned wave_sum_u(unsigned x) {
    x += (unsigned)__builtin_amdgcn_update_dpp(0, (int)x, 0xB1, 0xf, 0xf, true);
    x += (unsigned)__builtin_amdgcn_update_dpp(0, (int)x, 0x4E, 0xf, 0xf, true);
    x += (unsigned)__builtin_amdgcn_update_dpp(0, (int)x, 0x141, 0xf, 0xf, true);
    x += (unsigned)__builtin_amdgcn_update_dpp(0, (int)x, 0x140, 0xf, 0xf, true);
    return (unsigned)(__builtin_amdgcn_readlane((int)x, 0) + __builtin_amdgcn_readlane((int)x, 16) + __builtin_amdgcn_readlane((int)x, 32) + __builtin_amdgcn_readlane((int)x, 48));
}
__device__ __forceinline__ unsigned pk_add16(unsigned a, unsigned b) { return __builtin_bit_cast(unsigned, __builtin_bit_cast(u16x2, a) + __builtin_bit_cast(u16x2, b)); }
__device__ __forceinline__ unsigned mbcnt(unsigned long long m) { return __builtin_amdgcn_mbcnt_hi((unsigned)(m >> 32), __builtin_amdgcn_mbcnt_lo((unsigned)m, 0u)); }
__device__ __forceinline__ unsigned list_pos(unsigned i, bool rec) {
    return i + (rec ? 2u : 0u) + (__umul24(i + 1, rec ? 37u : 0u) >> 8);
}
__device__ __forceinline__ void encode_unit(unsigned (&w)[4][4], const char* dst, unsigned lst, unsigned rec, unsigned tag, int lane, bool raw) {
    unsigned h1;
    if (raw) {
        const unsigned vo = (unsigned)lane * 16u;
        gst128s_sys<0>(dst, vo, (u32x4){w[0][0], w[0][1], w[0][2], w[0][3]});
        gst128s_sys<1024>(dst, vo, (u32x4){w[1][0], w[1][1], w[1][2], w[1][3]});
        gst128s_sys<2048>(dst, vo, (u32x4){w[2][0], w[2][1], w[2][2], w[2][3]});
        gst128s_sys<3072>(dst, vo, (u32x4){w[3][0], w[3][1], w[3][2], w[3][3]});
        if (lane < 7) asm volatile("ds_write_b16 %0, %1" :: "v"(rec + (lane < 6 ? (unsigned)lane * 16u : 2u)), "v"(lane < 6 ? tag : RAWMODE << 8) : "memory");
        return;
    }
    unsigned sum = 0;
#pragma unroll
    for (int m = 0; m < 4; ++m) sum = __builtin_amdgcn_udot2(__builtin_bit_cast(u16x2, w[m][0] & 0x7f807f80u), (u16x2){1, 1}, sum, false);
    const unsigned T = (wave_sum_u(sum) + 185467u) >> 16;
    const unsigned par = ~T & 1u, top = (T + par) >> 1;
    if (par) {
#pragma unroll
        for (int m = 0; m < 4; ++m)
#pragma unroll
            for (int q = 0; q < 4; ++q) w[m][q] = pk_add16(w[m][q], 0x00800080u);
    }
    const unsigned bq = (top | 0x80u) * 0x01010101u;
    unsigned L[8], d[8], c[8], fl[4];
#pragma unroll
    for (int g = 0; g < 8; ++g) {
        const unsigned lo = w[g >> 1][(g & 1) * 2], hi = w[g >> 1][(g & 1) * 2 + 1];
        L[g] = vperm(hi, lo, 0x06040200u);
        const unsigned h = vperm(hi, lo, 0x07050301u);
        d[g] = bq - (h & 0x7f7f7f7fu);
        c[g] = (h >> 5 & 0x04040404u) | (d[g] & 0x03030303u);
        const unsigned f = (((d[g] >> 2 & 0x3f3f3f3fu) ^ 0x20202020u) + 0x7f7f7f7fu) & 0x80808080u;
        fl[g >> 1] = __builtin_amdgcn_udot4(f, g & 1 ? 0x80402010u : 0x08040201u, g & 1 ? fl[g >> 1] : 0u, false);
    }
    unsigned ex = fl[0] >> 7 | fl[1] << 1 | fl[2] << 9 | fl[3] << 17;
    const unsigned n = (unsigned)__builtin_popcount(ex);
    const unsigned long long b0 = __ballot(n & 1), b1 = __ballot(n & 2), b2 = __ballot(n & 4);
    unsigned cnt = (unsigned)(__builtin_popcountll(b0) + 2 * __builtin_popcountll(b1) + 4 * __builtin_popcountll(b2));
    if (FORCE_RAW || top > 123 || cnt > MAXEXC || __ballot(n >= 8) != 0) cnt = RAWMODE;
    if (!UNLIKELY(cnt == RAWMODE)) {
        const bool inrec = cnt <= RECN;
        const unsigned lb = inrec ? rec : lst;
        unsigned li = mbcnt(b0) + 2 * mbcnt(b1) + 4 * mbcnt(b2), inv = 0;
        while (__ballot(ex != 0) != 0) {
            const unsigned j = (unsigned)__builtin_ctz(ex | 0x80000000u) , sel = (j & 7) | 0x0c0c0c00u;
            const unsigned r0 = vperm(d[1], d[0], sel), r1 = vperm(d[3], d[2], sel), r2 = vperm(d[5], d[4], sel), r3 = vperm(d[7], d[6], sel);
            const unsigned ra = j & 8 ? r1 : r0, rb = j & 8 ? r3 : r2, db = j & 16 ? rb : ra;
            const unsigned q = ((db >> 2) + 1) ^ 0x20u;
            inv |= q;
            const unsigned en = (j & 16) << 6 | (unsigned)lane << 4 | (j & 15) | q << 11;
            if (ex) asm volatile("ds_write_b16 %0, %1" :: "v"(lb + 2 * list_pos(li, inrec)), "v"(en) : "memory");
            li += 1; ex &= ex - 1;
        }
        if (__ballot(inv > 31) != 0) cnt = RAWMODE;
    }
    const unsigned vo = (unsigned)lane * 16u;
    if (UNLIKELY(cnt == RAWMODE)) {
        const unsigned back = par ? 0xff80ff80u : 0u;
#pragma unroll
        for (int m = 0; m < 4; ++m)
#pragma unroll
            for (int q = 0; q < 4; ++q) w[m][q] = pk_add16(w[m][q], back);
        gst128s_sys<0>(dst, vo, (u32x4){w[0][0], w[0][1], w[0][2], w[0][3]});
        gst128s_sys<1024>(dst, vo, (u32x4){w[1][0], w[1][1], w[1][2], w[1][3]});
        gst128s_sys<2048>(dst, vo, (u32x4){w[2][0], w[2][1], w[2][2], w[2][3]});
        gst128s_sys<3072>(dst, vo, (u32x4){w[3][0], w[3][1], w[3][2], w[3][3]});
        h1 = RAWMODE << 8;
    } else {
        u32x4 lv;
        asm volatile("ds_read_b128 %0, %1" : "=&v"(lv) : "v"(lst + ((unsigned)lane & 7u) * 16u) : "memory");
        gst128s_sys<0>(dst, vo, (u32x4){L[0], L[1], L[2], L[3]});
        gst128s_sys<1024>(dst, vo, (u32x4){L[4], L[5], L[6], L[7]});
        const u32x3 cd = {c[0] | c[1] << 3 | (c[6] << 6 & 0xc0c0c0c0u), c[2] | c[3] << 3 | (c[7] << 6 & 0xc0c0c0c0u),
                          c[4] | c[5] << 3 | (c[6] << 4 & 0x40404040u) | (c[7] << 5 & 0x80808080u)};
        asm volatile("global_store_dwordx3 %0, %1, %2 offset:%c3 sc0 sc1\n\ts_nop 1" :: "v"((unsigned)lane * 12u), "v"(cd), "s"(dst), "i"(OFF_CODE) : "memory");
        asm volatile("s_waitcnt lgkmcnt(0)" : "+v"(lv) :: "memory");
        if (lane < 8) gst128s_sys<OFF_LIST>(dst, vo, lv);
        h1 = top | par << 7 | cnt << 8;
    }
    if (lane < 7) asm volatile("ds_write_b16 %0, %1" :: "v"(rec + (lane < 6 ? (unsigned)lane * 16u : 2u)), "v"(lane < 6 ? tag : h1) : "memory");
}
__device__ __forceinline__ void decode_high(unsigned d0, unsigned d1, unsigned d2, unsigned h1, unsigned (&H)[8]) {
    const unsigned lutl = ((h1 & 0x7fu) + 4) * 0x01010101u - 0x03020100u, luth = lutl | 0x80808080u;
    const unsigned c[8] = {d0 & 0x07070707u, d0 >> 3 & 0x07070707u, d1 & 0x07070707u, d1 >> 3 & 0x07070707u, d2 & 0x07070707u, d2 >> 3 & 0x07070707u,
                           (d0 >> 6 & 0x03030303u) | (d2 >> 4 & 0x04040404u), (d1 >> 6 & 0x03030303u) | (d2 >> 5 & 0x04040404u)};
#pragma unroll
    for (int g = 0; g < 8; ++g) H[g] = vperm(luth, lutl, c[g]) - H[g];
}

#define TRD(dst, addr, off) asm volatile("ds_read_b64_tr_b16 %0, %1 offset:%c2" : "=&v"(dst) : "v"(addr), "i"(off) : "memory")
#define PKB(x) (bf16x8){(x)[0][0], (x)[0][1], (x)[0][2], (x)[0][3], (x)[1][0], (x)[1][1], (x)[1][2], (x)[1][3]}
#define WAIT_V(n) asm volatile("s_waitcnt vmcnt(" #n ")" ::: "memory")
#define WAIT_L(n) asm volatile("s_waitcnt lgkmcnt(" #n ")" ::: "memory")
#define BAR __builtin_amdgcn_s_barrier()
#define SCHED __builtin_amdgcn_sched_barrier(0)
#define DSR128(dst, addr, off) asm volatile("ds_read_b128 %0, %1 offset:%c2" : "=&v"(dst) : "v"(addr), "i"(off) : "memory")
#define DSR32(dst, addr) asm volatile("ds_read_b32 %0, %1" : "=&v"(dst) : "v"(addr) : "memory")
#define DMA16(rs, ldsoff, voff, soff) __builtin_amdgcn_raw_ptr_buffer_load_lds(rs, (LAS void*)(lds + (ldsoff)), 16, voff, soff, 0, AUX_SYS)
#define TICKET(tk, ptr) asm volatile("s_mov_b64 exec, 1\n\tglobal_atomic_add %0, %1, %2, %3 sc0 sc1\n\ts_mov_b64 exec, -1" : "=&v"(tk) : "v"(0u), "v"(1u), "s"(ptr) : "memory")
__device__ __forceinline__ unsigned ticket_value(unsigned tk) {
    unsigned r;
    asm volatile("v_readfirstlane_b32 %0, %1" : "=s"(r) : "v"(tk) : "memory");
    return r;
}

__global__ __launch_bounds__(NTHR, 2) void gemm_rs_kernel(Params p) {
    extern __shared__ __attribute__((aligned(16))) unsigned char smem[];
    LAS unsigned char* lds = (LAS unsigned char*)smem;
    const int tid = threadIdx.x, wid = __builtin_amdgcn_readfirstlane(tid >> 6), lane = tid & 63;
    const int wr = wid >> 2, wc = wid & 3, fr = lane & 15, fq = lane >> 4;
    const int wg = blockIdx.x, rank = p.rank;
    const int bi = rank & 1, bj_ = (rank >> 1) & 1, bk = rank >> 2;
    char* myws = p.ws[rank];
    unsigned* ctl = (unsigned*)(myws + CTRL_OFF);
    unsigned* tmo = ctl + C_TMO / 4;

    int dslot = 0;
    STAMP(dslot); ++dslot;
    const unsigned ldsa = (unsigned)(size_t)lds;
    unsigned* cntp = ctl + C_CNT / 4 + wg * 8 + wid;

    if (wg < NGEMM) {
        const rsrc_t rx = mk_rsrc(p.x, MM * KL * 2), rw = mk_rsrc(p.w, KL * NN * 2);
        const unsigned vx = (unsigned)((tid >> 3) * (KL * 2) + (((tid & 7) ^ ((tid >> 4) & 7)) << 4));
        const unsigned vw = (unsigned)(fq * (NN * 2) + ((fr ^ ((fq << 2) | ((rank & 1) << 1))) << 4));
        auto pk_issue = [&](int idx, int buf) {
            const int g8 = idx >> 7, mat = (idx >> 6) & 1, kk = (idx >> 5) & 1, blk = idx & 31;
            if (mat == 0) {
#pragma unroll
                for (int i = 0; i < 4; ++i)
                    __builtin_amdgcn_raw_ptr_buffer_load_lds(rx, (LAS void*)(lds + buf + i * 8192 + wid * 1024), 16, vx,
                                                             (unsigned)((blk * 256 + i * 64) * (KL * 2) + kk * 1024 + g8 * 128), 0, XAUX);
            } else {
#pragma unroll
                for (int c = 0; c < 4; ++c)
                    __builtin_amdgcn_raw_ptr_buffer_load_lds(rw, (LAS void*)(lds + buf + wid * 4096 + c * 1024), 16, vw,
                                                             (unsigned)((kk * 512 + (g8 * 8 + wid) * 8 + (c & 1) * 4) * (NN * 2) + blk * 512 + (c >> 1) * 256), 0, XAUX);
            }
        };
        pk_issue(wg, 0);
        volatile LAS unsigned* const badf = (volatile LAS unsigned*)(lds + LDS_BAD);
        if (tid < 16) asm volatile("ds_write_b32 %0, %1" :: "v"(ldsa + LDS_BAD + tid * 4), "v"(0u) : "memory");
        const unsigned call = rfl(gld32_sys(cntp)) + 1;
        if (lane == 0) __hip_atomic_store(cntp, call, __ATOMIC_RELAXED, __HIP_MEMORY_SCOPE_AGENT);
        const unsigned par = call & 1;
        {
            const char* const pk = myws + OFF_PACK + (size_t)par * PACK_B;
            const rsrc_t rs = mk_rsrc(myws + OFF_STG, 2 * STG_B);
            int buf = 0;
            const unsigned tag = call & 0xffffu;
            auto pk_headers = [&](int idx, int b) {
                const int g8 = idx >> 7, mat = (idx >> 6) & 1, kk = (idx >> 5) & 1, blk = idx & 31;
                const int cb = mat == 0 ? ((blk >> 2) * 2 + (blk & 1)) : (blk & 15);
                const int c0 = mat == 0 ? ((blk >> 1) & 1) + 4 * kk : 2 * (blk >> 4) + 4 * kk, c1 = c0 + (mat == 0 ? 2 : 1);
                char* const w0 = p.ws[c0 & 7];
                char* const w1 = p.ws[c1 & 7];
                if (lane < 48) {
                    u32x4 h;
                    asm volatile("ds_read_b128 %0, %1\n\ts_waitcnt lgkmcnt(0)" : "=&v"(h) : "v"(ldsa + LDS_REC + (b >> 15) * (8 * REC_B) + lane * 16) : "memory");
                    const size_t o = OFF_IH + (size_t)par * IH_B + (size_t)((((mat * 16 + cb) * NR + rank) * 64 + g8 * 8) * REC_B);
                    if (c0 != rank) gst128s_sys<0>(w0 + o, lane * 16, h);
                    if (c1 != rank) gst128s_sys<0>(w1 + o, lane * 16, h);
                }
            };
            for (int idx = wg; idx < 1024; idx += NGEMM, buf ^= 32768) {
                const int g8 = idx >> 7, mat = (idx >> 6) & 1, kk = (idx >> 5) & 1, blk = idx & 31, t = g8 * 8 + wid;
                BAR;
                if (wid == 0 && idx != wg) pk_headers(idx - NGEMM, buf ^ 32768);
                const bool urgent = FASTP && idx < 128 && wid >= URG_LO && wid < URG_HI;
                const bool rawf = urgent && wid < RAWT && (mat == 0 ? !(blk & 1) : (blk & 15) < NFB);
                if (idx + NGEMM < 1024 && !rawf) pk_issue(idx + NGEMM, buf ^ 32768);
                if (urgent) __builtin_amdgcn_s_setprio(3);
                u32x4 v[4];
                if (mat == 0) {
                    const unsigned a = ldsa + buf + lane * 128 + ((wid ^ ((lane >> 1) & 7)) << 4);
                    DSR128(v[0], a, 0); DSR128(v[1], a, 8192); DSR128(v[2], a, 16384); DSR128(v[3], a, 24576);
                } else {
                    const unsigned a = ldsa + buf + wid * 4096 + lane * 16;
                    DSR128(v[0], a, 0); DSR128(v[1], a, 1024); DSR128(v[2], a, 2048); DSR128(v[3], a, 3072);
                }
                asm volatile("s_waitcnt lgkmcnt(0)" : "+v"(v[0]), "+v"(v[1]), "+v"(v[2]), "+v"(v[3]) :: "memory");
                const int cb = mat == 0 ? ((blk >> 2) * 2 + (blk & 1)) : (blk & 15);
                const int c0 = mat == 0 ? ((blk >> 1) & 1) + 4 * kk : 2 * (blk >> 4) + 4 * kk, c1 = c0 + (mat == 0 ? 2 : 1);
                char* const w0 = p.ws[c0 & 7];
                char* const w1 = p.ws[c1 & 7];
                const unsigned fo = (unsigned)(OFF_RDY + (((mat * 16 + cb) * 64 + t) * NR + rank) * 4);
                unsigned w[4][4];
#pragma unroll
                for (int c = 0; c < 4; ++c) { w[c][0] = v[c][0]; w[c][1] = v[c][1]; w[c][2] = v[c][2]; w[c][3] = v[c][3]; }
                const unsigned rec = ldsa + LDS_REC + (buf >> 15) * (8 * REC_B) + wid * REC_B;
                encode_unit(w, pk + (unsigned)(((mat * 2 + kk) * 32 + blk) * 64 + t) * UNIT_B, ldsa + LDS_AUX + wid * 1024, rec, tag, lane, rawf);
                const bool own = c0 == rank || c1 == rank;
                if (own) {
                    const unsigned dst = (unsigned)mat * STG_B + (unsigned)(2 * cb * 64 + t) * HTB + (unsigned)rank * 2048u;
#pragma unroll
                    for (int m = 0; m < 4; ++m) st128<AUX_SYS>(v[m], rs, lane * 16, dst + (m >> 1) * (64 * HTB) + (m & 1) * 1024);
                }
                WAIT_V(0);
                if (rawf) pk_issue(idx + NGEMM, buf ^ 32768);
                if (own) { if (lane == 0) __hip_atomic_store((unsigned*)(myws + fo), call, __ATOMIC_RELAXED, __HIP_MEMORY_SCOPE_SYSTEM); }
                if (urgent && lane < 12 && (lane < 6 ? c0 : c1) != rank) {
                    u32x4 h;
                    asm volatile("ds_read_b128 %0, %1\n\ts_waitcnt lgkmcnt(0)" : "=&v"(h) : "v"(rec + (lane < 6 ? lane : lane - 6) * 16) : "memory");
                    gst128_sys((lane < 6 ? w0 : w1) + OFF_IH + (size_t)par * IH_B + (size_t)((((mat * 16 + cb) * NR + rank) * 64 + t) * REC_B + (lane < 6 ? lane : lane - 6) * 16), h);
                }
                if (urgent) __builtin_amdgcn_s_setprio(0);
                if (rawf) WAIT_V(1);
                WAIT_L(0);
            }
            BAR;
            if (wid == 0) pk_headers(wg + (1023 - wg) / NGEMM * NGEMM, buf ^ 32768);
        }
        STAMP(dslot); ++dslot;
        const int pm = 2 * (wg & 7) + ((wg >> 3) & 1), pn = wg >> 4;
        const rsrc_t ra = mk_rsrc(myws + OFF_STG, STG_B), rb = mk_rsrc(myws + OFF_STG + STG_B, STG_B);
        const unsigned voff = (unsigned)(wid * 1024 + lane * 16);
        constexpr unsigned S_K = HTB, S_H = NT * HTB, S_T = 2 * NT * HTB;
        const unsigned ldsw = (unsigned)wid * 1024u;
        const int aoff = fq * 2048 + (wr * 64 + fr) * 16;
        unsigned boff[2];
        { const int q = (lane & 15) >> 2, pp = lane & 3, g = fq;
#pragma unroll
          for (int n = 0; n < 2; ++n) boff[n] = (unsigned)(size_t)lds + 4 * HTB + 256 * (8 * g + q) + 16 * (((wc ^ q) << 2) | ((n ^ (g & 1)) << 1) | (pp >> 1)) + 8 * (pp & 1); }
        const unsigned* rdyf = (const unsigned*)(myws + OFF_RDY);
#define SA(b, h) (((b) * 2 + (h)) * HTB)
#define SB(b, h) ((4 + (b) * 2 + (h)) * HTB)
#define STAGEX(rs, bufoff, soff, aux) do { _Pragma("unroll") for (int _i = 0; _i < 2; ++_i) \
        __builtin_amdgcn_raw_ptr_buffer_load_lds(rs, (LAS void*)(lds + (bufoff) + ldsw + _i * 8192), 16, voff, (soff) + _i * 8192, 0, aux); } while (0)
#define STAGE(rs, bufoff, soff) STAGEX(rs, bufoff, soff, GAUX)
#define STAGE2(rs, bufoff, soff) STAGEX(rs, bufoff, soff, GAUX2)
#define LDA(dst, b, h) do { _Pragma("unroll") for (int m = 0; m < 4; ++m) _Pragma("unroll") for (int k = 0; k < 2; ++k) dst[m][k] = *(const LAS bf16x8*)(lds + SA(b, h) + aoff + m * 256 + k * 8192); } while (0)
#define LDB(dst, b, h) do { _Pragma("unroll") for (int n = 0; n < 2; ++n) _Pragma("unroll") for (int k = 0; k < 2; ++k) { \
        TRD(dst[n][k][0], boff[n], ((b) * 2 + (h)) * HTB + k * 8192); TRD(dst[n][k][1], boff[n], ((b) * 2 + (h)) * HTB + k * 8192 + 1024); } } while (0)
#define MMA(ai, bj, At, Bt) do { __builtin_amdgcn_s_setprio(1); _Pragma("unroll") for (int m = 0; m < 4; ++m) _Pragma("unroll") for (int n = 0; n < 2; ++n) _Pragma("unroll") for (int k = 0; k < 2; ++k) \
        acc[ai][bj][m][n] = __builtin_amdgcn_mfma_f32_16x16x32_bf16(PKB(Bt[n][k]), At[m][k], acc[ai][bj][m][n], 0, 0, 0); __builtin_amdgcn_s_setprio(0); } while (0)
#define WAIT_B(X, Y) asm volatile("s_waitcnt lgkmcnt(0)" : "+v"(X[0][0][0]), "+v"(X[0][0][1]), "+v"(X[0][1][0]), "+v"(X[0][1][1]), "+v"(X[1][0][0]), "+v"(X[1][0][1]), "+v"(X[1][1][0]), "+v"(X[1][1][1]), \
        "+v"(Y[0][0][0]), "+v"(Y[0][0][1]), "+v"(Y[0][1][0]), "+v"(Y[0][1][1]), "+v"(Y[1][0][0]), "+v"(Y[1][0][1]), "+v"(Y[1][1][0]), "+v"(Y[1][1][1]) :: "memory")
#define WAIT_B1(X) asm volatile("s_waitcnt lgkmcnt(0)" : "+v"(X[0][0][0]), "+v"(X[0][0][1]), "+v"(X[0][1][0]), "+v"(X[0][1][1]), "+v"(X[1][0][0]), "+v"(X[1][0][1]), "+v"(X[1][1][0]), "+v"(X[1][1][1]) :: "memory")
        const rsrc_t rr = mk_rsrc(myws + OFF_RDY, 2 * 16 * 64 * 32);
        const unsigned pvo = (unsigned)(((lane >> 5) * 16 + (lane < 32 ? pm : pn)) * 2048 + (lane & 1) * 16);
        const unsigned pla = ldsa + LDS_AUX + wid * 1024 + lane * 16;
#define NOGATE 0
        int rdy = NOGATE ? NT : 0, pbase = 0;
        bool pol = false;
        u32x4 fv;
#define COUNT_RDY(v) do { const bool _ok = (int)((v)[0] - call) >= 0 && (int)((v)[1] - call) >= 0 && (int)((v)[2] - call) >= 0 && (int)((v)[3] - call) >= 0; \
            const unsigned long long _b = __ballot(_ok); \
            unsigned _m = (unsigned)_b & (unsigned)(_b >> 32); _m = ~(_m & (_m >> 1)) & 0x55555555u; \
            rdy = min(pbase + (_m ? (int)(__builtin_ctz(_m) >> 1) : 16), NT); } while (0)
#define GATE(tt) do { if (wid == 4) { \
            if (pol) { DSR128(fv, pla, 0); asm volatile("s_waitcnt lgkmcnt(0)" : "+v"(fv) :: "memory"); COUNT_RDY(fv); } \
            if (UNLIKELY((int)(tt) >= rdy)) { \
                const unsigned long long _t0 = __builtin_amdgcn_s_memrealtime(); \
                DIAG_BLK0; \
                for (;;) { \
                    pbase = rdy; \
                    const char* _p = (const char*)rdyf + pvo + min(rdy + ((lane >> 1) & 15), NT - 1) * 32; \
                    u32x4 _v; unsigned _tm; \
                    asm volatile("global_load_dwordx4 %0, %2, off sc0 sc1\n\tglobal_load_dword %1, %3, off sc0 sc1\n\ts_waitcnt vmcnt(0)" \
                                 : "=&v"(_v), "=&v"(_tm) : "v"(_p), "v"(tmo) : "memory"); \
                    COUNT_RDY(_v); \
                    if ((int)(tt) < rdy) break; \
                    if (rfl(_tm) == call || __builtin_amdgcn_s_memrealtime() - _t0 > DEADLINE) { \
                        __hip_atomic_store(tmo, call, __ATOMIC_RELAXED, __HIP_MEMORY_SCOPE_AGENT); *badf = 1; rdy = NT; break; } \
                    __builtin_amdgcn_s_sleep(GSLEEP); \
                } DIAG_BLK1(_t0); } \
            pol = rdy < min((int)(tt) + LOOK, NT); \
            if (pol) { pbase = rdy; DMA16(rr, LDS_AUX + wid * 1024, pvo + min(rdy + ((lane >> 1) & 15), NT - 1) * 32, 0); } } } while (0)
#define FLAGREAD do {} while (0)
#define GATE1(tt) GATE(tt)
#define GATE2(tt) do {} while (0)
#define WAIT_VP do { if (pol) WAIT_V(9); else WAIT_V(8); } while (0)
        const int owner = pm >> 1;
        char* chd = p.ws[owner] + OFF_CH;
        f32x4 acc[2][2][4][2];
#pragma unroll
        for (int a = 0; a < 2; ++a)
#pragma unroll
            for (int b = 0; b < 2; ++b)
#pragma unroll
                for (int m = 0; m < 4; ++m)
#pragma unroll
                    for (int n = 0; n < 2; ++n) acc[a][b][m][n] = (f32x4){0.f, 0.f, 0.f, 0.f};
        bf16x8 At[4][2]; s16x4 B0[2][2][2], B1[2][2][2];
        const unsigned cA = (unsigned)pm * S_T, cB = (unsigned)pn * S_T;
        if (!NOGEMM) { GATE(1); }
        BAR;
        STAGE(rb, SB(0, 0), cB); STAGE(rb, SB(0, 1), cB + S_H); STAGE(ra, SA(0, 0), cA); STAGE(ra, SA(0, 1), cA + S_H);
        if (wr == 1) BAR;
        WAIT_V(2); BAR;
        STAGE(rb, SB(1, 0), cB + S_K); STAGE(ra, SA(1, 0), cA + S_K); STAGE(rb, SB(1, 1), cB + S_H + S_K);
        WAIT_V(6); BAR;
        STAMP(dslot); ++dslot;
        for (int t = 0; t < (NOGEMM ? 0 : NT); t += 2) {
            const int t2 = min(t + 2, NT - 1), t3 = min(t + 3, NT - 1);
            const unsigned a1 = cA + (unsigned)(t + 1) * S_K;
            const unsigned a2 = cA + (unsigned)t2 * S_K, b2 = cB + (unsigned)t2 * S_K, a3 = cA + (unsigned)t3 * S_K, b3 = cB + (unsigned)t3 * S_K;
            LDB(B0, 0, 0); LDB(B1, 0, 1); SCHED; LDA(At, 0, 0); STAGE(ra, SA(1, 1), a1 + S_H);
            WAIT_V(8); WAIT_B(B0, B1); FLAGREAD; GATE1(t2); BAR; MMA(0, 0, At, B0); MMA(0, 1, At, B1); BAR; SCHED;
            GATE2(t2);
            LDA(At, 0, 1); STAGE(rb, SB(0, 0), b2); STAGE(rb, SB(0, 1), b2 + S_H); STAGE(ra, SA(0, 0), a2);
            WAIT_VP; WAIT_L(0); BAR; MMA(1, 0, At, B0); MMA(1, 1, At, B1); BAR; SCHED;
            LDB(B0, 1, 0); LDB(B1, 1, 1); SCHED; LDA(At, 1, 0); STAGE(ra, SA(0, 1), a2 + S_H);
            WAIT_V(8); WAIT_B(B0, B1); FLAGREAD; GATE1(t3); BAR; MMA(0, 0, At, B0); MMA(0, 1, At, B1); BAR; SCHED;
            GATE2(t3);
            LDA(At, 1, 1); STAGE(rb, SB(1, 0), b3); STAGE(rb, SB(1, 1), b3 + S_H); STAGE(ra, SA(1, 0), a3);
            WAIT_VP; WAIT_L(0); BAR; MMA(1, 0, At, B0); MMA(1, 1, At, B1); BAR; SCHED;
        }
        if (wr == 0) BAR;
        STAMP(dslot); ++dslot;
#define EPI_GROUP(A, pnx, g, slot) do { unsigned _w[4][4]; \
            _Pragma("unroll") for (int m = 0; m < 4; ++m) { \
                _w[m][0] = cvtpk(A[m][0][0], A[m][0][1]); _w[m][1] = cvtpk(A[m][0][2], A[m][0][3]); \
                _w[m][2] = cvtpk(A[m][1][0], A[m][1][1]); _w[m][3] = cvtpk(A[m][1][2], A[m][1][3]); } \
            encode_unit(_w, myws + OFF_CP + (unsigned)(((pm * 16 + (pnx)) * 8 + wid) * 4 + (g)) * UNIT_B, ldsa + LDS_AUX + wid * 1024, \
                        ldsa + LDS_REC + (slot) * (8 * REC_B) + wid * REC_B, call & 0xffffu, lane, false); } while (0)
        auto epi_hdr = [&](int pnx, int g, int slot) {
            unsigned old, bad;
            asm volatile("s_mov_b64 exec, 1\n\tds_add_rtn_u32 %0, %1, %2\n\ts_mov_b64 exec, -1\n\ts_waitcnt lgkmcnt(0)"
                         : "=&v"(old) : "v"(ldsa + LDS_BAD + 16 + slot * 4), "v"(1u) : "memory");
            if (rfl(old) != 7) return false;
            u32x4 hv;
            asm volatile("ds_read_b128 %0, %2\n\tds_read_b32 %1, %3\n\ts_waitcnt lgkmcnt(0)"
                         : "=&v"(hv), "=&v"(bad) : "v"(ldsa + LDS_REC + slot * (8 * REC_B) + min(lane, 47) * 16), "v"(ldsa + LDS_BAD) : "memory");
            if (bad && lane % 6 == 0) hv[0] = (hv[0] & 0xffffu) | POISON << 24;
            if (lane < 48) gst128s_sys<0>(chd + (unsigned)((((((bi * 2 + (pm & 1)) * 32 + bj_ * 16 + pnx) * 4 + g) * 2 + bk) * 8) * REC_B), lane * 16, hv);
            return true;
        };
#define KSPL 1
#define EPI1 1
        if (EPI1) {
            EPI_GROUP(acc[0][0], pn, 0, 0);
            WAIT_V(0);
            epi_hdr(pn, 0, 0);
            EPI_GROUP(acc[0][1], pn, 1, 1);
            EPI_GROUP(acc[1][0], pn, 2, 2);
            WAIT_V(4);
            const bool s1 = epi_hdr(pn, 1, 1);
            EPI_GROUP(acc[1][1], pn, 3, 3);
            if (s1) WAIT_V(5); else WAIT_V(4);
            const bool s2 = epi_hdr(pn, 2, 2);
            if (s2) WAIT_V(1); else WAIT_V(0);
            epi_hdr(pn, 3, 3);
        } else {
            EPI_GROUP(acc[0][0], pn, 0, 0);
            EPI_GROUP(acc[0][1], pn, 1, 1);
            WAIT_V(4);
            const bool s0 = epi_hdr(pn, 0, 0);
            EPI_GROUP(acc[1][0], pn, 2, 2);
            EPI_GROUP(acc[1][1], pn, 3, 3);
            if (s0) WAIT_V(9); else WAIT_V(8);
            const bool s1 = epi_hdr(pn, 1, 1);
            if (s1) WAIT_V(5); else WAIT_V(4);
            const bool s2 = epi_hdr(pn, 2, 2);
            if (s1 && s2) WAIT_V(2); else if (s1 || s2) WAIT_V(1); else WAIT_V(0);
            epi_hdr(pn, 3, 3);
        }
        STAMP(dslot); ++dslot;
        for (int e = pn; e < 4 * PN2; e += PN1) {
            const int ai = e & 1, bj = (e >> 1) & 1, pn2 = PN1 + (e >> 2);
            if (!NOGEMM && !NOGATE) {
                const unsigned long long t0 = __builtin_amdgcn_s_memrealtime();
                const u32x4* f = (const u32x4*)(rdyf + ((16 + pn2) * 64) * NR) + lane;
                for (;;) {
                    u32x4 v0, v1;
                    asm volatile("global_load_dwordx4 %0, %2, off sc0 sc1\n\tglobal_load_dwordx4 %1, %2, off offset:1024 sc0 sc1\n\ts_waitcnt vmcnt(0)"
                                 : "=&v"(v0), "=&v"(v1) : "v"(f) : "memory");
                    bool ok = true;
#pragma unroll
                    for (int q = 0; q < 4; ++q) ok = ok && (int)(v0[q] - call) >= 0 && (int)(v1[q] - call) >= 0;
                    if (__all(ok)) break;
                    if (timed_out(tmo, call, t0)) { *badf = 1; break; }
                    __builtin_amdgcn_s_sleep(16);
                }
            }
            BAR;
            STAMP(dslot); ++dslot;
            const unsigned gA = (unsigned)(2 * pm + ai) * S_H, gB = (unsigned)(2 * pn2 + bj) * S_H;
#pragma unroll
            for (int m = 0; m < 4; ++m)
#pragma unroll
                for (int n = 0; n < 2; ++n) acc[0][0][m][n] = acc[0][1][m][n] = (f32x4){0.f, 0.f, 0.f, 0.f};
#pragma unroll
            for (int q = 0; q < 3; ++q) { STAGE2(ra, q * HTB, gA + q * S_K); STAGE2(rb, (4 + q) * HTB, gB + q * S_K); }
            const unsigned kh = (unsigned)(wid & 1) * 8192u;
            unsigned bk2[2][2];
#pragma unroll
            for (int c = 0; c < 2; ++c)
#pragma unroll
                for (int n = 0; n < 2; ++n) bk2[c][n] = (boff[n] ^ (c ? 64u : 0u)) + kh;
            const unsigned ak2 = ldsa + (unsigned)aoff + kh;
#define RD2(Bx, k, q) do { _Pragma("unroll") for (int c = 0; c < 2; ++c) _Pragma("unroll") for (int n = 0; n < 2; ++n) { \
                TRD(Bx[n][c][0], bk2[c][n], (q) * HTB); TRD(Bx[n][c][1], bk2[c][n], (q) * HTB + 1024); } \
                _Pragma("unroll") for (int m = 0; m < 4; ++m) DSR128(At[m][k], ak2, (q) * HTB + m * 256); } while (0)
#define W2(X, k) asm volatile("s_waitcnt lgkmcnt(0)" : "+v"(X[0][0][0]), "+v"(X[0][0][1]), "+v"(X[0][1][0]), "+v"(X[0][1][1]), "+v"(X[1][0][0]), "+v"(X[1][0][1]), \
                "+v"(X[1][1][0]), "+v"(X[1][1][1]), "+v"(At[0][k]), "+v"(At[1][k]), "+v"(At[2][k]), "+v"(At[3][k]) :: "memory")
#define MM2(Bx, k) do { __builtin_amdgcn_s_setprio(1); _Pragma("unroll") for (int m = 0; m < 4; ++m) _Pragma("unroll") for (int c = 0; c < 2; ++c) \
                _Pragma("unroll") for (int n = 0; n < 2; ++n) \
                    acc[0][c][m][n] = __builtin_amdgcn_mfma_f32_16x16x32_bf16(PKB(Bx[n][c]), At[m][k], acc[0][c][m][n], 0, 0, 0); \
                __builtin_amdgcn_s_setprio(0); } while (0)
            STAGE2(ra, 3 * HTB, gA + 3 * S_K); STAGE2(rb, 7 * HTB, gB + 3 * S_K);
            WAIT_V(12); BAR;
            RD2(B0, 0, 0);
            for (int t = 0; t < (NOGEMM || NOGEMM2 ? 0 : NT); t += 4) {
#pragma unroll
                for (int q = 0; q < 4; ++q) {
                    const unsigned tn = (unsigned)min(t + q + 4, NT - 1) * S_K;
                    WAIT_V(8);
                    if (q & 1) W2(B1, 1); else W2(B0, 0);
                    BAR;
                    if (G2X != 1) { STAGE2(ra, q * HTB, gA + tn); if (G2X != 3) STAGE2(rb, (4 + q) * HTB, gB + tn); }
                    if (q & 1) RD2(B0, 0, (q + 1) & 3); else RD2(B1, 1, (q + 1) & 3);
                    SCHED;
                    if (q & 1) MM2(B1, 1); else MM2(B0, 0);
                    SCHED;
                }
            }
            W2(B0, 0);
            WAIT_V(0); BAR;
#pragma unroll
            for (int m = 0; m < 4; ++m)
#pragma unroll
                for (int n = 0; n < 2; ++n) *(LAS f32x4*)(lds + wid * 8192 + (m * 2 + n) * 1024 + lane * 16) = acc[0][1][m][n];
            WAIT_L(0); BAR;
#pragma unroll
            for (int m = 0; m < 4; ++m)
#pragma unroll
                for (int n = 0; n < 2; ++n) acc[0][0][m][n] += *(const LAS f32x4*)(lds + (wid ^ 1) * 8192 + (m * 2 + n) * 1024 + lane * 16);
            EPI_GROUP(acc[0][0], pn2, ai * 2 + bj, 4 + (e - pn) / PN1);
            WAIT_V(0);
            epi_hdr(pn2, ai * 2 + bj, 4 + (e - pn) / PN1);
            STAMP(dslot); ++dslot;
        }
#undef SA
#undef SB
    } else {
        unsigned cnt0;
        if (FASTC) asm volatile("global_load_dword %0, %1, off sc0 sc1" : "=&v"(cnt0) : "v"(cntp) : "memory");
        for (int q = tid; q <= Q_END; q += NTHR) {
            u32x4 en = {0u, 0u, 0u, 0u};
            if (q < Q_OUT) {
                const int c = in_job(q), e = c >> 6, st = c & 63, mat = e >> 4, blk = e & 15;
                const int gblk = mat == 0 ? (blk >> 1) * 4 + bi * 2 + (blk & 1) : bj_ * 16 + blk;
                en = (u32x4){(unsigned)(e * NR * 64 + st) * REC_B, (unsigned)(((mat * 2 + bk) * 32 + gblk) * 64 + st) * UNIT_B,
                             (unsigned)mat * STG_B + (unsigned)(2 * blk * 64 + st) * HTB, (unsigned)(c * NR * 4)};
            } else if (q < Q_END) {
                const int o = q - Q_OUT, y = out_tile(out_yg(o) >> 2), gg = out_yg(o) & 3, w8 = o & 7;
                en = (u32x4){(unsigned)((((y & 1) * 32 + (y >> 1)) * 4 + gg) * 16 + w8) * REC_B,
                             (unsigned)(((rank * 2 + (y & 1)) * 16 + (y >> 1)) * 32 + w8 * 4 + gg) * UNIT_B,
                             (unsigned)((((y & 1) * 256 + (gg >> 1) * 128 + (w8 >> 2) * 64) * NN + (y >> 1) * 256 + (gg & 1) * 128 + (w8 & 3) * 32) * 2), 0u};
            }
            asm volatile("ds_write_b128 %0, %1" :: "v"(ldsa + LDS_TAB + (unsigned)q * 16u), "v"(en) : "memory");
        }
        if (FASTC) asm volatile("s_waitcnt vmcnt(0)" : "+v"(cnt0) :: "memory"); else cnt0 = gld32_sys(cntp);
        const unsigned call = rfl(cnt0) + 1;
        if (lane == 0) __hip_atomic_store(cntp, call, __ATOMIC_RELAXED, __HIP_MEMORY_SCOPE_AGENT);
        const unsigned par = call & 1;
        unsigned* const tkt = ctl + C_TKT / 4 + par * 1024;
        if (wg == NGEMM && wid == 7 && lane < 16) __hip_atomic_store(ctl + C_TKT / 4 + (par ^ 1) * 1024 + lane * 64, 0u, __ATOMIC_RELAXED, __HIP_MEMORY_SCOPE_SYSTEM);
        WAIT_L(0);
        BAR;
        const rsrc_t rh = mk_rsrc(myws, (unsigned)WS_NEED);
        const unsigned lw = (unsigned)wid * 16384u, lh = LDS_AUX + (unsigned)wid * 1024u;
        const unsigned pat = ldsa + LDS_PAT + (unsigned)wid * 4096u;
        const unsigned vl16 = (unsigned)lane * 16u;
#define PULL_UNIT(rp, slot, so, h1) do { DMA16(rp, slot, vl16, so); DMA16(rp, (slot) + 1024, vl16, (so) + 1024); \
            if ((unsigned)lane < ((h1) >> 8 > RECN ? ((h1) >> 8 > MAXEXC ? 64u : 56u) : 48u)) DMA16(rp, (slot) + 2048, vl16, (so) + 2048); \
            if (UNLIKELY((h1) >> 8 == RAWMODE)) DMA16(rp, (slot) + 3072, vl16, (so) + 3072); } while (0)
#define READ_ENTRY(en, slot, rec, h1) asm volatile("ds_read_u16 %0, %1" : "=&v"(en) \
            : "v"(ldsa + ((h1) >> 8 <= RECN ? (rec) : (slot) + OFF_LIST) \
                  + 2 * list_pos(min((unsigned)lane, min(max((h1) >> 8, 1u), MAXEXC) - 1), (h1) >> 8 <= RECN)) : "memory")
#define READ_UNIT(d, cd, ce, slot, h1) asm volatile("ds_read_b128 %0, %6\n\tds_read_b128 %1, %6 offset:1024\n\t" \
            "s_cmp_eq_u32 %8, 255\n\ts_cbranch_scc1 1f\n\t" \
            "ds_read2_b32 %4, %7 offset1:1\n\tds_read_b32 %5, %7 offset:8\n\ts_branch 2f\n" \
            "1:\n\tds_read_b128 %2, %6 offset:2048\n\tds_read_b128 %3, %6 offset:3072\n" \
            "2:" : "=&v"(d[0]), "=&v"(d[1]), "=&v"(d[2]), "=&v"(d[3]), "=&v"(cd), "=&v"(ce) \
            : "v"(ldsa + (slot) + vl16), "v"(ldsa + (slot) + OFF_CODE + (unsigned)lane * 12u), "s"((h1) >> 8) : "scc", "memory")
#define PATCH_UNIT(d, en, u, h1) do { const unsigned _en = (h1) >> 8 == 0 ? 0x800u : (en), _ad = pat + (u) * 2048 + (_en & 0x7ffu); \
            asm volatile("s_cmp_eq_u32 %6, 255\n\ts_cbranch_scc1 1f\n\t" \
                         "ds_write_b8 %2, %3\n\tds_read_b128 %0, %4\n\tds_read_b128 %1, %4 offset:1024\n\tds_write_b8 %2, %5\n1:" \
                         : "+v"(d[2]), "+v"(d[3]) : "v"(_ad), "v"(_en >> 9 & 0x7cu), "v"(pat + (u) * 2048 + vl16), "v"(4u), "s"((h1) >> 8) : "scc", "memory"); } while (0)
#define WAIT_UNITS(d, cd, ce, hv, nx) asm volatile("s_waitcnt lgkmcnt(0)" : "+v"(d[0][0]), "+v"(d[0][1]), "+v"(d[0][2]), "+v"(d[0][3]), \
            "+v"(d[1][0]), "+v"(d[1][1]), "+v"(d[1][2]), "+v"(d[1][3]), "+v"(cd[0]), "+v"(cd[1]), "+v"(ce[0]), "+v"(ce[1]), \
            "+v"(hv), "+v"(nx) :: "memory")
#define NOCOMM 0
        if (const int cw = NACT == NCW ? (wg - NGEMM) * CW + wid : wid * NCOMM + (wg - NGEMM); wid < CW && cw < NACT && !NOCOMM) {
#pragma unroll
            for (int i = 0; i < 4; ++i) asm volatile("ds_write_b128 %0, %1" :: "v"(pat + i * 1024 + vl16), "v"((u32x4){0x04040404u, 0x04040404u, 0x04040404u, 0x04040404u}) : "memory");
            const int si = cw % (NR - 1), sq = cw / (NR - 1), s = (rank + 1 + si) & 7;
            constexpr int w_in = W_IN;
            const int oc = sq < NACT / 8 ? (s & 3) : (rank & 3);
            const int ti = oc & 1, tj = oc >> 1;
            const rsrc_t rp = mk_rsrc(p.ws[s] + OFF_PACK + (size_t)par * PACK_B, PACK_B);
            const rsrc_t rc0 = mk_rsrc(p.ws[oc] + OFF_CP, CP_B), rc1 = mk_rsrc(p.ws[oc + 4] + OFF_CP, CP_B);
            const rsrc_t ro = mk_rsrc(p.out, (MM / NR) * NN * 2);
            const unsigned hb_in = (unsigned)OFF_IH + par * IH_B + (unsigned)s * (64 * REC_B), hb_out = (unsigned)OFF_CH + (unsigned)((ti * 64 + tj * 16) * 64) * REC_B;
            const unsigned tag = call & 0xffffu;
            const char* const stg = myws + OFF_STG + (unsigned)s * 2048u;
            const char* const rdys = myws + OFF_RDY + (unsigned)s * 4u;
            const unsigned olane = (unsigned)(((ti * 512 + fr) * NN + tj * 4096 + (fq & 1) * 16 + (fq >> 1) * 8) * 2);
            auto hdr = [&](int q, unsigned x) {
                return lane < 12 ? (q < Q_OUT ? hb_in + (unsigned)lane * 16u : hb_out + (unsigned)lane * 16u + (lane < 6 ? 0u : 7 * REC_B)) + x
                                 : (unsigned)CTRL_OFF + C_TMO;
            };
            auto fetch_rec = [&](int q, unsigned x, unsigned slot) { if (lane < 13) DMA16(rh, slot, hdr(q, x), 0); };
            auto entry = [&](int q) {
                u32x4 en;
                asm volatile("ds_read_b128 %0, %1\n\ts_waitcnt lgkmcnt(0)" : "=&v"(en) : "v"(ldsa + LDS_TAB + (unsigned)q * 16u) : "memory");
                return (u32x4){rfl(en[0]), rfl(en[1]), rfl(en[2]), rfl(en[3])};
            };
            auto ready = [&](unsigned hv) { return ((unsigned)__ballot((hv & 0xffffu) == tag) & 0xfffu) == 0xfffu; };
            auto head = [&](unsigned hv, int u) { return (unsigned)__builtin_amdgcn_readlane((int)hv, 6 * u) >> 16; };
            auto gave_up = [&](unsigned hv) { return (unsigned)__builtin_amdgcn_readlane((int)hv, 12) == call; };
            auto pull = [&](int q, unsigned so, unsigned hv, unsigned slot) {
                const unsigned ha = head(hv, 0), hb = head(hv, 1);
                if (q < Q_OUT) {
                    PULL_UNIT(rp, slot, so, ha);
                    PULL_UNIT(rp, slot + 4096, so + UNIT_B, hb);
                } else {
                    PULL_UNIT(rc0, slot, so, ha);
                    PULL_UNIT(rc1, slot + 4096, so, hb);
                }
            };
            auto publish = [&](unsigned w) {
                if (lane < 2) asm volatile("global_store_dword %0, %1, %2 sc0 sc1" :: "v"(w + (unsigned)lane * 32u), "v"(call), "s"(rdys) : "memory");
            };
            auto nan_fill = [&](unsigned z) {
#pragma unroll
                for (int m = 0; m < 4; ++m)
                    st128<0>((u32x4){0x7fc07fc0u, 0x7fc07fc0u, 0x7fc07fc0u, 0x7fc07fc0u}, ro, olane + z, m * 16 * NN * 2);
            };
            const unsigned* const tki = tkt + si * 64;
            const unsigned* const tko = tkt + (8 + oc) * 64;
            int o1 = -1;
            unsigned tk, tk0;
            TICKET(tk0, tko);
            if (!FASTC || sq >= w_in) { WAIT_V(0); o1 = (int)min((unsigned)Q_OUT + ticket_value(tk0), (unsigned)Q_END); }
            static_assert(4 * (W_IN + XE) <= JE, "");
            const bool helper = XE && sq >= w_in && sq < w_in + XE;
            const int lim = helper ? JE : Q_OUT;
            const unsigned* const tkd = tkt + 15 * 64;
            auto next = [&](int qprev, unsigned t) {
                if (qprev >= Q_OUT) return (int)min((unsigned)Q_OUT + t, (unsigned)Q_END);
                if (XE && qprev >= lim) return o1;
                const int j = 4 * (w_in + XE) + (int)t;
                return j < Q_OUT ? j : o1;
            };
            int qA = sq, qB = sq + w_in, qC = sq + 2 * w_in;
            if (helper) {
                qA = 4 * w_in + (sq - w_in); qB = qA + XE; qC = qB + XE;
                asm volatile("v_mov_b32 %0, %1" : "=v"(tk) : "v"((unsigned)(sq - w_in - XE)));
            } else if (sq >= w_in) {
                unsigned tb, tc;
                TICKET(tb, tko); TICKET(tc, tko); TICKET(tk, tko); WAIT_V(0);
                qA = o1; qB = next(Q_OUT, ticket_value(tb)); qC = next(Q_OUT, ticket_value(tc));
            } else {
                asm volatile("v_mov_b32 %0, %1" : "=v"(tk) : "v"((unsigned)(sq - w_in - 4 * XE)));
            }
            u32x4 eA = entry(qA), eB = entry(qB), eC = entry(qC);
            bool dead = false;
            while (qA < Q_END) {
                unsigned hvA, hvB;
                gld32x2_sys(myws + hdr(qA, eA[0]), myws + hdr(qB, eB[0]), hvA, hvB);
                if (FASTC && o1 < 0) o1 = (int)min((unsigned)Q_OUT + ticket_value(tk0), (unsigned)Q_END);
                if (!ready(hvA)) {
                    const unsigned long long t0 = __builtin_amdgcn_s_memrealtime();
                    for (;;) {
                        __builtin_amdgcn_s_sleep(CSLEEP);
                        gld32x2_sys(myws + hdr(qA, eA[0]), myws + hdr(qB, eB[0]), hvA, hvB);
                        if (ready(hvA)) break;
                        if (gave_up(hvA) || __builtin_amdgcn_s_memrealtime() - t0 > DEADLINE) {
                            __hip_atomic_store(tmo, call, __ATOMIC_RELAXED, __HIP_MEMORY_SCOPE_AGENT); dead = true; break;
                        }
                    }
                    if (dead) break;
                }
                unsigned sl = lw, hs = 0;
                pull(qA, eA[1], hvA, sl);
                fetch_rec(qA, eA[0], lh); fetch_rec(qB, eB[0], lh + 256); fetch_rec(qC, eC[0], lh + 512);
                int nst = 0;
                unsigned pend = 0;
                unsigned pend2 = 0;
                bool have2 = false;
                bool soon = false, soon2 = false;
                for (;;) {
                    const bool more = qB < Q_END && ready(hvB);
                    if (more) {
                        if (nst == 8) WAIT_V(8); else if (nst == 4) WAIT_V(4); else WAIT_V(0);
                        pull(qB, eB[1], hvB, sl ^ 8192);
                        if (have2) publish(pend2);
                        have2 = nst == 8; pend2 = pend; soon2 = soon;
                    } else {
                        WAIT_V(0);
                        if (have2) publish(pend2);
                        have2 = false;
                        if (nst == 8) { publish(pend); nst = 0; }
                    }
                    const int qD = next(qC, ticket_value(tk));
                    unsigned hvC;
                    u32x4 d[2][4], eDv;
                    u32x2 cd[2];
                    unsigned ce[2], en0, en1;
                    const unsigned hu[2] = {head(hvA, 0), head(hvA, 1)};
                    READ_ENTRY(en0, sl, lh + hs, hu[0]);
                    READ_ENTRY(en1, sl + 4096, lh + hs + REC_B, hu[1]);
                    DSR128(eDv, ldsa + LDS_TAB + (unsigned)qD * 16u, 0);
                    DSR32(hvC, ldsa + lh + ((hs + 512) & 768) + min((unsigned)lane, 12u) * 16u);
                    READ_UNIT(d[0], cd[0], ce[0], sl, hu[0]);
                    READ_UNIT(d[1], cd[1], ce[1], sl + 4096, hu[1]);
                    asm volatile("s_waitcnt lgkmcnt(10)" : "+v"(en0), "+v"(en1) :: "memory");
                    PATCH_UNIT(d[0], en0, 0, hu[0]);
                    PATCH_UNIT(d[1], en1, 1, hu[1]);
                    WAIT_UNITS(d, cd, ce, hvC, eDv);
                    const u32x4 eD = {rfl(eDv[0]), rfl(eDv[1]), rfl(eDv[2]), rfl(eDv[3])};
                    TICKET(tk, qD < lim ? tki : XE && qD < Q_OUT ? tkd : tko);
                    if (more) fetch_rec(qD, eD[0], lh + ((hs + 768) & 768));
                    hs = (hs + 256) & 768;
                    if (qA < Q_OUT) {
                        const char* dst = stg + eA[2];
#pragma unroll
                        for (int u = 0; u < 2; ++u) {
                            const unsigned h1 = hu[u];
                            u32x4 o[4];
                            if (!UNLIKELY(h1 >> 8 == RAWMODE)) {
                                unsigned H[8] = {d[u][2][0], d[u][2][1], d[u][2][2], d[u][2][3], d[u][3][0], d[u][3][1], d[u][3][2], d[u][3][3]};
                                decode_high(cd[u][0], cd[u][1], ce[u], h1, H);
#pragma unroll
                                for (int m = 0; m < 4; ++m) {
                                    const unsigned l0 = d[u][m >> 1][(m & 1) * 2], l1 = d[u][m >> 1][(m & 1) * 2 + 1];
                                    o[m] = (u32x4){vperm(H[2 * m], l0, 0x05010400u), vperm(H[2 * m], l0, 0x07030602u),
                                                   vperm(H[2 * m + 1], l1, 0x05010400u), vperm(H[2 * m + 1], l1, 0x07030602u)};
                                }
                                if (h1 & 0x80u) {
#pragma unroll
                                    for (int m = 0; m < 4; ++m)
#pragma unroll
                                        for (int j = 0; j < 4; ++j) o[m][j] = pk_add16(o[m][j], 0xff80ff80u);
                                }
                            } else {
#pragma unroll
                                for (int m = 0; m < 4; ++m) o[m] = d[u][m];
                            }
                            if (!PUB2 && u == 0 && nst == 8) {
                                asm volatile("s_waitcnt vmcnt(8)" : "+v"(o[0]), "+v"(o[1]), "+v"(o[2]), "+v"(o[3]) :: "memory");
                                publish(pend);
                            }
                            if (JP0 < NJ1 && u == 0 && have2 && soon2) {
                                asm volatile("s_waitcnt vmcnt(8)" : "+v"(o[0]), "+v"(o[1]), "+v"(o[2]), "+v"(o[3]) :: "memory");
                                publish(pend2); have2 = false;
                            }
                            const char* d0 = dst + u * HTB;
                            gst128s_stg<0>(d0, vl16, o[0]); gst128s_stg<1024>(d0, vl16, o[1]);
                            gst128s_stg<0>(d0 + 64 * HTB, vl16, o[2]); gst128s_stg<1024>(d0 + 64 * HTB, vl16, o[3]);
                        }
                        pend = eA[3]; nst = 8;
                        soon = qA >= JP0 && qA < NJ1;
                    } else {
                        const bool bad = gave_up(hvA) || hu[0] >> 8 == POISON || hu[1] >> 8 == POISON;
                        if (!PUB2 && nst == 8) { WAIT_V(8); publish(pend); }
                        if (UNLIKELY(bad)) nan_fill(eA[2]);
                        else {
                            float a[4][8];
#pragma unroll
                            for (int k = 0; k < 2; ++k) {
                                const unsigned h1 = hu[k];
                                if (UNLIKELY(h1 >> 8 == RAWMODE)) {
#pragma unroll
                                    for (int m = 0; m < 4; ++m)
#pragma unroll
                                        for (int j = 0; j < 4; ++j) {
                                            const unsigned v = d[k][m][j];
                                            const float lo = __uint_as_float(v << 16), hi = __uint_as_float(v & 0xffff0000u);
                                            a[m][2 * j] = k == 0 ? lo : a[m][2 * j] + lo; a[m][2 * j + 1] = k == 0 ? hi : a[m][2 * j + 1] + hi;
                                        }
                                } else {
                                    unsigned H[8] = {d[k][2][0], d[k][2][1], d[k][2][2], d[k][2][3], d[k][3][0], d[k][3][1], d[k][3][2], d[k][3][3]};
                                    decode_high(cd[k][0], cd[k][1], ce[k], h1, H);
                                    unsigned vb[4][8];
#pragma unroll
                                    for (int m = 0; m < 4; ++m)
#pragma unroll
                                        for (int j = 0; j < 8; ++j) {
                                            const unsigned Lw = d[k][m >> 1][(m & 1) * 2 + (j >> 2)];
                                            vb[m][j] = vperm(H[2 * m + (j >> 2)], Lw, (4u + (j & 3)) << 24 | (unsigned)(j & 3) << 16 | 0x0c0cu);
                                        }
                                    if (h1 & 0x80u) {
#pragma unroll
                                        for (int m = 0; m < 4; ++m)
#pragma unroll
                                            for (int j = 0; j < 8; ++j) vb[m][j] -= 0x00800000u;
                                    }
#pragma unroll
                                    for (int m = 0; m < 4; ++m)
#pragma unroll
                                        for (int j = 0; j < 8; ++j) a[m][j] = k == 0 ? __uint_as_float(vb[m][j]) : a[m][j] + __uint_as_float(vb[m][j]);
                                }
                            }
#pragma unroll
                            for (int m = 0; m < 4; ++m) {
                                unsigned r0 = cvtpk(a[m][0], a[m][1]), r1 = cvtpk(a[m][2], a[m][3]), r2 = cvtpk(a[m][4], a[m][5]), r3 = cvtpk(a[m][6], a[m][7]);
                                auto s02 = __builtin_amdgcn_permlane16_swap(r0, r2, false, false);
                                auto s13 = __builtin_amdgcn_permlane16_swap(r1, r3, false, false);
                                u32x4 o; o.x = s02[0]; o.y = s13[0]; o.z = s02[1]; o.w = s13[1];
                                st128<OAUX>(o, ro, olane + eA[2], m * 16 * NN * 2);
                            }
                        }
                        STAMPJ(8 + oc, qA - Q_OUT);
                        if (qA == o1) STAMPW(32);
                        nst = 4;
                    }
                    qA = qB; qB = qC; qC = qD; eA = eB; eB = eC; eC = eD; hvA = hvB; hvB = hvC; sl ^= 8192;
                    if (!more) break;
                }
                if (nst == 8) { WAIT_V(0); publish(pend); }
            }
            if (dead) {
                while (qA < Q_END) {
                    if (qA >= Q_OUT) nan_fill(eA[2]);
                    WAIT_V(0);
                    const int qD = next(qC, ticket_value(tk));
                    TICKET(tk, qD < Q_OUT ? tki : tko);
                    qA = qB; qB = qC; qC = qD; eA = eB; eB = eC; eC = entry(qD);
                }
            }
            WAIT_V(0);
        }
        STAMPW(40);
    }
}

}

extern "C" void kernel_launch(int rank, int nranks, ncclComm_t comm,
                              hipStream_t stream,
                              void* const* d_in, const int* in_sizes, int n_in,
                              void* d_out, int out_size,
                              const int* in_dtypes, int out_dtype,
                              void* const* d_out_all,
                              void* const* d_ws_all, size_t ws_bytes) {
    (void)comm; (void)in_dtypes; (void)out_dtype; (void)d_out_all;
    if (nranks != NR || n_in < 2 || in_sizes[0] != MM * KL || in_sizes[1] != KL * NN || out_size != (MM / NR) * NN
        || ws_bytes < WS_NEED || (unsigned)rank >= (unsigned)NR) {
        fprintf(stderr, "kernel_launch: unexpected shapes\n");
        return;
    }
    static bool inited[NR] = {};
    if (!inited[rank]) {
        HIP_CHECK(hipFuncSetAttribute((const void*)gemm_rs_kernel, hipFuncAttributeMaxDynamicSharedMemorySize, LDS_BYTES));
        inited[rank] = true;
    }
    Params p;
    p.x = d_in[0]; p.w = d_in[1]; p.out = d_out; p.rank = rank;
    for (int i = 0; i < NR; ++i) p.ws[i] = (char*)d_ws_all[i];
    hipLaunchKernelGGL(gemm_rs_kernel, dim3(NWG), dim3(NTHR), LDS_BYTES, stream, p);
    HIP_CHECK(hipGetLastError());
}
```

```cpp
#include <hip/hip_runtime.h>
#include <rccl/rccl.h>
#include <cstdio>
#include <cstdint>

#define HIP_CHECK(expr) do { hipError_t _e = (expr); if (_e != hipSuccess) { \
    fprintf(stderr, "HIP error %s at %s:%d\n", hipGetErrorString(_e), __FILE__, __LINE__); \
    return; } } while (0)

#define LAS __attribute__((address_space(3)))
#define UNLIKELY(x) __builtin_expect(!!(x), 0)
typedef short bf16x8 __attribute__((ext_vector_type(8)));
typedef short s16x4 __attribute__((ext_vector_type(4)));
typedef float f32x4 __attribute__((ext_vector_type(4)));
typedef float f32x2 __attribute__((ext_vector_type(2)));
typedef unsigned u32x4 __attribute__((ext_vector_type(4)));
typedef unsigned u32x3 __attribute__((ext_vector_type(3)));
typedef unsigned u32x2 __attribute__((ext_vector_type(2)));
typedef unsigned short u16x2 __attribute__((ext_vector_type(2)));
typedef __bf16 bf16x2_t __attribute__((ext_vector_type(2)));
typedef __amdgpu_buffer_rsrc_t rsrc_t;

namespace {

constexpr int NR = 8;
constexpr int MM = 8192, NN = 8192, KL = 1024;
constexpr int NWG = 256, NTHR = 512;
#define PN1 13
#define LOOK 2
#define CW 3
#define LATE 1
#define PUB2 1
#define LEAD 1
#define GSLEEP 2
#define CSLEEP 2
#define FASTP 1
#define FASTC 1
#define URG_LO 0
#define URG_HI 4
#define RAWT 0
#define PKSLEEP 0
#define FORCE_RAW 0
#define NOGEMM 0
#define GAUX 17
#define GAUX2 GAUX
#define G2X 0
#define NOGEMM2 0
#define XAUX 2
#define OAUX 2
constexpr int NGEMM = 16 * PN1, NCOMM = NWG - NGEMM, NCW = NCOMM * CW;
constexpr int PN2 = 16 - PN1;
constexpr int HTB = 16384, STAGE_BYTES = 8 * HTB;
constexpr int NT = 64;
constexpr unsigned UNIT_B = 4096;
constexpr size_t MIB = 1u << 20;
constexpr size_t OFF_PACK = 0;
constexpr unsigned PACK_B = 32u << 20;
constexpr size_t OFF_STG = 64 * MIB;
constexpr unsigned STG_B = 32u << 20;
constexpr size_t OFF_CP = 128 * MIB;
constexpr unsigned CP_B = 32u << 20;
constexpr unsigned REC_B = 96;
constexpr size_t OFF_IH = 164 * MIB;
constexpr unsigned IH_B = 2 * 16 * 8 * 64 * REC_B;
constexpr size_t OFF_RDY = 161 * MIB;
constexpr size_t OFF_CH = 162 * MIB;
constexpr size_t CTRL_OFF = 256 * MIB;
constexpr size_t WS_NEED = CTRL_OFF + MIB;
constexpr unsigned C_TMO = 256, C_CNT = 4096;
constexpr unsigned C_TKT = 16384;
constexpr unsigned MAXEXC = 64, RECN = 41, RAWMODE = 255, POISON = 254;
constexpr int OFF_CODE = 2048, OFF_LIST = 2816;
constexpr unsigned long long DEADLINE = 30000000ull;
constexpr int LDS_AUX = STAGE_BYTES;
constexpr int LDS_BAD = STAGE_BYTES + 8192;
constexpr int LDS_REC = LDS_BAD + 128;
constexpr int LDS_BYTES = LDS_REC + 6 * 8 * REC_B;
constexpr int AUX_SYS = 17;
constexpr int E1 = 16 + PN1;
constexpr int NJ1 = (NT / 2) * E1, NJ_IN = (NT / 2) * 32;
#define NFB 4
#define SKD 3
constexpr int EF = 8 + NFB, ES = E1 - EF;
constexpr int Q_OUT = NJ_IN, Q_END = Q_OUT + 1024;
constexpr int LDS_PAT = 98304 + 2048;
constexpr int LDS_TAB = 65536;
__host__ __device__ constexpr int in_job(int j) {
    constexpr int R1 = SKD, R2 = NT / 2 - SKD;
    int i = 0, r = 0;
    if (j < R1 * EF) { i = j % EF; r = j / EF; }
    else if (j < R1 * EF + R2 * E1) { const int q = j - R1 * EF; i = q % E1; r = q / E1 + (i < EF ? SKD : 0); }
    else if (j < NJ1) { const int q = j - (R1 * EF + R2 * E1); i = EF + q % ES; r = NT / 2 - SKD + q / ES; }
    else { const int q = j - NJ1; i = E1 + q % PN2; r = q / PN2; }
    const int e = i < 8 ? 2 * i : i < EF ? 8 + i : i < 8 + EF ? 2 * (i - EF) + 1 : i;
    return e * 64 + 2 * r;
}
#define ORD 3
__host__ __device__ constexpr int out_yg(int o) {
    constexpr int NL = NFB, NM = 2 * PN1 - NFB;
    const int u = o >> 3;
    if ((ORD & 1) && u >= 4 * NL && u < 4 * (NL + NM)) return (NL + (u - 4 * NL) % NM) * 4 + (u - 4 * NL) / NM;
    if ((ORD & 2) && u < 4 * NL) return (u % NL) * 4 + u / NL;
    return u;
}
__host__ __device__ constexpr int out_tile(int y) { return y < NFB ? 2 * y : y < 2 * NFB ? 2 * (y - NFB) + 1 : y; }
#define W_IN 14
#define JP0 NJ1
#define XE 0
#define JE 128
#define NACT 120

#define STAMP(slot) do {} while (0)
#define STAMPW(slot) do {} while (0)
#define STAMPJ(row, j) do {} while (0)
#define DIAG_BLK0 do {} while (0)
#define DIAG_BLK1(t0) do {} while (0)
struct Params {
    const void* x; const void* w; void* out;
    char* ws[NR];
    int rank;
};

__device__ __forceinline__ unsigned cvtpk(float lo, float hi) { f32x2 v = {lo, hi}; bf16x2_t b = __builtin_convertvector(v, bf16x2_t); return __builtin_bit_cast(unsigned, b); }
__device__ __forceinline__ rsrc_t mk_rsrc(const void* p, unsigned bytes) { return __builtin_amdgcn_make_buffer_rsrc((void*)p, 0, bytes, 0x00020000); }

template <int AUX> __device__ __forceinline__ void st128(u32x4 v, rsrc_t r, unsigned voff, unsigned soff) {
    __builtin_amdgcn_raw_buffer_store_b128(v, r, voff, soff, AUX);
    asm volatile("s_nop 1" :: "v"(v) : "memory");
}
__device__ __forceinline__ void gst128_sys(void* ptr, u32x4 v) {
    asm volatile("global_store_dwordx4 %0, %1, off sc0 sc1\n\ts_nop 1" :: "v"(ptr), "v"(v) : "memory");
}
__device__ __forceinline__ unsigned gld32_sys(const void* ptr) {
    unsigned v;
    asm volatile("global_load_dword %0, %1, off sc0 sc1\n\ts_waitcnt vmcnt(0)" : "=&v"(v) : "v"(ptr) : "memory");
    return v;
}
__device__ __forceinline__ void gld32x2_sys(const void* pa, const void* pb, unsigned& a, unsigned& b) {
    asm volatile("global_load_dword %0, %2, off sc0 sc1\n\tglobal_load_dword %1, %3, off sc0 sc1\n\ts_waitcnt vmcnt(0)"
                 : "=&v"(a), "=&v"(b) : "v"(pa), "v"(pb) : "memory");
}
template <int OFF> __device__ __forceinline__ void gst128s_sys(const void* base, unsigned voff, u32x4 v) {
    asm volatile("global_store_dwordx4 %0, %1, %2 offset:%c3 sc0 sc1\n\ts_nop 1" :: "v"(voff), "v"(v), "s"(base), "i"(OFF) : "memory");
}
#define SPOL "sc0 sc1"
template <int OFF> __device__ __forceinline__ void gst128s_stg(const void* base, unsigned voff, u32x4 v) {
    asm volatile("global_store_dwordx4 %0, %1, %2 offset:%c3 " SPOL "\n\ts_nop 1" :: "v"(voff), "v"(v), "s"(base), "i"(OFF) : "memory");
}
__device__ __forceinline__ unsigned bfi(unsigned m, unsigned a, unsigned b) { return (a & m) | (b & ~m); }
__device__ __forceinline__ unsigned vperm(unsigned hi, unsigned lo, unsigned sel) { return __builtin_amdgcn_perm(hi, lo, sel); }
__device__ __forceinline__ unsigned rfl(unsigned v) { return __builtin_amdgcn_readfirstlane(v); }
__device__ __forceinline__ bool timed_out(unsigned* tmo, unsigned call, unsigned long long t0) {
    if (__hip_atomic_load(tmo, __ATOMIC_RELAXED, __HIP_MEMORY_SCOPE_AGENT) == call) return true;
    if (__builtin_amdgcn_s_memrealtime() - t0 > DEADLINE) { __hip_atomic_store(tmo, call, __ATOMIC_RELAXED, __HIP_MEMORY_SCOPE_AGENT); return true; }
    return false;
}

__device__ __forceinline__ unsigned wave_sum_u(unsigned x) {
    x += (unsigned)__builtin_amdgcn_update_dpp(0, (int)x, 0xB1, 0xf, 0xf, true);
    x += (unsigned)__builtin_amdgcn_update_dpp(0, (int)x, 0x4E, 0xf, 0xf, true);
    x += (unsigned)__builtin_amdgcn_update_dpp(0, (int)x, 0x141, 0xf, 0xf, true);
    x += (unsigned)__builtin_amdgcn_update_dpp(0, (int)x, 0x140, 0xf, 0xf, true);
    return (unsigned)(__builtin_amdgcn_readlane((int)x, 0) + __builtin_amdgcn_readlane((int)x, 16) + __builtin_amdgcn_readlane((int)x, 32) + __builtin_amdgcn_readlane((int)x, 48));
}
__device__ __forceinline__ unsigned pk_add16(unsigned a, unsigned b) { return __builtin_bit_cast(unsigned, __builtin_bit_cast(u16x2, a) + __builtin_bit_cast(u16x2, b)); }
__device__ __forceinline__ unsigned mbcnt(unsigned long long m) { return __builtin_amdgcn_mbcnt_hi((unsigned)(m >> 32), __builtin_amdgcn_mbcnt_lo((unsigned)m, 0u)); }
__device__ __forceinline__ unsigned list_pos(unsigned i, bool rec) {
    return i + (rec ? 2u : 0u) + (__umul24(i + 1, rec ? 37u : 0u) >> 8);
}
__device__ __forceinline__ void encode_unit(unsigned (&w)[4][4], const char* dst, unsigned lst, unsigned rec, unsigned tag, int lane, bool raw) {
    unsigned h1;
    if (raw) {
        const unsigned vo = (unsigned)lane * 16u;
        gst128s_sys<0>(dst, vo, (u32x4){w[0][0], w[0][1], w[0][2], w[0][3]});
        gst128s_sys<1024>(dst, vo, (u32x4){w[1][0], w[1][1], w[1][2], w[1][3]});
        gst128s_sys<2048>(dst, vo, (u32x4){w[2][0], w[2][1], w[2][2], w[2][3]});
        gst128s_sys<3072>(dst, vo, (u32x4){w[3][0], w[3][1], w[3][2], w[3][3]});
        if (lane < 7) asm volatile("ds_write_b16 %0, %1" :: "v"(rec + (lane < 6 ? (unsigned)lane * 16u : 2u)), "v"(lane < 6 ? tag : RAWMODE << 8) : "memory");
        return;
    }
    unsigned sum = 0;
#pragma unroll
    for (int m = 0; m < 4; ++m) sum = __builtin_amdgcn_udot2(__builtin_bit_cast(u16x2, w[m][0] & 0x7f807f80u), (u16x2){1, 1}, sum, false);
    const unsigned T = (wave_sum_u(sum) + 185467u) >> 16;
    const unsigned par = ~T & 1u, top = (T + par) >> 1;
    if (par) {
#pragma unroll
        for (int m = 0; m < 4; ++m)
#pragma unroll
            for (int q = 0; q < 4; ++q) w[m][q] = pk_add16(w[m][q], 0x00800080u);
    }
    const unsigned bq = (top | 0x80u) * 0x01010101u;
    unsigned L[8], d[8], c[8], fl[4];
#pragma unroll
    for (int g = 0; g < 8; ++g) {
        const unsigned lo = w[g >> 1][(g & 1) * 2], hi = w[g >> 1][(g & 1) * 2 + 1];
        L[g] = vperm(hi, lo, 0x06040200u);
        const unsigned h = vperm(hi, lo, 0x07050301u);
        d[g] = bq - (h & 0x7f7f7f7fu);
        c[g] = (h >> 5 & 0x04040404u) | (d[g] & 0x03030303u);
        const unsigned f = (((d[g] >> 2 & 0x3f3f3f3fu) ^ 0x20202020u) + 0x7f7f7f7fu) & 0x80808080u;
        fl[g >> 1] = __builtin_amdgcn_udot4(f, g & 1 ? 0x80402010u : 0x08040201u, g & 1 ? fl[g >> 1] : 0u, false);
    }
    unsigned ex = fl[0] >> 7 | fl[1] << 1 | fl[2] << 9 | fl[3] << 17;
    const unsigned n = (unsigned)__builtin_popcount(ex);
    const unsigned long long b0 = __ballot(n & 1), b1 = __ballot(n & 2), b2 = __ballot(n & 4);
    unsigned cnt = (unsigned)(__builtin_popcountll(b0) + 2 * __builtin_popcountll(b1) + 4 * __builtin_popcountll(b2));
    if (FORCE_RAW || top > 123 || cnt > MAXEXC || __ballot(n >= 8) != 0) cnt = RAWMODE;
    if (!UNLIKELY(cnt == RAWMODE)) {
        const bool inrec = cnt <= RECN;
        const unsigned lb = inrec ? rec : lst;
        unsigned li = mbcnt(b0) + 2 * mbcnt(b1) + 4 * mbcnt(b2), inv = 0;
        while (__ballot(ex != 0) != 0) {
            const unsigned j = (unsigned)__builtin_ctz(ex | 0x80000000u) , sel = (j & 7) | 0x0c0c0c00u;
            const unsigned r0 = vperm(d[1], d[0], sel), r1 = vperm(d[3], d[2], sel), r2 = vperm(d[5], d[4], sel), r3 = vperm(d[7], d[6], sel);
            const unsigned ra = j & 8 ? r1 : r0, rb = j & 8 ? r3 : r2, db = j & 16 ? rb : ra;
            const unsigned q = ((db >> 2) + 1) ^ 0x20u;
            inv |= q;
            const unsigned en = (j & 16) << 6 | (unsigned)lane << 4 | (j & 15) | q << 11;
            if (ex) asm volatile("ds_write_b16 %0, %1" :: "v"(lb + 2 * list_pos(li, inrec)), "v"(en) : "memory");
            li += 1; ex &= ex - 1;
        }
        if (__ballot(inv > 31) != 0) cnt = RAWMODE;
    }
    const unsigned vo = (unsigned)lane * 16u;
    if (UNLIKELY(cnt == RAWMODE)) {
        const unsigned back = par ? 0xff80ff80u : 0u;
#pragma unroll
        for (int m = 0; m < 4; ++m)
#pragma unroll
            for (int q = 0; q < 4; ++q) w[m][q] = pk_add16(w[m][q], back);
        gst128s_sys<0>(dst, vo, (u32x4){w[0][0], w[0][1], w[0][2], w[0][3]});
        gst128s_sys<1024>(dst, vo, (u32x4){w[1][0], w[1][1], w[1][2], w[1][3]});
        gst128s_sys<2048>(dst, vo, (u32x4){w[2][0], w[2][1], w[2][2], w[2][3]});
        gst128s_sys<3072>(dst, vo, (u32x4){w[3][0], w[3][1], w[3][2], w[3][3]});
        h1 = RAWMODE << 8;
    } else {
        u32x4 lv;
        asm volatile("ds_read_b128 %0, %1" : "=&v"(lv) : "v"(lst + ((unsigned)lane & 7u) * 16u) : "memory");
        gst128s_sys<0>(dst, vo, (u32x4){L[0], L[1], L[2], L[3]});
        gst128s_sys<1024>(dst, vo, (u32x4){L[4], L[5], L[6], L[7]});
        const u32x3 cd = {c[0] | c[1] << 3 | (c[6] << 6 & 0xc0c0c0c0u), c[2] | c[3] << 3 | (c[7] << 6 & 0xc0c0c0c0u),
                          c[4] | c[5] << 3 | (c[6] << 4 & 0x40404040u) | (c[7] << 5 & 0x80808080u)};
        asm volatile("global_store_dwordx3 %0, %1, %2 offset:%c3 sc0 sc1\n\ts_nop 1" :: "v"((unsigned)lane * 12u), "v"(cd), "s"(dst), "i"(OFF_CODE) : "memory");
        asm volatile("s_waitcnt lgkmcnt(0)" : "+v"(lv) :: "memory");
        if (lane < 8) gst128s_sys<OFF_LIST>(dst, vo, lv);
        h1 = top | par << 7 | cnt << 8;
    }
    if (lane < 7) asm volatile("ds_write_b16 %0, %1" :: "v"(rec + (lane < 6 ? (unsigned)lane * 16u : 2u)), "v"(lane < 6 ? tag : h1) : "memory");
}
__device__ __forceinline__ void decode_high(unsigned d0, unsigned d1, unsigned d2, unsigned h1, unsigned (&H)[8]) {
    const unsigned lutl = ((h1 & 0x7fu) + 4) * 0x01010101u - 0x03020100u, luth = lutl | 0x80808080u;
    const unsigned c[8] = {d0 & 0x07070707u, d0 >> 3 & 0x07070707u, d1 & 0x07070707u, d1 >> 3 & 0x07070707u, d2 & 0x07070707u, d2 >> 3 & 0x07070707u,
                           (d0 >> 6 & 0x03030303u) | (d2 >> 4 & 0x04040404u), (d1 >> 6 & 0x03030303u) | (d2 >> 5 & 0x04040404u)};
#pragma unroll
    for (int g = 0; g < 8; ++g) H[g] = vperm(luth, lutl, c[g]) - H[g];
}

#define TRD(dst, addr, off) asm volatile("ds_read_b64_tr_b16 %0, %1 offset:%c2" : "=&v"(dst) : "v"(addr), "i"(off) : "memory")
#define PKB(x) (bf16x8){(x)[0][0], (x)[0][1], (x)[0][2], (x)[0][3], (x)[1][0], (x)[1][1], (x)[1][2], (x)[1][3]}
#define WAIT_V(n) asm volatile("s_waitcnt vmcnt(" #n ")" ::: "memory")
#define WAIT_L(n) asm volatile("s_waitcnt lgkmcnt(" #n ")" ::: "memory")
#define BAR __builtin_amdgcn_s_barrier()
#define SCHED __builtin_amdgcn_sched_barrier(0)
#define DSR128(dst, addr, off) asm volatile("ds_read_b128 %0, %1 offset:%c2" : "=&v"(dst) : "v"(addr), "i"(off) : "memory")
#define DSR32(dst, addr) asm volatile("ds_read_b32 %0, %1" : "=&v"(dst) : "v"(addr) : "memory")
#define DMA16(rs, ldsoff, voff, soff) __builtin_amdgcn_raw_ptr_buffer_load_lds(rs, (LAS void*)(lds + (ldsoff)), 16, voff, soff, 0, AUX_SYS)
#define TICKET(tk, ptr) asm volatile("s_mov_b64 exec, 1\n\tglobal_atomic_add %0, %1, %2, %3 sc0 sc1\n\ts_mov_b64 exec, -1" : "=&v"(tk) : "v"(0u), "v"(1u), "s"(ptr) : "memory")
__device__ __forceinline__ unsigned ticket_value(unsigned tk) {
    unsigned r;
    asm volatile("v_readfirstlane_b32 %0, %1" : "=s"(r) : "v"(tk) : "memory");
    return r;
}

__global__ __launch_bounds__(NTHR, 2) void gemm_rs_kernel(Params p) {
    extern __shared__ __attribute__((aligned(16))) unsigned char smem[];
    LAS unsigned char* lds = (LAS unsigned char*)smem;
    const int tid = threadIdx.x, wid = __builtin_amdgcn_readfirstlane(tid >> 6), lane = tid & 63;
    const int wr = wid >> 2, wc = wid & 3, fr = lane & 15, fq = lane >> 4;
    const int wg = blockIdx.x, rank = p.rank;
    const int bi = rank & 1, bj_ = (rank >> 1) & 1, bk = rank >> 2;
    char* myws = p.ws[rank];
    unsigned* ctl = (unsigned*)(myws + CTRL_OFF);
    unsigned* tmo = ctl + C_TMO / 4;

    int dslot = 0;
    STAMP(dslot); ++dslot;
    const unsigned ldsa = (unsigned)(size_t)lds;
    unsigned* cntp = ctl + C_CNT / 4 + wg * 8 + wid;

    if (wg < NGEMM) {
        const rsrc_t rx = mk_rsrc(p.x, MM * KL * 2), rw = mk_rsrc(p.w, KL * NN * 2);
        const unsigned vx = (unsigned)((tid >> 3) * (KL * 2) + (((tid & 7) ^ ((tid >> 4) & 7)) << 4));
        const unsigned vw = (unsigned)(fq * (NN * 2) + ((fr ^ ((fq << 2) | ((rank & 1) << 1))) << 4));
        auto pk_issue = [&](int idx, int buf) {
            const int g8 = idx >> 7, mat = (idx >> 6) & 1, kk = (idx >> 5) & 1, blk = idx & 31;
            if (mat == 0) {
#pragma unroll
                for (int i = 0; i < 4; ++i)
                    __builtin_amdgcn_raw_ptr_buffer_load_lds(rx, (LAS void*)(lds + buf + i * 8192 + wid * 1024), 16, vx,
                                                             (unsigned)((blk * 256 + i * 64) * (KL * 2) + kk * 1024 + g8 * 128), 0, XAUX);
            } else {
#pragma unroll
                for (int c = 0; c < 4; ++c)
                    __builtin_amdgcn_raw_ptr_buffer_load_lds(rw, (LAS void*)(lds + buf + wid * 4096 + c * 1024), 16, vw,
                                                             (unsigned)((kk * 512 + (g8 * 8 + wid) * 8 + (c & 1) * 4) * (NN * 2) + blk * 512 + (c >> 1) * 256), 0, XAUX);
            }
        };
        pk_issue(wg, 0);
        volatile LAS unsigned* const badf = (volatile LAS unsigned*)(lds + LDS_BAD);
        if (tid < 16) asm volatile("ds_write_b32 %0, %1" :: "v"(ldsa + LDS_BAD + tid * 4), "v"(0u) : "memory");
        const unsigned call = rfl(gld32_sys(cntp)) + 1;
        if (lane == 0) __hip_atomic_store(cntp, call, __ATOMIC_RELAXED, __HIP_MEMORY_SCOPE_AGENT);
        const unsigned par = call & 1;
        {
            const char* const pk = myws + OFF_PACK + (size_t)par * PACK_B;
            const rsrc_t rs = mk_rsrc(myws + OFF_STG, 2 * STG_B);
            int buf = 0;
            const unsigned tag = call & 0xffffu;
            auto pk_headers = [&](int idx, int b) {
                const int g8 = idx >> 7, mat = (idx >> 6) & 1, kk = (idx >> 5) & 1, blk = idx & 31;
                const int cb = mat == 0 ? ((blk >> 2) * 2 + (blk & 1)) : (blk & 15);
                const int c0 = mat == 0 ? ((blk >> 1) & 1) + 4 * kk : 2 * (blk >> 4) + 4 * kk, c1 = c0 + (mat == 0 ? 2 : 1);
                char* const w0 = p.ws[c0 & 7];
                char* const w1 = p.ws[c1 & 7];
                if (lane < 48) {
                    u32x4 h;
                    asm volatile("ds_read_b128 %0, %1\n\ts_waitcnt lgkmcnt(0)" : "=&v"(h) : "v"(ldsa + LDS_REC + (b >> 15) * (8 * REC_B) + lane * 16) : "memory");
                    const size_t o = OFF_IH + (size_t)par * IH_B + (size_t)((((mat * 16 + cb) * NR + rank) * 64 + g8 * 8) * REC_B);
                    if (c0 != rank) gst128s_sys<0>(w0 + o, lane * 16, h);
                    if (c1 != rank) gst128s_sys<0>(w1 + o, lane * 16, h);
                }
            };
            for (int idx = wg; idx < 1024; idx += NGEMM, buf ^= 32768) {
                const int g8 = idx >> 7, mat = (idx >> 6) & 1, kk = (idx >> 5) & 1, blk = idx & 31, t = g8 * 8 + wid;
                BAR;
                if (wid == 0 && idx != wg) pk_headers(idx - NGEMM, buf ^ 32768);
                const bool urgent = FASTP && idx < 128 && wid >= URG_LO && wid < URG_HI;
                const bool rawf = urgent && wid < RAWT && (mat == 0 ? !(blk & 1) : (blk & 15) < NFB);
                if (idx + NGEMM < 1024 && !rawf) pk_issue(idx + NGEMM, buf ^ 32768);
                if (urgent) __builtin_amdgcn_s_setprio(3);
                u32x4 v[4];
                if (mat == 0) {
                    const unsigned a = ldsa + buf + lane * 128 + ((wid ^ ((lane >> 1) & 7)) << 4);
                    DSR128(v[0], a, 0); DSR128(v[1], a, 8192); DSR128(v[2], a, 16384); DSR128(v[3], a, 24576);
                } else {
                    const unsigned a = ldsa + buf + wid * 4096 + lane * 16;
                    DSR128(v[0], a, 0); DSR128(v[1], a, 1024); DSR128(v[2], a, 2048); DSR128(v[3], a, 3072);
                }
                asm volatile("s_waitcnt lgkmcnt(0)" : "+v"(v[0]), "+v"(v[1]), "+v"(v[2]), "+v"(v[3]) :: "memory");
                const int cb = mat == 0 ? ((blk >> 2) * 2 + (blk & 1)) : (blk & 15);
                const int c0 = mat == 0 ? ((blk >> 1) & 1) + 4 * kk : 2 * (blk >> 4) + 4 * kk, c1 = c0 + (mat == 0 ? 2 : 1);
                char* const w0 = p.ws[c0 & 7];
                char* const w1 = p.ws[c1 & 7];
                const unsigned fo = (unsigned)(OFF_RDY + (((mat * 16 + cb) * 64 + t) * NR + rank) * 4);
                unsigned w[4][4];
#pragma unroll
                for (int c = 0; c < 4; ++c) { w[c][0] = v[c][0]; w[c][1] = v[c][1]; w[c][2] = v[c][2]; w[c][3] = v[c][3]; }
                const unsigned rec = ldsa + LDS_REC + (buf >> 15) * (8 * REC_B) + wid * REC_B;
                encode_unit(w, pk + (unsigned)(((mat * 2 + kk) * 32 + blk) * 64 + t) * UNIT_B, ldsa + LDS_AUX + wid * 1024, rec, tag, lane, rawf);
                const bool own = c0 == rank || c1 == rank;
                if (own) {
                    const unsigned dst = (unsigned)mat * STG_B + (unsigned)(2 * cb * 64 + t) * HTB + (unsigned)rank * 2048u;
#pragma unroll
                    for (int m = 0; m < 4; ++m) st128<AUX_SYS>(v[m], rs, lane * 16, dst + (m >> 1) * (64 * HTB) + (m & 1) * 1024);
                }
                WAIT_V(0);
                if (rawf) pk_issue(idx + NGEMM, buf ^ 32768);
                if (own) { if (lane == 0) __hip_atomic_store((unsigned*)(myws + fo), call, __ATOMIC_RELAXED, __HIP_MEMORY_SCOPE_SYSTEM); }
                if (urgent && lane < 12 && (lane < 6 ? c0 : c1) != rank) {
                    u32x4 h;
                    asm volatile("ds_read_b128 %0, %1\n\ts_waitcnt lgkmcnt(0)" : "=&v"(h) : "v"(rec + (lane < 6 ? lane : lane - 6) * 16) : "memory");
                    gst128_sys((lane < 6 ? w0 : w1) + OFF_IH + (size_t)par * IH_B + (size_t)((((mat * 16 + cb) * NR + rank) * 64 + t) * REC_B + (lane < 6 ? lane : lane - 6) * 16), h);
                }
                if (urgent) __builtin_amdgcn_s_setprio(0);
                if (rawf) WAIT_V(1);
                WAIT_L(0);
            }
            BAR;
            if (wid == 0) pk_headers(wg + (1023 - wg) / NGEMM * NGEMM, buf ^ 32768);
        }
        STAMP(dslot); ++dslot;
        const int pm = 2 * (wg & 7) + ((wg >> 3) & 1), pn = wg >> 4;
        const rsrc_t ra = mk_rsrc(myws + OFF_STG, STG_B), rb = mk_rsrc(myws + OFF_STG + STG_B, STG_B);
        const unsigned voff = (unsigned)(wid * 1024 + lane * 16);
        constexpr unsigned S_K = HTB, S_H = NT * HTB, S_T = 2 * NT * HTB;
        const unsigned ldsw = (unsigned)wid * 1024u;
        const int aoff = fq * 2048 + (wr * 64 + fr) * 16;
        unsigned boff[2];
        { const int q = (lane & 15) >> 2, pp = lane & 3, g = fq;
#pragma unroll
          for (int n = 0; n < 2; ++n) boff[n] = (unsigned)(size_t)lds + 4 * HTB + 256 * (8 * g + q) + 16 * (((wc ^ q) << 2) | ((n ^ (g & 1)) << 1) | (pp >> 1)) + 8 * (pp & 1); }
        const unsigned* rdyf = (const unsigned*)(myws + OFF_RDY);
#define SA(b, h) (((b) * 2 + (h)) * HTB)
#define SB(b, h) ((4 + (b) * 2 + (h)) * HTB)
#define STAGEX(rs, bufoff, soff, aux) do { _Pragma("unroll") for (int _i = 0; _i < 2; ++_i) \
        __builtin_amdgcn_raw_ptr_buffer_load_lds(rs, (LAS void*)(lds + (bufoff) + ldsw + _i * 8192), 16, voff, (soff) + _i * 8192, 0, aux); } while (0)
#define STAGE(rs, bufoff, soff) STAGEX(rs, bufoff, soff, GAUX)
#define STAGE2(rs, bufoff, soff) STAGEX(rs, bufoff, soff, GAUX2)
#define LDA(dst, b, h) do { _Pragma("unroll") for (int m = 0; m < 4; ++m) _Pragma("unroll") for (int k = 0; k < 2; ++k) dst[m][k] = *(const LAS bf16x8*)(lds + SA(b, h) + aoff + m * 256 + k * 8192); } while (0)
#define LDB(dst, b, h) do { _Pragma("unroll") for (int n = 0; n < 2; ++n) _Pragma("unroll") for (int k = 0; k < 2; ++k) { \
        TRD(dst[n][k][0], boff[n], ((b) * 2 + (h)) * HTB + k * 8192); TRD(dst[n][k][1], boff[n], ((b) * 2 + (h)) * HTB + k * 8192 + 1024); } } while (0)
#define MMA(ai, bj, At, Bt) do { __builtin_amdgcn_s_setprio(1); _Pragma("unroll") for (int m = 0; m < 4; ++m) _Pragma("unroll") for (int n = 0; n < 2; ++n) _Pragma("unroll") for (int k = 0; k < 2; ++k) \
        acc[ai][bj][m][n] = __builtin_amdgcn_mfma_f32_16x16x32_bf16(PKB(Bt[n][k]), At[m][k], acc[ai][bj][m][n], 0, 0, 0); __builtin_amdgcn_s_setprio(0); } while (0)
#define WAIT_B(X, Y) asm volatile("s_waitcnt lgkmcnt(0)" : "+v"(X[0][0][0]), "+v"(X[0][0][1]), "+v"(X[0][1][0]), "+v"(X[0][1][1]), "+v"(X[1][0][0]), "+v"(X[1][0][1]), "+v"(X[1][1][0]), "+v"(X[1][1][1]), \
        "+v"(Y[0][0][0]), "+v"(Y[0][0][1]), "+v"(Y[0][1][0]), "+v"(Y[0][1][1]), "+v"(Y[1][0][0]), "+v"(Y[1][0][1]), "+v"(Y[1][1][0]), "+v"(Y[1][1][1]) :: "memory")
#define WAIT_B1(X) asm volatile("s_waitcnt lgkmcnt(0)" : "+v"(X[0][0][0]), "+v"(X[0][0][1]), "+v"(X[0][1][0]), "+v"(X[0][1][1]), "+v"(X[1][0][0]), "+v"(X[1][0][1]), "+v"(X[1][1][0]), "+v"(X[1][1][1]) :: "memory")
        const rsrc_t rr = mk_rsrc(myws + OFF_RDY, 2 * 16 * 64 * 32);
        const unsigned pvo = (unsigned)(((lane >> 5) * 16 + (lane < 32 ? pm : pn)) * 2048 + (lane & 1) * 16);
        const unsigned pla = ldsa + LDS_AUX + wid * 1024 + lane * 16;
#define NOGATE 0
        int rdy = NOGATE ? NT : 0, pbase = 0;
        bool pol = false;
        u32x4 fv;
#define COUNT_RDY(v) do { const bool _ok = (int)((v)[0] - call) >= 0 && (int)((v)[1] - call) >= 0 && (int)((v)[2] - call) >= 0 && (int)((v)[3] - call) >= 0; \
            const unsigned long long _b = __ballot(_ok); \
            unsigned _m = (unsigned)_b & (unsigned)(_b >> 32); _m = ~(_m & (_m >> 1)) & 0x55555555u; \
            rdy = min(pbase + (_m ? (int)(__builtin_ctz(_m) >> 1) : 16), NT); } while (0)
#define GATE(tt) do { if (wid == 4) { \
            if (pol) { DSR128(fv, pla, 0); asm volatile("s_waitcnt lgkmcnt(0)" : "+v"(fv) :: "memory"); COUNT_RDY(fv); } \
            if (UNLIKELY((int)(tt) >= rdy)) { \
                const unsigned long long _t0 = __builtin_amdgcn_s_memrealtime(); \
                DIAG_BLK0; \
                for (;;) { \
                    pbase = rdy; \
                    const char* _p = (const char*)rdyf + pvo + min(rdy + ((lane >> 1) & 15), NT - 1) * 32; \
                    u32x4 _v; unsigned _tm; \
                    asm volatile("global_load_dwordx4 %0, %2, off sc0 sc1\n\tglobal_load_dword %1, %3, off sc0 sc1\n\ts_waitcnt vmcnt(0)" \
                                 : "=&v"(_v), "=&v"(_tm) : "v"(_p), "v"(tmo) : "memory"); \
                    COUNT_RDY(_v); \
                    if ((int)(tt) < rdy) break; \
                    if (rfl(_tm) == call || __builtin_amdgcn_s_memrealtime() - _t0 > DEADLINE) { \
                        __hip_atomic_store(tmo, call, __ATOMIC_RELAXED, __HIP_MEMORY_SCOPE_AGENT); *badf = 1; rdy = NT; break; } \
                    __builtin_amdgcn_s_sleep(GSLEEP); \
                } DIAG_BLK1(_t0); } \
            pol = rdy < min((int)(tt) + LOOK, NT); \
            if (pol) { pbase = rdy; DMA16(rr, LDS_AUX + wid * 1024, pvo + min(rdy + ((lane >> 1) & 15), NT - 1) * 32, 0); } } } while (0)
#define FLAGREAD do {} while (0)
#define GATE1(tt) GATE(tt)
#define GATE2(tt) do {} while (0)
#define WAIT_VP do { if (pol) WAIT_V(9); else WAIT_V(8); } while (0)
        const int owner = pm >> 1;
        char* chd = p.ws[owner] + OFF_CH;
        f32x4 acc[2][2][4][2];
#pragma unroll
        for (int a = 0; a < 2; ++a)
#pragma unroll
            for (int b = 0; b < 2; ++b)
#pragma unroll
                for (int m = 0; m < 4; ++m)
#pragma unroll
                    for (int n = 0; n < 2; ++n) acc[a][b][m][n] = (f32x4){0.f, 0.f, 0.f, 0.f};
        bf16x8 At[4][2]; s16x4 B0[2][2][2], B1[2][2][2];
        const unsigned cA = (unsigned)pm * S_T, cB = (unsigned)pn * S_T;
        if (!NOGEMM) { GATE(1); }
        BAR;
        STAGE(rb, SB(0, 0), cB); STAGE(rb, SB(0, 1), cB + S_H); STAGE(ra, SA(0, 0), cA); STAGE(ra, SA(0, 1), cA + S_H);
        if (wr == 1) BAR;
        WAIT_V(2); BAR;
        STAGE(rb, SB(1, 0), cB + S_K); STAGE(ra, SA(1, 0), cA + S_K); STAGE(rb, SB(1, 1), cB + S_H + S_K);
        WAIT_V(6); BAR;
        STAMP(dslot); ++dslot;
        for (int t = 0; t < (NOGEMM ? 0 : NT); t += 2) {
            const int t2 = min(t + 2, NT - 1), t3 = min(t + 3, NT - 1);
            const unsigned a1 = cA + (unsigned)(t + 1) * S_K;
            const unsigned a2 = cA + (unsigned)t2 * S_K, b2 = cB + (unsigned)t2 * S_K, a3 = cA + (unsigned)t3 * S_K, b3 = cB + (unsigned)t3 * S_K;
            LDB(B0, 0, 0); LDB(B1, 0, 1); SCHED; LDA(At, 0, 0); STAGE(ra, SA(1, 1), a1 + S_H);
            WAIT_V(8); WAIT_B(B0, B1); FLAGREAD; GATE1(t2); BAR; MMA(0, 0, At, B0); MMA(0, 1, At, B1); BAR; SCHED;
            GATE2(t2);
            LDA(At, 0, 1); STAGE(rb, SB(0, 0), b2); STAGE(rb, SB(0, 1), b2 + S_H); STAGE(ra, SA(0, 0), a2);
            WAIT_VP; WAIT_L(0); BAR; MMA(1, 0, At, B0); MMA(1, 1, At, B1); BAR; SCHED;
            LDB(B0, 1, 0); LDB(B1, 1, 1); SCHED; LDA(At, 1, 0); STAGE(ra, SA(0, 1), a2 + S_H);
            WAIT_V(8); WAIT_B(B0, B1); FLAGREAD; GATE1(t3); BAR; MMA(0, 0, At, B0); MMA(0, 1, At, B1); BAR; SCHED;
            GATE2(t3);
            LDA(At, 1, 1); STAGE(rb, SB(1, 0), b3); STAGE(rb, SB(1, 1), b3 + S_H); STAGE(ra, SA(1, 0), a3);
            WAIT_VP; WAIT_L(0); BAR; MMA(1, 0, At, B0); MMA(1, 1, At, B1); BAR; SCHED;
        }
        if (wr == 0) BAR;
        STAMP(dslot); ++dslot;
#define EPI_GROUP(A, pnx, g, slot) do { unsigned _w[4][4]; \
            _Pragma("unroll") for (int m = 0; m < 4; ++m) { \
                _w[m][0] = cvtpk(A[m][0][0], A[m][0][1]); _w[m][1] = cvtpk(A[m][0][2], A[m][0][3]); \
                _w[m][2] = cvtpk(A[m][1][0], A[m][1][1]); _w[m][3] = cvtpk(A[m][1][2], A[m][1][3]); } \
            encode_unit(_w, myws + OFF_CP + (unsigned)(((pm * 16 + (pnx)) * 8 + wid) * 4 + (g)) * UNIT_B, ldsa + LDS_AUX + wid * 1024, \
                        ldsa + LDS_REC + (slot) * (8 * REC_B) + wid * REC_B, call & 0xffffu, lane, false); } while (0)
        auto epi_hdr = [&](int pnx, int g, int slot) {
            unsigned old, bad;
            asm volatile("s_mov_b64 exec, 1\n\tds_add_rtn_u32 %0, %1, %2\n\ts_mov_b64 exec, -1\n\ts_waitcnt lgkmcnt(0)"
                         : "=&v"(old) : "v"(ldsa + LDS_BAD + 16 + slot * 4), "v"(1u) : "memory");
            if (rfl(old) != 7) return false;
            u32x4 hv;
            asm volatile("ds_read_b128 %0, %2\n\tds_read_b32 %1, %3\n\ts_waitcnt lgkmcnt(0)"
                         : "=&v"(hv), "=&v"(bad) : "v"(ldsa + LDS_REC + slot * (8 * REC_B) + min(lane, 47) * 16), "v"(ldsa + LDS_BAD) : "memory");
            if (bad && lane % 6 == 0) hv[0] = (hv[0] & 0xffffu) | POISON << 24;
            if (lane < 48) gst128s_sys<0>(chd + (unsigned)((((((bi * 2 + (pm & 1)) * 32 + bj_ * 16 + pnx) * 4 + g) * 2 + bk) * 8) * REC_B), lane * 16, hv);
            return true;
        };
#define KSPL 1
#define EPI1 1
        if (EPI1) {
            EPI_GROUP(acc[0][0], pn, 0, 0);
            WAIT_V(0);
            epi_hdr(pn, 0, 0);
            EPI_GROUP(acc[0][1], pn, 1, 1);
            EPI_GROUP(acc[1][0], pn, 2, 2);
            WAIT_V(4);
            const bool s1 = epi_hdr(pn, 1, 1);
            EPI_GROUP(acc[1][1], pn, 3, 3);
            if (s1) WAIT_V(5); else WAIT_V(4);
            const bool s2 = epi_hdr(pn, 2, 2);
            if (s2) WAIT_V(1); else WAIT_V(0);
            epi_hdr(pn, 3, 3);
        } else {
            EPI_GROUP(acc[0][0], pn, 0, 0);
            EPI_GROUP(acc[0][1], pn, 1, 1);
            WAIT_V(4);
            const bool s0 = epi_hdr(pn, 0, 0);
            EPI_GROUP(acc[1][0], pn, 2, 2);
            EPI_GROUP(acc[1][1], pn, 3, 3);
            if (s0) WAIT_V(9); else WAIT_V(8);
            const bool s1 = epi_hdr(pn, 1, 1);
            if (s1) WAIT_V(5); else WAIT_V(4);
            const bool s2 = epi_hdr(pn, 2, 2);
            if (s1 && s2) WAIT_V(2); else if (s1 || s2) WAIT_V(1); else WAIT_V(0);
            epi_hdr(pn, 3, 3);
        }
        STAMP(dslot); ++dslot;
        for (int e = pn; e < 4 * PN2; e += PN1) {
            const int ai = e & 1, bj = (e >> 1) & 1, pn2 = PN1 + (e >> 2);
            if (!NOGEMM && !NOGATE) {
                const unsigned long long t0 = __builtin_amdgcn_s_memrealtime();
                const u32x4* f = (const u32x4*)(rdyf + ((16 + pn2) * 64) * NR) + lane;
                for (;;) {
                    u32x4 v0, v1;
                    asm volatile("global_load_dwordx4 %0, %2, off sc0 sc1\n\tglobal_load_dwordx4 %1, %2, off offset:1024 sc0 sc1\n\ts_waitcnt vmcnt(0)"
                                 : "=&v"(v0), "=&v"(v1) : "v"(f) : "memory");
                    bool ok = true;
#pragma unroll
                    for (int q = 0; q < 4; ++q) ok = ok && (int)(v0[q] - call) >= 0 && (int)(v1[q] - call) >= 0;
                    if (__all(ok)) break;
                    if (timed_out(tmo, call, t0)) { *badf = 1; break; }
                    __builtin_amdgcn_s_sleep(16);
                }
            }
            BAR;
            STAMP(dslot); ++dslot;
            const unsigned gA = (unsigned)(2 * pm + ai) * S_H, gB = (unsigned)(2 * pn2 + bj) * S_H;
#pragma unroll
            for (int m = 0; m < 4; ++m)
#pragma unroll
                for (int n = 0; n < 2; ++n) acc[0][0][m][n] = acc[0][1][m][n] = (f32x4){0.f, 0.f, 0.f, 0.f};
#pragma unroll
            for (int q = 0; q < 3; ++q) { STAGE2(ra, q * HTB, gA + q * S_K); STAGE2(rb, (4 + q) * HTB, gB + q * S_K); }
            const unsigned kh = (unsigned)(wid & 1) * 8192u;
            unsigned bk2[2][2];
#pragma unroll
            for (int c = 0; c < 2; ++c)
#pragma unroll
                for (int n = 0; n < 2; ++n) bk2[c][n] = (boff[n] ^ (c ? 64u : 0u)) + kh;
            const unsigned ak2 = ldsa + (unsigned)aoff + kh;
#define RD2(Bx, k, q) do { _Pragma("unroll") for (int c = 0; c < 2; ++c) _Pragma("unroll") for (int n = 0; n < 2; ++n) { \
                TRD(Bx[n][c][0], bk2[c][n], (q) * HTB); TRD(Bx[n][c][1], bk2[c][n], (q) * HTB + 1024); } \
                _Pragma("unroll") for (int m = 0; m < 4; ++m) DSR128(At[m][k], ak2, (q) * HTB + m * 256); } while (0)
#define W2(X, k) asm volatile("s_waitcnt lgkmcnt(0)" : "+v"(X[0][0][0]), "+v"(X[0][0][1]), "+v"(X[0][1][0]), "+v"(X[0][1][1]), "+v"(X[1][0][0]), "+v"(X[1][0][1]), \
                "+v"(X[1][1][0]), "+v"(X[1][1][1]), "+v"(At[0][k]), "+v"(At[1][k]), "+v"(At[2][k]), "+v"(At[3][k]) :: "memory")
#define MM2(Bx, k) do { __builtin_amdgcn_s_setprio(1); _Pragma("unroll") for (int m = 0; m < 4; ++m) _Pragma("unroll") for (int c = 0; c < 2; ++c) \
                _Pragma("unroll") for (int n = 0; n < 2; ++n) \
                    acc[0][c][m][n] = __builtin_amdgcn_mfma_f32_16x16x32_bf16(PKB(Bx[n][c]), At[m][k], acc[0][c][m][n], 0, 0, 0); \
                __builtin_amdgcn_s_setprio(0); } while (0)
#define ILV 0
#define MMH(Bx, k, m, c) do { _Pragma("unroll") for (int n = 0; n < 2; ++n) \
                acc[0][c][m][n] = __builtin_amdgcn_mfma_f32_16x16x32_bf16(PKB(Bx[n][c]), At[m][k], acc[0][c][m][n], 0, 0, 0); SCHED; } while (0)
#define MMC(Bx, k, m) do { MMH(Bx, k, m, 0); MMH(Bx, k, m, 1); } while (0)
#define RDBH(Bx, c, q) do { _Pragma("unroll") for (int n = 0; n < 2; ++n) { \
                TRD(Bx[n][c][0], bk2[c][n], (q) * HTB); TRD(Bx[n][c][1], bk2[c][n], (q) * HTB + 1024); } SCHED; } while (0)
#define RDAH(k, m0, q) do { DSR128(At[m0][k], ak2, (q) * HTB + (m0) * 256); DSR128(At[(m0) + 1][k], ak2, (q) * HTB + ((m0) + 1) * 256); SCHED; } while (0)
#define STA(q) do { if (G2X != 1) STAGE2(ra, (q) * HTB, gA + tn); SCHED; } while (0)
#define STB(q) do { if (G2X != 1 && G2X != 3) STAGE2(rb, (4 + (q)) * HTB, gB + tn); SCHED; } while (0)
#define STEP2(Bx, k, By, k1, q) do { const int q1 = ((q) + 1) & 3; SCHED; \
                if (ILV == 1) { MMC(Bx, k, 0); RDBH(By, 0, q1); RDBH(By, 1, q1); MMC(Bx, k, 1); RDAH(k1, 0, q1); RDAH(k1, 2, q1); MMC(Bx, k, 2); STA(q); STB(q); MMC(Bx, k, 3); } \
                else if (ILV == 2) { MMH(Bx, k, 0, 0); RDBH(By, 0, q1); MMH(Bx, k, 0, 1); RDBH(By, 1, q1); MMH(Bx, k, 1, 0); RDAH(k1, 0, q1); MMH(Bx, k, 1, 1); RDAH(k1, 2, q1); \
                                     MMC(Bx, k, 2); STA(q); MMH(Bx, k, 3, 0); STB(q); MMH(Bx, k, 3, 1); } \
                else if (ILV == 3) { MMH(Bx, k, 0, 0); STA(q); MMH(Bx, k, 0, 1); STB(q); MMH(Bx, k, 1, 0); RDBH(By, 0, q1); MMH(Bx, k, 1, 1); RDBH(By, 1, q1); \
                                     MMH(Bx, k, 2, 0); RDAH(k1, 0, q1); MMH(Bx, k, 2, 1); RDAH(k1, 2, q1); MMC(Bx, k, 3); } \
                else { MMH(Bx, k, 0, 0); RDBH(By, 0, q1); MMH(Bx, k, 0, 1); RDBH(By, 1, q1); MMH(Bx, k, 1, 0); RDAH(k1, 0, q1); MMH(Bx, k, 1, 1); RDAH(k1, 2, q1); \
                       MMH(Bx, k, 2, 0); STA(q); MMH(Bx, k, 2, 1); STB(q); MMC(Bx, k, 3); } } while (0)
            STAGE2(ra, 3 * HTB, gA + 3 * S_K); STAGE2(rb, 7 * HTB, gB + 3 * S_K);
            WAIT_V(12); BAR;
            RD2(B0, 0, 0);
            for (int t = 0; t < (NOGEMM || NOGEMM2 ? 0 : NT); t += 4) {
#pragma unroll
                for (int q = 0; q < 4; ++q) {
                    const unsigned tn = (unsigned)min(t + q + 4, NT - 1) * S_K;
                    WAIT_V(8);
                    if (q & 1) W2(B1, 1); else W2(B0, 0);
                    BAR;
                    if (G2X != 1) { STAGE2(ra, q * HTB, gA + tn); if (G2X != 3) STAGE2(rb, (4 + q) * HTB, gB + tn); }
                    if (q & 1) RD2(B0, 0, (q + 1) & 3); else RD2(B1, 1, (q + 1) & 3);
                    SCHED;
                    if (q & 1) MM2(B1, 1); else MM2(B0, 0);
                    SCHED;
                }
            }
            W2(B0, 0);
            WAIT_V(0); BAR;
#pragma unroll
            for (int m = 0; m < 4; ++m)
#pragma unroll
                for (int n = 0; n < 2; ++n) *(LAS f32x4*)(lds + wid * 8192 + (m * 2 + n) * 1024 + lane * 16) = acc[0][1][m][n];
            WAIT_L(0); BAR;
#pragma unroll
            for (int m = 0; m < 4; ++m)
#pragma unroll
                for (int n = 0; n < 2; ++n) acc[0][0][m][n] += *(const LAS f32x4*)(lds + (wid ^ 1) * 8192 + (m * 2 + n) * 1024 + lane * 16);
            EPI_GROUP(acc[0][0], pn2, ai * 2 + bj, 4 + (e - pn) / PN1);
            WAIT_V(0);
            epi_hdr(pn2, ai * 2 + bj, 4 + (e - pn) / PN1);
            STAMP(dslot); ++dslot;
        }
#undef SA
#undef SB
    } else {
        unsigned cnt0;
        if (FASTC) asm volatile("global_load_dword %0, %1, off sc0 sc1" : "=&v"(cnt0) : "v"(cntp) : "memory");
        for (int q = tid; q <= Q_END; q += NTHR) {
            u32x4 en = {0u, 0u, 0u, 0u};
            if (q < Q_OUT) {
                const int c = in_job(q), e = c >> 6, st = c & 63, mat = e >> 4, blk = e & 15;
                const int gblk = mat == 0 ? (blk >> 1) * 4 + bi * 2 + (blk & 1) : bj_ * 16 + blk;
                en = (u32x4){(unsigned)(e * NR * 64 + st) * REC_B, (unsigned)(((mat * 2 + bk) * 32 + gblk) * 64 + st) * UNIT_B,
                             (unsigned)mat * STG_B + (unsigned)(2 * blk * 64 + st) * HTB, (unsigned)(c * NR * 4)};
            } else if (q < Q_END) {
                const int o = q - Q_OUT, y = out_tile(out_yg(o) >> 2), gg = out_yg(o) & 3, w8 = o & 7;
                en = (u32x4){(unsigned)((((y & 1) * 32 + (y >> 1)) * 4 + gg) * 16 + w8) * REC_B,
                             (unsigned)(((rank * 2 + (y & 1)) * 16 + (y >> 1)) * 32 + w8 * 4 + gg) * UNIT_B,
                             (unsigned)((((y & 1) * 256 + (gg >> 1) * 128 + (w8 >> 2) * 64) * NN + (y >> 1) * 256 + (gg & 1) * 128 + (w8 & 3) * 32) * 2), 0u};
            }
            asm volatile("ds_write_b128 %0, %1" :: "v"(ldsa + LDS_TAB + (unsigned)q * 16u), "v"(en) : "memory");
        }
        if (FASTC) asm volatile("s_waitcnt vmcnt(0)" : "+v"(cnt0) :: "memory"); else cnt0 = gld32_sys(cntp);
        const unsigned call = rfl(cnt0) + 1;
        if (lane == 0) __hip_atomic_store(cntp, call, __ATOMIC_RELAXED, __HIP_MEMORY_SCOPE_AGENT);
        const unsigned par = call & 1;
        unsigned* const tkt = ctl + C_TKT / 4 + par * 1024;
        if (wg == NGEMM && wid == 7 && lane < 16) __hip_atomic_store(ctl + C_TKT / 4 + (par ^ 1) * 1024 + lane * 64, 0u, __ATOMIC_RELAXED, __HIP_MEMORY_SCOPE_SYSTEM);
        WAIT_L(0);
        BAR;
        const rsrc_t rh = mk_rsrc(myws, (unsigned)WS_NEED);
        const unsigned lw = (unsigned)wid * 16384u, lh = LDS_AUX + (unsigned)wid * 1024u;
        const unsigned pat = ldsa + LDS_PAT + (unsigned)wid * 4096u;
        const unsigned vl16 = (unsigned)lane * 16u;
#define PULL_UNIT(rp, slot, so, h1) do { DMA16(rp, slot, vl16, so); DMA16(rp, (slot) + 1024, vl16, (so) + 1024); \
            if ((unsigned)lane < ((h1) >> 8 > RECN ? ((h1) >> 8 > MAXEXC ? 64u : 56u) : 48u)) DMA16(rp, (slot) + 2048, vl16, (so) + 2048); \
            if (UNLIKELY((h1) >> 8 == RAWMODE)) DMA16(rp, (slot) + 3072, vl16, (so) + 3072); } while (0)
#define READ_ENTRY(en, slot, rec, h1) asm volatile("ds_read_u16 %0, %1" : "=&v"(en) \
            : "v"(ldsa + ((h1) >> 8 <= RECN ? (rec) : (slot) + OFF_LIST) \
                  + 2 * list_pos(min((unsigned)lane, min(max((h1) >> 8, 1u), MAXEXC) - 1), (h1) >> 8 <= RECN)) : "memory")
#define READ_UNIT(d, cd, ce, slot, h1) asm volatile("ds_read_b128 %0, %6\n\tds_read_b128 %1, %6 offset:1024\n\t" \
            "s_cmp_eq_u32 %8, 255\n\ts_cbranch_scc1 1f\n\t" \
            "ds_read2_b32 %4, %7 offset1:1\n\tds_read_b32 %5, %7 offset:8\n\ts_branch 2f\n" \
            "1:\n\tds_read_b128 %2, %6 offset:2048\n\tds_read_b128 %3, %6 offset:3072\n" \
            "2:" : "=&v"(d[0]), "=&v"(d[1]), "=&v"(d[2]), "=&v"(d[3]), "=&v"(cd), "=&v"(ce) \
            : "v"(ldsa + (slot) + vl16), "v"(ldsa + (slot) + OFF_CODE + (unsigned)lane * 12u), "s"((h1) >> 8) : "scc", "memory")
#define PATCH_UNIT(d, en, u, h1) do { const unsigned _en = (h1) >> 8 == 0 ? 0x800u : (en), _ad = pat + (u) * 2048 + (_en & 0x7ffu); \
            asm volatile("s_cmp_eq_u32 %6, 255\n\ts_cbranch_scc1 1f\n\t" \
                         "ds_write_b8 %2, %3\n\tds_read_b128 %0, %4\n\tds_read_b128 %1, %4 offset:1024\n\tds_write_b8 %2, %5\n1:" \
                         : "+v"(d[2]), "+v"(d[3]) : "v"(_ad), "v"(_en >> 9 & 0x7cu), "v"(pat + (u) * 2048 + vl16), "v"(4u), "s"((h1) >> 8) : "scc", "memory"); } while (0)
#define WAIT_UNITS(d, cd, ce, hv, nx) asm volatile("s_waitcnt lgkmcnt(0)" : "+v"(d[0][0]), "+v"(d[0][1]), "+v"(d[0][2]), "+v"(d[0][3]), \
            "+v"(d[1][0]), "+v"(d[1][1]), "+v"(d[1][2]), "+v"(d[1][3]), "+v"(cd[0]), "+v"(cd[1]), "+v"(ce[0]), "+v"(ce[1]), \
            "+v"(hv), "+v"(nx) :: "memory")
#define NOCOMM 0
        if (const int cw = NACT == NCW ? (wg - NGEMM) * CW + wid : wid * NCOMM + (wg - NGEMM); wid < CW && cw < NACT && !NOCOMM) {
#pragma unroll
            for (int i = 0; i < 4; ++i) asm volatile("ds_write_b128 %0, %1" :: "v"(pat + i * 1024 + vl16), "v"((u32x4){0x04040404u, 0x04040404u, 0x04040404u, 0x04040404u}) : "memory");
            const int si = cw % (NR - 1), sq = cw / (NR - 1), s = (rank + 1 + si) & 7;
            constexpr int w_in = W_IN;
            const int oc = sq < NACT / 8 ? (s & 3) : (rank & 3);
            const int ti = oc & 1, tj = oc >> 1;
            const rsrc_t rp = mk_rsrc(p.ws[s] + OFF_PACK + (size_t)par * PACK_B, PACK_B);
            const rsrc_t rc0 = mk_rsrc(p.ws[oc] + OFF_CP, CP_B), rc1 = mk_rsrc(p.ws[oc + 4] + OFF_CP, CP_B);
            const rsrc_t ro = mk_rsrc(p.out, (MM / NR) * NN * 2);
            const unsigned hb_in = (unsigned)OFF_IH + par * IH_B + (unsigned)s * (64 * REC_B), hb_out = (unsigned)OFF_CH + (unsigned)((ti * 64 + tj * 16) * 64) * REC_B;
            const unsigned tag = call & 0xffffu;
            const char* const stg = myws + OFF_STG + (unsigned)s * 2048u;
            const char* const rdys = myws + OFF_RDY + (unsigned)s * 4u;
            const unsigned olane = (unsigned)(((ti * 512 + fr) * NN + tj * 4096 + (fq & 1) * 16 + (fq >> 1) * 8) * 2);
            auto hdr = [&](int q, unsigned x) {
                return lane < 12 ? (q < Q_OUT ? hb_in + (unsigned)lane * 16u : hb_out + (unsigned)lane * 16u + (lane < 6 ? 0u : 7 * REC_B)) + x
                                 : (unsigned)CTRL_OFF + C_TMO;
            };
            auto fetch_rec = [&](int q, unsigned x, unsigned slot) { if (lane < 13) DMA16(rh, slot, hdr(q, x), 0); };
            auto entry = [&](int q) {
                u32x4 en;
                asm volatile("ds_read_b128 %0, %1\n\ts_waitcnt lgkmcnt(0)" : "=&v"(en) : "v"(ldsa + LDS_TAB + (unsigned)q * 16u) : "memory");
                return (u32x4){rfl(en[0]), rfl(en[1]), rfl(en[2]), rfl(en[3])};
            };
            auto ready = [&](unsigned hv) { return ((unsigned)__ballot((hv & 0xffffu) == tag) & 0xfffu) == 0xfffu; };
            auto head = [&](unsigned hv, int u) { return (unsigned)__builtin_amdgcn_readlane((int)hv, 6 * u) >> 16; };
            auto gave_up = [&](unsigned hv) { return (unsigned)__builtin_amdgcn_readlane((int)hv, 12) == call; };
            auto pull = [&](int q, unsigned so, unsigned hv, unsigned slot) {
                const unsigned ha = head(hv, 0), hb = head(hv, 1);
                if (q < Q_OUT) {
                    PULL_UNIT(rp, slot, so, ha);
                    PULL_UNIT(rp, slot + 4096, so + UNIT_B, hb);
                } else {
                    PULL_UNIT(rc0, slot, so, ha);
                    PULL_UNIT(rc1, slot + 4096, so, hb);
                }
            };
            auto publish = [&](unsigned w) {
                if (lane < 2) asm volatile("global_store_dword %0, %1, %2 sc0 sc1" :: "v"(w + (unsigned)lane * 32u), "v"(call), "s"(rdys) : "memory");
            };
            auto nan_fill = [&](unsigned z) {
#pragma unroll
                for (int m = 0; m < 4; ++m)
                    st128<0>((u32x4){0x7fc07fc0u, 0x7fc07fc0u, 0x7fc07fc0u, 0x7fc07fc0u}, ro, olane + z, m * 16 * NN * 2);
            };
            const unsigned* const tki = tkt + si * 64;
            const unsigned* const tko = tkt + (8 + oc) * 64;
            int o1 = -1;
            unsigned tk, tk0;
            TICKET(tk0, tko);
            if (!FASTC || sq >= w_in) { WAIT_V(0); o1 = (int)min((unsigned)Q_OUT + ticket_value(tk0), (unsigned)Q_END); }
            static_assert(4 * (W_IN + XE) <= JE, "");
            const bool helper = XE && sq >= w_in && sq < w_in + XE;
            const int lim = helper ? JE : Q_OUT;
            const unsigned* const tkd = tkt + 15 * 64;
            auto next = [&](int qprev, unsigned t) {
                if (qprev >= Q_OUT) return (int)min((unsigned)Q_OUT + t, (unsigned)Q_END);
                if (XE && qprev >= lim) return o1;
                const int j = 4 * (w_in + XE) + (int)t;
                return j < Q_OUT ? j : o1;
            };
            int qA = sq, qB = sq + w_in, qC = sq + 2 * w_in;
            if (helper) {
                qA = 4 * w_in + (sq - w_in); qB = qA + XE; qC = qB + XE;
                asm volatile("v_mov_b32 %0, %1" : "=v"(tk) : "v"((unsigned)(sq - w_in - XE)));
            } else if (sq >= w_in) {
                unsigned tb, tc;
                TICKET(tb, tko); TICKET(tc, tko); TICKET(tk, tko); WAIT_V(0);
                qA = o1; qB = next(Q_OUT, ticket_value(tb)); qC = next(Q_OUT, ticket_value(tc));
            } else {
                asm volatile("v_mov_b32 %0, %1" : "=v"(tk) : "v"((unsigned)(sq - w_in - 4 * XE)));
            }
            u32x4 eA = entry(qA), eB = entry(qB), eC = entry(qC);
            bool dead = false;
            while (qA < Q_END) {
                unsigned hvA, hvB;
                gld32x2_sys(myws + hdr(qA, eA[0]), myws + hdr(qB, eB[0]), hvA, hvB);
                if (FASTC && o1 < 0) o1 = (int)min((unsigned)Q_OUT + ticket_value(tk0), (unsigned)Q_END);
                if (!ready(hvA)) {
                    const unsigned long long t0 = __builtin_amdgcn_s_memrealtime();
                    for (;;) {
                        __builtin_amdgcn_s_sleep(CSLEEP);
                        gld32x2_sys(myws + hdr(qA, eA[0]), myws + hdr(qB, eB[0]), hvA, hvB);
                        if (ready(hvA)) break;
                        if (gave_up(hvA) || __builtin_amdgcn_s_memrealtime() - t0 > DEADLINE) {
                            __hip_atomic_store(tmo, call, __ATOMIC_RELAXED, __HIP_MEMORY_SCOPE_AGENT); dead = true; break;
                        }
                    }
                    if (dead) break;
                }
                unsigned sl = lw, hs = 0;
                pull(qA, eA[1], hvA, sl);
                fetch_rec(qA, eA[0], lh); fetch_rec(qB, eB[0], lh + 256); fetch_rec(qC, eC[0], lh + 512);
                int nst = 0;
                unsigned pend = 0;
                unsigned pend2 = 0;
                bool have2 = false;
                bool soon = false, soon2 = false;
                for (;;) {
                    const bool more = qB < Q_END && ready(hvB);
                    if (more) {
                        if (nst == 8) WAIT_V(8); else if (nst == 4) WAIT_V(4); else WAIT_V(0);
                        pull(qB, eB[1], hvB, sl ^ 8192);
                        if (have2) publish(pend2);
                        have2 = nst == 8; pend2 = pend; soon2 = soon;
                    } else {
                        WAIT_V(0);
                        if (have2) publish(pend2);
                        have2 = false;
                        if (nst == 8) { publish(pend); nst = 0; }
                    }
                    const int qD = next(qC, ticket_value(tk));
                    unsigned hvC;
                    u32x4 d[2][4], eDv;
                    u32x2 cd[2];
                    unsigned ce[2], en0, en1;
                    const unsigned hu[2] = {head(hvA, 0), head(hvA, 1)};
                    READ_ENTRY(en0, sl, lh + hs, hu[0]);
                    READ_ENTRY(en1, sl + 4096, lh + hs + REC_B, hu[1]);
                    DSR128(eDv, ldsa + LDS_TAB + (unsigned)qD * 16u, 0);
                    DSR32(hvC, ldsa + lh + ((hs + 512) & 768) + min((unsigned)lane, 12u) * 16u);
                    READ_UNIT(d[0], cd[0], ce[0], sl, hu[0]);
                    READ_UNIT(d[1], cd[1], ce[1], sl + 4096, hu[1]);
                    asm volatile("s_waitcnt lgkmcnt(10)" : "+v"(en0), "+v"(en1) :: "memory");
                    PATCH_UNIT(d[0], en0, 0, hu[0]);
                    PATCH_UNIT(d[1], en1, 1, hu[1]);
                    WAIT_UNITS(d, cd, ce, hvC, eDv);
                    const u32x4 eD = {rfl(eDv[0]), rfl(eDv[1]), rfl(eDv[2]), rfl(eDv[3])};
                    TICKET(tk, qD < lim ? tki : XE && qD < Q_OUT ? tkd : tko);
                    if (more) fetch_rec(qD, eD[0], lh + ((hs + 768) & 768));
                    hs = (hs + 256) & 768;
                    if (qA < Q_OUT) {
                        const char* dst = stg + eA[2];
#pragma unroll
                        for (int u = 0; u < 2; ++u) {
                            const unsigned h1 = hu[u];
                            u32x4 o[4];
                            if (!UNLIKELY(h1 >> 8 == RAWMODE)) {
                                unsigned H[8] = {d[u][2][0], d[u][2][1], d[u][2][2], d[u][2][3], d[u][3][0], d[u][3][1], d[u][3][2], d[u][3][3]};
                                decode_high(cd[u][0], cd[u][1], ce[u], h1, H);
#pragma unroll
                                for (int m = 0; m < 4; ++m) {
                                    const unsigned l0 = d[u][m >> 1][(m & 1) * 2], l1 = d[u][m >> 1][(m & 1) * 2 + 1];
                                    o[m] = (u32x4){vperm(H[2 * m], l0, 0x05010400u), vperm(H[2 * m], l0, 0x07030602u),
                                                   vperm(H[2 * m + 1], l1, 0x05010400u), vperm(H[2 * m + 1], l1, 0x07030602u)};
                                }
                                if (h1 & 0x80u) {
#pragma unroll
                                    for (int m = 0; m < 4; ++m)
#pragma unroll
                                        for (int j = 0; j < 4; ++j) o[m][j] = pk_add16(o[m][j], 0xff80ff80u);
                                }
                            } else {
#pragma unroll
                                for (int m = 0; m < 4; ++m) o[m] = d[u][m];
                            }
                            if (!PUB2 && u == 0 && nst == 8) {
                                asm volatile("s_waitcnt vmcnt(8)" : "+v"(o[0]), "+v"(o[1]), "+v"(o[2]), "+v"(o[3]) :: "memory");
                                publish(pend);
                            }
                            if (JP0 < NJ1 && u == 0 && have2 && soon2) {
                                asm volatile("s_waitcnt vmcnt(8)" : "+v"(o[0]), "+v"(o[1]), "+v"(o[2]), "+v"(o[3]) :: "memory");
                                publish(pend2); have2 = false;
                            }
                            const char* d0 = dst + u * HTB;
                            gst128s_stg<0>(d0, vl16, o[0]); gst128s_stg<1024>(d0, vl16, o[1]);
                            gst128s_stg<0>(d0 + 64 * HTB, vl16, o[2]); gst128s_stg<1024>(d0 + 64 * HTB, vl16, o[3]);
                        }
                        pend = eA[3]; nst = 8;
                        soon = qA >= JP0 && qA < NJ1;
                    } else {
                        const bool bad = gave_up(hvA) || hu[0] >> 8 == POISON || hu[1] >> 8 == POISON;
                        if (!PUB2 && nst == 8) { WAIT_V(8); publish(pend); }
                        if (UNLIKELY(bad)) nan_fill(eA[2]);
                        else {
                            float a[4][8];
#pragma unroll
                            for (int k = 0; k < 2; ++k) {
                                const unsigned h1 = hu[k];
                                if (UNLIKELY(h1 >> 8 == RAWMODE)) {
#pragma unroll
                                    for (int m = 0; m < 4; ++m)
#pragma unroll
                                        for (int j = 0; j < 4; ++j) {
                                            const unsigned v = d[k][m][j];
                                            const float lo = __uint_as_float(v << 16), hi = __uint_as_float(v & 0xffff0000u);
                                            a[m][2 * j] = k == 0 ? lo : a[m][2 * j] + lo; a[m][2 * j + 1] = k == 0 ? hi : a[m][2 * j + 1] + hi;
                                        }
                                } else {
                                    unsigned H[8] = {d[k][2][0], d[k][2][1], d[k][2][2], d[k][2][3], d[k][3][0], d[k][3][1], d[k][3][2], d[k][3][3]};
                                    decode_high(cd[k][0], cd[k][1], ce[k], h1, H);
                                    unsigned vb[4][8];
#pragma unroll
                                    for (int m = 0; m < 4; ++m)
#pragma unroll
                                        for (int j = 0; j < 8; ++j) {
                                            const unsigned Lw = d[k][m >> 1][(m & 1) * 2 + (j >> 2)];
                                            vb[m][j] = vperm(H[2 * m + (j >> 2)], Lw, (4u + (j & 3)) << 24 | (unsigned)(j & 3) << 16 | 0x0c0cu);
                                        }
                                    if (h1 & 0x80u) {
#pragma unroll
                                        for (int m = 0; m < 4; ++m)
#pragma unroll
                                            for (int j = 0; j < 8; ++j) vb[m][j] -= 0x00800000u;
                                    }
#pragma unroll
                                    for (int m = 0; m < 4; ++m)
#pragma unroll
                                        for (int j = 0; j < 8; ++j) a[m][j] = k == 0 ? __uint_as_float(vb[m][j]) : a[m][j] + __uint_as_float(vb[m][j]);
                                }
                            }
#pragma unroll
                            for (int m = 0; m < 4; ++m) {
                                unsigned r0 = cvtpk(a[m][0], a[m][1]), r1 = cvtpk(a[m][2], a[m][3]), r2 = cvtpk(a[m][4], a[m][5]), r3 = cvtpk(a[m][6], a[m][7]);
                                auto s02 = __builtin_amdgcn_permlane16_swap(r0, r2, false, false);
                                auto s13 = __builtin_amdgcn_permlane16_swap(r1, r3, false, false);
                                u32x4 o; o.x = s02[0]; o.y = s13[0]; o.z = s02[1]; o.w = s13[1];
                                st128<OAUX>(o, ro, olane + eA[2], m * 16 * NN * 2);
                            }
                        }
                        STAMPJ(8 + oc, qA - Q_OUT);
                        if (qA == o1) STAMPW(32);
                        nst = 4;
                    }
                    qA = qB; qB = qC; qC = qD; eA = eB; eB = eC; eC = eD; hvA = hvB; hvB = hvC; sl ^= 8192;
                    if (!more) break;
                }
                if (nst == 8) { WAIT_V(0); publish(pend); }
            }
            if (dead) {
                while (qA < Q_END) {
                    if (qA >= Q_OUT) nan_fill(eA[2]);
                    WAIT_V(0);
                    const int qD = next(qC, ticket_value(tk));
                    TICKET(tk, qD < Q_OUT ? tki : tko);
                    qA = qB; qB = qC; qC = qD; eA = eB; eB = eC; eC = entry(qD);
                }
            }
            WAIT_V(0);
        }
        STAMPW(40);
    }
}

}

extern "C" void kernel_launch(int rank, int nranks, ncclComm_t comm,
                              hipStream_t stream,
                              void* const* d_in, const int* in_sizes, int n_in,
                              void* d_out, int out_size,
                              const int* in_dtypes, int out_dtype,
                              void* const* d_out_all,
                              void* const* d_ws_all, size_t ws_bytes) {
    (void)comm; (void)in_dtypes; (void)out_dtype; (void)d_out_all;
    if (nranks != NR || n_in < 2 || in_sizes[0] != MM * KL || in_sizes[1] != KL * NN || out_size != (MM / NR) * NN
        || ws_bytes < WS_NEED || (unsigned)rank >= (unsigned)NR) {
        fprintf(stderr, "kernel_launch: unexpected shapes\n");
        return;
    }
    static bool inited[NR] = {};
    if (!inited[rank]) {
        HIP_CHECK(hipFuncSetAttribute((const void*)gemm_rs_kernel, hipFuncAttributeMaxDynamicSharedMemorySize, LDS_BYTES));
        inited[rank] = true;
    }
    Params p;
    p.x = d_in[0]; p.w = d_in[1]; p.out = d_out; p.rank = rank;
    for (int i = 0; i < NR; ++i) p.ws[i] = (char*)d_ws_all[i];
    hipLaunchKernelGGL(gemm_rs_kernel, dim3(NWG), dim3(NTHR), LDS_BYTES, stream, p);
    HIP_CHECK(hipGetLastError());
}
```
